```python
import numpy as np

kernel_name = "gpu_gemm_ar_m4096_k4096_n4096_bf16_gelu"


ABSMAX_RTOL = 0.008

COMPUTE_DTYPE = "bfloat16"
EPILOGUE = "gelu"
IN_QUANT = None
OUT_QUANT = None
IN_DTYPES = {"x": "bfloat16", "w_mat": "bfloat16"}
WORLD_SIZE = 8

def _to_bf16(x):
    u = np.ascontiguousarray(x, dtype=np.float32).view(np.uint32)
    return ((u + 0x7FFF + ((u >> 16) & 1)) & 0xFFFF0000).view(np.float32)

def setup_inputs():

    x = _to_bf16(np.random.randn(4096, 4096).astype(np.float32))
    w_mat = _to_bf16((np.random.randn(4096, 4096) / np.sqrt(4096)).astype(np.float32))
    return {"x": x, "w_mat": w_mat}

def _epilogue(y):
    c = 0.7978845608028654
    return (0.5 * y * (1.0 + np.tanh(c * (y + 0.044715 * y ** 3)))).astype(np.float32)

def reference(x, w_mat):
    y = (x @ w_mat).astype(np.float32)
    return _epilogue(y)
```

```cpp
#include <hip/hip_runtime.h>
#include <rccl/rccl.h>
#include <cstdio>
#include <cstdint>

#define HIP_CHECK(expr) do { hipError_t _e = (expr); if (_e != hipSuccess) { \
    fprintf(stderr, "HIP error %s at %s:%d\n", hipGetErrorString(_e), __FILE__, __LINE__); \
    return; } } while (0)

typedef unsigned u32;
typedef unsigned long long u64;
typedef u32 u32x4 __attribute__((ext_vector_type(4)));
typedef u32 u32x2 __attribute__((ext_vector_type(2)));
typedef float f32x4 __attribute__((ext_vector_type(4)));
typedef float f32x2 __attribute__((ext_vector_type(2)));
typedef __bf16 bf16x8 __attribute__((ext_vector_type(8)));
typedef __bf16 bf16x2 __attribute__((ext_vector_type(2)));
typedef short s16x4 __attribute__((ext_vector_type(4)));
#define LAS __attribute__((address_space(3)))
#define GAS __attribute__((address_space(1)))

constexpr int NR = 8;
constexpr int NGEMM = 128;
constexpr int PER_LINK = 16;
constexpr int NPULL = 7 * PER_LINK;
constexpr int GRID = NGEMM + NPULL;
constexpr int THREADS = 576;
constexpr int IN_ROUNDS = 12, OUT_ROUNDS = 16, ROUNDS = IN_ROUNDS + OUT_ROUNDS;
constexpr u32 SLOT = 16384 + 512;
constexpr u32 IMG = 16384;
constexpr int NSLOT = 8;
constexpr int WIN = 5;
constexpr unsigned long long DEADLINE = 100000000ull;

constexpr size_t OFF_CTR = 0;
constexpr size_t OFF_TMO = 32768;
constexpr size_t OFF_CNT = 36864;
constexpr size_t OFF_FLAG = 65536;
constexpr size_t OFF_DBG = 768 * 1024;
constexpr size_t OFF_PKIN = 1u << 20;
constexpr size_t OFF_PKOUT = 10u << 20;
constexpr size_t OFF_UX = 16u << 20;
constexpr size_t OFF_UW = 32u << 20;
constexpr size_t WS_NEED = 40u << 20;
static_assert(OFF_FLAG + (size_t)NR * ROUNDS * PER_LINK * 128 <= OFF_DBG, "flags");
static_assert(OFF_PKIN + 512 * (size_t)SLOT <= OFF_PKOUT, "pkin");
static_assert(OFF_PKOUT + 256 * (size_t)SLOT <= OFF_UX, "pkout");

constexpr u32 L_RING = 0;
constexpr u32 L_CTL = NSLOT * SLOT;
constexpr u32 L_FULL = L_CTL;
constexpr u32 L_FREE = L_CTL + 16;
constexpr u32 L_DONE = L_CTL + 16 + 4 * NSLOT;
constexpr u32 L_LEN = L_CTL + 16 + 8 * NSLOT;
constexpr u32 L_GBAD = 131072;
constexpr u32 LDS_BYTES = L_CTL + 256;
static_assert(LDS_BYTES >= 131072 + 64 && LDS_BYTES <= 163840, "lds");

constexpr u32 F_RAW = 0x80000000u;
constexpr u32 F_BAD = 0x40000000u;

struct Params {
    char* ws[NR];
    const char* x;
    const char* w;
    char* out;
    int rank;
    int pad;
};

#define STAMP(id) do {} while (0)

__device__ __forceinline__ void st16_wt(char* p, u32x4 v) {
    asm volatile("global_store_dwordx4 %0, %1, off sc0 sc1\n\ts_nop 1" :: "v"(p), "v"(v) : "memory");
}
__device__ __forceinline__ void vm_wait0() { asm volatile("s_waitcnt vmcnt(0)" ::: "memory"); }
__device__ __forceinline__ void lgkm_wait0() { asm volatile("s_waitcnt lgkmcnt(0)" ::: "memory"); }
__device__ __forceinline__ void bar() { asm volatile("s_waitcnt lgkmcnt(0)\n\ts_barrier" ::: "memory"); }
__device__ __forceinline__ void lds_st(u32 addr, u32 v) { asm volatile("ds_write_b32 %0, %1" :: "v"(addr), "v"(v) : "memory"); }
__device__ __forceinline__ u32 lds_ld(u32 addr) {
    u32 v; asm volatile("ds_read_b32 %0, %1\n\ts_waitcnt lgkmcnt(0)" : "=v"(v) : "v"(addr) : "memory"); return v;
}
__device__ __forceinline__ u32x4 lds_ld16(u32 addr) { return *(const LAS u32x4*)addr; }
__device__ __forceinline__ void dma1(const char* src, u32 lds_dst) {
    u32 keep;
    asm volatile("s_mov_b32 %0, m0\n\ts_mov_b32 m0, %2\n\ts_nop 0\n\tglobal_load_lds_dwordx4 %1, off\n\ts_mov_b32 m0, %0"
                 : "=&s"(keep) : "v"(src), "s"(lds_dst) : "memory");
}
__device__ __forceinline__ void dma1_sys(const char* src, u32 lds_dst) {
    u32 keep;
    asm volatile("s_mov_b32 %0, m0\n\ts_mov_b32 m0, %2\n\ts_nop 0\n\tglobal_load_lds_dwordx4 %1, off sc0 sc1\n\ts_mov_b32 m0, %0"
                 : "=&s"(keep) : "v"(src), "s"(lds_dst) : "memory");
}
__device__ __forceinline__ void dma2(const char* sbase, u32 voff, u32 lds_dst) {
    u32 keep;
    asm volatile("s_mov_b32 %0, m0\n\ts_mov_b32 m0, %3\n\ts_nop 0\n\t"
                 "global_load_lds_dwordx4 %1, %2\n\tglobal_load_lds_dwordx4 %1, %2 offset:1024\n\ts_mov_b32 m0, %0"
                 : "=&s"(keep) : "v"(voff), "s"(sbase), "s"(lds_dst) : "memory");
}

__device__ __forceinline__ float gelu(float y) {
    const float k0 = -2.0f * 0.7978845608028654f * 1.4426950408889634f;
    const float u = y * y;
    const float z = y * __builtin_fmaf(u, k0 * 0.044715f, k0);
    return y * __builtin_amdgcn_rcpf(1.0f + __builtin_amdgcn_exp2f(z));
}
__device__ __forceinline__ u32 pk_bf16(float a, float b) {
    f32x2 f = {a, b};
    return __builtin_bit_cast(u32, __builtin_convertvector(f, bf16x2));
}

__device__ __forceinline__ u32 emit_message(char* slot, u32x4 v0, u32x4 v1, int wave, int lane) {
    char* p = slot + wave * 2048 + lane * 16;
    st16_wt(p, v0);
    st16_wt(p + 1024, v1);
    return F_RAW | IMG;
}

__global__ void __launch_bounds__(THREADS, 1) fused_gemm_allgather(Params P) {
    extern __shared__ __attribute__((aligned(16))) char lds_raw[];
    const int lane = threadIdx.x & 63;
    const int wave = __builtin_amdgcn_readfirstlane(threadIdx.x >> 6);
    const int b = blockIdx.x;
    const int rank = P.rank;
    char* const myws = P.ws[rank];
    const u32 lds0 = (u32)(size_t)(LAS char*)lds_raw;
    GAS u32* const ctr = (GAS u32*)(myws + OFF_CTR + (size_t)b * 128);
    GAS u32* const tmo = (GAS u32*)(myws + OFF_TMO);

    if (b < NGEMM) {
        if (wave == 8) return;
        if (wave == 0) STAMP(0);
        const u32 call = __builtin_amdgcn_readfirstlane(__hip_atomic_load(ctr, __ATOMIC_RELAXED, __HIP_MEMORY_SCOPE_AGENT)) + 1u;
        int m_isw[4], m_grp[4], m_lp[4], m_kb[4], m_seq[4];
#pragma unroll
        for (int z = 0; z < 4; ++z) {
            const int rho = z * 128 + b;
            int isw, grp, lp, kb, t, u;
            if (rho < 384) {
                t = rho / 48; const int r = rho % 48;
                if (r < 16) { isw = 0; grp = r >> 3; kb = r & 7; lp = t; u = kb; }
                else { isw = 1; grp = (r - 16) >> 3; kb = r & 7; lp = t; u = 8 + kb; }
            } else {
                const int r2 = rho - 384; t = 8 + (r2 >> 5); const int r = r2 & 31;
                isw = 0; grp = r >> 4; kb = r & 7; const int pos = (r >> 3) & 1; lp = 8 + 2 * (t - 8) + pos; u = pos * 8 + kb;
            }
            m_isw[z] = isw; m_grp[z] = grp; m_lp[z] = lp; m_kb[z] = kb; m_seq[z] = t * 16 + u;
            const u32 dst = lds0 + z * IMG + wave * 2048;
            if (!isw) {
                const char* s = P.x + ((size_t)((grp * 16 + lp) * 128 + wave * 16 + (lane & 15)) * 512 + kb * 64 + (lane >> 4) * 8) * 2;
                dma1(s, dst); dma1(s + 64, dst + 1024);
            } else {
                const char* s = P.w + ((size_t)(kb * 64 + wave * 8 + (lane >> 4)) * 4096 + (grp * 8 + lp) * 128 + (lane & 15) * 8) * 2;
                dma1(s, dst); dma1(s + 4 * 8192, dst + 1024);
            }
        }
#pragma unroll
        for (int z = 0; z < 4; ++z) {
            if (z == 0) asm volatile("s_waitcnt vmcnt(6)" ::: "memory");
            bar();
            if (z == 0 && threadIdx.x == 0) __hip_atomic_store(ctr, call, __ATOMIC_RELAXED, __HIP_MEMORY_SCOPE_AGENT);
            u32x4 v0, v1;
            const u32 src = lds0 + z * IMG;
            if (!m_isw[z]) {
                v0 = lds_ld16(src + wave * 2048 + lane * 16);
                v1 = lds_ld16(src + wave * 2048 + 1024 + lane * 16);
            } else {
                const u32 a = src + ((lane >> 4) * 8 + ((lane & 15) >> 2)) * 256 + (32 * (wave >> 1) + 8 * (lane & 3) + 4 * (wave & 1)) * 2;
                const s16x4 t0 = __builtin_amdgcn_ds_read_tr16_b64_v4i16((LAS s16x4*)(a));
                const s16x4 t1 = __builtin_amdgcn_ds_read_tr16_b64_v4i16((LAS s16x4*)(a + 4 * 256));
                const s16x4 t2 = __builtin_amdgcn_ds_read_tr16_b64_v4i16((LAS s16x4*)(a + 32 * 256));
                const s16x4 t3 = __builtin_amdgcn_ds_read_tr16_b64_v4i16((LAS s16x4*)(a + 36 * 256));
                const u32x2 a0 = __builtin_bit_cast(u32x2, t0), a1 = __builtin_bit_cast(u32x2, t1);
                const u32x2 a2 = __builtin_bit_cast(u32x2, t2), a3 = __builtin_bit_cast(u32x2, t3);
                v0 = u32x4{a0.x, a0.y, a1.x, a1.y};
                v1 = u32x4{a2.x, a2.y, a3.x, a3.y};
            }
            const bool own = m_isw[z] ? (m_grp[z] == (rank & 3)) : (m_grp[z] == (rank >> 2));
            if (own) {
                char* img = myws + (m_isw[z] ? OFF_UW : OFF_UX) + (size_t)(m_lp[z] * 64 + rank * 8 + m_kb[z]) * IMG + wave * 2048 + lane * 16;
                st16_wt(img, v0); st16_wt(img + 1024, v1);
            }
            const int mid = (m_isw[z] ? 256 + (m_grp[z] * 8 + m_lp[z]) * 8 : (m_grp[z] * 16 + m_lp[z]) * 8) + m_kb[z];
            const u32 lo = emit_message(myws + OFF_PKIN + (size_t)mid * SLOT, v0, v1, wave, lane);
            vm_wait0();
            bar();
            if (wave == 0) {
                const bool dest = lane < NR && lane != rank && (m_isw[z] ? ((lane & 3) == m_grp[z]) : ((lane >> 2) == m_grp[z]));
                if (dest) {
                    GAS u64* f = (GAS u64*)(P.ws[lane & 7] + OFF_FLAG + ((size_t)rank * (ROUNDS * PER_LINK) + m_seq[z]) * 128);
                    __hip_atomic_store(f, ((u64)call << 32) | lo, __ATOMIC_RELAXED, __HIP_MEMORY_SCOPE_SYSTEM);
                }
                if (own && lane == 0)
                    __hip_atomic_fetch_add((GAS u32*)(myws + OFF_CNT + (size_t)((m_isw[z] ? 16 : 0) + m_lp[z]) * 256), 1u,
                                           __ATOMIC_RELAXED, __HIP_MEMORY_SCOPE_AGENT);
                STAMP(1 + z);
            }
        }
        const int ti = b >> 3, tj = b & 7;
        if (wave == 0) {
            GAS u32* c = (GAS u32*)(myws + OFF_CNT + (size_t)((lane & 1) ? 16 + tj : ti) * 256);
            const u32 want = call * 64u;
            const u64 t0 = __builtin_amdgcn_s_memrealtime();
            u32 bad = 0;
            while (!__all(__hip_atomic_load(c, __ATOMIC_RELAXED, __HIP_MEMORY_SCOPE_AGENT) >= want)) {
                if (__builtin_amdgcn_readfirstlane(__hip_atomic_load(tmo, __ATOMIC_RELAXED, __HIP_MEMORY_SCOPE_AGENT)) == call
                    || __builtin_amdgcn_s_memrealtime() - t0 > DEADLINE) {
                    __hip_atomic_store(tmo, call, __ATOMIC_RELAXED, __HIP_MEMORY_SCOPE_AGENT);
                    break;
                }
                __builtin_amdgcn_s_sleep(8);
            }
            __builtin_amdgcn_fence(__ATOMIC_ACQUIRE, "agent");
            bad = __hip_atomic_load(tmo, __ATOMIC_RELAXED, __HIP_MEMORY_SCOPE_AGENT) == call;
            vm_wait0();
            lds_st(lds0 + L_GBAD, bad);
            STAMP(5);
        }
        bar();
        const int kh = wave >> 2, wn = wave & 1, wm2 = (wave >> 1) & 1;
        const char* const xb = myws + OFF_UX + (size_t)ti * 64 * IMG;
        const char* const wb = myws + OFF_UW + (size_t)tj * 64 * IMG;
        const u32 voff = wave * 2048 + lane * 16;
        const u32 ldst = lds0 + wave * 2048;
        auto stage = [&](int kt) {
            const u32 kb = (u32)(rank * 8 + kt) & 63u;
            const u32 d = ldst + (kt & 3) * (2 * IMG);
            dma2(xb + (size_t)kb * IMG, voff, d);
            dma2(wb + (size_t)kb * IMG, voff, d + IMG);
        };
        f32x4 acc[4][4];
#pragma unroll
        for (int a = 0; a < 4; ++a)
#pragma unroll
            for (int c = 0; c < 4; ++c) acc[a][c] = f32x4{0.f, 0.f, 0.f, 0.f};
        const u32 xa = lds0 + (wm2 * 8 + kh) * 1024 + lane * 16;
        const u32 wa = lds0 + IMG + (wn * 8 + kh) * 1024 + lane * 16;
        auto compute = [&](int s) {
            bf16x8 wf[4], xf[4];
#pragma unroll
            for (int a = 0; a < 4; ++a) {
                wf[a] = *(const LAS bf16x8*)(wa + s * (2 * IMG) + a * 2048);
                xf[a] = *(const LAS bf16x8*)(xa + s * (2 * IMG) + a * 2048);
            }
#pragma unroll
            for (int a = 0; a < 4; ++a)
#pragma unroll
                for (int c = 0; c < 4; ++c)
                    acc[a][c] = __builtin_amdgcn_mfma_f32_16x16x32_bf16(wf[a], xf[c], acc[a][c], 0, 0, 0);
        };
        stage(0); stage(1); stage(2);
        for (int kt = 0; kt < 60; kt += 4) {
#pragma unroll
            for (int s = 0; s < 4; ++s) {
                asm volatile("s_waitcnt vmcnt(8)\n\ts_barrier" ::: "memory");
                stage(kt + s + 3);
                compute(s);
                lgkm_wait0();
            }
        }
        asm volatile("s_waitcnt vmcnt(8)\n\ts_barrier" ::: "memory");
        stage(63); compute(0); lgkm_wait0();
        asm volatile("s_waitcnt vmcnt(8)\n\ts_barrier" ::: "memory");
        compute(1); lgkm_wait0();
        asm volatile("s_waitcnt vmcnt(4)\n\ts_barrier" ::: "memory");
        compute(2); lgkm_wait0();
        asm volatile("s_waitcnt vmcnt(0)\n\ts_barrier" ::: "memory");
        compute(3);
        if (wave == 0) STAMP(6);
        {
            const u32 wr = lds0 + wave * 8192 + lane * 16;
#pragma unroll
            for (int a = 0; a < 4; ++a)
#pragma unroll
                for (int c = 0; c < 2; ++c)
                    *(LAS f32x4*)(wr + (a * 2 + c) * 1024) = kh ? acc[a][c] : acc[a][2 + c];
        }
        bar();
        const u32 bad = lds_ld(lds0 + L_GBAD);
        u32x4 ch[2][2];
        {
            const u32 rd = lds0 + (wave ^ 4) * 8192 + lane * 16;
#pragma unroll
            for (int c = 0; c < 2; ++c) {
                u32 o[8];
#pragma unroll
                for (int a = 0; a < 4; ++a) {
                    const f32x4 other = *(const LAS f32x4*)(rd + (a * 2 + c) * 1024);
                    const f32x4 y = (kh ? acc[a][2 + c] : acc[a][c]) + other;
                    o[a * 2] = pk_bf16(gelu(y.x), gelu(y.y));
                    o[a * 2 + 1] = pk_bf16(gelu(y.z), gelu(y.w));
                }
                ch[c][0] = u32x4{o[0], o[1], o[2], o[3]};
                ch[c][1] = u32x4{o[4], o[5], o[6], o[7]};
            }
        }
        if (bad) {
            const u32x4 nanv = {0x7fc07fc0u, 0x7fc07fc0u, 0x7fc07fc0u, 0x7fc07fc0u};
            ch[0][0] = nanv; ch[0][1] = nanv; ch[1][0] = nanv; ch[1][1] = nanv;
        }
        const int kmax = ti > tj ? ti : tj;
        const int ord = ti >= 8 ? 64 + (ti - 8) * 8 + tj : kmax * kmax + (ti == kmax ? tj : kmax + 1 + ti);
        u32 lo[2];
#pragma unroll
        for (int c = 0; c < 2; ++c)
            lo[c] = emit_message(myws + OFF_PKOUT + (size_t)(ord * 2 + c) * SLOT, ch[c][0], ch[c][1], wave, lane);
        vm_wait0();
        bar();
        if (wave == 0 && lane < 16 && (lane & 7) != rank) {
            const int c = lane >> 3;
            GAS u64* f = (GAS u64*)(P.ws[lane & 7] + OFF_FLAG + ((size_t)rank * (ROUNDS * PER_LINK) + IN_ROUNDS * PER_LINK + ord * 2 + c) * 128);
            __hip_atomic_store(f, ((u64)call << 32) | (c ? lo[1] : lo[0]), __ATOMIC_RELAXED, __HIP_MEMORY_SCOPE_SYSTEM);
        }
        {
            char* o = P.out + ((size_t)((rank >> 2) * 2048 + ti * 128 + wm2 * 64 + kh * 32 + (lane & 15)) * 4096
                               + (rank & 3) * 1024 + tj * 128 + wn * 64 + (lane >> 4) * 8) * 2;
#pragma unroll
            for (int c = 0; c < 2; ++c) {
                st16_wt(o + (size_t)c * 16 * 8192, ch[c][0]);
                st16_wt(o + (size_t)c * 16 * 8192 + 64, ch[c][1]);
            }
        }
        if (wave == 0) STAMP(7);
    } else {
        const int pu = b - NGEMM;
        const int u = pu / 7;
        const int src = (rank + 1 + pu % 7) & 7;
        const u32 call = __builtin_amdgcn_readfirstlane(__hip_atomic_load(ctr, __ATOMIC_RELAXED, __HIP_MEMORY_SCOPE_AGENT)) + 1u;
        if (wave == 0) lds_st(lds0 + L_CTL + lane * 4, 0u);
        if (wave == 8) STAMP(0);
        bar();
        if (threadIdx.x == 0) __hip_atomic_store(ctr, call, __ATOMIC_RELAXED, __HIP_MEMORY_SCOPE_AGENT);
        if (wave == 8) {
            vm_wait0();
            const char* const rws = P.ws[src];
            const char* const fp = myws + OFF_FLAG + ((size_t)src * (ROUNDS * PER_LINK) + (lane < ROUNDS ? lane : 0) * PER_LINK + u) * 128;
            u32x2 fl = {0u, 0u};
            bool dead = false;
            for (int t = 0; t < ROUNDS; ++t) {
                const int s = t % NSLOT;
                if (!dead && (u32)__builtin_amdgcn_readlane(fl.y, t) < call) {
                    vm_wait0();
                    lds_st(lds0 + L_FULL, t);
                    const u64 t0 = __builtin_amdgcn_s_memrealtime();
                    for (;;) {
                        asm volatile("global_load_dwordx2 %0, %1, off sc0 sc1\n\ts_waitcnt vmcnt(0)" : "=&v"(fl) : "v"(fp) : "memory");
                        if ((u32)__builtin_amdgcn_readlane(fl.y, t) >= call) break;
                        if (__builtin_amdgcn_readfirstlane(__hip_atomic_load(tmo, __ATOMIC_RELAXED, __HIP_MEMORY_SCOPE_AGENT)) == call
                            || __builtin_amdgcn_s_memrealtime() - t0 > DEADLINE) {
                            __hip_atomic_store(tmo, call, __ATOMIC_RELAXED, __HIP_MEMORY_SCOPE_AGENT);
                            dead = true;
                            break;
                        }
                        __builtin_amdgcn_s_sleep(2);
                    }
                }
                if (t >= NSLOT) {
                    const u32 need = 8u * (u32)(t / NSLOT);
                    if (__builtin_amdgcn_readfirstlane(lds_ld(lds0 + L_FREE + s * 4)) < need) {
                        vm_wait0();
                        lds_st(lds0 + L_FULL, t);
                        while (__builtin_amdgcn_readfirstlane(lds_ld(lds0 + L_FREE + s * 4)) < need) __builtin_amdgcn_s_sleep(1);
                    }
                }
                if (dead) {
                    lds_st(lds0 + L_LEN + s * 4, F_BAD);
                    lds_st(lds0 + L_FULL, t + 1);
                    continue;
                }
                const u32 lo = __builtin_amdgcn_readlane(fl.x, t);
                u32 len = lo & 0xffffu;
                len = len > SLOT ? SLOT : (len < 8192u ? 8192u : len);
                lds_st(lds0 + L_LEN + s * 4, lo);
                size_t moff;
                if (t < IN_ROUNDS) {
                    const int kb = u & 7;
                    int mid;
                    if (t < 8) mid = u < 8 ? ((rank >> 2) * 16 + t) * 8 + kb : 256 + ((rank & 3) * 8 + t) * 8 + kb;
                    else mid = ((rank >> 2) * 16 + 8 + 2 * (t - 8) + (u >> 3)) * 8 + kb;
                    moff = OFF_PKIN + (size_t)mid * SLOT;
                } else {
                    moff = OFF_PKOUT + (size_t)((t - IN_ROUNDS) * PER_LINK + u) * SLOT;
                }
                const char* sp = rws + moff + lane * 16;
                const u32 d = lds0 + L_RING + s * SLOT;
                u32x2 fn;
                asm volatile("global_load_dwordx2 %0, %1, off sc0 sc1" : "=v"(fn) : "v"(fp) : "memory");
#pragma unroll
                for (int k = 0; k < WIN; ++k) {
                    dma1_sys(sp + k * 1024, d + k * 1024);
                    asm volatile("s_waitcnt vmcnt(%0)" :: "n"(WIN - 1) : "memory");
                }
                asm volatile("" : "+v"(fn) :: "memory");
                fl = fn;
                lds_st(lds0 + L_FULL, t);
                for (u32 o = WIN * 1024; o < len; o += 1024) {
                    if (o + lane * 16 < len) dma1_sys(sp + o, d + o);
                    asm volatile("s_waitcnt vmcnt(%0)" :: "n"(WIN - 1) : "memory");
                }
            }
            vm_wait0();
            lds_st(lds0 + L_FULL, ROUNDS);
            STAMP(1);
        } else {
            const int kh = wave >> 2, wn = wave & 1, wm2 = (wave >> 1) & 1;
            for (int t = 0; t < ROUNDS; ++t) {
                const int s = t % NSLOT;
                while (__builtin_amdgcn_readfirstlane(lds_ld(lds0 + L_FULL)) <= (u32)t) __builtin_amdgcn_s_sleep(1);
                const u32 lo = __builtin_amdgcn_readfirstlane(lds_ld(lds0 + L_LEN + s * 4));
                const u32 sl = lds0 + L_RING + s * SLOT;
                u32x4 v0, v1;
                if (lo & F_BAD) {
                    v0 = u32x4{0x7fc07fc0u, 0x7fc07fc0u, 0x7fc07fc0u, 0x7fc07fc0u}; v1 = v0;
                } else {
                    v0 = lds_ld16(sl + wave * 2048 + lane * 16);
                    v1 = lds_ld16(sl + wave * 2048 + 1024 + lane * 16);
                }
                asm volatile("s_waitcnt lgkmcnt(0)" : "+v"(v0), "+v"(v1) :: "memory");
                if (lane == 0) asm volatile("ds_add_u32 %0, %1" :: "v"(lds0 + L_FREE + s * 4), "v"(1u) : "memory");
                if (t < IN_ROUNDS) {
                    const int kb = u & 7;
                    const bool isw = t < 8 && u >= 8;
                    const int lp = t < 8 ? t : 8 + 2 * (t - 8) + (u >> 3);
                    char* img = myws + (isw ? OFF_UW : OFF_UX) + (size_t)(lp * 64 + src * 8 + kb) * IMG + wave * 2048 + lane * 16;
                    st16_wt(img, v0); st16_wt(img + 1024, v1);
                    vm_wait0();
                    if (lane == 0) {
                        u32 old;
                        asm volatile("ds_add_rtn_u32 %0, %1, %2\n\ts_waitcnt lgkmcnt(0)" : "=&v"(old) : "v"(lds0 + L_DONE + s * 4), "v"(1u) : "memory");
                        if ((old & 7u) == 7u)
                            __hip_atomic_fetch_add((GAS u32*)(myws + OFF_CNT + (size_t)((isw ? 16 : 0) + lp) * 256), 1u,
                                                   __ATOMIC_RELAXED, __HIP_MEMORY_SCOPE_AGENT);
                    }
                } else {
                    const int o = (t - IN_ROUNDS) * PER_LINK + u, n = o >> 1, c = o & 1;
                    int ti, tj;
                    if (n < 64) {
                        int k = 0;
                        while ((k + 1) * (k + 1) <= n) ++k;
                        const int r = n - k * k;
                        ti = r <= k ? k : r - k - 1; tj = r <= k ? r : k;
                    } else { ti = 8 + ((n - 64) >> 3); tj = (n - 64) & 7; }
                    char* op = P.out + ((size_t)((src >> 2) * 2048 + ti * 128 + wm2 * 64 + kh * 32 + c * 16 + (lane & 15)) * 4096
                                        + (src & 3) * 1024 + tj * 128 + wn * 64 + (lane >> 4) * 8) * 2;
                    st16_wt(op, v0); st16_wt(op + 64, v1);
                }
            }
            if (wave == 0) { STAMP(2); }
        }
    }
}

extern "C" void kernel_launch(int rank, int nranks, ncclComm_t comm,
                              hipStream_t stream,
                              void* const* d_in, const int* in_sizes, int n_in,
                              void* d_out, int out_size,
                              const int* in_dtypes, int out_dtype,
                              void* const* d_out_all,
                              void* const* d_ws_all, size_t ws_bytes) {
    (void)comm; (void)in_dtypes; (void)out_dtype; (void)d_out_all;
    if (nranks != NR || (unsigned)rank >= (unsigned)NR || n_in != 2 || in_sizes[0] != 4096 * 512 || in_sizes[1] != 512 * 4096
        || out_size != 4096 * 4096 || ws_bytes < WS_NEED) {
        fprintf(stderr, "kernel_launch: unexpected problem shape\n");
        return;
    }
    static bool attr_set[NR] = {};
    if (!attr_set[rank]) {
        HIP_CHECK(hipFuncSetAttribute((const void*)fused_gemm_allgather, hipFuncAttributeMaxDynamicSharedMemorySize, LDS_BYTES));
        attr_set[rank] = true;
    }
    Params P;
    for (int p = 0; p < NR; ++p) P.ws[p] = (char*)d_ws_all[p];
    P.x = (const char*)d_in[0];
    P.w = (const char*)d_in[1];
    P.out = (char*)d_out;
    P.rank = rank;
    P.pad = 0;
    hipLaunchKernelGGL(fused_gemm_allgather, dim3(GRID), dim3(THREADS), LDS_BYTES, stream, P);
    HIP_CHECK(hipGetLastError());
}
```

```cpp
#include <hip/hip_runtime.h>
#include <rccl/rccl.h>
#include <cstdio>
#include <cstdint>
#include <type_traits>
#pragma clang diagnostic ignored "-Wint-to-pointer-cast"

#define HIP_CHECK(expr) do { hipError_t _e = (expr); if (_e != hipSuccess) { \
    fprintf(stderr, "HIP error %s at %s:%d\n", hipGetErrorString(_e), __FILE__, __LINE__); \
    return; } } while (0)

typedef unsigned u32;
typedef unsigned long long u64;
typedef u32 u32x4 __attribute__((ext_vector_type(4)));
typedef u32 u32x2 __attribute__((ext_vector_type(2)));
typedef float f32x4 __attribute__((ext_vector_type(4)));
typedef float f32x2 __attribute__((ext_vector_type(2)));
typedef __bf16 bf16x8 __attribute__((ext_vector_type(8)));
typedef __bf16 bf16x2 __attribute__((ext_vector_type(2)));
typedef short s16x4 __attribute__((ext_vector_type(4)));
#define LAS __attribute__((address_space(3)))
#define GAS __attribute__((address_space(1)))

constexpr int NR = 8;
constexpr int NGEMM = 128;
constexpr int PER_LINK = 16;
constexpr int NPULL = 7 * PER_LINK;
#define MAP_ 1
constexpr int NROLE = NGEMM + NPULL;
constexpr int GRID = MAP_ ? 256 : NROLE;
constexpr int THREADS = 576;
constexpr int IN_ROUNDS = 12, OUT_ROUNDS = 16, ROUNDS = IN_ROUNDS + OUT_ROUNDS;
constexpr u32 NMSG = ROUNDS * PER_LINK;
constexpr u32 SLOT = 16384 + 512;
constexpr u32 IMG = 16384;
constexpr int NSLOT = 8;
#define WIN_ 4
constexpr int WIN = WIN_;
constexpr unsigned long long DEADLINE = 100000000ull;

constexpr size_t OFF_CTR = 0;
constexpr size_t OFF_TMO = 32768;
constexpr size_t OFF_CNT = 36864;
constexpr size_t OFF_TKT = 45056;
constexpr size_t OFF_FLAG = 65536;
constexpr size_t OFF_DBG = 768 * 1024;
constexpr size_t OFF_PKIN = 1u << 20;
constexpr size_t OFF_PKOUT = 10u << 20;
constexpr size_t OFF_UX = 16u << 20;
constexpr size_t OFF_UW = 32u << 20;
constexpr size_t WS_NEED = 40u << 20;
static_assert(OFF_FLAG + (size_t)NR * ROUNDS * PER_LINK * 128 <= OFF_DBG, "flags");
static_assert(OFF_PKIN + 512 * (size_t)SLOT <= OFF_PKOUT, "pkin");
static_assert(OFF_PKOUT + 256 * (size_t)SLOT <= OFF_UX, "pkout");

constexpr u32 L_RING = 0;
constexpr u32 L_CTL = NSLOT * SLOT;
constexpr u32 L_FULL = L_CTL;
constexpr u32 L_FREE = L_CTL + 16;
constexpr u32 L_DONE = L_CTL + 16 + 4 * NSLOT;
constexpr u32 L_LEN = L_CTL + 16 + 8 * NSLOT;
constexpr u32 L_MSG = L_CTL + 16 + 12 * NSLOT;
constexpr u32 L_DTBL = L_CTL + 256;
constexpr u32 L_UNORD = L_DTBL + 512;
constexpr int NST = 6;
constexpr u32 STG = 8192 + IMG;
constexpr u32 L_VAR = NST * STG;
constexpr u32 L_TOT = L_VAR + 8192;
constexpr u32 L_ETBL = L_TOT + 64;
constexpr u32 LDS_BYTES = L_ETBL + 1024;
static_assert(LDS_BYTES >= L_UNORD + 256 && LDS_BYTES <= 163840 && 5 * IMG <= L_VAR, "lds");

constexpr int pack_passes(int b) {
    return (b >> 4) <= 1 && (b & 7) <= 1 ? 2 : ((b & 7) == 7 ? 5 : 4);
}
struct Sched { alignas(4) unsigned char unord[256]; unsigned short rho[128][5]; unsigned char ord[2][128]; };
constexpr Sched make_sched() {
    Sched s{};
    int r = 0;
    for (int z = 0; z < 5; ++z)
        for (int b = 0; b < 128; ++b) s.rho[b][z] = pack_passes(b) > z ? (unsigned short)r++ : (unsigned short)0xffff;
    int key[256] = {};
    for (int b = 0; b < 128; ++b) {
        const int i = b >> 4, j = b & 7, sh = i > j ? i : j;
        const int t1 = 30 * sh + 170, arr2 = 30 * (8 + (i >> 1));
        key[b] = t1 * 256 + b;
        key[128 + b] = ((t1 > arr2 ? t1 : arr2) + 170) * 256 + 128 + b;
    }
    for (int a = 0; a < 256; ++a) {
        int pos = 0;
        for (int c = 0; c < 256; ++c) pos += key[c] < key[a];
        s.ord[a >> 7][a & 127] = (unsigned char)pos;
        s.unord[pos] = (unsigned char)a;
    }
    return s;
}
__device__ constexpr Sched SCHED = make_sched();

constexpr u32 F_RAW = 0x80000000u;
constexpr u32 F_BAD = 0x40000000u;

struct Params {
    char* ws[NR];
    const char* x;
    const char* w;
    char* out;
    int rank;
    int pad;
};

#define STAMP(id) do {} while (0)

__device__ __forceinline__ void st16_wt(char* p, u32x4 v) {
    asm volatile("global_store_dwordx4 %0, %1, off sc0 sc1\n\ts_nop 1" :: "v"(p), "v"(v) : "memory");
}
__device__ __forceinline__ void vm_wait0() { asm volatile("s_waitcnt vmcnt(0)" ::: "memory"); }
__device__ __forceinline__ void lgkm_wait0() { asm volatile("s_waitcnt lgkmcnt(0)" ::: "memory"); }
__device__ __forceinline__ void bar() { asm volatile("s_waitcnt lgkmcnt(0)\n\ts_barrier" ::: "memory"); }
__device__ __forceinline__ void lds_st(u32 addr, u32 v) { asm volatile("ds_write_b32 %0, %1" :: "v"(addr), "v"(v) : "memory"); }
__device__ __forceinline__ u32 lds_ld(u32 addr) {
    u32 v; asm volatile("ds_read_b32 %0, %1\n\ts_waitcnt lgkmcnt(0)" : "=v"(v) : "v"(addr) : "memory"); return v;
}
__device__ __forceinline__ u32x4 lds_ld16(u32 addr) { return *(const LAS u32x4*)addr; }
template <int N> __device__ __forceinline__ u32 landed_first(u32 v) {
    u32 r; asm volatile("s_waitcnt vmcnt(%2)\n\tv_readfirstlane_b32 %0, %1\n\ts_nop 3" : "=s"(r) : "v"(v), "n"(N) : "memory"); return r;
}
__device__ __forceinline__ void dma1(const char* src, u32 lds_dst) {
    u32 keep;
    asm volatile("s_mov_b32 %0, m0\n\ts_mov_b32 m0, %2\n\ts_nop 0\n\tglobal_load_lds_dwordx4 %1, off\n\ts_mov_b32 m0, %0"
                 : "=&s"(keep) : "v"(src), "s"(lds_dst) : "memory");
}
__device__ __forceinline__ void dma1_sys(const char* src, u32 lds_dst) {
    u32 keep;
    asm volatile("s_mov_b32 %0, m0\n\ts_mov_b32 m0, %2\n\ts_nop 0\n\tglobal_load_lds_dwordx4 %1, off sc0 sc1\n\ts_mov_b32 m0, %0"
                 : "=&s"(keep) : "v"(src), "s"(lds_dst) : "memory");
}
__device__ __forceinline__ void dma_a(const char* sbase, u32 voff, u32 lds_dst) {
    asm volatile("s_mov_b32 m0, %2\n\ts_nop 0\n\tglobal_load_lds_dwordx4 %0, %1 sc0 sc1" :: "v"(voff), "s"(sbase), "s"(lds_dst) : "memory");
}
__device__ __forceinline__ void dma_b(const char* sbase, u32 voff) {
    asm volatile("global_load_lds_dwordx4 %0, %1 offset:1024 sc0 sc1" :: "v"(voff), "s"(sbase) : "memory");
}
__device__ __forceinline__ void wait_counters(GAS u32* c, u32 want, GAS u32* tmo, u32 call) {
    const u64 t0 = __builtin_amdgcn_s_memrealtime();
    for (u32 it = 1; !__all(__hip_atomic_load(c, __ATOMIC_RELAXED, __HIP_MEMORY_SCOPE_AGENT) >= want); ++it) {
        if ((it & 15u) == 0u
            && (__builtin_amdgcn_readfirstlane(__hip_atomic_load(tmo, __ATOMIC_RELAXED, __HIP_MEMORY_SCOPE_AGENT)) == call
                || __builtin_amdgcn_s_memrealtime() - t0 > DEADLINE)) {
            __hip_atomic_store(tmo, call, __ATOMIC_RELAXED, __HIP_MEMORY_SCOPE_AGENT);
            break;
        }
        __builtin_amdgcn_s_sleep(2);
    }
    asm volatile("s_waitcnt vmcnt(0)" ::: "memory");
}

__device__ __forceinline__ float gelu(float y) {
    const float k0 = -2.0f * 0.7978845608028654f * 1.4426950408889634f;
    const float u = y * y;
    const float z = y * __builtin_fmaf(u, k0 * 0.044715f, k0);
    return y * __builtin_amdgcn_rcpf(1.0f + __builtin_amdgcn_exp2f(z));
}
__device__ __forceinline__ u32 pk_bf16(float a, float b) {
    f32x2 f = {a, b};
    return __builtin_bit_cast(u32, __builtin_convertvector(f, bf16x2));
}

struct CodeTables { u32 code[2][64]; u32 inc[2][64]; u32 dec[2][64]; };
constexpr u32 code_entry(int c1, int q, bool term) {
    if (term) return (u32)c1 << 30;
    const int c2 = q < 3 ? q : 3, c3 = q < 3 ? 0 : (q < 6 ? q - 3 : 3), c4 = q < 6 ? 0 : q - 6;
    return ((u32)c1 << 30) | (u32)c2 | ((u32)c3 << 8) | ((u32)c4 << 16);
}
constexpr u32 inc_entry(int q, bool term) {
    return term ? 0u : 2u | (q >= 3 ? 2u << 8 : 0u) | (q >= 6 ? 3u << 16 : 0u);
}
constexpr CodeTables make_tables() {
    CodeTables t{};
    const int ord[2][13] = {{12, 11, 10, 9, 13, 8, 7, 6, 5, 4, 3, 2, 1}, {11, 10, 12, 9, 8, 7, 6, 5, 4, 13, 3, 2, 1}};
    for (int i = 0; i < 64; ++i) {
        t.code[0][i] = code_entry(0, 13, false); t.code[1][i] = code_entry(3, 13, false);
        t.inc[0][i] = inc_entry(13, false) | (1u << 24); t.inc[1][i] = t.inc[0][i];
        t.dec[0][i] = 1u; t.dec[1][i] = 1u;
    }
    for (int s = 0; s < 2; ++s) for (int m = 0; m < 2; ++m) for (int q = 0; q < 13; ++q) {
        t.code[0][(s << 5) | (ord[m][q] << 1) | m] = code_entry(0, q, false);
        t.inc[0][(s << 5) | (ord[m][q] << 1) | m] = inc_entry(q, false);
        t.dec[0][(((s << 1) | m) << 4) | q] = ((u32)s << 15) | ((u32)ord[m][q] << 7) | ((u32)m << 6);
    }
    const int rk[42] = {52, 51, 24, 26, 50, 25, 22, 23, 27, 20, 21, 49, 48, 18, 28, 19, 16, 47, 17, 46, 14, 15, 45, 44, 12, 13,
                        43, 42, 11, 10, 29, 41, 40, 9, 8, 39, 38, 7, 6, 37, 36, 5};
    for (int r = 0; r < 42; ++r) {
        const int c1 = r == 0 ? 0 : 1 + (r - 1) % 3, q = r == 0 ? 0 : (r - 1) / 3;
        const u32 sym = ((u32)(rk[r] >> 5) << 15) | ((u32)((rk[r] >> 1) & 15) << 7) | ((u32)(rk[r] & 1) << 6);
        t.code[1][rk[r]] = code_entry(c1, q, r == 0);
        t.inc[1][rk[r]] = inc_entry(q, r == 0);
        if (r == 0) for (int j = 0; j < 16; ++j) t.dec[1][j] = sym;
        else t.dec[1][(c1 << 4) | q] = sym;
    }
    return t;
}
__device__ constexpr CodeTables TBL = make_tables();
constexpr u32 OUT_E0 = 114;

__device__ __forceinline__ u32 wave_scan(u32 v) {
    v += (u32)__builtin_amdgcn_update_dpp(0, (int)v, 0x111, 0xf, 0xf, false);
    v += (u32)__builtin_amdgcn_update_dpp(0, (int)v, 0x112, 0xf, 0xf, false);
    v += (u32)__builtin_amdgcn_update_dpp(0, (int)v, 0x114, 0xf, 0xf, false);
    v += (u32)__builtin_amdgcn_update_dpp(0, (int)v, 0x118, 0xf, 0xf, false);
    v += (u32)__builtin_amdgcn_update_dpp(0, (int)v, 0x142, 0xa, 0xf, false);
    v += (u32)__builtin_amdgcn_update_dpp(0, (int)v, 0x143, 0xc, 0xf, false);
    return v;
}
__device__ __forceinline__ void put_enc_table(u32 l, int kind, int lane) { *(LAS u32x2*)(l + lane * 8) = u32x2{TBL.code[kind][lane], TBL.inc[kind][lane]}; }
__device__ __forceinline__ void put_dec_table(u32 l, int kind, int lane) {
    *(LAS unsigned short*)(l + (lane >> 4) * 64 + (lane & 15) * 2) = (unsigned short)TBL.dec[kind][lane];
}
__device__ __forceinline__ u32 ubfe(u32 v, u32 off, u32 w) { return __builtin_amdgcn_ubfe(v, off, w); }
__device__ __forceinline__ void lds_or_bits(u32 base, u32 pos, u32 bits) {
    const u64 x = (u64)bits << (pos & 31u);
    LAS u32* p = (LAS u32*)(base + (pos >> 5) * 4u);
    if ((u32)x) __hip_atomic_fetch_or(p, (u32)x, __ATOMIC_RELAXED, __HIP_MEMORY_SCOPE_WORKGROUP);
    if ((u32)(x >> 32)) __hip_atomic_fetch_or(p + 1, (u32)(x >> 32), __ATOMIC_RELAXED, __HIP_MEMORY_SCOPE_WORKGROUP);
}
__device__ __forceinline__ u32 lds_get_bits(u32 base, u32 pos) {
    const LAS u32* p = (const LAS u32*)(base + (pos >> 5) * 4u);
    return __builtin_amdgcn_alignbit(p[1], p[0], pos & 31u);
}
__device__ __forceinline__ u32 low_mask_even(u32 n) { return ~((0xffffffffu << (n >> 1)) << (n >> 1)); }

#define SDWA_SHL(B) \
    __device__ __forceinline__ u32 shl_byte##B(u32 sh, u32 v) {     \
        u32 r; asm("v_lshlrev_b32_sdwa %0, %1, %2 dst_sel:DWORD dst_unused:UNUSED_PAD src0_sel:BYTE_" #B " src1_sel:BYTE_" #B \
                   : "=v"(r) : "v"(sh), "v"(v)); return r; }
SDWA_SHL(0) SDWA_SHL(1) SDWA_SHL(2)

template <int B> __device__ __forceinline__ u32 and_byte(u32 v, u32 m) {
    u32 r;
    if constexpr (B == 0) asm("v_and_b32_sdwa %0, %1, %2 dst_sel:DWORD dst_unused:UNUSED_PAD src0_sel:BYTE_0 src1_sel:DWORD" : "=v"(r) : "v"(v), "v"(m));
    if constexpr (B == 1) asm("v_and_b32_sdwa %0, %1, %2 dst_sel:DWORD dst_unused:UNUSED_PAD src0_sel:BYTE_1 src1_sel:DWORD" : "=v"(r) : "v"(v), "v"(m));
    if constexpr (B == 2) asm("v_and_b32_sdwa %0, %1, %2 dst_sel:DWORD dst_unused:UNUSED_PAD src0_sel:BYTE_2 src1_sel:DWORD" : "=v"(r) : "v"(v), "v"(m));
    if constexpr (B == 3) asm("v_and_b32_sdwa %0, %1, %2 dst_sel:DWORD dst_unused:UNUSED_PAD src0_sel:BYTE_3 src1_sel:DWORD" : "=v"(r) : "v"(v), "v"(m));
    return r;
}

struct EncBlock { u32 bb[4], f2, f3, f4a, f4b, n4a, pos, sa, sb, N2, N3, escpos, NE, e0, esc[4]; };
template <bool OUT>
__device__ __forceinline__ void encode_block(EncBlock& E, const u32 (&d)[8], u32 ltbl) {
    u32 e0 = OUT_E0;
    if (!OUT) {
        const u32 s2 = ((d[0] >> 7) & 0x00ff00ffu) + ((d[5] >> 7) & 0x00ff00ffu);
        const u32 tot = __builtin_amdgcn_readlane(wave_scan((s2 & 0xffffu) + (s2 >> 16)), 63);
        const u32 anchor = (tot + 504u) >> 8;
        e0 = anchor < 12u ? 0u : (anchor > 252u ? 240u : anchor - 12u);
    }
    typedef unsigned short u16x2 __attribute__((ext_vector_type(2)));
    const u16x2 sub = {(unsigned short)(e0 * 16u), (unsigned short)(e0 * 16u)}, cap = {240, 240};
    u32 en[16], in[16], ix[8];
#pragma unroll
    for (int j = 0; j < 8; ++j) {
        const u16x2 t = __builtin_elementwise_min(__builtin_bit_cast(u16x2, (d[j] >> 3) & 0x0ff80ff8u) - sub, cap);
        ix[j] = __builtin_bit_cast(u32, t) | ((d[j] >> 7) & 0x01000100u);
    }
#pragma unroll
    for (int j = 0; j < 8; ++j) {
        const u32x2 a = *(const LAS u32x2*)(ltbl + (ix[j] & 0xffffu)), b = *(const LAS u32x2*)(ltbl + (ix[j] >> 16));
        en[2 * j] = a.x; in[2 * j] = a.y; en[2 * j + 1] = b.x; in[2 * j + 1] = b.y;
    }
    __builtin_amdgcn_sched_barrier(0);
    u32 f2 = 0, f3 = 0, f4[2] = {0, 0}, pos = 0, n4a = 0;
#pragma unroll
    for (int i = 0; i < 16; i += 2) {
        if (i == 8) { n4a = ubfe(pos, 16, 8); pos &= 0xff00ffffu; }
        const u32 pos1 = pos + in[i];
        if (OUT) f2 |= shl_byte0(pos, en[i]) | shl_byte0(pos1, en[i + 1]);
        else f2 |= ((en[i] & 3u) << (2 * i)) | ((en[i + 1] & 3u) << (2 * i + 2));
        f3 |= shl_byte1(pos, en[i]) | shl_byte1(pos1, en[i + 1]);
        f4[i >> 3] |= shl_byte2(pos, en[i]) | shl_byte2(pos1, en[i + 1]);
        pos = pos1 + in[i + 1];
    }
#pragma unroll
    for (int j = 0; j < 4; ++j) {
        if (OUT) {
            const u32 lowb = __builtin_amdgcn_perm(d[2 * j + 1], d[2 * j], 0x06040200u);
            const u32 c01 = __builtin_amdgcn_perm(en[4 * j + 1], en[4 * j], 0x0c0c0703u);
            const u32 c23 = __builtin_amdgcn_perm(en[4 * j + 3], en[4 * j + 2], 0x07030c0cu);
            E.bb[j] = (lowb & 0x3f3f3f3fu) | c01 | c23;
        } else {
            const u32 lowb = __builtin_amdgcn_perm(d[2 * j + 1], d[2 * j], 0x06040200u);
            const u32 higb = __builtin_amdgcn_perm(d[2 * j + 1], d[2 * j], 0x07050301u);
            E.bb[j] = (lowb & 0x7f7f7f7fu) | (higb & 0x80808080u);
        }
        E.esc[j] = __builtin_amdgcn_perm(in[4 * j + 1], in[4 * j], 0x0c0c0703u) | __builtin_amdgcn_perm(in[4 * j + 3], in[4 * j + 2], 0x07030c0cu);
    }
    const u32 n2 = pos & 0xffu, n3 = ubfe(pos, 8, 8), n4 = n4a + ubfe(pos, 16, 8), ne = pos >> 24;
    const u32 sa = wave_scan(n2 | (n3 << 16)), sb = wave_scan(n4 | (ne << 16));
    const u32 ta = __builtin_amdgcn_readlane(sa, 63), tb = __builtin_amdgcn_readlane(sb, 63);
    E.N2 = ta & 0xffffu; E.N3 = ta >> 16; E.NE = tb >> 16;
    E.escpos = (E.N2 + E.N3 + (tb & 0xffffu) + 15u) & ~15u;
    E.sa = sa - (n2 | (n3 << 16)); E.sb = sb - (n4 | (ne << 16));
    E.f2 = f2; E.f3 = f3; E.f4a = f4[0]; E.f4b = f4[1]; E.n4a = n4a; E.e0 = e0;
    asm volatile("" : "+v"(E.f2), "+v"(E.f3), "+v"(E.f4a), "+v"(E.f4b));
}
__device__ __forceinline__ u32 enc_dwords(const EncBlock& E) { return (E.escpos + 16u * E.NE + 31u) >> 5; }
constexpr u32 MAX_VAR = 2036;
template <bool OUT>
__device__ __forceinline__ u32 encode_place(const EncBlock& E, char* slot, const u32 (&d)[8], u32 lvar, u32 ltot, int wave, int lane) {
    const u32x4 t0 = lds_ld16(ltot), t1 = lds_ld16(ltot + 16);
    const u32 tw[8] = {t0.x, t0.y, t0.z, t0.w, t1.x, t1.y, t1.z, t1.w};
    u32 start = 0, total = 0;
#pragma unroll
    for (int j = 0; j < 8; ++j) {
        const u32 x = __builtin_amdgcn_readfirstlane(tw[j]);
        start += j < wave ? x : 0u; total += x;
    }
    if (total > MAX_VAR) {
        char* p = slot + wave * 2048 + lane * 16;
        st16_wt(p, u32x4{d[0], d[1], d[2], d[3]}); st16_wt(p + 1024, u32x4{d[4], d[5], d[6], d[7]});
        return total;
    }
    st16_wt(slot + wave * 1024 + lane * 16, u32x4{E.bb[0], E.bb[1], E.bb[2], E.bb[3]});
    const u32 base = lvar + 32u + start * 4u;
    if (lane == 0) *(LAS u32*)(lvar + wave * 4) = start | (E.e0 << 16);
    if (OUT) lds_or_bits(base, E.sa & 0xffffu, E.f2);
    else *(LAS u32*)(base + lane * 4) = E.f2;
    lds_or_bits(base, E.N2 + (E.sa >> 16), E.f3);
    const u32 p4 = E.N2 + E.N3 + (E.sb & 0xffffu);
    lds_or_bits(base, p4, E.f4a);
    lds_or_bits(base, p4 + E.n4a, E.f4b);
    if (E.NE) {
        u32 a = base + (E.escpos >> 3) + (E.sb >> 16) * 2u;
#pragma unroll
        for (int j = 0; j < 4; ++j)
            if (E.esc[j]) {
#pragma unroll
                for (int i = 0; i < 4; ++i)
                    if ((E.esc[j] >> (8 * i)) & 0xffu) { *(LAS unsigned short*)a = (unsigned short)(d[2 * j + (i >> 1)] >> (16 * (i & 1))); a += 2u; }
            }
    }
    return total;
}
__device__ __forceinline__ u32 encode_flush(u32 total, char* slot, u32 lvar, int wave, int lane) {
    if (total > MAX_VAR) return F_RAW | IMG;
    const u32 vbytes = 32u + total * 4u, o = wave * 1024 + lane * 16;
    if (o < vbytes) st16_wt(slot + 8192 + o, lds_ld16(lvar + o));
    return 8192u + ((vbytes + 15u) & ~15u);
}
template <bool OUT>
__device__ __forceinline__ u32 encode_message(char* slot, u32x4 v0, u32x4 v1, u32 ltbl, u32 lvar, u32 ltot, int wave, int lane) {
    *(LAS u32x4*)(lvar + wave * 1024 + lane * 16) = u32x4{0u, 0u, 0u, 0u};
    const u32 d[8] = {v0.x, v0.y, v0.z, v0.w, v1.x, v1.y, v1.z, v1.w};
    EncBlock E;
    encode_block<OUT>(E, d, ltbl);
    if (lane == 0) *(LAS u32*)(ltot + wave * 4) = enc_dwords(E);
    bar();
    const u32 total = encode_place<OUT>(E, slot, d, lvar, ltot, wave, lane);
    bar();
    return encode_flush(total, slot, lvar, wave, lane);
}

template <bool OUT>
__device__ __forceinline__ void decode_block(u32 sl, u32 ltbl, int wave, int lane, u32x4& v0, u32x4& v1) {
    const u32x4 B = lds_ld16(sl + wave * 1024 + lane * 16);
    const u32 hdr = __builtin_amdgcn_readfirstlane(*(const LAS u32*)(sl + 8192 + wave * 4));
    const u32 start = (hdr & 0xffffu) > 2040u ? 2040u : (hdr & 0xffffu);
    const u32 base = sl + 8192 + 32 + start * 4u;
    const u32 eadd = ((hdr >> 16) & 0xffu) * 0x00800080u;
    const u32 bb[4] = {B.x, B.y, B.z, B.w};
    u32 F2, N2;
    if (OUT) {
        u32 c = 0;
#pragma unroll
        for (int j = 0; j < 4; ++j) c += __builtin_popcount((bb[j] | (bb[j] << 1)) & 0x80808080u);
        const u32 n2 = 2u * c, s2 = wave_scan(n2);
        N2 = __builtin_amdgcn_readlane(s2, 63);
        F2 = lds_get_bits(base, s2 - n2) & low_mask_even(n2);
    } else {
        F2 = *(const LAS u32*)(base + lane * 4);
        N2 = 2048u;
    }
    const u32 W3 = F2 & (F2 << 1) & 0xaaaaaaaau;
    const u32 n3 = 2u * __builtin_popcount(W3), s3 = wave_scan(n3);
    const u32 N3 = __builtin_amdgcn_readlane(s3, 63);
    const u32 F3 = lds_get_bits(base, N2 + s3 - n3) & low_mask_even(n3);
    const u32 K4 = F3 & (F3 >> 1) & 0x55555555u, W4 = K4 * 3u;
    const u32 n4 = 3u * __builtin_popcount(K4), s4 = wave_scan(n4);
    const u32 N4 = __builtin_amdgcn_readlane(s4, 63);
    const u32 P4 = N2 + N3 + s4 - n4;
    const LAS u32* q4 = (const LAS u32*)(base + (P4 >> 5) * 4u);
    const u32 q40 = q4[0], q41 = q4[1], q42 = q4[2];
    u32 F4 = __builtin_amdgcn_alignbit(q41, q40, P4 & 31u);
    const u32 F4h = __builtin_amdgcn_alignbit(q42, q41, P4 & 31u);
    u32 t[16], p2 = 0, p3 = 0, p4 = 0;
    const u32 c0 = 0xc0u;
#pragma unroll
    for (int i = 0; i < 16; ++i) {
        if (i == 8) { F4 = __builtin_amdgcn_alignbit(F4h, F4, p4); p4 = 0; }
        const u32 c1 = (i & 3) == 0 ? and_byte<0>(bb[i >> 2], c0) : (i & 3) == 1 ? and_byte<1>(bb[i >> 2], c0)
                     : (i & 3) == 2 ? and_byte<2>(bb[i >> 2], c0) : and_byte<3>(bb[i >> 2], c0);
        u32 c2, w3;
        if (OUT) { const u32 w2 = c1 < 2u ? c1 : 2u; c2 = ubfe(F2, p2, w2); w3 = ubfe(W3, p2, w2); p2 += w2; }
        else { c2 = ubfe(F2, 2 * i, 2); w3 = ubfe(W3, 2 * i, 2); }
        const u32 c3 = ubfe(F3, p3, w3), w4 = ubfe(W4, p3, w3); p3 += w3;
        const u32 c4 = ubfe(F4, p4, w4); p4 += w4;
        t[i] = c1 + ((c2 + c3 + c4) << 1);
    }
    u32 o[8];
#pragma unroll
    for (int j = 0; j < 8; ++j) {
        typedef unsigned short u16x2 __attribute__((ext_vector_type(2)));
        const u16x2 pr = {*(const LAS unsigned short*)(ltbl + t[2 * j]), *(const LAS unsigned short*)(ltbl + t[2 * j + 1])};
        o[j] = __builtin_bit_cast(u32, pr) + eadd;
    }
    const u32 any = (o[0] | o[1] | o[2]) | (o[3] | o[4] | o[5]) | (o[6] | o[7]);
    if (__builtin_amdgcn_ballot_w64((any & 0x00010001u) != 0u)) {
        u32 c = 0;
#pragma unroll
        for (int j = 0; j < 8; ++j) c += o[j] & 0x00010001u;
        const u32 ne = (c & 0xffffu) + (c >> 16);
        u32 a = base + (((N2 + N3 + N4 + 15u) & ~15u) >> 3) + (wave_scan(ne) - ne) * 2u;
        const u32 amax = sl + SLOT - 4u;
#pragma unroll
        for (int j = 0; j < 8; ++j) {
            if (o[j] & 0x00010001u) {
                a = a < amax ? a : amax;
                if (o[j] & 1u) { o[j] = (o[j] & 0xffff0000u) | (*(const LAS unsigned short*)a & 0xffc0u); a += 2u; }
                if (o[j] & 0x10000u) { o[j] = (o[j] & 0xffffu) | ((u32)(*(const LAS unsigned short*)a & 0xffc0u) << 16); a += 2u; }
            }
        }
    }
#pragma unroll
    for (int j = 0; j < 8; ++j)
        o[j] |= __builtin_amdgcn_perm(0u, bb[j >> 1] & 0x3f3f3f3fu, (j & 1) ? 0x0c030c02u : 0x0c010c00u);
    v0 = u32x4{o[0], o[1], o[2], o[3]};
    v1 = u32x4{o[4], o[5], o[6], o[7]};
}

__global__ void __launch_bounds__(THREADS, 1) fused_gemm_allgather(Params P) {
    const int lane = threadIdx.x & 63;
    const int wave = __builtin_amdgcn_readfirstlane(threadIdx.x >> 6);
    int b = blockIdx.x;
    if (MAP_ == 1) {
        const int x = b & 7, y = b >> 3;
        if (x >= 4 && y >= 28) return;
        b = x < 4 ? (y >> 1) * 8 + 2 * x + (y & 1) : NGEMM + (x - 4) * 28 + y;
    }
    if (MAP_ == 2) {
        const int x = b & 7, y = b >> 3;
        if ((x & 1) && y >= 28) return;
        b = !(x & 1) ? (y >> 1) * 8 + x + (y & 1) : NGEMM + (x >> 1) * 28 + y;
    }
    const int rank = P.rank;
    char* const myws = P.ws[rank];
    constexpr u32 lds0 = 0;
    GAS u32* const ctr = (GAS u32*)(myws + OFF_CTR + (size_t)b * 128);
    GAS u32* const tmo = (GAS u32*)(myws + OFF_TMO);

    if (b < NGEMM) {
        const int tj = b & 7, ti0 = b >> 4, half = (b >> 3) & 1;
        const int npass = pack_passes(b);
        constexpr int KBAR = 65, EBAR = 4;
        if (wave == 8) {
            const u32 call = __builtin_amdgcn_readfirstlane(__hip_atomic_load(ctr, __ATOMIC_RELAXED, __HIP_MEMORY_SCOPE_AGENT)) + 1u;
            put_enc_table(L_ETBL, 0, lane);
            put_enc_table(L_ETBL + 512, 1, lane);
            for (int i = 0; i < 4 * npass; ++i) bar();
            for (int tile = 0; tile < 2; ++tile) {
                GAS u32* c = (GAS u32*)(myws + OFF_CNT + (size_t)((lane & 1) ? 16 + tj : ti0 + 8 * tile) * 256);
                wait_counters(c + 32, call * 8u, tmo, call);
                for (int i = 0; i < 2 + 8 - NST; ++i) bar();
                u32 dummy = 0;
                auto prefetch = [&]() {
                    const char* pw = myws + OFF_UW + ((size_t)tj * 64 + (u32)((rank * 8 + 8 + (b >> 3) * 4) & 63)) * IMG + lane * 128;
                    if ((b >> 3) < 14) {
#pragma unroll
                        for (int j = 0; j < 8; ++j)
                            asm volatile("global_load_dword %0, %1, off sc1" : "+v"(dummy) : "v"(pw + j * 8192) : "memory");
                    }
                };
                const bool early = tile == 0 && ti0 > tj;
                if (early) {
                    wait_counters((GAS u32*)(myws + OFF_CNT + (size_t)(16 + tj) * 256), call * 56u, tmo, call);
                    prefetch();
                }
                wait_counters(c, call * 56u, tmo, call);
                STAMP(6 + 6 * tile);
                bar();
                if (tile == 0 && !early) prefetch();
                if (tile == 0)
                    for (int i = 0; i < KBAR - (2 + 8 - NST) + EBAR; ++i) bar();
                asm volatile("s_waitcnt vmcnt(0)" : "+v"(dummy) :: "memory");
            }
            return;
        }
        if (wave == 0) STAMP(0);
        u32 cv;
        asm volatile("global_load_dword %0, %1, off sc1" : "=v"(cv) : "v"(ctr) : "memory");
        auto msg_of = [&](int rho, int& isw, int& grp, int& lp, int& kb, int& seq) {
            if (rho < 384) {
                const int t = rho / 48, r = rho % 48;
                isw = r >= 16; grp = isw ? (r - 16) >> 3 : r >> 3; kb = r & 7; lp = t; seq = t * 16 + (isw ? 8 : 0) + kb;
            } else {
                const int t = 8 + ((rho - 384) >> 5), r = rho & 31, pos = (r >> 3) & 1;
                isw = 0; grp = r >> 4; kb = r & 7; lp = 8 + 2 * (t - 8) + pos; seq = t * 16 + pos * 8 + kb;
            }
        };
        auto fetch = [&](int z, int rho) {
            int isw, grp, lp, kb, seq;
            msg_of(rho, isw, grp, lp, kb, seq);
            const u32 dst = lds0 + z * IMG + wave * 2048;
            if (!isw) {
                const char* s = P.x + ((size_t)((grp * 16 + lp) * 128 + wave * 16 + (lane & 15)) * 512 + kb * 64 + (lane >> 4) * 8) * 2;
                dma1(s, dst); dma1(s + 64, dst + 1024);
            } else {
                const char* s = P.w + ((size_t)(kb * 64 + wave * 8 + (lane >> 4)) * 4096 + (grp * 8 + lp) * 128 + (lane & 15) * 8) * 2;
                dma1(s, dst); dma1(s + 4 * 8192, dst + 1024);
            }
        };
        fetch(0, b); fetch(1, 128 + b);
        const u32 call = landed_first<2>(cv) + 1u;
        char* const wsl = P.ws[lane & 7];
        for (int z = 0; z < npass; ++z) {
            int isw, grp, lp, kb, seq;
            msg_of(z < 2 ? z * 128 + b : (int)SCHED.rho[b][z], isw, grp, lp, kb, seq);
            bar();
            if (z == 1) for (int y = 2; y < npass; ++y) fetch(y, SCHED.rho[b][y]);
            if (z == 0 && threadIdx.x == 0) __hip_atomic_store(ctr, call, __ATOMIC_RELAXED, __HIP_MEMORY_SCOPE_AGENT);
            u32x4 v0, v1;
            const u32 src = lds0 + z * IMG;
            if (!isw) {
                v0 = lds_ld16(src + wave * 2048 + lane * 16);
                v1 = lds_ld16(src + wave * 2048 + 1024 + lane * 16);
            } else {
                const u32 a = src + ((lane >> 4) * 8 + ((lane & 15) >> 2)) * 256 + (32 * (wave >> 1) + 8 * (lane & 3) + 4 * (wave & 1)) * 2;
                const s16x4 t0 = __builtin_amdgcn_ds_read_tr16_b64_v4i16((LAS s16x4*)(a));
                const s16x4 t1 = __builtin_amdgcn_ds_read_tr16_b64_v4i16((LAS s16x4*)(a + 4 * 256));
                const s16x4 t2 = __builtin_amdgcn_ds_read_tr16_b64_v4i16((LAS s16x4*)(a + 32 * 256));
                const s16x4 t3 = __builtin_amdgcn_ds_read_tr16_b64_v4i16((LAS s16x4*)(a + 36 * 256));
                const u32x2 a0 = __builtin_bit_cast(u32x2, t0), a1 = __builtin_bit_cast(u32x2, t1);
                const u32x2 a2 = __builtin_bit_cast(u32x2, t2), a3 = __builtin_bit_cast(u32x2, t3);
                v0 = u32x4{a0.x, a0.y, a1.x, a1.y};
                v1 = u32x4{a2.x, a2.y, a3.x, a3.y};
            }
            const int mid = (isw ? 256 + (grp * 8 + lp) * 8 : (grp * 16 + lp) * 8) + kb;
            const u32 lo = encode_message<false>(myws + OFF_PKIN + (size_t)mid * SLOT, v0, v1, L_ETBL, L_VAR, L_TOT, wave, lane);
            const bool own = isw ? (grp == (rank & 3)) : (grp == (rank >> 2));
            if (own) {
                char* img = myws + (isw ? OFF_UW : OFF_UX) + (size_t)(lp * 64 + rank * 8 + kb) * IMG + wave * 2048 + lane * 16;
                st16_wt(img, v0); st16_wt(img + 1024, v1);
            }
            vm_wait0();
            bar();
            if (wave == 0) {
                const bool dest = lane < NR && lane != rank && (isw ? ((lane & 3) == grp) : ((lane >> 2) == grp));
                if (dest) {
                    GAS u64* f = (GAS u64*)(wsl + OFF_FLAG + ((size_t)rank * (ROUNDS * PER_LINK) + seq) * 128);
                    __hip_atomic_store(f, ((u64)call << 32) | lo, __ATOMIC_RELAXED, __HIP_MEMORY_SCOPE_SYSTEM);
                }
                if (own && lane == 0)
                    __hip_atomic_fetch_add((GAS u32*)(myws + OFF_CNT + (size_t)((isw ? 16 : 0) + lp) * 256 + 128), 1u,
                                           __ATOMIC_RELAXED, __HIP_MEMORY_SCOPE_AGENT);
                STAMP(1 + z);
            }
        }
        u32 ords = __builtin_amdgcn_readfirstlane((u32)SCHED.ord[0][b] | ((u32)SCHED.ord[1][b] << 8));
        asm volatile("s_waitcnt vmcnt(0)" : "+s"(ords) :: "memory");
        const int kh = wave >> 2, wn = wave & 1, wm = (wave >> 1) & 1;
        const char* const wb = myws + OFF_UW + (size_t)tj * 64 * IMG;
        const u32 voffx = wave * 1024 + lane * 16, voffw = wave * 2048 + lane * 16;
        const u32 xa = lds0 + (wm * 4 + kh) * 1024 + lane * 16;
        const u32 wa = lds0 + 8192 + (wn * 8 + kh) * 1024 + lane * 16;
#define SB __builtin_amdgcn_sched_barrier(0)
        for (int tile = 0; tile < 2; ++tile) {
            const int ti = ti0 + 8 * tile;
            const char* const xb = myws + OFF_UX + (size_t)ti * 64 * IMG + half * 8192;
            f32x4 acc[4][2];
#pragma unroll
            for (int a = 0; a < 4; ++a) { acc[a][0] = f32x4{0.f, 0.f, 0.f, 0.f}; acc[a][1] = acc[a][0]; }
            bf16x8 wA[4], xA[2], wB[4], xB[2];
            auto mma = [&](int a, const bf16x8* wf, const bf16x8* xf) {
                acc[a][0] = __builtin_amdgcn_mfma_f32_16x16x32_bf16(wf[a], xf[0], acc[a][0], 0, 0, 0);
                acc[a][1] = __builtin_amdgcn_mfma_f32_16x16x32_bf16(wf[a], xf[1], acc[a][1], 0, 0, 0);
            };
            u32 buf = 0;
#define ROT_ 7
            u32 kq = tj * ROT_;
            auto step = [&](int kt, auto dma_tag, auto vm_tag, const bf16x8* wc, const bf16x8* xc, bf16x8* wx, bf16x8* xx) {
                constexpr bool DMA = decltype(dma_tag)::value;
                constexpr int VM = decltype(vm_tag)::value;
                const u32 kb = (u32)(rank * 8 + (kt + NST < 8 ? kt + NST : 8 + kq)) & 63u;
                if (kt + NST >= 8) kq = kq == 55u ? 0u : kq + 1u;
                const char* gx = xb + (size_t)kb * IMG; const char* gw = wb + (size_t)kb * IMG;
                const u32 d = buf;
                const u32 nb = buf + STG == NST * STG ? 0u : buf + STG;
                const u32 ra = xa + nb, rb = wa + nb;
                buf = nb;
                asm volatile("s_waitcnt lgkmcnt(0)\n\ts_waitcnt vmcnt(%0)\n\ts_barrier" :: "n"(VM) : "memory"); SB;
                if (DMA) dma_a(gx, voffx, d + wave * 1024); SB;
                mma(0, wc, xc); SB;
                xx[0] = *(const LAS bf16x8*)(ra); wx[0] = *(const LAS bf16x8*)(rb); SB;
                mma(1, wc, xc); SB;
                if (DMA) dma_a(gw, voffw, d + 8192 + wave * 2048); SB;
                xx[1] = *(const LAS bf16x8*)(ra + 2048); wx[1] = *(const LAS bf16x8*)(rb + 2048); SB;
                mma(2, wc, xc); SB;
                wx[2] = *(const LAS bf16x8*)(rb + 4096); wx[3] = *(const LAS bf16x8*)(rb + 6144); SB;
                if (DMA) dma_b(gw, voffw); SB;
                mma(3, wc, xc); SB;
            };
            using T_ = std::true_type; using F_ = std::false_type;
            bar();
#pragma unroll
            for (int st = 0; st < NST; ++st) {
                const u32 kb = (u32)(rank * 8 + st) & 63u;
                dma_a(xb + (size_t)kb * IMG, voffx, st * STG + wave * 1024);
                dma_a(wb + (size_t)kb * IMG, voffw, st * STG + 8192 + wave * 2048);
                dma_b(wb + (size_t)kb * IMG, voffw);
            }
            asm volatile("s_waitcnt vmcnt(%0)\n\ts_barrier" :: "n"((NST - 1) * 3) : "memory");
#pragma unroll
            for (int a = 0; a < 4; ++a) wA[a] = *(const LAS bf16x8*)(wa + a * 2048);
            xA[0] = *(const LAS bf16x8*)(xa); xA[1] = *(const LAS bf16x8*)(xa + 2048);
            static_assert(NST == 6, "tail below");
            using VMS = std::integral_constant<int, (NST - 2) * 3>;
            for (int kt = 0; kt < 64 - NST; kt += 2) {
                step(kt, T_{}, VMS{}, wA, xA, wB, xB);
                step(kt + 1, T_{}, VMS{}, wB, xB, wA, xA);
            }
            step(58, F_{}, std::integral_constant<int, 12>{}, wA, xA, wB, xB);
            step(59, F_{}, std::integral_constant<int, 9>{}, wB, xB, wA, xA);
            step(60, F_{}, std::integral_constant<int, 6>{}, wA, xA, wB, xB);
            step(61, F_{}, std::integral_constant<int, 3>{}, wB, xB, wA, xA);
            step(62, F_{}, std::integral_constant<int, 0>{}, wA, xA, wB, xB);
            u32 tv;
            asm volatile("global_load_dword %0, %1, off sc1" : "=v"(tv) : "v"(tmo) : "memory");
            bar();
            mma(0, wB, xB); mma(1, wB, xB); mma(2, wB, xB); mma(3, wB, xB);
            if (wave == 0) STAMP(7 + 6 * tile);
            {
                const u32 wr = lds0 + wave * 4096 + lane * 16;
#pragma unroll
                for (int a = 0; a < 4; ++a) *(LAS f32x4*)(wr + a * 1024) = kh ? acc[a][0] : acc[a][1];
                *(LAS u32x4*)(L_VAR + wave * 1024 + lane * 16) = u32x4{0u, 0u, 0u, 0u};
            }
            bar();
            u32 d[8];
            {
                const u32 rd = lds0 + (wave ^ 4) * 4096 + lane * 16;
#pragma unroll
                for (int a = 0; a < 4; ++a) {
                    const f32x4 y = (kh ? acc[a][1] : acc[a][0]) + *(const LAS f32x4*)(rd + a * 1024);
                    d[a * 2] = pk_bf16(gelu(y.x), gelu(y.y));
                    d[a * 2 + 1] = pk_bf16(gelu(y.z), gelu(y.w));
                }
            }
            if (landed_first<0>(tv) == call) {
#pragma unroll
                for (int j = 0; j < 8; ++j) d[j] = 0x7fc07fc0u;
            }
            if (wave == 0) STAMP(8 + 6 * tile);
            const u32 ord = (ords >> (8 * tile)) & 0xffu;
            char* const oslot = myws + OFF_PKOUT + (size_t)ord * SLOT;
            EncBlock E;
            encode_block<true>(E, d, L_ETBL + 512);
            if (lane == 0) *(LAS u32*)(L_TOT + wave * 4) = enc_dwords(E);
            bar();
            const u32 tot = encode_place<true>(E, oslot, d, L_VAR, L_TOT, wave, lane);
            bar();
            const u32 lo = encode_flush(tot, oslot, L_VAR, wave, lane);
            if (wave == 0) STAMP(9 + 6 * tile);
            vm_wait0();
            bar();
            if (wave == 0 && lane < NR && lane != rank) {
                GAS u64* f = (GAS u64*)(wsl + OFF_FLAG + ((size_t)rank * (ROUNDS * PER_LINK) + IN_ROUNDS * PER_LINK + ord) * 128);
                __hip_atomic_store(f, ((u64)call << 32) | lo, __ATOMIC_RELAXED, __HIP_MEMORY_SCOPE_SYSTEM);
            }
            char* o = P.out + ((size_t)((rank >> 2) * 2048 + ti * 128 + half * 64 + wm * 32 + kh * 16 + (lane & 15)) * 4096
                               + (rank & 3) * 1024 + tj * 128 + wn * 64 + (lane >> 4) * 8) * 2;
            st16_wt(o, u32x4{d[0], d[1], d[2], d[3]});
            st16_wt(o + 64, u32x4{d[4], d[5], d[6], d[7]});
            if (wave == 0) STAMP(10 + 6 * tile);
            if (tile == 0) vm_wait0();
        }
    } else {
        const int pu = b - NGEMM;
        const int u = pu / 7;
        const int src = (rank + 1 + pu % 7) & 7;
        const u32 call = __builtin_amdgcn_readfirstlane(__hip_atomic_load(ctr, __ATOMIC_RELAXED, __HIP_MEMORY_SCOPE_AGENT)) + 1u;
        if (wave == 0) lds_st(lds0 + L_CTL + lane * 4, 0u);
        if (wave == 1) put_dec_table(L_DTBL, 0, lane);
        if (wave == 2) put_dec_table(L_DTBL + 256, 1, lane);
        if (wave == 3) *(LAS u32*)(L_UNORD + lane * 4) = ((const u32*)SCHED.unord)[lane];
        if (wave == 8) STAMP(0);
        bar();
        if (threadIdx.x == 0) __hip_atomic_store(ctr, call, __ATOMIC_RELAXED, __HIP_MEMORY_SCOPE_AGENT);
        if (wave == 8) {
            vm_wait0();
            const char* const rws = P.ws[src];
            const char* const fbase = myws + OFF_FLAG + (size_t)src * NMSG * 128;
            char* const tkt = myws + OFF_TKT + (size_t)(pu % 7) * 256;
            const u32 tbase = (call - 1u) * NMSG - PER_LINK;
            u32 m = u, n = 0;
            u32 fl_lo = 0u, fl_hi = 0u;
            bool dead = false;
            for (;; ++n) {
                const u32 s = n % NSLOT;
                if (m < NMSG && !dead && fl_hi < call) {
                    vm_wait0();
                    lds_st(lds0 + L_FULL, n);
                    const u64 t0 = __builtin_amdgcn_s_memrealtime();
                    for (u32 it = 1;; ++it) {
                        u32x2 fl;
                        asm volatile("global_load_dwordx2 %0, %1, off sc0 sc1\n\ts_waitcnt vmcnt(0)" : "=&v"(fl) : "v"(fbase + m * 128) : "memory");
                        fl_lo = __builtin_amdgcn_readfirstlane(fl.x); fl_hi = __builtin_amdgcn_readfirstlane(fl.y);
                        if (fl_hi >= call) break;
                        if ((it & 15u) == 0u
                            && (__builtin_amdgcn_readfirstlane(__hip_atomic_load(tmo, __ATOMIC_RELAXED, __HIP_MEMORY_SCOPE_AGENT)) == call
                                || __builtin_amdgcn_s_memrealtime() - t0 > DEADLINE)) {
                            __hip_atomic_store(tmo, call, __ATOMIC_RELAXED, __HIP_MEMORY_SCOPE_AGENT);
                            dead = true;
                            break;
                        }
                        __builtin_amdgcn_s_sleep(1);
                    }
                }
                if (n >= NSLOT) {
                    const u32 need = 8u * (n / NSLOT);
                    if (__builtin_amdgcn_readfirstlane(lds_ld(lds0 + L_FREE + s * 4)) < need) {
                        vm_wait0();
                        lds_st(lds0 + L_FULL, n);
                        while (__builtin_amdgcn_readfirstlane(lds_ld(lds0 + L_FREE + s * 4)) < need) __builtin_amdgcn_s_sleep(1);
                    }
                }
                if (m >= NMSG) {
                    lds_st(lds0 + L_MSG + s * 4, NMSG);
                    vm_wait0();
                    lds_st(lds0 + L_FULL, n + 1);
                    break;
                }
                const u32 lo = dead ? F_BAD : fl_lo;
                lds_st(lds0 + L_LEN + s * 4, lo);
                lds_st(lds0 + L_MSG + s * 4, m);
                u32 nt = 1u;
                u64 keep;
                asm volatile("s_mov_b64 %1, exec\n\ts_mov_b64 exec, 1\n\tglobal_atomic_add %0, %2, %0, off sc0\n\ts_mov_b64 exec, %1"
                             : "+v"(nt), "=&s"(keep) : "v"(tkt) : "memory");
                if (dead) {
                    m = landed_first<0>(nt) - tbase;
                    lds_st(lds0 + L_FULL, n + 1);
                    continue;
                }
                u32 len = lo & 0xffffu;
                len = len > SLOT ? SLOT : (len < 8208u ? 8208u : len);
                const u32 t = m / PER_LINK, v = m % PER_LINK;
                size_t moff;
                if (t < IN_ROUNDS) {
                    const u32 kb = v & 7;
                    u32 mid;
                    if (t < 8) mid = v < 8 ? ((rank >> 2) * 16 + t) * 8 + kb : 256 + ((rank & 3) * 8 + t) * 8 + kb;
                    else mid = ((rank >> 2) * 16 + 8 + 2 * (t - 8) + (v >> 3)) * 8 + kb;
                    moff = OFF_PKIN + (size_t)mid * SLOT;
                } else {
                    moff = OFF_PKOUT + (size_t)(m - IN_ROUNDS * PER_LINK) * SLOT;
                }
                const char* sp = rws + moff + lane * 16;
                const u32 d = lds0 + L_RING + s * SLOT;
#pragma unroll
                for (int k = 0; k < WIN - 1; ++k) {
                    dma1_sys(sp + k * 1024, d + k * 1024);
                    asm volatile("s_waitcnt vmcnt(%0)" :: "n"(WIN - 1) : "memory");
                    if (k == WIN - 3) lds_st(lds0 + L_FULL, n);
                }
                m = landed_first<WIN - 1>(nt) - tbase;
                if (m >= NMSG) {
                    for (u32 o = (WIN - 1) * 1024; o < len; o += 1024)
                        if (o + lane * 16 < len) dma1_sys(sp + o, d + o);
                } else {
                    u32x2 fn;
                    asm volatile("global_load_dwordx2 %0, %1, off sc0 sc1" : "=v"(fn) : "v"(fbase + m * 128) : "memory");
                    for (u32 o = (WIN - 1) * 1024; o < len; o += 1024) {
                        if (o + lane * 16 < len) dma1_sys(sp + o, d + o);
                        asm volatile("s_waitcnt vmcnt(%0)" :: "n"(WIN - 1) : "memory");
                    }
                    fl_lo = landed_first<10 - WIN>(fn.x); fl_hi = landed_first<10 - WIN>(fn.y);
                }
            }
            STAMP(1);
        } else {
            const int kh = wave >> 2, wn = wave & 1, wm = (wave >> 1) & 1;
            for (u32 n = 0;; ++n) {
                const u32 s = n % NSLOT;
                while (__builtin_amdgcn_readfirstlane(lds_ld(lds0 + L_FULL)) <= n) __builtin_amdgcn_s_sleep(1);
                const u32 m = __builtin_amdgcn_readfirstlane(lds_ld(lds0 + L_MSG + s * 4));
                if (m >= NMSG) break;
                const u32 t = m / PER_LINK, v = m % PER_LINK;
                const u32 lo = __builtin_amdgcn_readfirstlane(lds_ld(lds0 + L_LEN + s * 4));
                const u32 sl = lds0 + L_RING + s * SLOT;
                u32x4 v0, v1;
                if (lo & F_BAD) {
                    v0 = u32x4{0x7fc07fc0u, 0x7fc07fc0u, 0x7fc07fc0u, 0x7fc07fc0u}; v1 = v0;
                } else if (lo & F_RAW) {
                    v0 = lds_ld16(sl + wave * 2048 + lane * 16);
                    v1 = lds_ld16(sl + wave * 2048 + 1024 + lane * 16);
                } else if (t < IN_ROUNDS) {
                    decode_block<false>(sl, L_DTBL, wave, lane, v0, v1);
                } else {
                    decode_block<true>(sl, L_DTBL + 256, wave, lane, v0, v1);
                }
                asm volatile("s_waitcnt lgkmcnt(0)" : "+v"(v0), "+v"(v1) :: "memory");
                if (lane == 0) asm volatile("ds_add_u32 %0, %1" :: "v"(lds0 + L_FREE + s * 4), "v"(1u) : "memory");
                if (t < IN_ROUNDS) {
                    const u32 kb = v & 7;
                    const bool isw = t < 8 && v >= 8;
                    const u32 lp = t < 8 ? t : 8 + 2 * (t - 8) + (v >> 3);
                    char* img = myws + (isw ? OFF_UW : OFF_UX) + (size_t)(lp * 64 + src * 8 + kb) * IMG + wave * 2048 + lane * 16;
                    st16_wt(img, v0); st16_wt(img + 1024, v1);
                    vm_wait0();
                    if (lane == 0) {
                        u32 old;
                        asm volatile("ds_add_rtn_u32 %0, %1, %2\n\ts_waitcnt lgkmcnt(0)" : "=&v"(old) : "v"(lds0 + L_DONE + s * 4), "v"(1u) : "memory");
                        if ((old & 7u) == 7u) {
                            __hip_atomic_fetch_add((GAS u32*)(myws + OFF_CNT + (size_t)((isw ? 16 : 0) + lp) * 256), 1u,
                                                   __ATOMIC_RELAXED, __HIP_MEMORY_SCOPE_AGENT);
                        }
                    }
                } else {
                    const int g = __builtin_amdgcn_readfirstlane(*(const LAS unsigned char*)(L_UNORD + m - IN_ROUNDS * PER_LINK));
                    const int ti = ((g >> 4) & 7) + 8 * (g >> 7), hf = (g >> 3) & 1, tj = g & 7;
                    char* op = P.out + ((size_t)((src >> 2) * 2048 + ti * 128 + hf * 64 + wm * 32 + kh * 16 + (lane & 15)) * 4096
                                        + (src & 3) * 1024 + tj * 128 + wn * 64 + (lane >> 4) * 8) * 2;
                    st16_wt(op, v0); st16_wt(op + 64, v1);
                }
            }
            if (wave == 0) { STAMP(2); }
        }
    }
}

extern "C" void kernel_launch(int rank, int nranks, ncclComm_t comm,
                              hipStream_t stream,
                              void* const* d_in, const int* in_sizes, int n_in,
                              void* d_out, int out_size,
                              const int* in_dtypes, int out_dtype,
                              void* const* d_out_all,
                              void* const* d_ws_all, size_t ws_bytes) {
    (void)comm; (void)in_dtypes; (void)out_dtype; (void)d_out_all;
    if (nranks != NR || (unsigned)rank >= (unsigned)NR || n_in != 2 || in_sizes[0] != 4096 * 512 || in_sizes[1] != 512 * 4096
        || out_size != 4096 * 4096 || ws_bytes < WS_NEED) {
        fprintf(stderr, "kernel_launch: unexpected problem shape\n");
        return;
    }
    static bool attr_set[NR] = {};
    if (!attr_set[rank]) {
        HIP_CHECK(hipFuncSetAttribute((const void*)fused_gemm_allgather, hipFuncAttributeMaxDynamicSharedMemorySize, LDS_BYTES));
        attr_set[rank] = true;
    }
    Params P;
    for (int p = 0; p < NR; ++p) P.ws[p] = (char*)d_ws_all[p];
    P.x = (const char*)d_in[0];
    P.w = (const char*)d_in[1];
    P.out = (char*)d_out;
    P.rank = rank;
    P.pad = 0;
    hipLaunchKernelGGL(fused_gemm_allgather, dim3(GRID), dim3(THREADS), LDS_BYTES, stream, P);
    HIP_CHECK(hipGetLastError());
}
```

```cpp
#include <hip/hip_runtime.h>
#include <rccl/rccl.h>
#include <cstdio>
#include <cstdint>
#include <type_traits>
#pragma clang diagnostic ignored "-Wint-to-pointer-cast"

#define HIP_CHECK(expr) do { hipError_t _e = (expr); if (_e != hipSuccess) { \
    fprintf(stderr, "HIP error %s at %s:%d\n", hipGetErrorString(_e), __FILE__, __LINE__); \
    return; } } while (0)

typedef unsigned u32;
typedef unsigned long long u64;
typedef u32 u32x4 __attribute__((ext_vector_type(4)));
typedef u32 u32x2 __attribute__((ext_vector_type(2)));
typedef float f32x4 __attribute__((ext_vector_type(4)));
typedef float f32x2 __attribute__((ext_vector_type(2)));
typedef __bf16 bf16x8 __attribute__((ext_vector_type(8)));
typedef __bf16 bf16x2 __attribute__((ext_vector_type(2)));
typedef short s16x4 __attribute__((ext_vector_type(4)));
#define INL __attribute__((always_inline))
#define LAS __attribute__((address_space(3)))
#define GAS __attribute__((address_space(1)))

constexpr int NR = 8;
constexpr int NGEMM = 128;
constexpr int PER_LINK = 16;
constexpr int NPULL = 7 * PER_LINK;
#define MAP_ 1
constexpr int NROLE = NGEMM + NPULL;
constexpr int GRID = MAP_ ? 256 : NROLE;
constexpr int THREADS = 576;
constexpr int IN_ROUNDS = 12, OUT_ROUNDS = 16, ROUNDS = IN_ROUNDS + OUT_ROUNDS;
#define NSPLIT_ 0
constexpr u32 NSPLIT = NSPLIT_;
constexpr u32 OUT0 = IN_ROUNDS * PER_LINK;
constexpr u32 HALF0 = ROUNDS * PER_LINK - NSPLIT;
constexpr u32 NMSG = HALF0 + 2 * NSPLIT;
constexpr u32 HOFF = 4352;
constexpr u32 SLOT = 16384 + 512;
constexpr u32 GSLOT = 2 * SLOT;
constexpr u32 IMG = 16384;
constexpr int NSLOT = 8;
#define WIN_ 4
#define WIN2_ 5
constexpr int WIN2 = WIN2_;
constexpr int WIN = WIN_;
constexpr unsigned long long DEADLINE = 100000000ull;

constexpr size_t OFF_CTR = 0;
constexpr size_t OFF_TMO = 32768;
constexpr size_t OFF_CNT = 36864;
constexpr size_t OFF_TKT = 45056;
constexpr size_t OFF_FLAG = 65536;
constexpr size_t OFF_DBG = 768 * 1024;
constexpr size_t OFF_PKIN = 1u << 20;
constexpr size_t OFF_PKOUT = 18u << 20;
constexpr size_t OFF_UX = 28u << 20;
constexpr size_t OFF_UW = 44u << 20;
constexpr size_t WS_NEED = 52u << 20;
static_assert(OFF_FLAG + (size_t)NR * NMSG * 128 <= OFF_DBG, "flags");
static_assert(OFF_PKIN + 512 * (size_t)GSLOT <= OFF_PKOUT, "pkin");
static_assert(OFF_PKOUT + 256 * (size_t)GSLOT <= OFF_UX, "pkout");

constexpr u32 L_RING = 0;
constexpr u32 L_CTL = NSLOT * SLOT;
constexpr u32 L_FULL = L_CTL;
constexpr u32 L_DESC = L_CTL + 4;
constexpr u32 L_F12 = L_CTL + 8;
constexpr u32 L_TRDY = L_CTL + 240;
constexpr u32 L_FREE = L_CTL + 16;
constexpr u32 L_DONE = L_CTL + 16 + 4 * NSLOT;
constexpr u32 L_LEN = L_CTL + 16 + 8 * NSLOT;
constexpr u32 L_MSG = L_CTL + 16 + 12 * NSLOT;
constexpr u32 L_DTBL = L_CTL + 256;
constexpr u32 L_UNORD = L_DTBL + 768;
constexpr u32 L_STMP = L_UNORD + 256;
constexpr int NST = 6;
constexpr u32 STG = 8192 + IMG;
constexpr u32 L_VAR = 5 * IMG;
constexpr u32 L_ETIN = L_VAR + 8192;
constexpr u32 L_ETOUT = NST * STG;
constexpr u32 L_TOT = L_ETOUT + 2048;
constexpr u32 LDS_BYTES = L_TOT + 64;
static_assert(LDS_BYTES >= L_STMP + 5120 && LDS_BYTES <= 163840 && L_ETIN + 4096 <= L_ETOUT, "lds");

constexpr int pack_passes(int b) {
    return (b >> 4) <= 1 && (b & 7) <= 1 ? 2 : ((b & 7) == 7 ? 5 : 4);
}
#define WFRESH_ 0
struct Sched { alignas(4) unsigned char unord[256]; unsigned short rho[128][5]; unsigned char ord[2][128]; };
constexpr Sched make_sched() {
    Sched s{};
    int r = 0;
    for (int z = 0; z < 5; ++z)
        for (int b = 0; b < 128; ++b) s.rho[b][z] = pack_passes(b) > z ? (unsigned short)r++ : (unsigned short)0xffff;
    int key[256] = {};
    for (int b = 0; b < 128; ++b) {
        const int i = b >> 4, j = b & 7, sh = i > j ? i : j;
        const int t1 = 30 * sh + 170 + (j > i ? WFRESH_ : 0), arr2 = 30 * (8 + (i >> 1));
        key[b] = t1 * 256 + b;
        key[128 + b] = ((t1 > arr2 ? t1 : arr2) + 170) * 256 + 128 + b;
    }
    for (int a = 0; a < 256; ++a) {
        int pos = 0;
        for (int c = 0; c < 256; ++c) pos += key[c] < key[a];
        s.ord[a >> 7][a & 127] = (unsigned char)pos;
        s.unord[pos] = (unsigned char)a;
    }
    return s;
}
__device__ constexpr Sched SCHED = make_sched();

constexpr u32 F_RAW = 0x80000000u;
constexpr u32 F_BAD = 0x40000000u;

struct Ranges { u32 a0, a1, b0, b1; };
__device__ __forceinline__ Ranges msg_ranges(u32 m, u32 lo) {
    const u32 half = m >= HALF0, hb = half ? (m - HALF0) & 1u : 0u, vmax = half ? HOFF : 8192u;
    u32 v = lo & 0xffffu;
    v = v < 16u ? 16u : (v > vmax ? vmax : v);
    if (lo & F_RAW) return Ranges{hb * 8192u, half ? hb * 8192u + 8192u : IMG, 0u, 0u};
    return Ranges{8192u + hb * HOFF, 8192u + hb * HOFF + v, hb * 4096u, half ? hb * 4096u + 4096u : 8192u};
}
template <int N, int I = 0, class F> __device__ __forceinline__ void static_for(F f) {
    if constexpr (I < N) { f(std::integral_constant<int, I>{}); static_for<N, I + 1>(f); }
}

struct Params {
    const char* x;
    const char* w;
    char* myws;
    char* out;
    int rank;
    int pad;
    char* ws[NR];
};

#define STAMP(id) do {} while (0)
#define TS(i) do {} while (0)
#define LSTAMP(base, cnt, wgrow, id) do {} while (0)
#define LFLUSH(base, cnt) do {} while (0)

__device__ __forceinline__ void st16_wt(char* p, u32x4 v) {
    asm volatile("global_store_dwordx4 %0, %1, off sc0 sc1\n\ts_nop 1" :: "v"(p), "v"(v) : "memory");
}
__device__ __forceinline__ void st4_wt(char* p, u32 v) {
    asm volatile("global_store_dword %0, %1, off sc0 sc1" :: "v"(p), "v"(v) : "memory");
}
__device__ __forceinline__ void vm_wait0() { asm volatile("s_waitcnt vmcnt(0)" ::: "memory"); }
__device__ __forceinline__ void lgkm_wait0() { asm volatile("s_waitcnt lgkmcnt(0)" ::: "memory"); }
__device__ __forceinline__ void bar() { asm volatile("s_waitcnt lgkmcnt(0)\n\ts_barrier" ::: "memory"); }
__device__ __forceinline__ void lds_st(u32 addr, u32 v) { asm volatile("ds_write_b32 %0, %1" :: "v"(addr), "v"(v) : "memory"); }
__device__ __forceinline__ u32 lds_ld(u32 addr) {
    u32 v; asm volatile("ds_read_b32 %0, %1\n\ts_waitcnt lgkmcnt(0)" : "=v"(v) : "v"(addr) : "memory"); return v;
}
__device__ __forceinline__ u32x4 lds_ld16(u32 addr) { return *(const LAS u32x4*)addr; }
template <int N> __device__ __forceinline__ u32 landed_first(u32 v) {
    u32 r; asm volatile("s_waitcnt vmcnt(%2)\n\tv_readfirstlane_b32 %0, %1\n\ts_nop 3" : "=s"(r) : "v"(v), "n"(N) : "memory"); return r;
}
__device__ __forceinline__ void dma1(const char* src, u32 lds_dst) {
    u32 keep;
    asm volatile("s_mov_b32 %0, m0\n\ts_mov_b32 m0, %2\n\ts_nop 0\n\tglobal_load_lds_dwordx4 %1, off\n\ts_mov_b32 m0, %0"
                 : "=&s"(keep) : "v"(src), "s"(lds_dst) : "memory");
}
__device__ __forceinline__ void dma1_sys(const char* src, u32 lds_dst) {
    u32 keep;
    asm volatile("s_mov_b32 %0, m0\n\ts_mov_b32 m0, %2\n\ts_nop 0\n\tglobal_load_lds_dwordx4 %1, off sc0 sc1\n\ts_mov_b32 m0, %0"
                 : "=&s"(keep) : "v"(src), "s"(lds_dst) : "memory");
}
__device__ __forceinline__ void dma_a(const char* sbase, u32 voff, u32 lds_dst) {
    asm volatile("s_mov_b32 m0, %2\n\ts_nop 0\n\tglobal_load_lds_dwordx4 %0, %1 sc0 sc1" :: "v"(voff), "s"(sbase), "s"(lds_dst) : "memory");
}
__device__ __forceinline__ void dma_b(const char* sbase, u32 voff) {
    asm volatile("global_load_lds_dwordx4 %0, %1 offset:1024 sc0 sc1" :: "v"(voff), "s"(sbase) : "memory");
}
__device__ __forceinline__ void wait_counters(GAS u32* c, u32 want, GAS u32* tmo, u32 call) {
    const u64 t0 = __builtin_amdgcn_s_memrealtime();
    for (u32 it = 1; !__all(__hip_atomic_load(c, __ATOMIC_RELAXED, __HIP_MEMORY_SCOPE_SYSTEM) >= want); ++it) {
        if ((it & 15u) == 0u
            && (__builtin_amdgcn_readfirstlane(__hip_atomic_load(tmo, __ATOMIC_RELAXED, __HIP_MEMORY_SCOPE_AGENT)) == call
                || __builtin_amdgcn_s_memrealtime() - t0 > DEADLINE)) {
            __hip_atomic_store(tmo, call, __ATOMIC_RELAXED, __HIP_MEMORY_SCOPE_AGENT);
            break;
        }
        __builtin_amdgcn_s_sleep(2);
    }
    asm volatile("s_waitcnt vmcnt(0)" ::: "memory");
}

__device__ __forceinline__ float gelu(float y) {
    const float k0 = -2.0f * 0.7978845608028654f * 1.4426950408889634f;
    const float u = y * y;
    const float z = y * __builtin_fmaf(u, k0 * 0.044715f, k0);
    return y * __builtin_amdgcn_rcpf(1.0f + __builtin_amdgcn_exp2f(z));
}
__device__ __forceinline__ u32 pk_bf16(float a, float b) {
    f32x2 f = {a, b};
    return __builtin_bit_cast(u32, __builtin_convertvector(f, bf16x2));
}

constexpr u64 IN_REL0 = 0x1234567'8d9abcull, IN_REL1 = 0x123d456'789cabull;
constexpr u32 OUT_E0 = 114, OUT_NSYM = 184;
__device__ constexpr unsigned char OUT_RANK[192] = {
    210, 209, 208, 207, 104, 206, 96, 105, 205, 97, 204, 98, 203, 106, 99, 202, 100, 201, 88, 101, 107, 200, 89,
    90, 102, 91, 103, 92, 108, 93, 94, 95, 109, 112, 80, 81, 82, 83, 84, 110, 85, 86, 87, 199, 198, 111,
    197, 196, 195, 194, 193, 192, 113, 72, 73, 74, 75, 76, 77, 78, 79, 114, 65, 191, 64, 66, 190, 67, 189,
    68, 188, 69, 187, 70, 71, 186, 185, 184, 115, 57, 58, 56, 59, 60, 61, 62, 63, 183, 182, 181, 180, 179,
    178, 177, 176, 116, 49, 50, 51, 52, 53, 48, 54, 55, 175, 174, 173, 172, 171, 170, 169, 168, 41, 42, 43,
    44, 45, 46, 47, 40, 117, 167, 166, 165, 164, 163, 162, 161, 160, 33, 34, 35, 36, 37, 38, 39, 32, 118,
    159, 158, 157, 156, 155, 154, 153, 152, 25, 26, 27, 28, 29, 30, 31, 24, 151, 150, 149, 147, 148, 146, 145,
    144, 119, 17, 18, 19, 20, 21, 23, 22, 16, 137, 138, 139, 140, 141, 142, 143, 136, 11, 9, 10, 12, 13};
__device__ __forceinline__ constexpr u32x2 enc_words(u32 c1, u32 q) {
    const u32 c2 = q < 3u ? q : 3u, c3 = q < 3u ? 0u : (q < 6u ? q - 3u : 3u), c4 = q < 6u ? 0u : q - 6u;
    return u32x2{(c1 << 28) | c2 | (c3 << 8) | (c4 << 16), 2u | (q >= 3u ? 2u << 8 : 0u) | (q >= 6u ? 3u << 16 : 0u)};
}
template <bool OUT> __device__ __forceinline__ void put_enc_table(u32 l, int lane) {
    const u32x2 esc = enc_words(OUT ? 15u : 0u, 13u);
#pragma unroll
    for (int j = 0; j < (OUT ? 2 : 4); ++j)
        *(LAS u32x4*)(l + j * 1024 + lane * 16) = u32x4{esc.x, esc.y | (1u << 24), esc.x, esc.y | (1u << 24)};
    if (OUT) {
#pragma unroll
        for (u32 r = lane; r < 192u; r += 64u)
            if (r < OUT_NSYM) *(LAS u32x2*)(l + OUT_RANK[r] * 8u) = r < 3u ? u32x2{r << 28, 0u} : enc_words(3u + (r - 3u) % 13u, (r - 3u) / 13u);
    } else {
        const u32 q = lane & 15u, m = ((u32)lane >> 4) & 1u;
        if (lane < 32 && q < 13u) *(LAS u32x2*)(l + ((((u32)((m ? IN_REL1 : IN_REL0) >> (4u * q)) & 15u) << 1) | m) * 8u) = enc_words(0u, q);
    }
}
template <bool OUT> __device__ __forceinline__ void put_dec_table(u32 l, int lane) {
    if (OUT) {
#pragma unroll
        for (u32 i = lane; i < 256u; i += 64u) {
            const u32 c1 = i >> 4, q = i & 15u, r = c1 < 3u ? c1 : 13u * q + c1, x = OUT_RANK[r < OUT_NSYM ? r : 0u];
            *(LAS unsigned short*)(l + i * 2u) =
                (unsigned short)(r < OUT_NSYM ? ((x >> 7) << 15) | ((((x >> 3) & 15u) + OUT_E0) << 7) | ((x & 7u) << 4) : 1u);
        }
    } else {
        const u32 c1 = (u32)lane >> 4, q = lane & 15u;
        *(LAS unsigned short*)(l + c1 * 64u + q * 2u) = (unsigned short)(
            q < 13u ? ((c1 >> 1) << 15) | (((u32)(((c1 & 1u) ? IN_REL1 : IN_REL0) >> (4u * q)) & 15u) << 7) | ((c1 & 1u) << 6) : 1u);
    }
}

__device__ __forceinline__ u32 wave_scan(u32 v) {
    v += (u32)__builtin_amdgcn_update_dpp(0, (int)v, 0x111, 0xf, 0xf, false);
    v += (u32)__builtin_amdgcn_update_dpp(0, (int)v, 0x112, 0xf, 0xf, false);
    v += (u32)__builtin_amdgcn_update_dpp(0, (int)v, 0x114, 0xf, 0xf, false);
    v += (u32)__builtin_amdgcn_update_dpp(0, (int)v, 0x118, 0xf, 0xf, false);
    v += (u32)__builtin_amdgcn_update_dpp(0, (int)v, 0x142, 0xa, 0xf, false);
    v += (u32)__builtin_amdgcn_update_dpp(0, (int)v, 0x143, 0xc, 0xf, false);
    return v;
}
__device__ __forceinline__ u32 ubfe(u32 v, u32 off, u32 w) { return __builtin_amdgcn_ubfe(v, off, w); }
__device__ __forceinline__ void lds_or_bits(u32 base, u32 pos, u32 bits) {
    if (bits) {
        LAS u32* p = (LAS u32*)(base + (pos >> 5) * 4u);
        __hip_atomic_fetch_or(p, bits << (pos & 31u), __ATOMIC_RELAXED, __HIP_MEMORY_SCOPE_WORKGROUP);
        __hip_atomic_fetch_or(p + 1, (bits >> 1) >> (~pos & 31u), __ATOMIC_RELAXED, __HIP_MEMORY_SCOPE_WORKGROUP);
    }
}
__device__ __forceinline__ u32 lds_get_bits(u32 base, u32 pos) {
    const LAS u32* p = (const LAS u32*)(base + (pos >> 5) * 4u);
    return __builtin_amdgcn_alignbit(p[1], p[0], pos & 31u);
}
__device__ __forceinline__ u32 low_mask_even(u32 n) { return ~((0xffffffffu << (n >> 1)) << (n >> 1)); }

#define SDWA_SHL(B) \
    __device__ __forceinline__ u32 shl_byte##B(u32 sh, u32 v) {     \
        u32 r; asm("v_lshlrev_b32_sdwa %0, %1, %2 dst_sel:DWORD dst_unused:UNUSED_PAD src0_sel:BYTE_" #B " src1_sel:BYTE_" #B \
                   : "=v"(r) : "v"(sh), "v"(v)); return r; }
SDWA_SHL(0) SDWA_SHL(1) SDWA_SHL(2)

template <int B> __device__ __forceinline__ u32 and_byte(u32 v, u32 m) {
    u32 r;
    if constexpr (B == 0) asm("v_and_b32_sdwa %0, %1, %2 dst_sel:DWORD dst_unused:UNUSED_PAD src0_sel:BYTE_0 src1_sel:DWORD" : "=v"(r) : "v"(v), "v"(m));
    if constexpr (B == 1) asm("v_and_b32_sdwa %0, %1, %2 dst_sel:DWORD dst_unused:UNUSED_PAD src0_sel:BYTE_1 src1_sel:DWORD" : "=v"(r) : "v"(v), "v"(m));
    if constexpr (B == 2) asm("v_and_b32_sdwa %0, %1, %2 dst_sel:DWORD dst_unused:UNUSED_PAD src0_sel:BYTE_2 src1_sel:DWORD" : "=v"(r) : "v"(v), "v"(m));
    if constexpr (B == 3) asm("v_and_b32_sdwa %0, %1, %2 dst_sel:DWORD dst_unused:UNUSED_PAD src0_sel:BYTE_3 src1_sel:DWORD" : "=v"(r) : "v"(v), "v"(m));
    return r;
}

__device__ __forceinline__ void store_input_bytes(char* slot, const u32 (&d)[8], int wave, int lane) {
    u32 bb[4];
#pragma unroll
    for (int j = 0; j < 4; ++j) {
        const u32 lowb = __builtin_amdgcn_perm(d[2 * j + 1], d[2 * j], 0x06040200u);
        const u32 higb = __builtin_amdgcn_perm(d[2 * j + 1], d[2 * j], 0x07050301u);
        bb[j] = (lowb & 0x7f7f7f7fu) | (higb & 0x80808080u);
    }
    st16_wt(slot + wave * 1024 + lane * 16, u32x4{bb[0], bb[1], bb[2], bb[3]});
}
constexpr u32 RAW_OFF = SLOT;
template <bool OUT>
__device__ __forceinline__ void encode_block(char* slot, const u32 (&d)[8], u32 ltbl, u32 lpriv, u32 ltot, int wave, int lane) {
    *(LAS u32x4*)(lpriv + lane * 16) = u32x4{0u, 0u, 0u, 0u};
    u32 e0 = 0u;
    u32 en[16], in[16];
    if (!OUT) {
        const u32 s2 = ((d[0] >> 7) & 0x00ff00ffu) + ((d[5] >> 7) & 0x00ff00ffu);
        const u32 tot = __builtin_amdgcn_readlane(wave_scan((s2 & 0xffffu) + (s2 >> 16)), 63);
        const u32 anchor = (tot + 504u) >> 8;
        e0 = anchor < 12u ? 0u : (anchor > 252u ? 240u : anchor - 12u);
    }
    typedef unsigned short u16x2 __attribute__((ext_vector_type(2)));
#pragma unroll
    for (int j = 0; j < 8; ++j) {
        u32 i0, i1;
        if (OUT) {
            const u16x2 sub = {OUT_E0 << 6, OUT_E0 << 6}, cap = {15 << 6, 15 << 6};
            const u16x2 t = __builtin_elementwise_min(__builtin_bit_cast(u16x2, (d[j] >> 1) & 0x3ff83ff8u) - sub, cap);
            const u32 x = __builtin_bit_cast(u32, t) | ((d[j] >> 5) & 0x04000400u);
            i0 = x & 0xffffu; i1 = x >> 16;
        } else {
            const u16x2 sub = {(unsigned short)(e0 << 7), (unsigned short)(e0 << 7)};
            const u32 x = __builtin_bit_cast(u32, __builtin_bit_cast(u16x2, d[j]) - sub);
            i0 = (x >> 3) & 0xff8u; i1 = (x >> 19) & 0xff8u;
        }
        const u32x2 a = *(const LAS u32x2*)(ltbl + i0), b = *(const LAS u32x2*)(ltbl + i1);
        en[2 * j] = a.x; in[2 * j] = a.y; en[2 * j + 1] = b.x; in[2 * j + 1] = b.y;
    }
    __builtin_amdgcn_sched_barrier(0);
    if (OUT) {
        u32 bb[4];
#pragma unroll
        for (int j = 0; j < 4; ++j) {
            const u32 lowb = __builtin_amdgcn_perm(d[2 * j + 1], d[2 * j], 0x06040200u);
            const u32 c01 = __builtin_amdgcn_perm(en[4 * j + 1], en[4 * j], 0x0c0c0703u);
            const u32 c23 = __builtin_amdgcn_perm(en[4 * j + 3], en[4 * j + 2], 0x07030c0cu);
            bb[j] = (lowb & 0x0f0f0f0fu) | c01 | c23;
        }
        st16_wt(slot + wave * 1024 + lane * 16, u32x4{bb[0], bb[1], bb[2], bb[3]});
    }
    u32 f2 = 0, f3 = 0, f4[2] = {0, 0}, pos = 0, n4a = 0;
#pragma unroll
    for (int i = 0; i < 16; i += 2) {
        if (i == 8) { n4a = ubfe(pos, 16, 8); pos &= 0xff00ffffu; }
        const u32 pos1 = pos + in[i];
        f2 |= shl_byte0(pos, en[i]) | shl_byte0(pos1, en[i + 1]);
        f3 |= shl_byte1(pos, en[i]) | shl_byte1(pos1, en[i + 1]);
        f4[i >> 3] |= shl_byte2(pos, en[i]) | shl_byte2(pos1, en[i + 1]);
        pos = pos1 + in[i + 1];
    }
    const u32 n4 = n4a + ubfe(pos, 16, 8), ne = pos >> 24;
    const u32 ca = (pos & 0xffu) | ((pos << 8) & 0xff0000u), cb = n4 | (ne << 16);
    const u32 sa = wave_scan(ca), sb = wave_scan(cb);
    const u32 ta = __builtin_amdgcn_readlane(sa, 63), tb = __builtin_amdgcn_readlane(sb, 63);
    const u32 N2 = ta & 0xffffu, N3 = ta >> 16, NE = tb >> 16;
    const u32 escpos = (N2 + N3 + (tb & 0xffffu) + 15u) & ~15u;
    const u32 dw = (escpos + 16u * NE + 31u) >> 5;
    if (lane == 0) *(LAS u32*)(ltot + wave * 4) = (dw <= 252u ? dw : 0x1000u) | (e0 << 16);
    if (dw <= 252u) {
        const u32 xa = sa - ca, xb = sb - cb;
        if (OUT) lds_or_bits(lpriv, xa & 0xffffu, f2);
        else *(LAS u32*)(lpriv + lane * 4) = f2;
        lds_or_bits(lpriv, N2 + (xa >> 16), f3);
        const u32 p4 = N2 + N3 + (xb & 0xffffu);
        if (!__builtin_amdgcn_ballot_w64(n4 > 32u)) {
            lds_or_bits(lpriv, p4, f4[0] | (f4[1] << n4a));
        } else {
            lds_or_bits(lpriv, p4, f4[0]);
            lds_or_bits(lpriv, p4 + n4a, f4[1]);
        }
        if (NE) {
            u32 a = lpriv + (escpos >> 3) + (xb >> 16) * 2u;
#pragma unroll
            for (int j = 0; j < 4; ++j)
                if ((in[4 * j] | in[4 * j + 1] | in[4 * j + 2] | in[4 * j + 3]) >> 24) {
#pragma unroll
                    for (int i = 4 * j; i < 4 * j + 4; ++i)
                        if (in[i] >> 24) { *(LAS unsigned short*)a = (unsigned short)(d[i >> 1] >> (16 * (i & 1))); a += 2u; }
                }
        }
    }
}
__device__ __forceinline__ u32 encode_finish(char* slot, const u32 (&d)[8], u32 lpriv, u32 ltot, int wave, int lane, bool split, u32& lo2) {
    const u32 mine = *(const LAS u32*)(ltot + (lane & 7) * 4);
    const u32 w0 = *(const LAS u32*)(lpriv + lane * 4), w1 = *(const LAS u32*)(lpriv + 256 + lane * 4);
    u32 inc = mine & 0xffffu;
    inc += (u32)__builtin_amdgcn_update_dpp(0, (int)inc, 0x111, 0xf, 0xf, false);
    inc += (u32)__builtin_amdgcn_update_dpp(0, (int)inc, 0x112, 0xf, 0xf, false);
    inc += (u32)__builtin_amdgcn_update_dpp(0, (int)inc, 0x114, 0xf, 0xf, false);
    const u32 total = __builtin_amdgcn_readlane(inc, 7);
    if (total >= 0x1000u) {
        char* p = slot + RAW_OFF + wave * 2048 + lane * 16;
        st16_wt(p, u32x4{d[0], d[1], d[2], d[3]}); st16_wt(p + 1024, u32x4{d[4], d[5], d[6], d[7]});
        lo2 = F_RAW;
        return F_RAW;
    }
    const u32 first = split ? __builtin_amdgcn_readlane(inc, 3) : 0u;
    const u32 hdr = split ? 16u : 32u;
    const u32 dw = __builtin_amdgcn_readlane(mine, wave) & 0xffffu;
    const u32 start = __builtin_amdgcn_readlane(inc, wave) - dw - (wave >= 4 ? first : 0u);
    char* p = slot + 8192 + (split && wave >= 4 ? HOFF : 0u) + hdr + (start + lane) * 4u;
    if ((u32)lane < dw) st4_wt(p, w0);
    if ((u32)lane + 64u < dw) st4_wt(p + 256, w1);
    if (dw > 128u) {
        if ((u32)lane + 128u < dw) st4_wt(p + 512, *(const LAS u32*)(lpriv + 512 + lane * 4));
        if ((u32)lane + 192u < dw) st4_wt(p + 768, *(const LAS u32*)(lpriv + 768 + lane * 4));
    }
    if (wave == 0 && lane < 8)
        st4_wt(slot + 8192 + lane * 4 + (split && lane >= 4 ? HOFF - 16u : 0u),
               (inc - (mine & 0xffffu) - (lane >= 4 ? first : 0u)) | (mine & 0xffff0000u));
    lo2 = (hdr + (total - first) * 4u + 15u) & ~15u;
    return (hdr + (split ? first : total) * 4u + 15u) & ~15u;
}

template <bool OUT>
__device__ __forceinline__ void decode_block(u32 sl, u32 blk, u32 lhdr, u32 lvar, u32 ltbl, int lane, u32x4& v0, u32x4& v1) {
    const u32x4 B = lds_ld16(sl + blk * 1024 + lane * 16);
    const u32 hdr = __builtin_amdgcn_readfirstlane(*(const LAS u32*)lhdr);
    const u32 start = (hdr & 0xffffu) > 2040u ? 2040u : (hdr & 0xffffu);
    const u32 base = lvar + start * 4u;
    const u32 eadd = ((hdr >> 16) & 0xffu) * 0x00800080u;
    const u32 bb[4] = {B.x, B.y, B.z, B.w};
    u32 F2, N2, W2[4];
    if (OUT) {
        u32 c = 0;
#pragma unroll
        for (int j = 0; j < 4; ++j) {
            W2[j] = ((((bb[j] >> 4) & 0x0f0f0f0fu) + 0x0d0d0d0du) >> 3) & 0x02020202u;
            c += __builtin_popcount(W2[j]);
        }
        const u32 n2 = 2u * c, s2 = wave_scan(n2);
        N2 = __builtin_amdgcn_readlane(s2, 63);
        F2 = lds_get_bits(base, s2 - n2) & low_mask_even(n2);
    } else {
        F2 = *(const LAS u32*)(base + lane * 4);
        N2 = 2048u;
    }
    const u32 W3 = F2 & (F2 << 1) & 0xaaaaaaaau;
    const u32 n3 = 2u * __builtin_popcount(W3), s3 = wave_scan(n3);
    const u32 N3 = __builtin_amdgcn_readlane(s3, 63);
    const u32 F3 = lds_get_bits(base, N2 + s3 - n3) & low_mask_even(n3);
    const u32 K4 = F3 & (F3 >> 1) & 0x55555555u, W4 = K4 * 3u;
    const u32 n4 = 3u * __builtin_popcount(K4), s4 = wave_scan(n4);
    const u32 N4 = __builtin_amdgcn_readlane(s4, 63);
    const u32 P4 = N2 + N3 + s4 - n4;
    const LAS u32* q4 = (const LAS u32*)(base + (P4 >> 5) * 4u);
    const u32 q40 = q4[0], q41 = q4[1], q42 = q4[2];
    u32 F4 = __builtin_amdgcn_alignbit(q41, q40, P4 & 31u);
    const u32 F4h = __builtin_amdgcn_alignbit(q42, q41, P4 & 31u);
    u32 t[16], p2 = 0, p3 = 0, p4 = 0;
    const u32 cm = OUT ? 0xf0u : 0xc0u;
    static_for<16>([&](auto ii) INL {
        constexpr int i = decltype(ii)::value;
        if (i == 8) { F4 = __builtin_amdgcn_alignbit(F4h, F4, p4); p4 = 0; }
        const u32 c1 = and_byte<i & 3>(bb[i >> 2], cm);
        u32 c2, w3;
        if (OUT) { const u32 w2 = and_byte<i & 3>(W2[i >> 2], cm - 1u); c2 = ubfe(F2, p2, w2); w3 = ubfe(W3, p2, w2); p2 += w2; }
        else { c2 = ubfe(F2, 2 * i, 2); w3 = ubfe(W3, 2 * i, 2); }
        const u32 c3 = ubfe(F3, p3, w3), w4 = ubfe(W4, p3, w3); p3 += w3;
        const u32 c4 = ubfe(F4, p4, w4); p4 += w4;
        t[i] = OUT ? (c1 + c2 + c3 + c4) << 1 : c1 + ((c2 + c3 + c4) << 1);
    });
    u32 o[8];
#pragma unroll
    for (int j = 0; j < 8; ++j) {
        typedef unsigned short u16x2 __attribute__((ext_vector_type(2)));
        const u16x2 pr = {*(const LAS unsigned short*)(ltbl + t[2 * j]), *(const LAS unsigned short*)(ltbl + t[2 * j + 1])};
        o[j] = __builtin_bit_cast(u32, pr) + (OUT ? 0u : eadd);
    }
    const u32 any = (o[0] | o[1] | o[2]) | (o[3] | o[4] | o[5]) | (o[6] | o[7]);
    if (__builtin_amdgcn_ballot_w64((any & 0x00010001u) != 0u)) {
        u32 c = 0;
#pragma unroll
        for (int j = 0; j < 8; ++j) c += o[j] & 0x00010001u;
        const u32 ne = (c & 0xffffu) + (c >> 16);
        u32 a = base + (((N2 + N3 + N4 + 15u) & ~15u) >> 3) + (wave_scan(ne) - ne) * 2u;
        const u32 amax = sl + SLOT - 4u;
#pragma unroll
        for (int j = 0; j < 8; ++j) {
            if (o[j] & 0x00010001u) {
                a = a < amax ? a : amax;
                if (o[j] & 1u) { o[j] = (o[j] & 0xffff0000u) | *(const LAS unsigned short*)a; a += 2u; }
                if (o[j] & 0x10000u) { o[j] = (o[j] & 0xffffu) | ((u32)*(const LAS unsigned short*)a << 16); a += 2u; }
            }
        }
    }
#pragma unroll
    for (int j = 0; j < 8; ++j)
        o[j] |= __builtin_amdgcn_perm(0u, bb[j >> 1] & (OUT ? 0x0f0f0f0fu : 0x3f3f3f3fu), (j & 1) ? 0x0c030c02u : 0x0c010c00u);
    v0 = u32x4{o[0], o[1], o[2], o[3]};
    v1 = u32x4{o[4], o[5], o[6], o[7]};
}

__global__ void __launch_bounds__(THREADS, 1) fused_gemm_allgather(Params P) {
    const int lane = threadIdx.x & 63;
    const int wave = __builtin_amdgcn_readfirstlane(threadIdx.x >> 6);
    int b = blockIdx.x;
    if (MAP_ == 1) {
        const int x = b & 7, y = b >> 3;
        if (x >= 4 && y >= 28) return;
        b = x < 4 ? (y >> 1) * 8 + 2 * x + (y & 1) : NGEMM + (x - 4) * 28 + y;
    }
    if (MAP_ == 2) {
        const int x = b & 7, y = b >> 3;
        if ((x & 1) && y >= 28) return;
        b = !(x & 1) ? (y >> 1) * 8 + x + (y & 1) : NGEMM + (x >> 1) * 28 + y;
    }
    const int rank = P.rank;
    char* myws = P.myws;
    const char* px = P.x;
    const char* pw = P.w;
    asm volatile("" : "+s"(myws), "+s"(px), "+s"(pw));
    constexpr u32 lds0 = 0;
    GAS u32* const ctr = (GAS u32*)(myws + OFF_CTR + (size_t)b * 128);
    GAS u32* const tmo = (GAS u32*)(myws + OFF_TMO);

    if (__builtin_expect(b < NGEMM, 1)) {
        const int tj = b & 7, ti0 = b >> 4, half = (b >> 3) & 1;
        const int npass = pack_passes(b);
        constexpr int KBAR = 65, EBAR = 3;
        if (wave == 8) {
            u32 cv;
            asm volatile("global_load_dword %0, %1, off sc1" : "=v"(cv) : "v"(ctr) : "memory");
            bar();
            put_enc_table<false>(L_ETIN, lane);
            const u32 call = landed_first<0>(cv) + 1u;
            bar();
            put_enc_table<true>(L_ETOUT, lane);
            for (int i = 0; i < 3 * npass - 1; ++i) bar();
            for (int tile = 0; tile < 2; ++tile) {
                GAS u32* c = (GAS u32*)(myws + OFF_CNT + (size_t)((lane & 1) ? 16 + tj : ti0 + 8 * tile) * 256);
                wait_counters(c + 32, call * 8u, tmo, call);
                for (int i = 0; i < 2 + 8 - NST; ++i) bar();
                u32 dummy = 0;
                auto prefetch = [&]() INL {
                    const char* pf = myws + OFF_UW + ((size_t)tj * 64 + (u32)((rank * 8 + 8 + (b >> 3) * 4) & 63)) * IMG + lane * 128;
                    if ((b >> 3) < 14) {
#pragma unroll
                        for (int j = 0; j < 8; ++j)
                            asm volatile("global_load_dword %0, %1, off sc1" : "+v"(dummy) : "v"(pf + j * 8192) : "memory");
                    }
                };
                const bool early = tile == 0 && ti0 >= tj;
                if (early) {
                    wait_counters((GAS u32*)(myws + OFF_CNT + (size_t)(16 + tj) * 256), call * 56u, tmo, call);
                    prefetch();
                }
                wait_counters(c, call * 56u, tmo, call);
                STAMP(6 + 6 * tile);
                bar();
                if (tile == 0 && !early) prefetch();
                if (tile == 0)
                    for (int i = 0; i < KBAR - (2 + 8 - NST) + EBAR; ++i) bar();
                asm volatile("s_waitcnt vmcnt(0)" : "+v"(dummy) :: "memory");
            }
            return;
        }
        if (wave == 0) STAMP(0);
        u32 cv;
        asm volatile("global_load_dword %0, %1, off sc1" : "=v"(cv) : "v"(ctr) : "memory");
        auto msg_of = [&](int rho, int& isw, int& grp, int& lp, int& kb, int& seq) INL {
            if (rho < 384) {
                const int t = rho / 48, r = rho % 48;
                isw = r < 32; grp = isw ? r >> 3 : (r - 32) >> 3; kb = r & 7; lp = t; seq = t * 16 + (isw ? 0 : 8) + kb;
            } else {
                const int t = 8 + ((rho - 384) >> 5), r = rho & 31, pos = (r >> 3) & 1;
                isw = 0; grp = r >> 4; kb = r & 7; lp = 8 + 2 * (t - 8) + pos; seq = t * 16 + pos * 8 + kb;
            }
        };
        auto fetch = [&](int z, int rho) INL {
            int isw, grp, lp, kb, seq;
            msg_of(rho, isw, grp, lp, kb, seq);
            const u32 dst = lds0 + z * IMG + wave * 2048;
            if (!isw) {
                const char* s = px + ((size_t)((grp * 16 + lp) * 128 + wave * 16 + (lane & 15)) * 512 + kb * 64 + (lane >> 4) * 8) * 2;
                dma1(s, dst); dma1(s + 64, dst + 1024);
            } else {
                const char* s = pw + ((size_t)(kb * 64 + wave * 8 + (lane >> 4)) * 4096 + (grp * 8 + lp) * 128 + (lane & 15) * 8) * 2;
                dma1(s, dst); dma1(s + 4 * 8192, dst + 1024);
            }
        };
        [[maybe_unused]] u64 tq[3];
        fetch(0, b);
        const u32 call = landed_first<0>(cv) + 1u;
        char* const wsl = P.ws[lane & 7];
        for (int z = 0; z < npass; ++z) {
            int isw, grp, lp, kb, seq;
            msg_of(z < 2 ? z * 128 + b : (int)SCHED.rho[b][z], isw, grp, lp, kb, seq);
            bar();
            if (z == 1) for (int y = 2; y < npass; ++y) fetch(y, SCHED.rho[b][y]);
            if (z == 1 && threadIdx.x == 0) __hip_atomic_store(ctr, call, __ATOMIC_RELAXED, __HIP_MEMORY_SCOPE_AGENT);
            u32x4 v0, v1;
            const u32 src = lds0 + z * IMG;
            if (!isw) {
                v0 = lds_ld16(src + wave * 2048 + lane * 16);
                v1 = lds_ld16(src + wave * 2048 + 1024 + lane * 16);
            } else {
                const u32 a = src + ((lane >> 4) * 8 + ((lane & 15) >> 2)) * 256 + (32 * (wave >> 1) + 8 * (lane & 3) + 4 * (wave & 1)) * 2;
                const s16x4 t0 = __builtin_amdgcn_ds_read_tr16_b64_v4i16((LAS s16x4*)(a));
                const s16x4 t1 = __builtin_amdgcn_ds_read_tr16_b64_v4i16((LAS s16x4*)(a + 4 * 256));
                const s16x4 t2 = __builtin_amdgcn_ds_read_tr16_b64_v4i16((LAS s16x4*)(a + 32 * 256));
                const s16x4 t3 = __builtin_amdgcn_ds_read_tr16_b64_v4i16((LAS s16x4*)(a + 36 * 256));
                const u32x2 a0 = __builtin_bit_cast(u32x2, t0), a1 = __builtin_bit_cast(u32x2, t1);
                const u32x2 a2 = __builtin_bit_cast(u32x2, t2), a3 = __builtin_bit_cast(u32x2, t3);
                v0 = u32x4{a0.x, a0.y, a1.x, a1.y};
                v1 = u32x4{a2.x, a2.y, a3.x, a3.y};
            }
            const int mid = (isw ? 256 + (grp * 8 + lp) * 8 : (grp * 16 + lp) * 8) + kb;
            [[maybe_unused]] u64 ts[4];
            TS(0);
            const size_t imgoff = (isw ? OFF_UW : OFF_UX) + (size_t)(lp * 64 + rank * 8 + kb) * IMG + wave * 2048 + lane * 16;
            char* const slot = myws + OFF_PKIN + (size_t)mid * GSLOT;
            const u32 d[8] = {v0.x, v0.y, v0.z, v0.w, v1.x, v1.y, v1.z, v1.w};
            store_input_bytes(slot, d, wave, lane);
            const bool dest = lane < NR && lane != rank && (isw ? ((lane & 3) == grp) : ((lane >> 2) == grp));
            GAS u64* const flag = (GAS u64*)(wsl + OFF_FLAG + ((size_t)rank * NMSG + seq) * 128);
            if (z == 0) {
                if (seq < PER_LINK) vm_wait0();
                bar();
                if (seq < PER_LINK && wave == 0 && dest)
                    __hip_atomic_store((GAS u32*)(flag + 1), call, __ATOMIC_RELAXED, __HIP_MEMORY_SCOPE_SYSTEM);
                fetch(1, 128 + b);
            }
            TS(1);
            encode_block<false>(slot, d, L_ETIN, L_VAR + wave * 1024, L_TOT, wave, lane);
            bar();
            TS(2);
            u32 lo2;
            const u32 lo = encode_finish(slot, d, L_VAR + wave * 1024, L_TOT, wave, lane, false, lo2);
            TS(3);
            const bool own = isw ? (grp == (rank & 3)) : (grp == (rank >> 2));
            if (own) { st16_wt(myws + imgoff, v0); st16_wt(myws + imgoff + 1024, v1); }
            vm_wait0();
            bar();
            if (wave == 0) {
                if (dest) __hip_atomic_store(flag, ((u64)call << 32) | lo, __ATOMIC_RELAXED, __HIP_MEMORY_SCOPE_SYSTEM);
                if (own && lane == 0)
                    __hip_atomic_fetch_add((GAS u32*)(myws + OFF_CNT + (size_t)((isw ? 16 : 0) + lp) * 256 + 128), 1u,
                                           __ATOMIC_RELAXED, __HIP_MEMORY_SCOPE_AGENT);
                STAMP(1 + z);
            }
        }
        u32 ords = __builtin_amdgcn_readfirstlane((u32)SCHED.ord[0][b] | ((u32)SCHED.ord[1][b] << 8));
        asm volatile("s_waitcnt vmcnt(0)" : "+s"(ords) :: "memory");
        const int kh = wave >> 2, wn = wave & 1, wm = (wave >> 1) & 1;
        const char* const wb = myws + OFF_UW + (size_t)tj * 64 * IMG;
        const u32 voffx = wave * 1024 + lane * 16, voffw = wave * 2048 + lane * 16;
        const u32 xa = lds0 + (wm * 4 + kh) * 1024 + lane * 16;
        const u32 wa = lds0 + 8192 + (wn * 8 + kh) * 1024 + lane * 16;
#define SB __builtin_amdgcn_sched_barrier(0)
        for (int tile = 0; tile < 2; ++tile) {
            const int ti = ti0 + 8 * tile;
            const char* const xb = myws + OFF_UX + (size_t)ti * 64 * IMG + half * 8192;
            f32x4 acc[4][2];
#pragma unroll
            for (int a = 0; a < 4; ++a) { acc[a][0] = f32x4{0.f, 0.f, 0.f, 0.f}; acc[a][1] = acc[a][0]; }
            bf16x8 wA[4], xA[2], wB[4], xB[2];
            auto mma = [&](int a, const bf16x8* wf, const bf16x8* xf) INL {
                acc[a][0] = __builtin_amdgcn_mfma_f32_16x16x32_bf16(wf[a], xf[0], acc[a][0], 0, 0, 0);
                acc[a][1] = __builtin_amdgcn_mfma_f32_16x16x32_bf16(wf[a], xf[1], acc[a][1], 0, 0, 0);
            };
            u32 buf = 0;
#define ROT_ 7
            u32 kq = tj * ROT_;
            auto step = [&](int kt, auto dma_tag, auto vm_tag, const bf16x8* wc, const bf16x8* xc, bf16x8* wx, bf16x8* xx) INL {
                constexpr bool DMA = decltype(dma_tag)::value;
                constexpr int VM = decltype(vm_tag)::value;
                const u32 kb = (u32)(rank * 8 + (kt + NST < 8 ? kt + NST : 8 + kq)) & 63u;
                if (kt + NST >= 8) kq = kq == 55u ? 0u : kq + 1u;
                const char* gx = xb + (size_t)kb * IMG; const char* gw = wb + (size_t)kb * IMG;
                const u32 d = buf;
                const u32 nb = buf + STG == NST * STG ? 0u : buf + STG;
                const u32 ra = xa + nb, rb = wa + nb;
                buf = nb;
                asm volatile("s_waitcnt lgkmcnt(0)\n\ts_waitcnt vmcnt(%0)\n\ts_barrier" :: "n"(VM) : "memory"); SB;
                if (DMA) dma_a(gx, voffx, d + wave * 1024); SB;
                mma(0, wc, xc); SB;
                xx[0] = *(const LAS bf16x8*)(ra); wx[0] = *(const LAS bf16x8*)(rb); SB;
                mma(1, wc, xc); SB;
                if (DMA) dma_a(gw, voffw, d + 8192 + wave * 2048); SB;
                xx[1] = *(const LAS bf16x8*)(ra + 2048); wx[1] = *(const LAS bf16x8*)(rb + 2048); SB;
                mma(2, wc, xc); SB;
                wx[2] = *(const LAS bf16x8*)(rb + 4096); wx[3] = *(const LAS bf16x8*)(rb + 6144); SB;
                if (DMA) dma_b(gw, voffw); SB;
                mma(3, wc, xc); SB;
            };
            using T_ = std::true_type; using F_ = std::false_type;
            bar();
#pragma unroll
            for (int st = 0; st < NST; ++st) {
                const u32 kb = (u32)(rank * 8 + st) & 63u;
                dma_a(xb + (size_t)kb * IMG, voffx, st * STG + wave * 1024);
                dma_a(wb + (size_t)kb * IMG, voffw, st * STG + 8192 + wave * 2048);
                dma_b(wb + (size_t)kb * IMG, voffw);
            }
            asm volatile("s_waitcnt vmcnt(%0)\n\ts_barrier" :: "n"((NST - 1) * 3) : "memory");
#pragma unroll
            for (int a = 0; a < 4; ++a) wA[a] = *(const LAS bf16x8*)(wa + a * 2048);
            xA[0] = *(const LAS bf16x8*)(xa); xA[1] = *(const LAS bf16x8*)(xa + 2048);
            static_assert(NST == 6, "tail below");
            using VMS = std::integral_constant<int, (NST - 2) * 3>;
            for (int kt = 0; kt < 64 - NST; kt += 2) {
                step(kt, T_{}, VMS{}, wA, xA, wB, xB);
                step(kt + 1, T_{}, VMS{}, wB, xB, wA, xA);
            }
            step(58, F_{}, std::integral_constant<int, 12>{}, wA, xA, wB, xB);
            step(59, F_{}, std::integral_constant<int, 9>{}, wB, xB, wA, xA);
            step(60, F_{}, std::integral_constant<int, 6>{}, wA, xA, wB, xB);
            step(61, F_{}, std::integral_constant<int, 3>{}, wB, xB, wA, xA);
            step(62, F_{}, std::integral_constant<int, 0>{}, wA, xA, wB, xB);
            u32 tv;
            asm volatile("global_load_dword %0, %1, off sc1" : "=v"(tv) : "v"(tmo) : "memory");
            bar();
            mma(0, wB, xB); mma(1, wB, xB); mma(2, wB, xB); mma(3, wB, xB);
            if (wave == 0) STAMP(7 + 6 * tile);
            {
                const u32 wr = lds0 + wave * 4096 + lane * 16;
#pragma unroll
                for (int a = 0; a < 4; ++a) *(LAS f32x4*)(wr + a * 1024) = kh ? acc[a][0] : acc[a][1];
            }
            bar();
            u32 d[8];
            {
                const u32 rd = lds0 + (wave ^ 4) * 4096 + lane * 16;
#pragma unroll
                for (int a = 0; a < 4; ++a) {
                    const f32x4 y = (kh ? acc[a][1] : acc[a][0]) + *(const LAS f32x4*)(rd + a * 1024);
                    d[a * 2] = pk_bf16(gelu(y.x), gelu(y.y));
                    d[a * 2 + 1] = pk_bf16(gelu(y.z), gelu(y.w));
                }
            }
            if (landed_first<0>(tv) == call) {
#pragma unroll
                for (int j = 0; j < 8; ++j) d[j] = 0x7fc07fc0u;
            }
            if (wave == 0) STAMP(8 + 6 * tile);
            const u32 ord = (ords >> (8 * tile)) & 0xffu;
            char* const oslot = myws + OFF_PKOUT + (size_t)ord * GSLOT;
            encode_block<true>(oslot, d, L_ETOUT, L_VAR + wave * 1024, L_TOT, wave, lane);
            bar();
            const bool split = OUT0 + ord >= HALF0;
            u32 lo2;
            const u32 lo = encode_finish(oslot, d, L_VAR + wave * 1024, L_TOT, wave, lane, split, lo2);
            if (wave == 0) STAMP(9 + 6 * tile);
            vm_wait0();
            bar();
            if (wave == 0 && lane < (split ? 16 : 8) && (lane & 7) != rank) {
                const u32 m = split ? HALF0 + (OUT0 + ord - HALF0) * 2u + (lane >> 3) : OUT0 + ord;
                GAS u64* f = (GAS u64*)(wsl + OFF_FLAG + ((size_t)rank * NMSG + m) * 128);
                __hip_atomic_store(f, ((u64)call << 32) | (lane < 8 ? lo : lo2), __ATOMIC_RELAXED, __HIP_MEMORY_SCOPE_SYSTEM);
            }
            char* o = P.out + ((size_t)((rank >> 2) * 2048 + ti * 128 + half * 64 + wm * 32 + kh * 16 + (lane & 15)) * 4096
                               + (rank & 3) * 1024 + tj * 128 + wn * 64 + (lane >> 4) * 8) * 2;
            st16_wt(o, u32x4{d[0], d[1], d[2], d[3]});
            st16_wt(o + 64, u32x4{d[4], d[5], d[6], d[7]});
            if (wave == 0) STAMP(10 + 6 * tile);
            if (tile == 0) vm_wait0();
        }
    } else {
        const int pu = b - NGEMM;
        const int u = pu / 7;
        const int src = (rank + 1 + pu % 7) & 7;
        const u32 call = __builtin_amdgcn_readfirstlane(__hip_atomic_load(ctr, __ATOMIC_RELAXED, __HIP_MEMORY_SCOPE_AGENT)) + 1u;
        if (wave == 0) lds_st(lds0 + L_CTL + lane * 4, 0u);
        if (wave == 1) put_dec_table<false>(L_DTBL, lane);
        if (wave == 8) STAMP(0);
        bar();
        if (threadIdx.x == 0) __hip_atomic_store(ctr, call, __ATOMIC_RELAXED, __HIP_MEMORY_SCOPE_AGENT);
        if (wave == 8) {
            vm_wait0();
            const char* const rws = P.ws[src];
            const char* const fbase = myws + OFF_FLAG + (size_t)src * NMSG * 128;
            char* const tkt = myws + OFF_TKT + (size_t)(pu % 7) * 256;
            const u32 tbase = (call - 1u) * (NMSG - PER_LINK) - 2 * PER_LINK;
            u32 m = u, n = 0, np = 0;
            u32 fl_lo = 0u, fl_hi = 0u;
            u32 pub = 0u;
            bool dead = false;
            [[maybe_unused]] u32 nst = 0;
            auto publish = [&](u32 k) INL {
                if (np > pub + k) { pub = np - k; lds_st(lds0 + L_FULL, pub); }
            };
            auto drain = [&](auto top) INL {
                constexpr int TOP = decltype(top)::value;
                static_for<TOP + 1>([&](auto i) INL {
                    constexpr int K = TOP - decltype(i)::value;
                    asm volatile("s_waitcnt vmcnt(%0)" :: "n"(K) : "memory");
                    publish(K);
                });
            };
            auto poll = [&](const char* fp) INL {
                const u64 t0 = __builtin_amdgcn_s_memrealtime();
                for (u32 it = 1;; ++it) {
                    u32x2 fl;
                    asm volatile("global_load_dwordx2 %0, %1, off sc0 sc1\n\ts_waitcnt vmcnt(0)" : "=&v"(fl) : "v"(fp) : "memory");
                    fl_lo = __builtin_amdgcn_readfirstlane(fl.x); fl_hi = __builtin_amdgcn_readfirstlane(fl.y);
                    if (fl_hi >= call) break;
                    if ((it & 15u) == 0u
                        && (__builtin_amdgcn_readfirstlane(__hip_atomic_load(tmo, __ATOMIC_RELAXED, __HIP_MEMORY_SCOPE_AGENT)) == call
                            || __builtin_amdgcn_s_memrealtime() - t0 > DEADLINE)) {
                        __hip_atomic_store(tmo, call, __ATOMIC_RELAXED, __HIP_MEMORY_SCOPE_AGENT);
                        dead = true;
                        break;
                    }
                    __builtin_amdgcn_s_sleep(1);
                }
            };
            auto remote_slot = [&](u32 mm) INL {
                const u32 t = mm / PER_LINK, v = mm % PER_LINK;
                size_t moff;
                if (t < IN_ROUNDS) {
                    const u32 kb = v & 7;
                    u32 mid;
                    if (t < 8) mid = v >= 8 ? ((rank >> 2) * 16 + t) * 8 + kb : 256 + ((rank & 3) * 8 + t) * 8 + kb;
                    else mid = ((rank >> 2) * 16 + 8 + 2 * (t - 8) + (v >> 3)) * 8 + kb;
                    moff = OFF_PKIN + (size_t)mid * GSLOT;
                } else {
                    moff = OFF_PKOUT + (size_t)((mm < HALF0 ? mm : HALF0 + ((mm - HALF0) >> 1)) - OUT0) * GSLOT;
                }
                return rws + moff + lane * 16;
            };
            {
                poll(fbase + m * 128 + 4);
                fl_hi = 0u;
                if (!dead) {
                    const char* sp = remote_slot(m);
#pragma unroll
                    for (int k = 0; k < 8; ++k) {
                        dma1_sys(sp + k * 1024, lds0 + L_RING + k * 1024);
                        asm volatile("s_waitcnt vmcnt(%0)" :: "n"(WIN - 1) : "memory");
                    }
                    np = 8u;
                }
            }
            for (;; ++n) {
                const u32 s = n % NSLOT;
                if (n < 2 && !dead) {
                    fl_lo = __builtin_amdgcn_readfirstlane(lds_ld(lds0 + L_F12 + n * 4));
                    if (!fl_lo) {
                        if (n) drain(std::integral_constant<int, WIN - 2>{});
                        while (!(fl_lo = __builtin_amdgcn_readfirstlane(lds_ld(lds0 + L_F12 + n * 4)))) __builtin_amdgcn_s_sleep(1);
                    }
                    dead = fl_lo & F_BAD;
                } else if (m < NMSG && !dead && fl_hi < call) {
                    drain(std::integral_constant<int, WIN - 2>{});
                    poll(fbase + m * 128);
                }
                if (n >= NSLOT) {
                    const u32 need = 8u * (n / NSLOT);
                    if (__builtin_amdgcn_readfirstlane(lds_ld(lds0 + L_FREE + s * 4)) < need) {
                        drain(std::integral_constant<int, WIN - 2>{});
                        while (__builtin_amdgcn_readfirstlane(lds_ld(lds0 + L_FREE + s * 4)) < need) __builtin_amdgcn_s_sleep(1);
                    }
                }
                if (m >= NMSG) {
                    lds_st(lds0 + L_MSG + s * 4, NMSG);
                    lds_st(lds0 + L_DESC, n + 1);
                    break;
                }
                const u32 lo = dead ? F_BAD : fl_lo;
                lds_st(lds0 + L_LEN + s * 4, lo);
                lds_st(lds0 + L_MSG + s * 4, m | (np << 16));
                lds_st(lds0 + L_DESC, n + 1);
                const Ranges R = msg_ranges(m, lo);
                const bool half = m >= HALF0;
                [[maybe_unused]] const u32 t = m / PER_LINK < ROUNDS ? m / PER_LINK : ROUNDS - 1, v = m % PER_LINK;
                const char* sp = remote_slot(m) + ((lo & F_RAW) ? RAW_OFF : 0u);
                if (n == 0) {
                    if (!dead) {
                        if (lo & F_RAW) vm_wait0();
                        for (u32 o = R.a0; o < R.a1; o += 1024u) {
                            if (o + lane * 16 < R.a1) dma1_sys(sp + o, lds0 + L_RING + o);
                            ++np;
                            asm volatile("s_waitcnt vmcnt(%0)" :: "n"(WIN - 1) : "memory");
                            publish(WIN - 1);
                        }
                    }
                    m += PER_LINK;
                    LSTAMP(L_STMP, nst, NGEMM + v * 7 + pu % 7, 8 + t);
                    continue;
                }
                u32 nt = 1u;
                u64 keep;
                asm volatile("s_mov_b64 %1, exec\n\ts_mov_b64 exec, 1\n\tglobal_atomic_add %0, %2, %0, off sc0\n\ts_mov_b64 exec, %1"
                             : "+v"(nt), "=&s"(keep) : "v"(tkt) : "memory");
                if (dead) {
                    m = landed_first<0>(nt) - tbase;
                    publish(0);
                    continue;
                }
                const u32 d = lds0 + L_RING + s * SLOT;
                const u32 cnt = ((R.a1 - R.a0 + 1023u) >> 10) + ((R.b1 - R.b0) >> 10);
                u32 o = R.a0, oend = R.a1;
                auto piece = [&]() INL {
                    if (o + lane * 16 < oend) dma1_sys(sp + o, d + o);
                    o += 1024u;
                    if (o >= oend) { o = R.b0; oend = R.b1; }
                    ++np;
                };
#pragma unroll
                for (int k = 0; k < WIN - 1; ++k) {
                    piece();
                    asm volatile("s_waitcnt vmcnt(%0)" :: "n"(WIN - 1) : "memory");
                    publish(WIN - 1);
                }
                m = landed_first<WIN - 1>(nt) - tbase;
                if (m >= NMSG) {
                    for (u32 k = WIN - 1; k < cnt; ++k) {
                        piece();
                        asm volatile("s_waitcnt vmcnt(%0)" :: "n"(WIN2 - 1) : "memory");
                        publish(WIN2 - 1);
                    }
                    drain(std::integral_constant<int, WIN2 - 2>{});
                } else {
                    u32x2 fn;
                    asm volatile("global_load_dwordx2 %0, %1, off sc0 sc1" : "=v"(fn) : "v"(fbase + m * 128) : "memory");
                    for (u32 k = WIN - 1; k < cnt; ++k) {
                        piece();
                        asm volatile("s_waitcnt vmcnt(%0)" :: "n"(WIN - 1) : "memory");
                        publish(WIN - 1);
                    }
                    if (half) asm volatile("s_waitcnt vmcnt(%0)" :: "n"(6 - WIN) : "memory");
                    fl_lo = landed_first<10 - WIN>(fn.x); fl_hi = landed_first<10 - WIN>(fn.y);
                }
                LSTAMP(L_STMP, nst, NGEMM + v * 7 + pu % 7, 8 + t);
            }
            STAMP(1);
            LFLUSH(L_STMP, nst);
        } else {
            if (wave == 0) {
                const char* fp = myws + OFF_FLAG + ((size_t)src * NMSG + u + (lane & 1) * PER_LINK) * 128;
                const u64 t0 = __builtin_amdgcn_s_memrealtime();
                for (u32 it = 1;; ++it) {
                    u32x2 fl;
                    asm volatile("global_load_dwordx2 %0, %1, off sc0 sc1\n\ts_waitcnt vmcnt(0)" : "=&v"(fl) : "v"(fp) : "memory");
                    const bool ok = fl.y >= call;
                    if (ok && lane < 2) lds_st(lds0 + L_F12 + lane * 4, fl.x & ~F_BAD);
                    if (__all(ok)) break;
                    if ((it & 15u) == 0u
                        && (__builtin_amdgcn_readfirstlane(__hip_atomic_load(tmo, __ATOMIC_RELAXED, __HIP_MEMORY_SCOPE_AGENT)) == call
                            || __builtin_amdgcn_s_memrealtime() - t0 > DEADLINE)) {
                        __hip_atomic_store(tmo, call, __ATOMIC_RELAXED, __HIP_MEMORY_SCOPE_AGENT);
                        if (!ok && lane < 2) lds_st(lds0 + L_F12 + lane * 4, F_BAD);
                        break;
                    }
                    __builtin_amdgcn_s_sleep(1);
                }
            }
            if (wave == 2) {
                put_dec_table<true>(L_DTBL + 256, lane);
                *(LAS u32*)(L_UNORD + lane * 4) = ((const u32*)SCHED.unord)[lane];
                lgkm_wait0();
                lds_st(lds0 + L_TRDY, 1u);
            }
            [[maybe_unused]] u32 nst = 0;
            u32 nhalf = 0;
            bool tables = false;
            for (u32 n = 0;; ++n) {
                const u32 s = n % NSLOT;
                while (__builtin_amdgcn_readfirstlane(lds_ld(lds0 + L_DESC)) <= n) __builtin_amdgcn_s_sleep(1);
                const u32 md = __builtin_amdgcn_readfirstlane(lds_ld(lds0 + L_MSG + s * 4)), m = md & 0xffffu;
                if (m >= NMSG) break;
                const u32 t = m / PER_LINK, v = m % PER_LINK;
                const u32 lo = __builtin_amdgcn_readfirstlane(lds_ld(lds0 + L_LEN + s * 4));
                const u32 sl = lds0 + L_RING + s * SLOT;
                if (m >= OUT0 && !tables) {
                    while (!__builtin_amdgcn_readfirstlane(lds_ld(lds0 + L_TRDY))) __builtin_amdgcn_s_sleep(1);
                    tables = true;
                }
                const bool half = m >= HALF0;
                if (half && (u32)(wave >> 2) != (nhalf++ & 1u)) {
                    if (lane == 0) asm volatile("ds_add_u32 %0, %1" :: "v"(lds0 + L_FREE + s * 4), "v"(1u) : "memory");
                    continue;
                }
                const u32 lb = half ? wave & 3 : wave, blk = half ? ((m - HALF0) & 1u) * 4u + lb : lb;
                const Ranges R = msg_ranges(m, lo);
                if (!(lo & F_BAD)) {
                    const u32 need = (md >> 16) + ((lo & F_RAW) ? 2u * lb + 2u : ((R.a1 - R.a0 + 1023u) >> 10) + (n ? lb + 1u : 0u));
                    while (__builtin_amdgcn_readfirstlane(lds_ld(lds0 + L_FULL)) < need) __builtin_amdgcn_s_sleep(1);
                }
                if (t == 3 && wave == 7) LSTAMP(L_STMP + 1024 + wave * 512, nst, NGEMM + v * 7 + pu % 7, 56);
                u32x4 v0, v1;
                if (lo & F_BAD) {
                    v0 = u32x4{0x7fc07fc0u, 0x7fc07fc0u, 0x7fc07fc0u, 0x7fc07fc0u}; v1 = v0;
                } else if (lo & F_RAW) {
                    v0 = lds_ld16(sl + blk * 2048 + lane * 16);
                    v1 = lds_ld16(sl + blk * 2048 + 1024 + lane * 16);
                } else if (t < IN_ROUNDS) {
                    decode_block<false>(sl, blk, sl + 8192 + lb * 4, sl + 8192 + 32, L_DTBL, lane, v0, v1);
                } else {
                    decode_block<true>(sl, blk, sl + R.a0 + lb * 4, sl + R.a0 + (half ? 16u : 32u), L_DTBL + 256, lane, v0, v1);
                }
                asm volatile("s_waitcnt lgkmcnt(0)" : "+v"(v0), "+v"(v1) :: "memory");
                if (lane == 0) asm volatile("ds_add_u32 %0, %1" :: "v"(lds0 + L_FREE + s * 4), "v"(1u) : "memory");
                if (t == 3 && wave == 7) LSTAMP(L_STMP + 1024 + wave * 512, nst, NGEMM + v * 7 + pu % 7, 57);
                if (t < IN_ROUNDS) {
                    const u32 kb = v & 7;
                    const bool isw = t < 8 && v < 8;
                    const u32 lp = t < 8 ? t : 8 + 2 * (t - 8) + (v >> 3);
                    char* img = myws + (isw ? OFF_UW : OFF_UX) + (size_t)(lp * 64 + src * 8 + kb) * IMG + wave * 2048 + lane * 16;
                    st16_wt(img, v0); st16_wt(img + 1024, v1);
                    vm_wait0();
                    u32 old = 0;
                    if (lane == 0) {
                        asm volatile("ds_add_rtn_u32 %0, %1, %2\n\ts_waitcnt lgkmcnt(0)" : "=&v"(old) : "v"(lds0 + L_DONE + s * 4), "v"(1u) : "memory");
                        if ((old & 7u) == 7u)
                            __hip_atomic_fetch_add((GAS u32*)(myws + OFF_CNT + (size_t)((isw ? 16 : 0) + lp) * 256), 1u,
                                                   __ATOMIC_RELAXED, __HIP_MEMORY_SCOPE_AGENT);
                    }
                } else {
                    const int g = __builtin_amdgcn_readfirstlane(*(const LAS unsigned char*)(L_UNORD + (half ? HALF0 + ((m - HALF0) >> 1) : m) - OUT0));
                    const u32 kh = blk >> 2, wn = blk & 1, wm = (blk >> 1) & 1;
                    const int ti = ((g >> 4) & 7) + 8 * (g >> 7), hf = (g >> 3) & 1, tj = g & 7;
                    char* op = P.out + ((size_t)((src >> 2) * 2048 + ti * 128 + hf * 64 + wm * 32 + kh * 16 + (lane & 15)) * 4096
                                        + (src & 3) * 1024 + tj * 128 + wn * 64 + (lane >> 4) * 8) * 2;
                    st16_wt(op, v0); st16_wt(op + 64, v1);
                }
            }
            STAMP(2);
            LFLUSH(L_STMP + 1024 + wave * 512, nst);
        }
    }
}

extern "C" void kernel_launch(int rank, int nranks, ncclComm_t comm,
                              hipStream_t stream,
                              void* const* d_in, const int* in_sizes, int n_in,
                              void* d_out, int out_size,
                              const int* in_dtypes, int out_dtype,
                              void* const* d_out_all,
                              void* const* d_ws_all, size_t ws_bytes) {
    (void)comm; (void)in_dtypes; (void)out_dtype; (void)d_out_all;
    if (nranks != NR || (unsigned)rank >= (unsigned)NR || n_in != 2 || in_sizes[0] != 4096 * 512 || in_sizes[1] != 512 * 4096
        || out_size != 4096 * 4096 || ws_bytes < WS_NEED) {
        fprintf(stderr, "kernel_launch: unexpected problem shape\n");
        return;
    }
    static bool attr_set[NR] = {};
    if (!attr_set[rank]) {
        HIP_CHECK(hipFuncSetAttribute((const void*)fused_gemm_allgather, hipFuncAttributeMaxDynamicSharedMemorySize, LDS_BYTES));
        attr_set[rank] = true;
    }
    Params P;
    for (int p = 0; p < NR; ++p) P.ws[p] = (char*)d_ws_all[p];
    P.x = (const char*)d_in[0];
    P.w = (const char*)d_in[1];
    P.out = (char*)d_out;
    P.myws = P.ws[rank];
    P.rank = rank;
    P.pad = 0;
    hipLaunchKernelGGL(fused_gemm_allgather, dim3(GRID), dim3(THREADS), LDS_BYTES, stream, P);
    HIP_CHECK(hipGetLastError());
}
```

```cpp
#include <hip/hip_runtime.h>
#include <rccl/rccl.h>
#include <cstdio>
#include <cstdint>
#include <cstddef>
#include <type_traits>
#pragma clang diagnostic ignored "-Wint-to-pointer-cast"

#define HIP_CHECK(expr) do { hipError_t _e = (expr); if (_e != hipSuccess) { \
    fprintf(stderr, "HIP error %s at %s:%d\n", hipGetErrorString(_e), __FILE__, __LINE__); \
    return; } } while (0)

typedef unsigned u32;
typedef unsigned long long u64;
typedef u32 u32x4 __attribute__((ext_vector_type(4)));
typedef u32 u32x2 __attribute__((ext_vector_type(2)));
typedef float f32x4 __attribute__((ext_vector_type(4)));
typedef float f32x2 __attribute__((ext_vector_type(2)));
typedef __bf16 bf16x8 __attribute__((ext_vector_type(8)));
typedef __bf16 bf16x2 __attribute__((ext_vector_type(2)));
typedef short s16x4 __attribute__((ext_vector_type(4)));
#define INL __attribute__((always_inline))
#define LAS __attribute__((address_space(3)))
#define GAS __attribute__((address_space(1)))

constexpr int NR = 8;
constexpr int NGEMM = 128;
constexpr int PER_LINK = 16;
constexpr int NPULL = 7 * PER_LINK;
#define MAP_ 1
constexpr int NROLE = NGEMM + NPULL;
constexpr int GRID = MAP_ ? 256 : NROLE;
constexpr int THREADS = 576;
constexpr int IN_ROUNDS = 12, OUT_ROUNDS = 16, ROUNDS = IN_ROUNDS + OUT_ROUNDS;
constexpr u32 OUT0 = IN_ROUNDS * PER_LINK;
constexpr u32 NMSG = ROUNDS * PER_LINK;
constexpr u32 SLOT = 16384 + 512;
constexpr u32 GSLOT = 2 * SLOT;
constexpr u32 NEARF = 3 * PER_LINK, SFLAG = SLOT - 128;
#define OBATCH_ 144
#define INBATCH_ 1
constexpr u32 OBATCH = OBATCH_;
constexpr bool INBATCH = INBATCH_;
static_assert(OBATCH % 8 == 0 && NEARF % 8 == 0, "whole groups");
constexpr u32 IMG = 16384;
constexpr int NSLOT = 8;
#define WIN_ 4
#define WIN2_ 5
constexpr int WIN2 = WIN2_;
static_assert(2 * WIN_ <= 9, "a message has at least 9 pieces");
constexpr int WIN = WIN_;
constexpr unsigned long long DEADLINE = 100000000ull;

constexpr size_t OFF_CTR = 0;
constexpr size_t OFF_TMO = 32768;
constexpr size_t OFF_CNT = 36864;
constexpr size_t OFF_TKT = 45056;
constexpr size_t OFF_GRP = 49152;
constexpr size_t OFF_FLAG = 65536;
constexpr size_t OFF_DBG = 768 * 1024;
constexpr size_t OFF_PKIN = 1u << 20;
constexpr size_t OFF_PKOUT = 18u << 20;
constexpr size_t OFF_UX = 28u << 20;
constexpr size_t OFF_UW = 44u << 20;
constexpr size_t WS_NEED = 52u << 20;
static_assert(OFF_FLAG + (size_t)NR * NMSG * 128 <= OFF_DBG, "flags");
static_assert(OFF_PKIN + 512 * (size_t)GSLOT <= OFF_PKOUT, "pkin");
static_assert(OFF_PKOUT + 256 * (size_t)GSLOT <= OFF_UX, "pkout");

constexpr u32 L_RING = 0;
constexpr u32 L_CTL = NSLOT * SLOT;
constexpr u32 L_FULL = L_CTL;
constexpr u32 L_DESC = L_CTL + 4;
constexpr u32 L_TRDY = L_CTL + 240;
constexpr u32 L_FREE = L_CTL + 16;
constexpr u32 L_DONE = L_CTL + 16 + 4 * NSLOT;
constexpr u32 L_LEN = L_CTL + 16 + 8 * NSLOT;
constexpr u32 L_MSG = L_CTL + 16 + 12 * NSLOT;
constexpr u32 L_DTBL = L_CTL + 256;
constexpr u32 L_UNORD = L_DTBL + 768;
constexpr u32 L_STMP = L_UNORD + 256;
constexpr int NST = 6;
constexpr u32 STG = 8192 + IMG;
constexpr u32 L_VAR = 5 * IMG;
constexpr u32 L_ETIN = L_VAR + 8192;
constexpr u32 L_ETOUT = NST * STG;
constexpr u32 L_TOT = L_ETOUT + 2048;
constexpr u32 LDS_BYTES = L_TOT + 64;
static_assert(LDS_BYTES >= L_STMP + 5120 && LDS_BYTES <= 163840 && L_ETIN + 4096 <= L_ETOUT, "lds");

constexpr int pack_passes(int b) {
    return (b >> 4) <= 1 && (b & 7) <= 1 ? 2 : ((b & 7) == 7 ? 5 : 4);
}
constexpr int pack_msg(int b, int z) {
    const int early = b < 32 ? 2 * (b >> 3) + ((b & 7) < 2 ? (b & 7) : 2) : 8;
    return z < 2 ? z * 128 + b : (z < 4 ? 256 + (z - 2) * 120 + b - early : 496 + (b >> 3));
}
__device__ __forceinline__ void msg_of(int rho, int& isw, int& grp, int& lp, int& kb, int& seq) {
    if (rho < 384) {
        const int t = rho / 48, r = rho % 48;
        isw = r < 32; grp = isw ? r >> 3 : (r - 32) >> 3; kb = r & 7; lp = t; seq = t * 16 + (isw ? 0 : 8) + kb;
    } else {
        const int t = 8 + ((rho - 384) >> 5), r = rho & 31, pos = (r >> 3) & 1;
        isw = 0; grp = r >> 4; kb = r & 7; lp = 8 + 2 * (t - 8) + pos; seq = t * 16 + pos * 8 + kb;
    }
}
#define WFRESH_ 0
struct Sched { alignas(4) unsigned char unord[256]; unsigned char ord[2][128]; bool ok; };
constexpr Sched make_sched() {
    Sched s{};
    int r = 0;
    s.ok = true;
    for (int z = 0; z < 5; ++z)
        for (int b = 0; b < 128; ++b)
            if (pack_passes(b) > z && pack_msg(b, z) != r++) s.ok = false;
    if (r != 512) s.ok = false;
    int key[256] = {};
    for (int b = 0; b < 128; ++b) {
        const int i = b >> 4, j = b & 7, sh = i > j ? i : j;
        const int t1 = 30 * sh + 170 + (j > i ? WFRESH_ : 0), arr2 = 30 * (8 + (i >> 1));
        key[b] = t1 * 256 + b;
        key[128 + b] = ((t1 > arr2 ? t1 : arr2) + 170) * 256 + 128 + b;
    }
    for (int a = 0; a < 256; ++a) {
        int pos = 0;
        for (int c = 0; c < 256; ++c) pos += key[c] < key[a];
        s.ord[a >> 7][a & 127] = (unsigned char)pos;
        s.unord[pos] = (unsigned char)a;
    }
    return s;
}
__device__ constexpr Sched SCHED = make_sched();
static_assert(make_sched().ok, "every input message is packed once");

constexpr u32 F_RAW = 0x80000000u;
constexpr u32 F_BAD = 0x40000000u;

__device__ __forceinline__ u32 var_bytes(u32 lo) {
    const u32 v = lo & 0xffffu;
    return (lo & F_RAW) || v > 8192u ? 8192u : (v < 16u ? 16u : v);
}
template <int N, int I = 0, class F> __device__ __forceinline__ void static_for(F f) {
    if constexpr (I < N) { f(std::integral_constant<int, I>{}); static_for<N, I + 1>(f); }
}

struct Params {
    const char* x;
    const char* w;
    char* myws;
    char* out;
    int rank;
    int pad;
    char* ws[NR];
};

#define STAMP(id) do {} while (0)
#define TS(i) do {} while (0)
#define LSTAMP(base, cnt, wgrow, id) do {} while (0)
#define LFLUSH(base, cnt) do {} while (0)

__device__ __forceinline__ void st16_wt(char* p, u32x4 v) {
    asm volatile("global_store_dwordx4 %0, %1, off sc0 sc1\n\ts_nop 1" :: "v"(p), "v"(v) : "memory");
}
#define OUTB 5
#define OUTBITS " sc0 sc1 nt"
__device__ __forceinline__ void st16_out(char* p, u32x4 v) {
    asm volatile("global_store_dwordx4 %0, %1, off" OUTBITS "\n\ts_nop 1" :: "v"(p), "v"(v) : "memory");
}
__device__ __forceinline__ void st4_wt(char* p, u32 v) {
    asm volatile("global_store_dword %0, %1, off sc0 sc1" :: "v"(p), "v"(v) : "memory");
}
__device__ __forceinline__ void vm_wait0() { asm volatile("s_waitcnt vmcnt(0)" ::: "memory"); }
__device__ __forceinline__ void lgkm_wait0() { asm volatile("s_waitcnt lgkmcnt(0)" ::: "memory"); }
__device__ __forceinline__ void bar() { asm volatile("s_waitcnt lgkmcnt(0)\n\ts_barrier" ::: "memory"); }
__device__ __forceinline__ void lds_st(u32 addr, u32 v) { asm volatile("ds_write_b32 %0, %1" :: "v"(addr), "v"(v) : "memory"); }
__device__ __forceinline__ u32 lds_ld(u32 addr) {
    u32 v; asm volatile("ds_read_b32 %0, %1\n\ts_waitcnt lgkmcnt(0)" : "=v"(v) : "v"(addr) : "memory"); return v;
}
__device__ __forceinline__ u32x4 lds_ld16(u32 addr) { return *(const LAS u32x4*)addr; }
template <int N> __device__ __forceinline__ u32 landed_first(u32 v) {
    u32 r; asm volatile("s_waitcnt vmcnt(%2)\n\tv_readfirstlane_b32 %0, %1\n\ts_nop 3" : "=s"(r) : "v"(v), "n"(N) : "memory"); return r;
}
__device__ __forceinline__ void dma1(const char* src, u32 lds_dst) {
    u32 keep;
    asm volatile("s_mov_b32 %0, m0\n\ts_mov_b32 m0, %2\n\ts_nop 0\n\tglobal_load_lds_dwordx4 %1, off\n\ts_mov_b32 m0, %0"
                 : "=&s"(keep) : "v"(src), "s"(lds_dst) : "memory");
}
__device__ __forceinline__ void dma1_sys(const char* src, u32 lds_dst) {
    u32 keep;
    asm volatile("s_mov_b32 %0, m0\n\ts_mov_b32 m0, %2\n\ts_nop 0\n\tglobal_load_lds_dwordx4 %1, off sc0 sc1\n\ts_mov_b32 m0, %0"
                 : "=&s"(keep) : "v"(src), "s"(lds_dst) : "memory");
}
__device__ __forceinline__ void dma1s(const char* sbase, u32 voff, u32 lds_dst) {
    asm volatile("s_mov_b32 m0, %2\n\ts_nop 0\n\tglobal_load_lds_dwordx4 %0, %1" :: "v"(voff), "s"(sbase), "s"(lds_dst) : "memory");
}
__device__ __forceinline__ void dma_a(const char* sbase, u32 voff, u32 lds_dst) {
    asm volatile("s_mov_b32 m0, %2\n\ts_nop 0\n\tglobal_load_lds_dwordx4 %0, %1 sc0 sc1" :: "v"(voff), "s"(sbase), "s"(lds_dst) : "memory");
}
__device__ __forceinline__ void dma_b(const char* sbase, u32 voff) {
    asm volatile("global_load_lds_dwordx4 %0, %1 offset:1024 sc0 sc1" :: "v"(voff), "s"(sbase) : "memory");
}
__device__ __forceinline__ void announce(char* myws, char* wsl, int rank, u32 seq, u64 val, u32 destmask, int grp, int lane) {
    const size_t line = OFF_FLAG + ((size_t)rank * NMSG + (seq & ~7u)) * 128;
    if (grp < 0) {
        if (lane < NR && ((destmask >> lane) & 1u))
            __hip_atomic_store((GAS u64*)(wsl + line + (seq & 7u) * 8), val, __ATOMIC_RELAXED, __HIP_MEMORY_SCOPE_SYSTEM);
        return;
    }
    char* g = myws + OFF_GRP + (size_t)grp * 128;
    u32 old = 0;
    if (lane == 0) {
        __hip_atomic_store((GAS u64*)(g + (seq & 7u) * 8), val, __ATOMIC_RELAXED, __HIP_MEMORY_SCOPE_AGENT);
        asm volatile("s_waitcnt vmcnt(0)" ::: "memory");
        old = __hip_atomic_fetch_add((GAS u32*)(g + 64), 1u, __ATOMIC_RELAXED, __HIP_MEMORY_SCOPE_AGENT);
    }
    if ((__builtin_amdgcn_readfirstlane(old) & 7u) == 7u) {
        const u64 e = __hip_atomic_load((GAS u64*)(g + (lane & 7) * 8), __ATOMIC_RELAXED, __HIP_MEMORY_SCOPE_AGENT);
        const u64 wl = (u64)wsl;
        char* wp = (char*)((u64)(u32)__builtin_amdgcn_ds_bpermute((lane >> 3) * 4, (int)(u32)wl)
                           | (u64)(u32)__builtin_amdgcn_ds_bpermute((lane >> 3) * 4, (int)(u32)(wl >> 32)) << 32);
        if ((destmask >> (lane >> 3)) & 1u)
            __hip_atomic_store((GAS u64*)(wp + line + (lane & 7) * 8), e, __ATOMIC_RELAXED, __HIP_MEMORY_SCOPE_SYSTEM);
    }
}
__device__ __forceinline__ void wait_counters(GAS u32* c, u32 want, GAS u32* tmo, u32 call) {
    const u64 t0 = __builtin_amdgcn_s_memrealtime();
    for (u32 it = 1; !__all(__hip_atomic_load(c, __ATOMIC_RELAXED, __HIP_MEMORY_SCOPE_SYSTEM) >= want); ++it) {
        if ((it & 15u) == 0u
            && (__builtin_amdgcn_readfirstlane(__hip_atomic_load(tmo, __ATOMIC_RELAXED, __HIP_MEMORY_SCOPE_AGENT)) == call
                || __builtin_amdgcn_s_memrealtime() - t0 > DEADLINE)) {
            __hip_atomic_store(tmo, call, __ATOMIC_RELAXED, __HIP_MEMORY_SCOPE_AGENT);
            break;
        }
        __builtin_amdgcn_s_sleep(2);
    }
    asm volatile("s_waitcnt vmcnt(0)" ::: "memory");
}

__device__ __forceinline__ float gelu(float y) {
    const float k0 = -2.0f * 0.7978845608028654f * 1.4426950408889634f;
    const float u = y * y;
    const float z = y * __builtin_fmaf(u, k0 * 0.044715f, k0);
    return y * __builtin_amdgcn_rcpf(1.0f + __builtin_amdgcn_exp2f(z));
}
__device__ __forceinline__ u32 pk_bf16(float a, float b) {
    f32x2 f = {a, b};
    return __builtin_bit_cast(u32, __builtin_convertvector(f, bf16x2));
}

constexpr u64 IN_REL0 = 0x1234567'8d9abcull, IN_REL1 = 0x123d456'789cabull;
constexpr u32 OUT_E0 = 114, OUT_NSYM = 184;
__device__ constexpr unsigned char OUT_RANK[192] = {
    210, 209, 208, 207, 104, 206, 96, 105, 205, 97, 204, 98, 203, 106, 99, 202, 100, 201, 88, 101, 107, 200, 89,
    90, 102, 91, 103, 92, 108, 93, 94, 95, 109, 112, 80, 81, 82, 83, 84, 110, 85, 86, 87, 199, 198, 111,
    197, 196, 195, 194, 193, 192, 113, 72, 73, 74, 75, 76, 77, 78, 79, 114, 65, 191, 64, 66, 190, 67, 189,
    68, 188, 69, 187, 70, 71, 186, 185, 184, 115, 57, 58, 56, 59, 60, 61, 62, 63, 183, 182, 181, 180, 179,
    178, 177, 176, 116, 49, 50, 51, 52, 53, 48, 54, 55, 175, 174, 173, 172, 171, 170, 169, 168, 41, 42, 43,
    44, 45, 46, 47, 40, 117, 167, 166, 165, 164, 163, 162, 161, 160, 33, 34, 35, 36, 37, 38, 39, 32, 118,
    159, 158, 157, 156, 155, 154, 153, 152, 25, 26, 27, 28, 29, 30, 31, 24, 151, 150, 149, 147, 148, 146, 145,
    144, 119, 17, 18, 19, 20, 21, 23, 22, 16, 137, 138, 139, 140, 141, 142, 143, 136, 11, 9, 10, 12, 13};
__device__ __forceinline__ constexpr u32x2 enc_words(u32 c1, u32 q) {
    const u32 c2 = q < 3u ? q : 3u, c3 = q < 3u ? 0u : (q < 6u ? q - 3u : 3u), c4 = q < 6u ? 0u : q - 6u;
    return u32x2{(c1 << 28) | c2 | (c3 << 8) | (c4 << 16), 2u | (q >= 3u ? 2u << 8 : 0u) | (q >= 6u ? 3u << 16 : 0u)};
}
template <bool OUT> __device__ __forceinline__ void put_enc_table(u32 l, int lane) {
    const u32x2 esc = enc_words(OUT ? 15u : 0u, 13u);
#pragma unroll
    for (int j = 0; j < (OUT ? 2 : 4); ++j)
        *(LAS u32x4*)(l + j * 1024 + lane * 16) = u32x4{esc.x, esc.y | (1u << 24), esc.x, esc.y | (1u << 24)};
    if (OUT) {
#pragma unroll
        for (u32 r = lane; r < 192u; r += 64u)
            if (r < OUT_NSYM) *(LAS u32x2*)(l + OUT_RANK[r] * 8u) = r < 3u ? u32x2{r << 28, 0u} : enc_words(3u + (r - 3u) % 13u, (r - 3u) / 13u);
    } else {
        const u32 q = lane & 15u, m = ((u32)lane >> 4) & 1u;
        if (lane < 32 && q < 13u) *(LAS u32x2*)(l + ((((u32)((m ? IN_REL1 : IN_REL0) >> (4u * q)) & 15u) << 1) | m) * 8u) = enc_words(0u, q);
    }
}
template <bool OUT> __device__ __forceinline__ void put_dec_table(u32 l, int lane) {
    if (OUT) {
#pragma unroll
        for (u32 i = lane; i < 256u; i += 64u) {
            const u32 c1 = i >> 4, q = i & 15u, r = c1 < 3u ? c1 : 13u * q + c1, x = OUT_RANK[r < OUT_NSYM ? r : 0u];
            *(LAS unsigned short*)(l + i * 2u) =
                (unsigned short)(r < OUT_NSYM ? ((x >> 7) << 15) | ((((x >> 3) & 15u) + OUT_E0) << 7) | ((x & 7u) << 4) : 1u);
        }
    } else {
        const u32 c1 = (u32)lane >> 4, q = lane & 15u;
        *(LAS unsigned short*)(l + c1 * 64u + q * 2u) = (unsigned short)(
            q < 13u ? ((c1 >> 1) << 15) | (((u32)(((c1 & 1u) ? IN_REL1 : IN_REL0) >> (4u * q)) & 15u) << 7) | ((c1 & 1u) << 6) : 1u);
    }
}

__device__ __forceinline__ u32 wave_scan(u32 v) {
    v += (u32)__builtin_amdgcn_update_dpp(0, (int)v, 0x111, 0xf, 0xf, false);
    v += (u32)__builtin_amdgcn_update_dpp(0, (int)v, 0x112, 0xf, 0xf, false);
    v += (u32)__builtin_amdgcn_update_dpp(0, (int)v, 0x114, 0xf, 0xf, false);
    v += (u32)__builtin_amdgcn_update_dpp(0, (int)v, 0x118, 0xf, 0xf, false);
    v += (u32)__builtin_amdgcn_update_dpp(0, (int)v, 0x142, 0xa, 0xf, false);
    v += (u32)__builtin_amdgcn_update_dpp(0, (int)v, 0x143, 0xc, 0xf, false);
    return v;
}
__device__ __forceinline__ u32 ubfe(u32 v, u32 off, u32 w) { return __builtin_amdgcn_ubfe(v, off, w); }
__device__ __forceinline__ void lds_or_bits(u32 base, u32 pos, u32 bits) {
    if (bits) {
        LAS u32* p = (LAS u32*)(base + (pos >> 5) * 4u);
        __hip_atomic_fetch_or(p, bits << (pos & 31u), __ATOMIC_RELAXED, __HIP_MEMORY_SCOPE_WORKGROUP);
        __hip_atomic_fetch_or(p + 1, (bits >> 1) >> (~pos & 31u), __ATOMIC_RELAXED, __HIP_MEMORY_SCOPE_WORKGROUP);
    }
}
__device__ __forceinline__ u32 lds_get_bits(u32 base, u32 pos) {
    const LAS u32* p = (const LAS u32*)(base + (pos >> 5) * 4u);
    return __builtin_amdgcn_alignbit(p[1], p[0], pos & 31u);
}
__device__ __forceinline__ u32 low_mask_even(u32 n) { return ~((0xffffffffu << (n >> 1)) << (n >> 1)); }

#define SDWA_SHL(B) \
    __device__ __forceinline__ u32 shl_byte##B(u32 sh, u32 v) {     \
        u32 r; asm("v_lshlrev_b32_sdwa %0, %1, %2 dst_sel:DWORD dst_unused:UNUSED_PAD src0_sel:BYTE_" #B " src1_sel:BYTE_" #B \
                   : "=v"(r) : "v"(sh), "v"(v)); return r; }
SDWA_SHL(0) SDWA_SHL(1) SDWA_SHL(2)

template <int B> __device__ __forceinline__ u32 and_byte(u32 v, u32 m) {
    u32 r;
    if constexpr (B == 0) asm("v_and_b32_sdwa %0, %1, %2 dst_sel:DWORD dst_unused:UNUSED_PAD src0_sel:BYTE_0 src1_sel:DWORD" : "=v"(r) : "v"(v), "v"(m));
    if constexpr (B == 1) asm("v_and_b32_sdwa %0, %1, %2 dst_sel:DWORD dst_unused:UNUSED_PAD src0_sel:BYTE_1 src1_sel:DWORD" : "=v"(r) : "v"(v), "v"(m));
    if constexpr (B == 2) asm("v_and_b32_sdwa %0, %1, %2 dst_sel:DWORD dst_unused:UNUSED_PAD src0_sel:BYTE_2 src1_sel:DWORD" : "=v"(r) : "v"(v), "v"(m));
    if constexpr (B == 3) asm("v_and_b32_sdwa %0, %1, %2 dst_sel:DWORD dst_unused:UNUSED_PAD src0_sel:BYTE_3 src1_sel:DWORD" : "=v"(r) : "v"(v), "v"(m));
    return r;
}

__device__ __forceinline__ void store_input_bytes(char* slot, const u32 (&d)[8], int wave, int lane) {
    u32 bb[4];
#pragma unroll
    for (int j = 0; j < 4; ++j) {
        const u32 lowb = __builtin_amdgcn_perm(d[2 * j + 1], d[2 * j], 0x06040200u);
        const u32 higb = __builtin_amdgcn_perm(d[2 * j + 1], d[2 * j], 0x07050301u);
        bb[j] = (lowb & 0x7f7f7f7fu) | (higb & 0x80808080u);
    }
    st16_wt(slot + wave * 1024 + lane * 16, u32x4{bb[0], bb[1], bb[2], bb[3]});
}
constexpr u32 RAW_OFF = SLOT;
__device__ __forceinline__ u32 input_e0(u32x2 sample) {
    const u32 s2 = ((sample.x >> 7) & 0x00ff00ffu) + ((sample.y >> 7) & 0x00ff00ffu);
    const u32 tot = __builtin_amdgcn_readlane(wave_scan((s2 & 0xffffu) + (s2 >> 16)), 63);
    const u32 anchor = (tot + 504u) >> 8;
    return anchor < 12u ? 0u : (anchor > 252u ? 240u : anchor - 12u);
}
template <bool OUT>
__device__ __forceinline__ void encode_block(char* slot, const u32 (&d)[8], u32 e0, u32 ltbl, u32 lpriv, u32 ltot, int wave, int lane) {
    *(LAS u32x4*)(lpriv + lane * 16) = u32x4{0u, 0u, 0u, 0u};
    u32 en[16], in[16];
    typedef unsigned short u16x2 __attribute__((ext_vector_type(2)));
#pragma unroll
    for (int j = 0; j < 8; ++j) {
        u32 i0, i1;
        if (OUT) {
            const u16x2 sub = {OUT_E0 << 6, OUT_E0 << 6}, cap = {15 << 6, 15 << 6};
            const u16x2 t = __builtin_elementwise_min(__builtin_bit_cast(u16x2, (d[j] >> 1) & 0x3ff83ff8u) - sub, cap);
            const u32 x = __builtin_bit_cast(u32, t) | ((d[j] >> 5) & 0x04000400u);
            i0 = x & 0xffffu; i1 = x >> 16;
        } else {
            const u16x2 sub = {(unsigned short)(e0 << 7), (unsigned short)(e0 << 7)};
            const u32 x = __builtin_bit_cast(u32, __builtin_bit_cast(u16x2, d[j]) - sub);
            i0 = (x >> 3) & 0xff8u; i1 = (x >> 19) & 0xff8u;
        }
        const u32x2 a = *(const LAS u32x2*)(ltbl + i0), b = *(const LAS u32x2*)(ltbl + i1);
        en[2 * j] = a.x; in[2 * j] = a.y; en[2 * j + 1] = b.x; in[2 * j + 1] = b.y;
    }
    __builtin_amdgcn_sched_barrier(0);
    if (OUT) {
        u32 bb[4];
#pragma unroll
        for (int j = 0; j < 4; ++j) {
            const u32 lowb = __builtin_amdgcn_perm(d[2 * j + 1], d[2 * j], 0x06040200u);
            const u32 c01 = __builtin_amdgcn_perm(en[4 * j + 1], en[4 * j], 0x0c0c0703u);
            const u32 c23 = __builtin_amdgcn_perm(en[4 * j + 3], en[4 * j + 2], 0x07030c0cu);
            bb[j] = (lowb & 0x0f0f0f0fu) | c01 | c23;
        }
        st16_wt(slot + wave * 1024 + lane * 16, u32x4{bb[0], bb[1], bb[2], bb[3]});
    }
    u32 f2 = 0, f3 = 0, f4[2] = {0, 0}, pos = 0, n4a = 0;
#pragma unroll
    for (int i = 0; i < 16; i += 2) {
        if (i == 8) { n4a = ubfe(pos, 16, 8); pos &= 0xff00ffffu; }
        const u32 pos1 = pos + in[i];
        f2 |= shl_byte0(pos, en[i]) | shl_byte0(pos1, en[i + 1]);
        f3 |= shl_byte1(pos, en[i]) | shl_byte1(pos1, en[i + 1]);
        f4[i >> 3] |= shl_byte2(pos, en[i]) | shl_byte2(pos1, en[i + 1]);
        pos = pos1 + in[i + 1];
    }
    const u32 n4 = n4a + ubfe(pos, 16, 8), ne = pos >> 24;
    const u32 ca = (pos & 0xffu) | ((pos << 8) & 0xff0000u), cb = n4 | (ne << 16);
    const u32 sa = wave_scan(ca), sb = wave_scan(cb);
    const u32 ta = __builtin_amdgcn_readlane(sa, 63), tb = __builtin_amdgcn_readlane(sb, 63);
    const u32 N2 = ta & 0xffffu, N3 = ta >> 16, NE = tb >> 16;
    const u32 escpos = (N2 + N3 + (tb & 0xffffu) + 15u) & ~15u;
    const u32 dw = (escpos + 16u * NE + 31u) >> 5;
    if (lane == 0) *(LAS u32*)(ltot + wave * 4) = dw <= 252u ? dw : 0x1000u;
    if (dw <= 252u) {
        const u32 xa = sa - ca, xb = sb - cb;
        if (OUT) lds_or_bits(lpriv, xa & 0xffffu, f2);
        else *(LAS u32*)(lpriv + lane * 4) = f2;
        lds_or_bits(lpriv, N2 + (xa >> 16), f3);
        const u32 p4 = N2 + N3 + (xb & 0xffffu);
        if (!__builtin_amdgcn_ballot_w64(n4 > 32u)) {
            lds_or_bits(lpriv, p4, f4[0] | (f4[1] << n4a));
        } else {
            lds_or_bits(lpriv, p4, f4[0]);
            lds_or_bits(lpriv, p4 + n4a, f4[1]);
        }
        if (NE) {
            u32 a = lpriv + (escpos >> 3) + (xb >> 16) * 2u;
#pragma unroll
            for (int j = 0; j < 4; ++j)
                if ((in[4 * j] | in[4 * j + 1] | in[4 * j + 2] | in[4 * j + 3]) >> 24) {
#pragma unroll
                    for (int i = 4 * j; i < 4 * j + 4; ++i)
                        if (in[i] >> 24) { *(LAS unsigned short*)a = (unsigned short)(d[i >> 1] >> (16 * (i & 1))); a += 2u; }
                }
        }
    }
}
__device__ __forceinline__ u32 encode_finish(char* slot, const u32 (&d)[8], u32 lpriv, u32 ltot, int wave, int lane) {
    const u32 mine = *(const LAS u32*)(ltot + (lane & 7) * 4);
    const u32 w0 = *(const LAS u32*)(lpriv + lane * 4), w1 = *(const LAS u32*)(lpriv + 256 + lane * 4);
    u32 inc = mine;
    inc += (u32)__builtin_amdgcn_update_dpp(0, (int)inc, 0x111, 0xf, 0xf, false);
    inc += (u32)__builtin_amdgcn_update_dpp(0, (int)inc, 0x112, 0xf, 0xf, false);
    inc += (u32)__builtin_amdgcn_update_dpp(0, (int)inc, 0x114, 0xf, 0xf, false);
    const u32 total = __builtin_amdgcn_readlane(inc, 7);
    if (total >= 0x1000u) {
        char* p = slot + RAW_OFF + wave * 1024 + lane * 16;
        st16_wt(p + 8192, u32x4{d[0], d[1], d[2], d[3]}); st16_wt(p, u32x4{d[4], d[5], d[6], d[7]});
        return F_RAW;
    }
    const u32 dw = __builtin_amdgcn_readlane(mine, wave), start = __builtin_amdgcn_readlane(inc, wave) - dw;
    char* p = slot + 8192 + 16 + (start + lane) * 4u;
    if ((u32)lane < dw) st4_wt(p, w0);
    if ((u32)lane + 64u < dw) st4_wt(p + 256, w1);
    if (dw > 128u) {
        if ((u32)lane + 128u < dw) st4_wt(p + 512, *(const LAS u32*)(lpriv + 512 + lane * 4));
        if ((u32)lane + 192u < dw) st4_wt(p + 768, *(const LAS u32*)(lpriv + 768 + lane * 4));
    }
    if (wave == 0 && lane < 8 && !(lane & 1)) st4_wt(slot + 8192 + lane * 2, (inc - mine) | (inc << 16));
    return (16u + total * 4u + 15u) & ~15u;
}

template <bool OUT>
__device__ __forceinline__ void decode_block(u32 sl, u32 e0, u32 ltbl, int wave, int lane, u32x4& v0, u32x4& v1) {
    const u32x4 B = lds_ld16(sl + wave * 1024 + lane * 16);
    const u32 hdr = __builtin_amdgcn_readfirstlane(*(const LAS unsigned short*)(sl + 8192 + wave * 2));
    const u32 base = sl + 8192 + 16 + (hdr > 2040u ? 2040u : hdr) * 4u;
    const u32 eadd = e0 * 0x00800080u;
    const u32 bb[4] = {B.x, B.y, B.z, B.w};
    u32 F2, N2, W2[4];
    if (OUT) {
        u32 c = 0;
#pragma unroll
        for (int j = 0; j < 4; ++j) {
            W2[j] = ((((bb[j] >> 4) & 0x0f0f0f0fu) + 0x0d0d0d0du) >> 3) & 0x02020202u;
            c += __builtin_popcount(W2[j]);
        }
        const u32 n2 = 2u * c, s2 = wave_scan(n2);
        N2 = __builtin_amdgcn_readlane(s2, 63);
        F2 = lds_get_bits(base, s2 - n2) & low_mask_even(n2);
    } else {
        F2 = *(const LAS u32*)(base + lane * 4);
        N2 = 2048u;
    }
    const u32 W3 = F2 & (F2 << 1) & 0xaaaaaaaau;
    const u32 n3 = 2u * __builtin_popcount(W3), s3 = wave_scan(n3);
    const u32 N3 = __builtin_amdgcn_readlane(s3, 63);
    const u32 F3 = lds_get_bits(base, N2 + s3 - n3) & low_mask_even(n3);
    const u32 K4 = F3 & (F3 >> 1) & 0x55555555u, W4 = K4 * 3u;
    const u32 n4 = 3u * __builtin_popcount(K4), s4 = wave_scan(n4);
    const u32 N4 = __builtin_amdgcn_readlane(s4, 63);
    const u32 P4 = N2 + N3 + s4 - n4;
    const LAS u32* q4 = (const LAS u32*)(base + (P4 >> 5) * 4u);
    const u32 q40 = q4[0], q41 = q4[1], q42 = q4[2];
    u32 F4 = __builtin_amdgcn_alignbit(q41, q40, P4 & 31u);
    const u32 F4h = __builtin_amdgcn_alignbit(q42, q41, P4 & 31u);
    u32 t[16], p2 = 0, p3 = 0, p4 = 0;
    const u32 cm = OUT ? 0xf0u : 0xc0u;
    static_for<16>([&](auto ii) INL {
        constexpr int i = decltype(ii)::value;
        if (i == 8) { F4 = __builtin_amdgcn_alignbit(F4h, F4, p4); p4 = 0; }
        const u32 c1 = and_byte<i & 3>(bb[i >> 2], cm);
        u32 c2, w3;
        if (OUT) { const u32 w2 = and_byte<i & 3>(W2[i >> 2], cm - 1u); c2 = ubfe(F2, p2, w2); w3 = ubfe(W3, p2, w2); p2 += w2; }
        else { c2 = ubfe(F2, 2 * i, 2); w3 = ubfe(W3, 2 * i, 2); }
        const u32 c3 = ubfe(F3, p3, w3), w4 = ubfe(W4, p3, w3); p3 += w3;
        const u32 c4 = ubfe(F4, p4, w4); p4 += w4;
        t[i] = OUT ? (c1 + c2 + c3 + c4) << 1 : c1 + ((c2 + c3 + c4) << 1);
    });
    u32 o[8];
#pragma unroll
    for (int j = 0; j < 8; ++j) {
        typedef unsigned short u16x2 __attribute__((ext_vector_type(2)));
        const u16x2 pr = {*(const LAS unsigned short*)(ltbl + t[2 * j]), *(const LAS unsigned short*)(ltbl + t[2 * j + 1])};
        o[j] = __builtin_bit_cast(u32, pr) + (OUT ? 0u : eadd);
    }
    const u32 any = (o[0] | o[1] | o[2]) | (o[3] | o[4] | o[5]) | (o[6] | o[7]);
    if (__builtin_amdgcn_ballot_w64((any & 0x00010001u) != 0u)) {
        u32 c = 0;
#pragma unroll
        for (int j = 0; j < 8; ++j) c += o[j] & 0x00010001u;
        const u32 ne = (c & 0xffffu) + (c >> 16);
        u32 a = base + (((N2 + N3 + N4 + 15u) & ~15u) >> 3) + (wave_scan(ne) - ne) * 2u;
        const u32 amax = sl + SLOT - 4u;
#pragma unroll
        for (int j = 0; j < 8; ++j) {
            if (o[j] & 0x00010001u) {
                a = a < amax ? a : amax;
                if (o[j] & 1u) { o[j] = (o[j] & 0xffff0000u) | *(const LAS unsigned short*)a; a += 2u; }
                if (o[j] & 0x10000u) { o[j] = (o[j] & 0xffffu) | ((u32)*(const LAS unsigned short*)a << 16); a += 2u; }
            }
        }
    }
#pragma unroll
    for (int j = 0; j < 8; ++j)
        o[j] |= __builtin_amdgcn_perm(0u, bb[j >> 1] & (OUT ? 0x0f0f0f0fu : 0x3f3f3f3fu), (j & 1) ? 0x0c030c02u : 0x0c010c00u);
    v0 = u32x4{o[0], o[1], o[2], o[3]};
    v1 = u32x4{o[4], o[5], o[6], o[7]};
}

__global__ void __launch_bounds__(THREADS, 1) fused_gemm_allgather(Params P) {
    const int lane = threadIdx.x & 63;
    const int wave = __builtin_amdgcn_readfirstlane(threadIdx.x >> 6);
    int b = blockIdx.x;
    if (MAP_ == 1) {
        const int x = b & 7, y = b >> 3;
        if (x >= 4 && y >= 28) return;
        b = x < 4 ? (y >> 1) * 8 + 2 * x + (y & 1) : NGEMM + (x - 4) * 28 + y;
    }
    if (MAP_ == 2) {
        const int x = b & 7, y = b >> 3;
        if ((x & 1) && y >= 28) return;
        b = !(x & 1) ? (y >> 1) * 8 + x + (y & 1) : NGEMM + (x >> 1) * 28 + y;
    }
    const int rank = P.rank;
    char* myws = P.myws;
    const char* px = P.x;
    const char* pw = P.w;
    asm volatile("" : "+s"(myws), "+s"(px), "+s"(pw));
    constexpr u32 lds0 = 0;
    GAS u32* const ctr = (GAS u32*)(myws + OFF_CTR + (size_t)b * 128);
    GAS u32* const tmo = (GAS u32*)(myws + OFF_TMO);

    if (__builtin_expect(b < NGEMM, 1)) {
        const int tj = b & 7, ti0 = b >> 4, half = (b >> 3) & 1;
        const int npass = pack_passes(b);
        constexpr int KBAR = 65, EBAR = 3;
        if (wave == 8) {
            u32 cv;
            char* wsl;
            asm volatile("global_load_dword %0, %2, off sc1\n\tglobal_load_dwordx2 %1, %3, %4 offset:%5"
                         : "=&v"(cv), "=&v"(wsl)
                         : "v"(ctr), "v"((lane & 7) * 8), "s"(__builtin_amdgcn_kernarg_segment_ptr()), "n"(offsetof(Params, ws)) : "memory");
            bar();
            put_enc_table<false>(L_ETIN, lane);
            const u32 call = landed_first<0>(cv) + 1u;
            asm volatile("" : "+v"(wsl));
            bar();
            int isw, grp, lp, kb, seq;
            msg_of(b, isw, grp, lp, kb, seq);
            if (seq < 2 * PER_LINK && lane < NR && lane != rank && (isw ? ((lane & 3) == grp) : ((lane >> 2) == grp)))
                __hip_atomic_store((GAS u32*)(wsl + OFF_FLAG + ((size_t)rank * NMSG + seq) * 128 + 68), call, __ATOMIC_RELAXED, __HIP_MEMORY_SCOPE_SYSTEM);
            if (lane == 0) __hip_atomic_store(ctr, call, __ATOMIC_RELAXED, __HIP_MEMORY_SCOPE_AGENT);
            for (int z = 0; z < npass; ++z) {
                if (z + 1 < npass) {
                    msg_of(pack_msg(b, z + 1), isw, grp, lp, kb, seq);
                    const char* sb = isw ? pw + ((size_t)kb * 64 * 4096 + (grp * 8 + lp) * 128) * 2 : px + ((size_t)(grp * 16 + lp) * 128 * 512 + kb * 64) * 2;
                    const u32 vo = isw ? (lane >> 4) * 8192 + (lane & 15) * 16 : (lane & 15) * 1024 + (lane >> 4) * 16;
                    const u32 sw = isw ? 65536u : 16384u, sh = isw ? 32768u : 64u;
#pragma unroll
                    for (int j = 0; j < 16; ++j) dma1s(sb + (j >> 1) * sw + (j & 1) * sh, vo, lds0 + (z + 1) * IMG + j * 1024);
                }
                bar();
                vm_wait0();
                bar();
                msg_of(pack_msg(b, z), isw, grp, lp, kb, seq);
                const u32 lo = lds_ld(L_TOT + 32);
                if (seq < (int)NEARF) {
                    if (lane == 0)
                        __hip_atomic_store((GAS u64*)(myws + OFF_PKIN + (size_t)((isw ? 256 + (grp * 8 + lp) * 8 : (grp * 16 + lp) * 8) + kb) * GSLOT + SFLAG),
                                           ((u64)call << 32) | lo, __ATOMIC_RELAXED, __HIP_MEMORY_SCOPE_SYSTEM);
                } else {
                    announce(myws, wsl, rank, seq, ((u64)call << 32) | lo, (isw ? 0x11u << grp : 0xfu << (4 * grp)) & ~(1u << rank),
                             INBATCH ? (isw ? 32 + grp * 8 + lp : grp * 16 + lp) : -1, lane);
                }
                if (lane == 0 && (isw ? (grp == (rank & 3)) : (grp == (rank >> 2))))
                    __hip_atomic_fetch_add((GAS u32*)(myws + OFF_CNT + (size_t)((isw ? 16 : 0) + lp) * 256 + 128), 1u,
                                           __ATOMIC_RELAXED, __HIP_MEMORY_SCOPE_AGENT);
                STAMP(1 + z);
            }
            bool own_ok = false, rem_ok = false;
            for (int tile = 0; tile < 2; ++tile) {
                GAS u32* c = (GAS u32*)(myws + OFF_CNT + (size_t)((lane & 1) ? 16 + tj : ti0 + 8 * tile) * 256);
                if (!own_ok) wait_counters(c + 32, call * 8u, tmo, call);
                for (int i = 0; i < 2 + 8 - NST; ++i) bar();
                u32 dummy = 0;
                auto prefetch = [&]() INL {
                    const char* pf = myws + OFF_UW + ((size_t)tj * 64 + (u32)((rank * 8 + 8 + (b >> 3) * 4) & 63)) * IMG + lane * 128;
                    if ((b >> 3) < 14) {
#pragma unroll
                        for (int j = 0; j < 8; ++j)
                            asm volatile("global_load_dword %0, %1, off sc1" : "+v"(dummy) : "v"(pf + j * 8192) : "memory");
                    }
                };
                if (tile == 0) put_enc_table<true>(L_ETOUT, lane);
                const bool early = tile == 0 && ti0 >= tj;
                if (early) {
                    wait_counters((GAS u32*)(myws + OFF_CNT + (size_t)(16 + tj) * 256), call * 56u, tmo, call);
                    prefetch();
                }
                if (!rem_ok) wait_counters(c, call * 56u, tmo, call);
                STAMP(6 + 6 * tile);
                bar();
                if (tile == 0 && !early) prefetch();
                if (tile == 0) {
                    const u32 ord = SCHED.ord[0][b];
                    for (int i = 0; i < KBAR - (2 + 8 - NST) - 4; ++i) bar();
                    u32 pk;
                    asm volatile("global_load_dword %0, %1, off sc0 sc1" : "=v"(pk)
                                 : "v"(myws + OFF_CNT + (size_t)(ti0 + 8) * 256 + (lane & 1) * 128) : "memory");
                    for (int i = 0; i < 4 + EBAR - 1; ++i) bar();
                    u32 k0, k1;
                    asm volatile("s_waitcnt vmcnt(0)\n\tv_readlane_b32 %0, %2, 0\n\tv_readlane_b32 %1, %2, 1\n\ts_nop 3"
                                 : "=&s"(k0), "=&s"(k1) : "v"(pk) : "memory");
                    rem_ok = k0 >= call * 56u; own_ok = k1 >= call * 8u;
                    bar();
                    const u32 lo = lds_ld(L_TOT + 32);
                    announce(myws, wsl, rank, OUT0 + ord, ((u64)call << 32) | lo, 0xffu & ~(1u << rank), ord >= OBATCH ? 64 + (int)(ord >> 3) : -1, lane);
                }
                asm volatile("" :: "v"(dummy));
            }
            return;
        }
        if (wave == 0) STAMP(0);
        u32 cv;
        char* wsl;
        asm volatile("global_load_dword %0, %2, off sc1\n\tglobal_load_dwordx2 %1, %3, %4 offset:%5"
                     : "=&v"(cv), "=&v"(wsl)
                     : "v"(ctr), "v"((lane & 7) * 8), "s"(__builtin_amdgcn_kernarg_segment_ptr()), "n"(offsetof(Params, ws)) : "memory");
        auto fetch = [&](int z, int rho) INL {
            int isw, grp, lp, kb, seq;
            msg_of(rho, isw, grp, lp, kb, seq);
            const u32 dst = lds0 + z * IMG + wave * 2048;
            if (!isw) {
                const char* s = px + ((size_t)((grp * 16 + lp) * 128 + wave * 16 + (lane & 15)) * 512 + kb * 64 + (lane >> 4) * 8) * 2;
                dma1(s, dst); dma1(s + 64, dst + 1024);
            } else {
                const char* s = pw + ((size_t)(kb * 64 + wave * 8 + (lane >> 4)) * 4096 + (grp * 8 + lp) * 128 + (lane & 15) * 8) * 2;
                dma1(s, dst); dma1(s + 4 * 8192, dst + 1024);
            }
        };
        [[maybe_unused]] u64 tq[3];
        fetch(0, b);
        const u32 call = landed_first<0>(cv) + 1u;
        int isw, grp, lp, kb, seq;
        char* slot;
        u32 d[8];
        u32x2 sample;
        auto take = [&](int z) INL {
            msg_of(pack_msg(b, z), isw, grp, lp, kb, seq);
            const u32 src = lds0 + z * IMG;
            if (!isw) {
                const u32x4 v0 = lds_ld16(src + wave * 2048 + lane * 16), v1 = lds_ld16(src + wave * 2048 + 1024 + lane * 16);
                d[0] = v0.x; d[1] = v0.y; d[2] = v0.z; d[3] = v0.w; d[4] = v1.x; d[5] = v1.y; d[6] = v1.z; d[7] = v1.w;
            } else {
                const u32 a = src + ((lane >> 4) * 8 + ((lane & 15) >> 2)) * 256 + (32 * (wave >> 1) + 8 * (lane & 3) + 4 * (wave & 1)) * 2;
                const u32x2 a0 = __builtin_bit_cast(u32x2, __builtin_amdgcn_ds_read_tr16_b64_v4i16((LAS s16x4*)(a)));
                const u32x2 a1 = __builtin_bit_cast(u32x2, __builtin_amdgcn_ds_read_tr16_b64_v4i16((LAS s16x4*)(a + 4 * 256)));
                const u32x2 a2 = __builtin_bit_cast(u32x2, __builtin_amdgcn_ds_read_tr16_b64_v4i16((LAS s16x4*)(a + 32 * 256)));
                const u32x2 a3 = __builtin_bit_cast(u32x2, __builtin_amdgcn_ds_read_tr16_b64_v4i16((LAS s16x4*)(a + 36 * 256)));
                d[0] = a0.x; d[1] = a0.y; d[2] = a1.x; d[3] = a1.y; d[4] = a2.x; d[5] = a2.y; d[6] = a3.x; d[7] = a3.y;
            }
            sample = *(const LAS u32x2*)(src + lane * 248);
            slot = myws + OFF_PKIN + (size_t)((isw ? 256 + (grp * 8 + lp) * 8 : (grp * 16 + lp) * 8) + kb) * GSLOT;
            store_input_bytes(slot, d, wave, lane);
        };
        [[maybe_unused]] u64 ts[4], tsave[4] = {};
        bar();
        TS(0);
        take(0);
        asm volatile("s_waitcnt vmcnt(0)" : "+v"(wsl) :: "memory");
        bar();
        for (int z = 0;;) {
            TS(1);
            const u32 e0 = input_e0(sample);
            encode_block<false>(slot, d, e0, L_ETIN, L_VAR + wave * 1024, L_TOT, wave, lane);
            bar();
            TS(2);
            const u32 lo = encode_finish(slot, d, L_VAR + wave * 1024, L_TOT, wave, lane) | (e0 << 16);
            TS(3);
            const bool own = isw ? (grp == (rank & 3)) : (grp == (rank >> 2));
            if (own) {
                char* img = myws + (isw ? OFF_UW : OFF_UX) + (size_t)(lp * 64 + rank * 8 + kb) * IMG + wave * 2048 + lane * 16;
                st16_wt(img, u32x4{d[0], d[1], d[2], d[3]}); st16_wt(img + 1024, u32x4{d[4], d[5], d[6], d[7]});
            }
            if (threadIdx.x == 0) lds_st(L_TOT + 32, lo);
            vm_wait0();
            bar();
            if (++z == npass) break;
            TS(0);
            take(z);
        }
        u32 ords = __builtin_amdgcn_readfirstlane((u32)SCHED.ord[0][b] | ((u32)SCHED.ord[1][b] << 8));
        asm volatile("s_waitcnt vmcnt(0)" : "+s"(ords) :: "memory");
        const int kh = wave >> 2, wn = wave & 1, wm = (wave >> 1) & 1;
        const char* const wb = myws + OFF_UW + (size_t)tj * 64 * IMG;
        const u32 voffx = wave * 1024 + lane * 16, voffw = wave * 2048 + lane * 16;
        const u32 xa = lds0 + (wm * 4 + kh) * 1024 + lane * 16;
        const u32 wa = lds0 + 8192 + (wn * 8 + kh) * 1024 + lane * 16;
#define SB __builtin_amdgcn_sched_barrier(0)
        for (int tile = 0; tile < 2; ++tile) {
            const int ti = ti0 + 8 * tile;
            const char* const xb = myws + OFF_UX + (size_t)ti * 64 * IMG + half * 8192;
            f32x4 acc[4][2];
#pragma unroll
            for (int a = 0; a < 4; ++a) { acc[a][0] = f32x4{0.f, 0.f, 0.f, 0.f}; acc[a][1] = acc[a][0]; }
            bf16x8 wA[4], xA[2], wB[4], xB[2];
            auto mma = [&](int a, const bf16x8* wf, const bf16x8* xf) INL {
                acc[a][0] = __builtin_amdgcn_mfma_f32_16x16x32_bf16(wf[a], xf[0], acc[a][0], 0, 0, 0);
                acc[a][1] = __builtin_amdgcn_mfma_f32_16x16x32_bf16(wf[a], xf[1], acc[a][1], 0, 0, 0);
            };
            u32 buf = 0;
#define ROT_ 7
            u32 kq = tj * ROT_;
            auto step = [&](int kt, auto dma_tag, auto vm_tag, const bf16x8* wc, const bf16x8* xc, bf16x8* wx, bf16x8* xx) INL {
                constexpr bool DMA = decltype(dma_tag)::value;
                constexpr int VM = decltype(vm_tag)::value;
                const u32 kb = (u32)(rank * 8 + (kt + NST < 8 ? kt + NST : 8 + kq)) & 63u;
                if (kt + NST >= 8) kq = kq == 55u ? 0u : kq + 1u;
                const char* gx = xb + (size_t)kb * IMG; const char* gw = wb + (size_t)kb * IMG;
                const u32 d = buf;
                const u32 nb = buf + STG == NST * STG ? 0u : buf + STG;
                const u32 ra = xa + nb, rb = wa + nb;
                buf = nb;
                asm volatile("s_waitcnt lgkmcnt(0)\n\ts_waitcnt vmcnt(%0)\n\ts_barrier" :: "n"(VM) : "memory"); SB;
                if (DMA) dma_a(gx, voffx, d + wave * 1024); SB;
                mma(0, wc, xc); SB;
                xx[0] = *(const LAS bf16x8*)(ra); wx[0] = *(const LAS bf16x8*)(rb); SB;
                mma(1, wc, xc); SB;
                if (DMA) dma_a(gw, voffw, d + 8192 + wave * 2048); SB;
                xx[1] = *(const LAS bf16x8*)(ra + 2048); wx[1] = *(const LAS bf16x8*)(rb + 2048); SB;
                mma(2, wc, xc); SB;
                wx[2] = *(const LAS bf16x8*)(rb + 4096); wx[3] = *(const LAS bf16x8*)(rb + 6144); SB;
                if (DMA) dma_b(gw, voffw); SB;
                mma(3, wc, xc); SB;
            };
            using T_ = std::true_type; using F_ = std::false_type;
            bar();
#pragma unroll
            for (int st = 0; st < NST; ++st) {
                const u32 kb = (u32)(rank * 8 + st) & 63u;
                dma_a(xb + (size_t)kb * IMG, voffx, st * STG + wave * 1024);
                dma_a(wb + (size_t)kb * IMG, voffw, st * STG + 8192 + wave * 2048);
                dma_b(wb + (size_t)kb * IMG, voffw);
            }
            asm volatile("s_waitcnt vmcnt(%0)\n\ts_barrier" :: "n"((NST - 1) * 3) : "memory");
#pragma unroll
            for (int a = 0; a < 4; ++a) wA[a] = *(const LAS bf16x8*)(wa + a * 2048);
            xA[0] = *(const LAS bf16x8*)(xa); xA[1] = *(const LAS bf16x8*)(xa + 2048);
            static_assert(NST == 6, "tail below");
            using VMS = std::integral_constant<int, (NST - 2) * 3>;
            for (int kt = 0; kt < 64 - NST; kt += 2) {
                step(kt, T_{}, VMS{}, wA, xA, wB, xB);
                step(kt + 1, T_{}, VMS{}, wB, xB, wA, xA);
            }
            step(58, F_{}, std::integral_constant<int, 12>{}, wA, xA, wB, xB);
            step(59, F_{}, std::integral_constant<int, 9>{}, wB, xB, wA, xA);
            step(60, F_{}, std::integral_constant<int, 6>{}, wA, xA, wB, xB);
            step(61, F_{}, std::integral_constant<int, 3>{}, wB, xB, wA, xA);
            step(62, F_{}, std::integral_constant<int, 0>{}, wA, xA, wB, xB);
            u32 tv;
            asm volatile("global_load_dword %0, %1, off sc1" : "=v"(tv) : "v"(tmo) : "memory");
            bar();
            mma(0, wB, xB); mma(1, wB, xB); mma(2, wB, xB); mma(3, wB, xB);
            if (wave == 0) STAMP(7 + 6 * tile);
            {
                const u32 wr = lds0 + wave * 4096 + lane * 16;
#pragma unroll
                for (int a = 0; a < 4; ++a) *(LAS f32x4*)(wr + a * 1024) = kh ? acc[a][0] : acc[a][1];
            }
            bar();
            {
                const u32 rd = lds0 + (wave ^ 4) * 4096 + lane * 16;
#pragma unroll
                for (int a = 0; a < 4; ++a) {
                    const f32x4 y = (kh ? acc[a][1] : acc[a][0]) + *(const LAS f32x4*)(rd + a * 1024);
                    d[a * 2] = pk_bf16(gelu(y.x), gelu(y.y));
                    d[a * 2 + 1] = pk_bf16(gelu(y.z), gelu(y.w));
                }
            }
            if (landed_first<0>(tv) == call) {
#pragma unroll
                for (int j = 0; j < 8; ++j) d[j] = 0x7fc07fc0u;
            }
            if (wave == 0) STAMP(8 + 6 * tile);
            const u32 ord = (ords >> (8 * tile)) & 0xffu;
            char* const oslot = myws + OFF_PKOUT + (size_t)ord * GSLOT;
            encode_block<true>(oslot, d, 0u, L_ETOUT, L_VAR + wave * 1024, L_TOT, wave, lane);
            bar();
            const u32 lo = encode_finish(oslot, d, L_VAR + wave * 1024, L_TOT, wave, lane);
            if (wave == 0) STAMP(9 + 6 * tile);
            if (threadIdx.x == 0) lds_st(L_TOT + 32, lo);
            vm_wait0();
            bar();
            if (tile == 1 && wave == 0)
                announce(myws, wsl, rank, OUT0 + ord, ((u64)call << 32) | lo, 0xffu & ~(1u << rank), ord >= OBATCH ? 64 + (int)(ord >> 3) : -1, lane);
            char* o = P.out + ((size_t)((rank >> 2) * 2048 + ti * 128 + half * 64 + wm * 32 + kh * 16 + (lane & 15)) * 4096
                               + (rank & 3) * 1024 + tj * 128 + wn * 64 + (lane >> 4) * 8) * 2;
            st16_out(o, u32x4{d[0], d[1], d[2], d[3]});
            st16_out(o + 64, u32x4{d[4], d[5], d[6], d[7]});
            if (wave == 0) STAMP(10 + 6 * tile);
            if (tile == 0) vm_wait0();
        }
    } else {
        const int pu = b - NGEMM;
        const int u = pu / 7;
        const int src = (rank + 1 + pu % 7) & 7;
        const u32 call = __builtin_amdgcn_readfirstlane(__hip_atomic_load(ctr, __ATOMIC_RELAXED, __HIP_MEMORY_SCOPE_AGENT)) + 1u;
        if (wave == 0) lds_st(lds0 + L_CTL + lane * 4, 0u);
        if (wave == 1) put_dec_table<false>(L_DTBL, lane);
        if (wave == 8) STAMP(0);
        bar();
        if (threadIdx.x == 0) __hip_atomic_store(ctr, call, __ATOMIC_RELAXED, __HIP_MEMORY_SCOPE_AGENT);
        if (wave == 8) {
            vm_wait0();
            const char* const rws = P.ws[src];
            const char* const fbase = myws + OFF_FLAG + (size_t)src * NMSG * 128;
            char* const tkt = myws + OFF_TKT + (size_t)(pu % 7) * 256;
            const u32 tbase = (call - 1u) * (NMSG - 2 * PER_LINK) - 3 * PER_LINK;
            u32 m = u, n = 0, np = 0;
            u32 fl_lo = 0u, fl_hi = 0u;
            u32 pub = 0u;
            bool dead = false;
            [[maybe_unused]] u32 nst = 0;
            auto publish = [&](u32 k) INL {
                if (np > pub + k) { pub = np - k; lds_st(lds0 + L_FULL, pub); }
            };
            auto drain = [&](auto top) INL {
                constexpr int TOP = decltype(top)::value;
                static_for<TOP + 1>([&](auto i) INL {
                    constexpr int K = TOP - decltype(i)::value;
                    asm volatile("s_waitcnt vmcnt(%0)" :: "n"(K) : "memory");
                    publish(K);
                });
            };
            auto poll = [&](const char* fp) INL {
                const u64 t0 = __builtin_amdgcn_s_memrealtime();
                for (u32 it = 1;; ++it) {
                    u32x2 fl;
                    asm volatile("global_load_dwordx2 %0, %1, off sc0 sc1\n\ts_waitcnt vmcnt(0)" : "=&v"(fl) : "v"(fp) : "memory");
                    fl_lo = __builtin_amdgcn_readfirstlane(fl.x); fl_hi = __builtin_amdgcn_readfirstlane(fl.y);
                    if (fl_hi >= call) break;
                    if ((it & 15u) == 0u
                        && (__builtin_amdgcn_readfirstlane(__hip_atomic_load(tmo, __ATOMIC_RELAXED, __HIP_MEMORY_SCOPE_AGENT)) == call
                            || __builtin_amdgcn_s_memrealtime() - t0 > DEADLINE)) {
                        __hip_atomic_store(tmo, call, __ATOMIC_RELAXED, __HIP_MEMORY_SCOPE_AGENT);
                        dead = true;
                        break;
                    }
                    __builtin_amdgcn_s_sleep(1);
                }
            };
            auto remote_slot = [&](u32 mm) INL {
                const u32 t = mm / PER_LINK, v = mm % PER_LINK;
                size_t moff;
                if (t < IN_ROUNDS) {
                    const u32 kb = v & 7;
                    u32 mid;
                    if (t < 8) mid = v >= 8 ? ((rank >> 2) * 16 + t) * 8 + kb : 256 + ((rank & 3) * 8 + t) * 8 + kb;
                    else mid = ((rank >> 2) * 16 + 8 + 2 * (t - 8) + (v >> 3)) * 8 + kb;
                    moff = OFF_PKIN + (size_t)mid * GSLOT;
                } else {
                    moff = OFF_PKOUT + (size_t)(mm - OUT0) * GSLOT;
                }
                return rws + moff;
            };
            auto flag_of = [&](u32 mm) INL { return mm < NEARF ? remote_slot(mm) + SFLAG : fbase + (mm & ~7u) * 128 + (mm & 7u) * 8; };
            u32 f2_lo = 0u, f2_hi = 0u, f3_lo = 0u, f3_hi = 0u;
            LSTAMP(L_STMP, nst, b, 59);
            poll(fbase + m * 128 + 64);
            LSTAMP(L_STMP, nst, b, 60);
            fl_hi = 0u;
            if (!dead) {
                u32 e2;
                asm volatile("global_load_dword %0, %1, off offset:%2 sc0 sc1" : "=v"(e2) : "v"(fbase + m * 128), "n"(PER_LINK * 128 + 68) : "memory");
                const char* sp = remote_slot(m) + lane * 16;
#pragma unroll
                for (int k = 0; k < 8; ++k) {
                    dma1_sys(sp + k * 1024, lds0 + L_RING + k * 1024);
                    asm volatile("s_waitcnt vmcnt(%0)" :: "n"(WIN - 1) : "memory");
                }
                np = 8u;
                if (landed_first<WIN - 1>(e2) < call) { poll(fbase + (m + PER_LINK) * 128 + 64); fl_hi = 0u; }
            }
            if (!dead) {
                u32x2 fa, fb;
                asm volatile("global_load_dwordx2 %0, %2, off sc0 sc1\n\tglobal_load_dwordx2 %1, %3, off sc0 sc1"
                             : "=&v"(fa), "=&v"(fb) : "v"(flag_of(m)), "v"(flag_of(m + PER_LINK)) : "memory");
                const char* sp = remote_slot(m + PER_LINK) + lane * 16;
#pragma unroll
                for (int k = 0; k < WIN - 1; ++k) {
                    dma1_sys(sp + k * 1024, lds0 + L_RING + SLOT + k * 1024);
                    if (k < WIN - 2) asm volatile("s_waitcnt vmcnt(%0)" :: "n"(WIN + 1) : "memory");
                }
                fl_lo = landed_first<WIN - 1>(fa.x); fl_hi = landed_first<WIN - 1>(fa.y);
                f2_lo = landed_first<WIN - 1>(fb.x); f2_hi = landed_first<WIN - 1>(fb.y);
                np += WIN - 1;
                LSTAMP(L_STMP, nst, b, 61);
            }
            for (;; ++n) {
                const u32 s = n % NSLOT;
                if (m < NMSG && !dead && fl_hi < call) {
                    drain(std::integral_constant<int, WIN - 2>{});
                    poll(flag_of(m));
                }
                if (n < 2) LSTAMP(L_STMP, nst, b, 62 + n);
                if (n >= NSLOT) {
                    const u32 need = 8u * (n / NSLOT);
                    if (__builtin_amdgcn_readfirstlane(lds_ld(lds0 + L_FREE + s * 4)) < need) {
                        drain(std::integral_constant<int, WIN - 2>{});
                        while (__builtin_amdgcn_readfirstlane(lds_ld(lds0 + L_FREE + s * 4)) < need) __builtin_amdgcn_s_sleep(1);
                    }
                }
                if (m >= NMSG) {
                    lds_st(lds0 + L_MSG + s * 4, NMSG);
                    lds_st(lds0 + L_DESC, n + 1);
                    break;
                }
                const u32 lo = dead ? F_BAD : fl_lo;
                lds_st(lds0 + L_LEN + s * 4, lo);
                lds_st(lds0 + L_MSG + s * 4, m | (np << 16));
                lds_st(lds0 + L_DESC, n + 1);
                const u32 vb = var_bytes(lo), nv = (vb + 1023u) >> 10;
                [[maybe_unused]] const u32 t = m / PER_LINK, v = m % PER_LINK;
                const char* sp = remote_slot(m) + ((lo & F_RAW) ? RAW_OFF : 0u) + lane * 16;
                if (n < 2) {
                    if (!dead) {
                        if (lo & F_RAW) vm_wait0();
                        for (u32 o = (lo & F_RAW) ? 0u : 8192u; o < 8192u + vb; o += 1024u) {
                            if (o + lane * 16 < 8192u + vb) dma1_sys(sp + o, lds0 + L_RING + n * SLOT + o);
                            ++np;
                            asm volatile("s_waitcnt vmcnt(%0)" :: "n"(WIN - 1) : "memory");
                            publish(WIN - 1);
                        }
                        if (n == 0) {
                            u32x2 fc;
                            asm volatile("global_load_dwordx2 %0, %1, off sc0 sc1" : "=v"(fc) : "v"(flag_of(m + 2 * PER_LINK)) : "memory");
                            sp = remote_slot(m + PER_LINK) + lane * 16;
#pragma unroll
                            for (int k = WIN - 1; k < 8; ++k) {
                                dma1_sys(sp + k * 1024, lds0 + L_RING + SLOT + k * 1024);
                                ++np;
                                asm volatile("s_waitcnt vmcnt(%0)" :: "n"(WIN - 1) : "memory");
                                publish(WIN - 1);
                            }
                            f3_lo = landed_first<WIN - 1>(fc.x); f3_hi = landed_first<WIN - 1>(fc.y);
                        }
                    }
                    m += PER_LINK; fl_lo = f2_lo; fl_hi = f2_hi; f2_lo = f3_lo; f2_hi = f3_hi;
                    LSTAMP(L_STMP, nst, NGEMM + v * 7 + pu % 7, 8 + t);
                    continue;
                }
                u32 nt = 1u;
                u64 keep;
                asm volatile("s_mov_b64 %1, exec\n\ts_mov_b64 exec, 1\n\tglobal_atomic_add %0, %2, %0, off sc0\n\ts_mov_b64 exec, %1"
                             : "+v"(nt), "=&s"(keep) : "v"(tkt) : "memory");
                if (dead) {
                    m = landed_first<0>(nt) - tbase;
                    publish(0);
                    continue;
                }
                const u32 d = lds0 + L_RING + s * SLOT;
                {
                    const u32 o = 8192u + (nv - 1u) * 1024u;
                    if (o + lane * 16 < 8192u + vb) dma1_sys(sp + o, d + o);
                    ++np;
                }
                u32 left = nv + 7u, o = nv > 1u ? 8192u + (nv - 2u) * 1024u : 0u;
                auto piece = [&]() INL {
                    dma1_sys(sp + o, d + o);
                    o = o > 8192u ? o - 1024u : (o == 8192u ? 0u : o + 1024u);
                    ++np; --left;
                };
                asm volatile("s_waitcnt vmcnt(%0)" :: "n"(WIN) : "memory");
#pragma unroll
                for (int k = 0; k < WIN - 2; ++k) {
                    piece();
                    asm volatile("s_waitcnt vmcnt(%0)" :: "n"(WIN) : "memory");
                    publish(WIN - 1);
                }
                piece();
                m = landed_first<WIN - 1>(nt) - tbase;
                publish(WIN - 1);
                if (m >= NMSG) {
                    while (left) {
                        piece();
                        asm volatile("s_waitcnt vmcnt(%0)" :: "n"(WIN2 - 1) : "memory");
                        publish(WIN2 - 1);
                    }
                    drain(std::integral_constant<int, WIN2 - 2>{});
                } else {
                    u32x2 fn;
                    asm volatile("global_load_dwordx2 %0, %1, off sc0 sc1" : "=v"(fn) : "v"(flag_of(m)) : "memory");
#pragma unroll
                    for (int k = 0; k < WIN - 1; ++k) {
                        piece();
                        asm volatile("s_waitcnt vmcnt(%0)" :: "n"(WIN) : "memory");
                        publish(WIN - 1);
                    }
                    piece();
                    fl_lo = landed_first<WIN - 1>(fn.x); fl_hi = landed_first<WIN - 1>(fn.y);
                    publish(WIN - 1);
                    while (left) {
                        piece();
                        asm volatile("s_waitcnt vmcnt(%0)" :: "n"(WIN - 1) : "memory");
                        publish(WIN - 1);
                    }
                }
                LSTAMP(L_STMP, nst, NGEMM + v * 7 + pu % 7, 8 + t);
            }
            STAMP(1);
            LFLUSH(L_STMP, nst);
        } else {
            if (wave == 2) {
                put_dec_table<true>(L_DTBL + 256, lane);
                *(LAS u32*)(L_UNORD + lane * 4) = ((const u32*)SCHED.unord)[lane];
                lgkm_wait0();
                lds_st(lds0 + L_TRDY, 1u);
            }
            [[maybe_unused]] u32 nst = 0;
            bool tables = false;
            for (u32 n = 0;; ++n) {
                const u32 s = n % NSLOT;
                while (__builtin_amdgcn_readfirstlane(lds_ld(lds0 + L_DESC)) <= n) __builtin_amdgcn_s_sleep(1);
                const u32 md = __builtin_amdgcn_readfirstlane(lds_ld(lds0 + L_MSG + s * 4)), m = md & 0xffffu;
                if (m >= NMSG) break;
                const u32 t = m / PER_LINK, v = m % PER_LINK;
                const u32 lo = __builtin_amdgcn_readfirstlane(lds_ld(lds0 + L_LEN + s * 4));
                const u32 sl = lds0 + L_RING + s * SLOT;
                if (m >= OUT0 && !tables) {
                    while (!__builtin_amdgcn_readfirstlane(lds_ld(lds0 + L_TRDY))) __builtin_amdgcn_s_sleep(1);
                    tables = true;
                }
                if (!(lo & F_BAD)) {
                    const u32 nv = (var_bytes(lo) + 1023u) >> 10;
                    const u32 need = (md >> 16) + (n >= 2 ? nv + wave + 1u : ((lo & F_RAW) ? 16u : nv));
                    while (__builtin_amdgcn_readfirstlane(lds_ld(lds0 + L_FULL)) < need) __builtin_amdgcn_s_sleep(1);
                }
                if (t == 3 && wave == 7) LSTAMP(L_STMP + 1024 + wave * 512, nst, NGEMM + v * 7 + pu % 7, 56);
                u32x4 v0, v1;
                if (lo & F_BAD) {
                    v0 = u32x4{0x7fc07fc0u, 0x7fc07fc0u, 0x7fc07fc0u, 0x7fc07fc0u}; v1 = v0;
                } else if (lo & F_RAW) {
                    v0 = lds_ld16(sl + 8192 + wave * 1024 + lane * 16);
                    v1 = lds_ld16(sl + wave * 1024 + lane * 16);
                } else if (t < IN_ROUNDS) {
                    decode_block<false>(sl, (lo >> 16) & 0xffu, L_DTBL, wave, lane, v0, v1);
                } else {
                    decode_block<true>(sl, 0u, L_DTBL + 256, wave, lane, v0, v1);
                }
                asm volatile("s_waitcnt lgkmcnt(0)" : "+v"(v0), "+v"(v1) :: "memory");
                if (lane == 0) asm volatile("ds_add_u32 %0, %1" :: "v"(lds0 + L_FREE + s * 4), "v"(1u) : "memory");
                if (t == 3 && wave == 7) LSTAMP(L_STMP + 1024 + wave * 512, nst, NGEMM + v * 7 + pu % 7, 57);
                if (t < IN_ROUNDS) {
                    const u32 kb = v & 7;
                    const bool isw = t < 8 && v < 8;
                    const u32 lp = t < 8 ? t : 8 + 2 * (t - 8) + (v >> 3);
                    char* img = myws + (isw ? OFF_UW : OFF_UX) + (size_t)(lp * 64 + src * 8 + kb) * IMG + wave * 2048 + lane * 16;
                    st16_wt(img, v0); st16_wt(img + 1024, v1);
                    vm_wait0();
                    u32 old = 0;
                    if (lane == 0) {
                        asm volatile("ds_add_rtn_u32 %0, %1, %2\n\ts_waitcnt lgkmcnt(0)" : "=&v"(old) : "v"(lds0 + L_DONE + s * 4), "v"(1u) : "memory");
                        if ((old & 7u) == 7u)
                            __hip_atomic_fetch_add((GAS u32*)(myws + OFF_CNT + (size_t)((isw ? 16 : 0) + lp) * 256), 1u,
                                                   __ATOMIC_RELAXED, __HIP_MEMORY_SCOPE_AGENT);
                    }
                } else {
                    const int g = __builtin_amdgcn_readfirstlane(*(const LAS unsigned char*)(L_UNORD + m - OUT0));
                    const int kh = wave >> 2, wn = wave & 1, wm = (wave >> 1) & 1;
                    const int ti = ((g >> 4) & 7) + 8 * (g >> 7), hf = (g >> 3) & 1, tj = g & 7;
                    char* op = P.out + ((size_t)((src >> 2) * 2048 + ti * 128 + hf * 64 + wm * 32 + kh * 16 + (lane & 15)) * 4096
                                        + (src & 3) * 1024 + tj * 128 + wn * 64 + (lane >> 4) * 8) * 2;
                    st16_out(op, v0); st16_out(op + 64, v1);
                }
            }
            STAMP(2);
            LFLUSH(L_STMP + 1024 + wave * 512, nst);
        }
    }
}

extern "C" void kernel_launch(int rank, int nranks, ncclComm_t comm,
                              hipStream_t stream,
                              void* const* d_in, const int* in_sizes, int n_in,
                              void* d_out, int out_size,
                              const int* in_dtypes, int out_dtype,
                              void* const* d_out_all,
                              void* const* d_ws_all, size_t ws_bytes) {
    (void)comm; (void)in_dtypes; (void)out_dtype; (void)d_out_all;
    if (nranks != NR || (unsigned)rank >= (unsigned)NR || n_in != 2 || in_sizes[0] != 4096 * 512 || in_sizes[1] != 512 * 4096
        || out_size != 4096 * 4096 || ws_bytes < WS_NEED) {
        fprintf(stderr, "kernel_launch: unexpected problem shape\n");
        return;
    }
    static bool attr_set[NR] = {};
    if (!attr_set[rank]) {
        HIP_CHECK(hipFuncSetAttribute((const void*)fused_gemm_allgather, hipFuncAttributeMaxDynamicSharedMemorySize, LDS_BYTES));
        attr_set[rank] = true;
    }
    Params P;
    for (int p = 0; p < NR; ++p) P.ws[p] = (char*)d_ws_all[p];
    P.x = (const char*)d_in[0];
    P.w = (const char*)d_in[1];
    P.out = (char*)d_out;
    P.myws = P.ws[rank];
    P.rank = rank;
    P.pad = 0;
    hipLaunchKernelGGL(fused_gemm_allgather, dim3(GRID), dim3(THREADS), LDS_BYTES, stream, P);
    HIP_CHECK(hipGetLastError());
}
```

```cpp
#include <hip/hip_runtime.h>
#include <rccl/rccl.h>
#include <cstdio>
#include <cstdint>
#include <cstddef>
#include <type_traits>
#pragma clang diagnostic ignored "-Wint-to-pointer-cast"

#define HIP_CHECK(expr) do { hipError_t _e = (expr); if (_e != hipSuccess) { \
    fprintf(stderr, "HIP error %s at %s:%d\n", hipGetErrorString(_e), __FILE__, __LINE__); \
    return; } } while (0)

typedef unsigned u32;
typedef unsigned long long u64;
typedef u32 u32x4 __attribute__((ext_vector_type(4)));
typedef u32 u32x2 __attribute__((ext_vector_type(2)));
typedef float f32x4 __attribute__((ext_vector_type(4)));
typedef float f32x2 __attribute__((ext_vector_type(2)));
typedef __bf16 bf16x8 __attribute__((ext_vector_type(8)));
typedef __bf16 bf16x2 __attribute__((ext_vector_type(2)));
typedef short s16x4 __attribute__((ext_vector_type(4)));
#define INL __attribute__((always_inline))
#define LAS __attribute__((address_space(3)))
#define GAS __attribute__((address_space(1)))

constexpr int NR = 8;
constexpr int NGEMM = 128;
constexpr int PER_LINK = 16;
constexpr int NPULL = 7 * PER_LINK;
#define MAP_ 1
constexpr int NROLE = NGEMM + NPULL;
constexpr int GRID = MAP_ ? 256 : NROLE;
constexpr int THREADS = 576;
constexpr int IN_ROUNDS = 12, OUT_ROUNDS = 16, ROUNDS = IN_ROUNDS + OUT_ROUNDS;
constexpr u32 OUT0 = IN_ROUNDS * PER_LINK;
constexpr u32 NMSG = ROUNDS * PER_LINK;
constexpr u32 SLOT = 16384 + 512;
constexpr u32 GSLOT = 2 * SLOT;
constexpr u32 NEARF = 3 * PER_LINK, SFLAG = SLOT - 128;
#define OBATCH_ 144
#define INBATCH_ 1
constexpr u32 OBATCH = OBATCH_;
constexpr bool INBATCH = INBATCH_;
static_assert(OBATCH % 8 == 0 && NEARF % 8 == 0, "whole groups");
#define RK_ 20
#define RHYST_ 1
#define RTHR_ 2
#define RMARG_ 2
constexpr u32 RK = RK_, RW0 = OUT0 + 32, RN = 7 * RK, REMAX = 31;
constexpr u32 TICKETS = NMSG - 2 * PER_LINK + REMAX + 1;
static_assert(RK < 32 && RW0 + RN <= NMSG, "window");
constexpr u32 IMG = 16384;
constexpr int NSLOT = 8;
#define WIN_ 4
#define WIN2_ 8
#define ENDK_ 12
constexpr unsigned ENDK = ENDK_;
constexpr int WIN2 = WIN2_;
static_assert(2 * WIN_ + 1 <= 9, "a message has at least 9 pieces");
constexpr int WIN = WIN_;
constexpr unsigned long long DEADLINE = 100000000ull;

constexpr size_t OFF_CTR = 0;
constexpr size_t OFF_TMO = 32768;
constexpr size_t OFF_CNT = 36864;
constexpr size_t OFF_TKT = 45056;
constexpr size_t OFF_KEEP = 47104;
constexpr size_t OFF_GRP = 49152;
constexpr size_t OFF_FLAG = 65536;
constexpr size_t OFF_DBG = 768 * 1024;
constexpr size_t OFF_PKIN = 1u << 20;
constexpr size_t OFF_PKOUT = 18u << 20;
constexpr size_t OFF_UX = 28u << 20;
constexpr size_t OFF_UW = 44u << 20;
constexpr size_t OFF_RLY = 52u << 20;
constexpr size_t WS_NEED = 96u << 20;
static_assert(OFF_RLY + (size_t)7 * RN * GSLOT <= WS_NEED, "copies");
static_assert(OFF_FLAG + (size_t)NR * NMSG * 128 <= OFF_DBG, "flags");
static_assert(OFF_PKIN + 512 * (size_t)GSLOT <= OFF_PKOUT, "pkin");
static_assert(OFF_PKOUT + 256 * (size_t)GSLOT <= OFF_UX, "pkout");

constexpr u32 L_RING = 0;
constexpr u32 L_CTL = NSLOT * SLOT;
constexpr u32 L_FULL = L_CTL;
constexpr u32 L_DESC = L_CTL + 4;
constexpr u32 L_PLAN = L_CTL + 192;
constexpr u32 L_TRDY = L_CTL + 240;
constexpr u32 L_FREE = L_CTL + 16;
constexpr u32 L_DONE = L_CTL + 16 + 4 * NSLOT;
constexpr u32 L_LEN = L_CTL + 16 + 8 * NSLOT;
constexpr u32 L_MSG = L_CTL + 16 + 12 * NSLOT;
constexpr u32 L_DTBL = L_CTL + 256;
constexpr u32 L_UNORD = L_DTBL + 768;
constexpr u32 L_STMP = L_UNORD + 256;
constexpr int NST = 6;
constexpr u32 STG = 8192 + IMG;
constexpr u32 L_XCH = 4 * STG;
constexpr u32 L_VAR = L_XCH + 32768;
constexpr u32 L_ETIN = L_VAR + 8192;
constexpr u32 L_ETOUT = NST * STG;
constexpr u32 L_TOT = L_ETOUT + 2048;
constexpr u32 L_SRC = L_TOT + 48;
constexpr u32 LDS_BYTES = L_TOT + 64;
static_assert(LDS_BYTES >= L_STMP + 5120 && LDS_BYTES <= 163840 && L_ETIN + 4096 <= L_ETOUT, "lds");

constexpr int pack_passes(int b) {
    return (b >> 4) <= 1 && (b & 7) <= 1 ? 2 : ((b & 7) == 7 ? 5 : 4);
}
constexpr int pack_msg(int b, int z) {
    const int early = b < 32 ? 2 * (b >> 3) + ((b & 7) < 2 ? (b & 7) : 2) : 8;
    return z < 2 ? z * 128 + b : (z < 4 ? 256 + (z - 2) * 120 + b - early : 496 + (b >> 3));
}
__device__ __forceinline__ void msg_of(int rho, int& isw, int& grp, int& lp, int& kb, int& seq) {
    if (rho < 384) {
        const int t = rho / 48, r = rho % 48;
        isw = r < 32; grp = isw ? r >> 3 : (r - 32) >> 3; kb = r & 7; lp = t; seq = t * 16 + (isw ? 0 : 8) + kb;
    } else {
        const int t = 8 + ((rho - 384) >> 5), r = rho & 31, pos = (r >> 3) & 1;
        isw = 0; grp = r >> 4; kb = r & 7; lp = 8 + 2 * (t - 8) + pos; seq = t * 16 + pos * 8 + kb;
    }
}
#define WFRESH_ 0
struct Sched { alignas(4) unsigned char unord[256]; unsigned char ord[2][128]; bool ok; };
constexpr Sched make_sched() {
    Sched s{};
    int r = 0;
    s.ok = true;
    for (int z = 0; z < 5; ++z)
        for (int b = 0; b < 128; ++b)
            if (pack_passes(b) > z && pack_msg(b, z) != r++) s.ok = false;
    if (r != 512) s.ok = false;
    int key[256] = {};
    for (int b = 0; b < 128; ++b) {
        const int i = b >> 4, j = b & 7, sh = i > j ? i : j;
        const int t1 = 30 * sh + 170 + (j > i ? WFRESH_ : 0), arr2 = 30 * (8 + (i >> 1));
        key[b] = t1 * 256 + b;
        key[128 + b] = ((t1 > arr2 ? t1 : arr2) + 170) * 256 + 128 + b;
    }
    for (int a = 0; a < 256; ++a) {
        int pos = 0;
        for (int c = 0; c < 256; ++c) pos += key[c] < key[a];
        s.ord[a >> 7][a & 127] = (unsigned char)pos;
        s.unord[pos] = (unsigned char)a;
    }
    return s;
}
__device__ constexpr Sched SCHED = make_sched();
static_assert(make_sched().ok, "every input message is packed once");

constexpr u32 F_RAW = 0x80000000u;
constexpr u32 F_BAD = 0x40000000u;

__device__ __forceinline__ u32 var_bytes(u32 lo) {
    const u32 v = lo & 0xffffu;
    return (lo & F_RAW) || v > 8192u ? 8192u : (v < 16u ? 16u : v);
}
template <int N, int I = 0, class F> __device__ __forceinline__ void static_for(F f) {
    if constexpr (I < N) { f(std::integral_constant<int, I>{}); static_for<N, I + 1>(f); }
}

struct Params {
    const char* x;
    const char* w;
    char* myws;
    char* out;
    int rank;
    int pad;
    char* ws[NR];
};

#define STAMP(id) do {} while (0)
#define TS(i) do {} while (0)
#define LSTAMP(base, cnt, wgrow, id) do {} while (0)
#define LFLUSH(base, cnt) do {} while (0)

__device__ __forceinline__ void st16_wt(char* p, u32x4 v) {
    asm volatile("global_store_dwordx4 %0, %1, off sc0 sc1\n\ts_nop 1" :: "v"(p), "v"(v) : "memory");
}
#define OUTB 5
#define OUTBITS " sc0 sc1 nt"
__device__ __forceinline__ void st16_out(char* p, u32x4 v) {
    asm volatile("global_store_dwordx4 %0, %1, off" OUTBITS "\n\ts_nop 1" :: "v"(p), "v"(v) : "memory");
}
__device__ __forceinline__ void st4_wt(char* p, u32 v) {
    asm volatile("global_store_dword %0, %1, off sc0 sc1" :: "v"(p), "v"(v) : "memory");
}
__device__ __forceinline__ void vm_wait0() { asm volatile("s_waitcnt vmcnt(0)" ::: "memory"); }
__device__ __forceinline__ void lgkm_wait0() { asm volatile("s_waitcnt lgkmcnt(0)" ::: "memory"); }
__device__ __forceinline__ void bar() { asm volatile("s_waitcnt lgkmcnt(0)\n\ts_barrier" ::: "memory"); }
__device__ __forceinline__ void lds_st(u32 addr, u32 v) { asm volatile("ds_write_b32 %0, %1" :: "v"(addr), "v"(v) : "memory"); }
__device__ __forceinline__ u32 lds_ld(u32 addr) {
    u32 v; asm volatile("ds_read_b32 %0, %1\n\ts_waitcnt lgkmcnt(0)" : "=v"(v) : "v"(addr) : "memory"); return v;
}
__device__ __forceinline__ u32x4 lds_ld16(u32 addr) { return *(const LAS u32x4*)addr; }
template <int N> __device__ __forceinline__ u32 landed_first(u32 v) {
    u32 r; asm volatile("s_waitcnt vmcnt(%2)\n\tv_readfirstlane_b32 %0, %1\n\ts_nop 3" : "=s"(r) : "v"(v), "n"(N) : "memory"); return r;
}
__device__ __forceinline__ void dma1(const char* src, u32 lds_dst) {
    u32 keep;
    asm volatile("s_mov_b32 %0, m0\n\ts_mov_b32 m0, %2\n\ts_nop 0\n\tglobal_load_lds_dwordx4 %1, off\n\ts_mov_b32 m0, %0"
                 : "=&s"(keep) : "v"(src), "s"(lds_dst) : "memory");
}
__device__ __forceinline__ void dma1_sys(const char* src, u32 lds_dst) {
    u32 keep;
    asm volatile("s_mov_b32 %0, m0\n\ts_mov_b32 m0, %2\n\ts_nop 0\n\tglobal_load_lds_dwordx4 %1, off sc0 sc1\n\ts_mov_b32 m0, %0"
                 : "=&s"(keep) : "v"(src), "s"(lds_dst) : "memory");
}
__device__ __forceinline__ void dma1s(const char* sbase, u32 voff, u32 lds_dst) {
    u64 t;
    asm volatile("s_mov_b32 m0, %3\n\ts_mov_b64 %0, %2\n\tglobal_load_lds_dwordx4 %1, %0" : "=&s"(t) : "v"(voff), "s"(sbase), "s"(lds_dst) : "memory");
}
__device__ __forceinline__ void dma_a(const char* sbase, u32 voff, u32 lds_dst) {
    u64 t;
    asm volatile("s_mov_b32 m0, %3\n\ts_mov_b64 %0, %2\n\tglobal_load_lds_dwordx4 %1, %0 sc0 sc1" : "=&s"(t) : "v"(voff), "s"(sbase), "s"(lds_dst) : "memory");
}
__device__ __forceinline__ void dma_b(const char* sbase, u32 voff) {
    u64 t;
    asm volatile("s_mov_b64 %0, %2\n\tglobal_load_lds_dwordx4 %1, %0 offset:1024 sc0 sc1" : "=&s"(t) : "v"(voff), "s"(sbase) : "memory");
}
__device__ __forceinline__ void announce(char* myws, char* wsl, int rank, u32 seq, u64 val, u32 destmask, int grp, int lane) {
    const size_t line = OFF_FLAG + ((size_t)rank * NMSG + (seq & ~7u)) * 128;
    if (grp < 0) {
        if (lane < NR && ((destmask >> lane) & 1u))
            __hip_atomic_store((GAS u64*)(wsl + line + (seq & 7u) * 8), val, __ATOMIC_RELAXED, __HIP_MEMORY_SCOPE_SYSTEM);
        return;
    }
    char* g = myws + OFF_GRP + (size_t)grp * 128;
    u32 old = 0;
    if (lane == 0) {
        __hip_atomic_store((GAS u64*)(g + (seq & 7u) * 8), val, __ATOMIC_RELAXED, __HIP_MEMORY_SCOPE_AGENT);
        asm volatile("s_waitcnt vmcnt(0)" ::: "memory");
        old = __hip_atomic_fetch_add((GAS u32*)(g + 64), 1u, __ATOMIC_RELAXED, __HIP_MEMORY_SCOPE_AGENT);
    }
    if ((__builtin_amdgcn_readfirstlane(old) & 7u) == 7u) {
        const u64 e = __hip_atomic_load((GAS u64*)(g + (lane & 7) * 8), __ATOMIC_RELAXED, __HIP_MEMORY_SCOPE_AGENT);
        const u64 wl = (u64)wsl;
        char* wp = (char*)((u64)(u32)__builtin_amdgcn_ds_bpermute((lane >> 3) * 4, (int)(u32)wl)
                           | (u64)(u32)__builtin_amdgcn_ds_bpermute((lane >> 3) * 4, (int)(u32)(wl >> 32)) << 32);
        if ((destmask >> (lane >> 3)) & 1u)
            __hip_atomic_store((GAS u64*)(wp + line + (lane & 7) * 8), e, __ATOMIC_RELAXED, __HIP_MEMORY_SCOPE_SYSTEM);
    }
}
__device__ __forceinline__ u32 fetch_plan(char* tkt, u32 call, int lane) {
    const u32 h = __builtin_amdgcn_readfirstlane(__hip_atomic_load((GAS u32*)(tkt + 128), __ATOMIC_RELAXED, __HIP_MEMORY_SCOPE_AGENT));
    if ((h >> 15) != (call & 0x1ffffu)) return 0u;
    const u32 w = __hip_atomic_load((GAS u32*)(tkt + 132 + (lane & 7) * 4), __ATOMIC_RELAXED, __HIP_MEMORY_SCOPE_AGENT);
    if (lane < 8) lds_st(L_PLAN + lane * 4, w);
    lds_st(L_PLAN + 32, (h & 0x7fffu) | 0x8000u);
    return (h & 0x7fffu) | 0x8000u;
}
__device__ __forceinline__ void wait_counters(GAS u32* c, u32 want, GAS u32* tmo, u32 call) {
    const u64 t0 = __builtin_amdgcn_s_memrealtime();
    for (u32 it = 1; !__all(__hip_atomic_load(c, __ATOMIC_RELAXED, __HIP_MEMORY_SCOPE_SYSTEM) >= want); ++it) {
        if ((it & 15u) == 0u
            && (__builtin_amdgcn_readfirstlane(__hip_atomic_load(tmo, __ATOMIC_RELAXED, __HIP_MEMORY_SCOPE_AGENT)) == call
                || __builtin_amdgcn_s_memrealtime() - t0 > DEADLINE)) {
            __hip_atomic_store(tmo, call, __ATOMIC_RELAXED, __HIP_MEMORY_SCOPE_AGENT);
            break;
        }
        __builtin_amdgcn_s_sleep(2);
    }
    asm volatile("s_waitcnt vmcnt(0)" ::: "memory");
}

__device__ __forceinline__ float gelu(float y) {
    const float k0 = -2.0f * 0.7978845608028654f * 1.4426950408889634f;
    const float u = y * y;
    const float z = y * __builtin_fmaf(u, k0 * 0.044715f, k0);
    return y * __builtin_amdgcn_rcpf(1.0f + __builtin_amdgcn_exp2f(z));
}
__device__ __forceinline__ u32 pk_bf16(float a, float b) {
    f32x2 f = {a, b};
    return __builtin_bit_cast(u32, __builtin_convertvector(f, bf16x2));
}

constexpr u64 IN_REL0 = 0x1234567'8d9abcull, IN_REL1 = 0x123d456'789cabull;
constexpr u32 OUT_E0 = 114, OUT_NSYM = 184;
__device__ constexpr unsigned char OUT_RANK[192] = {
    210, 209, 208, 207, 104, 206, 96, 105, 205, 97, 204, 98, 203, 106, 99, 202, 100, 201, 88, 101, 107, 200, 89,
    90, 102, 91, 103, 92, 108, 93, 94, 95, 109, 112, 80, 81, 82, 83, 84, 110, 85, 86, 87, 199, 198, 111,
    197, 196, 195, 194, 193, 192, 113, 72, 73, 74, 75, 76, 77, 78, 79, 114, 65, 191, 64, 66, 190, 67, 189,
    68, 188, 69, 187, 70, 71, 186, 185, 184, 115, 57, 58, 56, 59, 60, 61, 62, 63, 183, 182, 181, 180, 179,
    178, 177, 176, 116, 49, 50, 51, 52, 53, 48, 54, 55, 175, 174, 173, 172, 171, 170, 169, 168, 41, 42, 43,
    44, 45, 46, 47, 40, 117, 167, 166, 165, 164, 163, 162, 161, 160, 33, 34, 35, 36, 37, 38, 39, 32, 118,
    159, 158, 157, 156, 155, 154, 153, 152, 25, 26, 27, 28, 29, 30, 31, 24, 151, 150, 149, 147, 148, 146, 145,
    144, 119, 17, 18, 19, 20, 21, 23, 22, 16, 137, 138, 139, 140, 141, 142, 143, 136, 11, 9, 10, 12, 13};
__device__ __forceinline__ constexpr u32x2 enc_words(u32 c1, u32 q) {
    const u32 c2 = q < 3u ? q : 3u, c3 = q < 3u ? 0u : (q < 6u ? q - 3u : 3u), c4 = q < 6u ? 0u : q - 6u;
    return u32x2{(c1 << 28) | c2 | (c3 << 8) | (c4 << 16), 2u | (q >= 3u ? 2u << 8 : 0u) | (q >= 6u ? 3u << 16 : 0u)};
}
template <bool OUT> __device__ __forceinline__ void put_enc_table(u32 l, int lane) {
    const u32x2 esc = enc_words(OUT ? 15u : 0u, 13u);
#pragma unroll
    for (int j = 0; j < (OUT ? 2 : 4); ++j)
        *(LAS u32x4*)(l + j * 1024 + lane * 16) = u32x4{esc.x, esc.y | (1u << 24), esc.x, esc.y | (1u << 24)};
    if (OUT) {
#pragma unroll
        for (u32 r = lane; r < 192u; r += 64u)
            if (r < OUT_NSYM) *(LAS u32x2*)(l + OUT_RANK[r] * 8u) = r < 3u ? u32x2{r << 28, 0u} : enc_words(3u + (r - 3u) % 13u, (r - 3u) / 13u);
    } else {
        const u32 q = lane & 15u, m = ((u32)lane >> 4) & 1u;
        if (lane < 32 && q < 13u) *(LAS u32x2*)(l + ((((u32)((m ? IN_REL1 : IN_REL0) >> (4u * q)) & 15u) << 1) | m) * 8u) = enc_words(0u, q);
    }
}
template <bool OUT> __device__ __forceinline__ void put_dec_table(u32 l, int lane) {
    if (OUT) {
#pragma unroll
        for (u32 i = lane; i < 256u; i += 64u) {
            const u32 c1 = i >> 4, q = i & 15u, r = c1 < 3u ? c1 : 13u * q + c1, x = OUT_RANK[r < OUT_NSYM ? r : 0u];
            *(LAS unsigned short*)(l + i * 2u) =
                (unsigned short)(r < OUT_NSYM ? ((x >> 7) << 15) | ((((x >> 3) & 15u) + OUT_E0) << 7) | ((x & 7u) << 4) : 1u);
        }
    } else {
        const u32 c1 = (u32)lane >> 4, q = lane & 15u;
        *(LAS unsigned short*)(l + c1 * 64u + q * 2u) = (unsigned short)(
            q < 13u ? ((c1 >> 1) << 15) | (((u32)(((c1 & 1u) ? IN_REL1 : IN_REL0) >> (4u * q)) & 15u) << 7) | ((c1 & 1u) << 6) : 1u);
    }
}

__device__ __forceinline__ u32 wave_scan(u32 v) {
    v += (u32)__builtin_amdgcn_update_dpp(0, (int)v, 0x111, 0xf, 0xf, false);
    v += (u32)__builtin_amdgcn_update_dpp(0, (int)v, 0x112, 0xf, 0xf, false);
    v += (u32)__builtin_amdgcn_update_dpp(0, (int)v, 0x114, 0xf, 0xf, false);
    v += (u32)__builtin_amdgcn_update_dpp(0, (int)v, 0x118, 0xf, 0xf, false);
    v += (u32)__builtin_amdgcn_update_dpp(0, (int)v, 0x142, 0xa, 0xf, false);
    v += (u32)__builtin_amdgcn_update_dpp(0, (int)v, 0x143, 0xc, 0xf, false);
    return v;
}
__device__ __forceinline__ u32 ubfe(u32 v, u32 off, u32 w) { return __builtin_amdgcn_ubfe(v, off, w); }
__device__ __forceinline__ void lds_or_bits(u32 base, u32 pos, u32 bits) {
    if (bits) {
        LAS u32* p = (LAS u32*)(base + (pos >> 5) * 4u);
        __hip_atomic_fetch_or(p, bits << (pos & 31u), __ATOMIC_RELAXED, __HIP_MEMORY_SCOPE_WORKGROUP);
        __hip_atomic_fetch_or(p + 1, (bits >> 1) >> (~pos & 31u), __ATOMIC_RELAXED, __HIP_MEMORY_SCOPE_WORKGROUP);
    }
}
__device__ __forceinline__ u32 lds_get_bits(u32 base, u32 pos) {
    const LAS u32* p = (const LAS u32*)(base + (pos >> 5) * 4u);
    return __builtin_amdgcn_alignbit(p[1], p[0], pos & 31u);
}
__device__ __forceinline__ u32 low_mask_even(u32 n) { return ~((0xffffffffu << (n >> 1)) << (n >> 1)); }

#define SDWA_SHL(B) \
    __device__ __forceinline__ u32 shl_byte##B(u32 sh, u32 v) {     \
        u32 r; asm("v_lshlrev_b32_sdwa %0, %1, %2 dst_sel:DWORD dst_unused:UNUSED_PAD src0_sel:BYTE_" #B " src1_sel:BYTE_" #B \
                   : "=v"(r) : "v"(sh), "v"(v)); return r; }
SDWA_SHL(0) SDWA_SHL(1) SDWA_SHL(2)

template <int B> __device__ __forceinline__ u32 and_byte(u32 v, u32 m) {
    u32 r;
    if constexpr (B == 0) asm("v_and_b32_sdwa %0, %1, %2 dst_sel:DWORD dst_unused:UNUSED_PAD src0_sel:BYTE_0 src1_sel:DWORD" : "=v"(r) : "v"(v), "v"(m));
    if constexpr (B == 1) asm("v_and_b32_sdwa %0, %1, %2 dst_sel:DWORD dst_unused:UNUSED_PAD src0_sel:BYTE_1 src1_sel:DWORD" : "=v"(r) : "v"(v), "v"(m));
    if constexpr (B == 2) asm("v_and_b32_sdwa %0, %1, %2 dst_sel:DWORD dst_unused:UNUSED_PAD src0_sel:BYTE_2 src1_sel:DWORD" : "=v"(r) : "v"(v), "v"(m));
    if constexpr (B == 3) asm("v_and_b32_sdwa %0, %1, %2 dst_sel:DWORD dst_unused:UNUSED_PAD src0_sel:BYTE_3 src1_sel:DWORD" : "=v"(r) : "v"(v), "v"(m));
    return r;
}

__device__ __forceinline__ void store_input_bytes(char* slot, const u32 (&d)[8], int wave, int lane) {
    u32 bb[4];
#pragma unroll
    for (int j = 0; j < 4; ++j) {
        const u32 lowb = __builtin_amdgcn_perm(d[2 * j + 1], d[2 * j], 0x06040200u);
        const u32 higb = __builtin_amdgcn_perm(d[2 * j + 1], d[2 * j], 0x07050301u);
        bb[j] = (lowb & 0x7f7f7f7fu) | (higb & 0x80808080u);
    }
    st16_wt(slot + wave * 1024 + lane * 16, u32x4{bb[0], bb[1], bb[2], bb[3]});
}
constexpr u32 RAW_OFF = SLOT;
__device__ __forceinline__ u32 input_e0(u32x2 sample) {
    const u32 s2 = ((sample.x >> 7) & 0x00ff00ffu) + ((sample.y >> 7) & 0x00ff00ffu);
    const u32 tot = __builtin_amdgcn_readlane(wave_scan((s2 & 0xffffu) + (s2 >> 16)), 63);
    const u32 anchor = (tot + 504u) >> 8;
    return anchor < 12u ? 0u : (anchor > 252u ? 240u : anchor - 12u);
}
template <bool OUT>
__device__ __forceinline__ void encode_block(char* slot, const u32 (&d)[8], u32 e0, u32 ltbl, u32 lpriv, u32 ltot, int wave, int lane) {
    *(LAS u32x4*)(lpriv + lane * 16) = u32x4{0u, 0u, 0u, 0u};
    u32 en[16], in[16];
    typedef unsigned short u16x2 __attribute__((ext_vector_type(2)));
#pragma unroll
    for (int j = 0; j < 8; ++j) {
        u32 i0, i1;
        if (OUT) {
            const u16x2 sub = {OUT_E0 << 6, OUT_E0 << 6}, cap = {15 << 6, 15 << 6};
            const u16x2 t = __builtin_elementwise_min(__builtin_bit_cast(u16x2, (d[j] >> 1) & 0x3ff83ff8u) - sub, cap);
            const u32 x = __builtin_bit_cast(u32, t) | ((d[j] >> 5) & 0x04000400u);
            i0 = x & 0xffffu; i1 = x >> 16;
        } else {
            const u16x2 sub = {(unsigned short)(e0 << 7), (unsigned short)(e0 << 7)};
            const u32 x = __builtin_bit_cast(u32, __builtin_bit_cast(u16x2, d[j]) - sub);
            i0 = (x >> 3) & 0xff8u; i1 = (x >> 19) & 0xff8u;
        }
        const u32x2 a = *(const LAS u32x2*)(ltbl + i0), b = *(const LAS u32x2*)(ltbl + i1);
        en[2 * j] = a.x; in[2 * j] = a.y; en[2 * j + 1] = b.x; in[2 * j + 1] = b.y;
    }
    __builtin_amdgcn_sched_barrier(0);
    if (OUT) {
        u32 bb[4];
#pragma unroll
        for (int j = 0; j < 4; ++j) {
            const u32 lowb = __builtin_amdgcn_perm(d[2 * j + 1], d[2 * j], 0x06040200u);
            const u32 c01 = __builtin_amdgcn_perm(en[4 * j + 1], en[4 * j], 0x0c0c0703u);
            const u32 c23 = __builtin_amdgcn_perm(en[4 * j + 3], en[4 * j + 2], 0x07030c0cu);
            bb[j] = (lowb & 0x0f0f0f0fu) | c01 | c23;
        }
        st16_wt(slot + wave * 1024 + lane * 16, u32x4{bb[0], bb[1], bb[2], bb[3]});
    }
    u32 f2 = 0, f3 = 0, f4[2] = {0, 0}, pos = 0, n4a = 0;
#pragma unroll
    for (int i = 0; i < 16; i += 2) {
        if (i == 8) { n4a = ubfe(pos, 16, 8); pos &= 0xff00ffffu; }
        const u32 pos1 = pos + in[i];
        f2 |= shl_byte0(pos, en[i]) | shl_byte0(pos1, en[i + 1]);
        f3 |= shl_byte1(pos, en[i]) | shl_byte1(pos1, en[i + 1]);
        f4[i >> 3] |= shl_byte2(pos, en[i]) | shl_byte2(pos1, en[i + 1]);
        pos = pos1 + in[i + 1];
    }
    const u32 n4 = n4a + ubfe(pos, 16, 8), ne = pos >> 24;
    const u32 ca = (pos & 0xffu) | ((pos << 8) & 0xff0000u), cb = n4 | (ne << 16);
    const u32 sa = wave_scan(ca), sb = wave_scan(cb);
    const u32 ta = __builtin_amdgcn_readlane(sa, 63), tb = __builtin_amdgcn_readlane(sb, 63);
    const u32 N2 = ta & 0xffffu, N3 = ta >> 16, NE = tb >> 16;
    const u32 escpos = (N2 + N3 + (tb & 0xffffu) + 15u) & ~15u;
    const u32 dw = (escpos + 16u * NE + 31u) >> 5;
    if (lane == 0) *(LAS u32*)(ltot + wave * 4) = dw <= 252u ? dw : 0x1000u;
    if (dw <= 252u) {
        const u32 xa = sa - ca, xb = sb - cb;
        if (OUT) lds_or_bits(lpriv, xa & 0xffffu, f2);
        else *(LAS u32*)(lpriv + lane * 4) = f2;
        lds_or_bits(lpriv, N2 + (xa >> 16), f3);
        const u32 p4 = N2 + N3 + (xb & 0xffffu);
        if (!__builtin_amdgcn_ballot_w64(n4 > 32u)) {
            lds_or_bits(lpriv, p4, f4[0] | (f4[1] << n4a));
        } else {
            lds_or_bits(lpriv, p4, f4[0]);
            lds_or_bits(lpriv, p4 + n4a, f4[1]);
        }
        if (NE) {
            u32 a = lpriv + (escpos >> 3) + (xb >> 16) * 2u;
#pragma unroll
            for (int j = 0; j < 4; ++j)
                if ((in[4 * j] | in[4 * j + 1] | in[4 * j + 2] | in[4 * j + 3]) >> 24) {
#pragma unroll
                    for (int i = 4 * j; i < 4 * j + 4; ++i)
                        if (in[i] >> 24) { *(LAS unsigned short*)a = (unsigned short)(d[i >> 1] >> (16 * (i & 1))); a += 2u; }
                }
        }
    }
}
__device__ __forceinline__ u32 encode_finish(char* slot, const u32 (&d)[8], u32 lpriv, u32 ltot, int wave, int lane) {
    const u32 mine = *(const LAS u32*)(ltot + (lane & 7) * 4);
    const u32 w0 = *(const LAS u32*)(lpriv + lane * 4), w1 = *(const LAS u32*)(lpriv + 256 + lane * 4);
    u32 inc = mine;
    inc += (u32)__builtin_amdgcn_update_dpp(0, (int)inc, 0x111, 0xf, 0xf, false);
    inc += (u32)__builtin_amdgcn_update_dpp(0, (int)inc, 0x112, 0xf, 0xf, false);
    inc += (u32)__builtin_amdgcn_update_dpp(0, (int)inc, 0x114, 0xf, 0xf, false);
    const u32 total = __builtin_amdgcn_readlane(inc, 7);
    if (total >= 0x1000u) {
        char* p = slot + RAW_OFF + wave * 1024 + lane * 16;
        st16_wt(p + 8192, u32x4{d[0], d[1], d[2], d[3]}); st16_wt(p, u32x4{d[4], d[5], d[6], d[7]});
        return F_RAW;
    }
    const u32 dw = __builtin_amdgcn_readlane(mine, wave), start = __builtin_amdgcn_readlane(inc, wave) - dw;
    char* p = slot + 8192 + 16 + (start + lane) * 4u;
    if ((u32)lane < dw) st4_wt(p, w0);
    if ((u32)lane + 64u < dw) st4_wt(p + 256, w1);
    if (dw > 128u) {
        if ((u32)lane + 128u < dw) st4_wt(p + 512, *(const LAS u32*)(lpriv + 512 + lane * 4));
        if ((u32)lane + 192u < dw) st4_wt(p + 768, *(const LAS u32*)(lpriv + 768 + lane * 4));
    }
    if (wave == 0 && lane < 8 && !(lane & 1)) st4_wt(slot + 8192 + lane * 2, (inc - mine) | (inc << 16));
    return (16u + total * 4u + 15u) & ~15u;
}

template <bool OUT>
__device__ __forceinline__ void decode_block(u32 sl, u32 e0, u32 ltbl, int wave, int lane, u32x4& v0, u32x4& v1) {
    const u32x4 B = lds_ld16(sl + wave * 1024 + lane * 16);
    const u32 hdr = __builtin_amdgcn_readfirstlane(*(const LAS unsigned short*)(sl + 8192 + wave * 2));
    const u32 base = sl + 8192 + 16 + (hdr > 2040u ? 2040u : hdr) * 4u;
    const u32 eadd = e0 * 0x00800080u;
    const u32 bb[4] = {B.x, B.y, B.z, B.w};
    u32 F2, N2, W2[4];
    if (OUT) {
        u32 c = 0;
#pragma unroll
        for (int j = 0; j < 4; ++j) {
            W2[j] = ((((bb[j] >> 4) & 0x0f0f0f0fu) + 0x0d0d0d0du) >> 3) & 0x02020202u;
            c += __builtin_popcount(W2[j]);
        }
        const u32 n2 = 2u * c, s2 = wave_scan(n2);
        N2 = __builtin_amdgcn_readlane(s2, 63);
        F2 = lds_get_bits(base, s2 - n2) & low_mask_even(n2);
    } else {
        F2 = *(const LAS u32*)(base + lane * 4);
        N2 = 2048u;
    }
    const u32 W3 = F2 & (F2 << 1) & 0xaaaaaaaau;
    const u32 n3 = 2u * __builtin_popcount(W3), s3 = wave_scan(n3);
    const u32 N3 = __builtin_amdgcn_readlane(s3, 63);
    const u32 F3 = lds_get_bits(base, N2 + s3 - n3) & low_mask_even(n3);
    const u32 K4 = F3 & (F3 >> 1) & 0x55555555u, W4 = K4 * 3u;
    const u32 n4 = 3u * __builtin_popcount(K4), s4 = wave_scan(n4);
    const u32 N4 = __builtin_amdgcn_readlane(s4, 63);
    const u32 P4 = N2 + N3 + s4 - n4;
    const LAS u32* q4 = (const LAS u32*)(base + (P4 >> 5) * 4u);
    const u32 q40 = q4[0], q41 = q4[1], q42 = q4[2];
    u32 F4 = __builtin_amdgcn_alignbit(q41, q40, P4 & 31u);
    const u32 F4h = __builtin_amdgcn_alignbit(q42, q41, P4 & 31u);
    u32 t[16], p2 = 0, p3 = 0, p4 = 0;
    const u32 cm = OUT ? 0xf0u : 0xc0u;
    static_for<16>([&](auto ii) INL {
        constexpr int i = decltype(ii)::value;
        if (i == 8) { F4 = __builtin_amdgcn_alignbit(F4h, F4, p4); p4 = 0; }
        const u32 c1 = and_byte<i & 3>(bb[i >> 2], cm);
        u32 c2, w3;
        if (OUT) {
            const u32 w2 = W2[i >> 2] >> (8 * (i & 3)); c2 = ubfe(F2, p2, w2); w3 = ubfe(W3, p2, w2); p2 += w2;
        }
        else { c2 = ubfe(F2, 2 * i, 2); w3 = ubfe(W3, 2 * i, 2); }
        const u32 c3 = ubfe(F3, p3, w3), w4 = ubfe(W4, p3, w3); p3 += w3;
        const u32 c4 = ubfe(F4, p4, w4); p4 += w4;
        t[i] = OUT ? (c1 + c2 + c3 + c4) << 1 : c1 + ((c2 + c3 + c4) << 1);
    });
    u32 o[8];
#pragma unroll
    for (int j = 0; j < 8; ++j) {
        typedef unsigned short u16x2 __attribute__((ext_vector_type(2)));
        const u16x2 pr = {*(const LAS unsigned short*)(ltbl + t[2 * j]), *(const LAS unsigned short*)(ltbl + t[2 * j + 1])};
        o[j] = __builtin_bit_cast(u32, pr) + (OUT ? 0u : eadd);
    }
    const u32 any = (o[0] | o[1] | o[2]) | (o[3] | o[4] | o[5]) | (o[6] | o[7]);
    if (__builtin_amdgcn_ballot_w64((any & 0x00010001u) != 0u)) {
        u32 c = 0;
#pragma unroll
        for (int j = 0; j < 8; ++j) c += o[j] & 0x00010001u;
        const u32 ne = (c & 0xffffu) + (c >> 16);
        u32 a = base + (((N2 + N3 + N4 + 15u) & ~15u) >> 3) + (wave_scan(ne) - ne) * 2u;
        const u32 amax = sl + SLOT - 4u;
#pragma unroll
        for (int j = 0; j < 8; ++j) {
            if (o[j] & 0x00010001u) {
                a = a < amax ? a : amax;
                if (o[j] & 1u) { o[j] = (o[j] & 0xffff0000u) | *(const LAS unsigned short*)a; a += 2u; }
                if (o[j] & 0x10000u) { o[j] = (o[j] & 0xffffu) | ((u32)*(const LAS unsigned short*)a << 16); a += 2u; }
            }
        }
    }
#pragma unroll
    for (int j = 0; j < 8; ++j)
        o[j] |= __builtin_amdgcn_perm(0u, bb[j >> 1] & (OUT ? 0x0f0f0f0fu : 0x3f3f3f3fu), (j & 1) ? 0x0c030c02u : 0x0c010c00u);
    v0 = u32x4{o[0], o[1], o[2], o[3]};
    v1 = u32x4{o[4], o[5], o[6], o[7]};
}

__global__ void __launch_bounds__(THREADS, 1) fused_gemm_allgather(Params P) {
    const int lane = threadIdx.x & 63;
    const int wave = __builtin_amdgcn_readfirstlane(threadIdx.x >> 6);
    int b = blockIdx.x;
    if (MAP_ == 1) {
        const int x = b & 7, y = b >> 3;
        if (x >= 4 && y >= 28 && b != 4 + 28 * 8) return;
        b = y >= 28 && x >= 4 ? NROLE : x < 4 ? (y >> 1) * 8 + 2 * x + (y & 1) : NGEMM + (x - 4) * 28 + y;
    }
    if (MAP_ == 2) {
        const int x = b & 7, y = b >> 3;
        if ((x & 1) && y >= 28) return;
        b = !(x & 1) ? (y >> 1) * 8 + x + (y & 1) : NGEMM + (x >> 1) * 28 + y;
    }
    const int rank = P.rank;
    char* myws = P.myws;
    const char* px = P.x;
    const char* pw = P.w;
    asm volatile("" : "+s"(myws), "+s"(px), "+s"(pw));
    constexpr u32 lds0 = 0;
    GAS u32* const ctr = (GAS u32*)(myws + OFF_CTR + (size_t)b * 128);
    GAS u32* const tmo = (GAS u32*)(myws + OFF_TMO);

    if (b == NROLE) {
        if (wave) return;
        const u32 call = __builtin_amdgcn_readfirstlane(__hip_atomic_load(ctr, __ATOMIC_RELAXED, __HIP_MEMORY_SCOPE_AGENT)) + 1u;
        if (lane == 0) __hip_atomic_store(ctr, call, __ATOMIC_RELAXED, __HIP_MEMORY_SCOPE_AGENT);
        GAS u32* tk = (GAS u32*)(myws + OFF_TKT + (size_t)(lane < 7 ? lane : 0) * 256);
        u32 sum = 0, cnt = 0;
        const u64 t0 = __builtin_amdgcn_s_memrealtime();
        for (;;) {
            const u32 pos = __hip_atomic_load(tk, __ATOMIC_RELAXED, __HIP_MEMORY_SCOPE_AGENT) - (call - 1u) * TICKETS + 3 * PER_LINK;
            if (__any(pos >= 9 * PER_LINK)) break;
            if (__any(pos >= 4 * PER_LINK)) { sum += pos; ++cnt; }
            if (__builtin_amdgcn_s_memrealtime() - t0 > DEADLINE) { cnt = 0; break; }
            __builtin_amdgcn_s_sleep(8);
        }
        int eff = cnt ? -(int)((4u * sum) / cnt) : 0, mean = 0;
        for (int q = 0; q < 7; ++q) mean += __builtin_amdgcn_readlane(eff, q);
        eff -= mean / 7;
        const int kw = (eff + 3) / 4 + RMARG_;
        const u32 keep = eff < RTHR_ ? 0u : (kw > (int)RK ? RK : (u32)kw);
        u64 kv = (u64)(call & 0xfffffu) << 40;
        for (int q = 0; q < 7; ++q) kv |= (u64)(u32)__builtin_amdgcn_readlane(keep, q) << (5 * q);
        if (lane == 0) __hip_atomic_store((GAS u64*)(myws + OFF_KEEP), kv, __ATOMIC_RELAXED, __HIP_MEMORY_SCOPE_SYSTEM);
        STAMP(0);
        u64 pk = 0;
        for (;;) {
            if (lane < 7 && eff < 0) pk = __hip_atomic_load((GAS u64*)(P.ws[(rank + 1 + lane) & 7] + OFF_KEEP), __ATOMIC_RELAXED, __HIP_MEMORY_SCOPE_SYSTEM);
            if (!__any(lane < 7 && eff < 0 && (u32)(pk >> 40) != (call & 0xfffffu))) break;
            if (__builtin_amdgcn_s_memrealtime() - t0 > DEADLINE) { pk = 0; break; }
            __builtin_amdgcn_s_sleep(8);
        }
        u32 kq = 0, eq = 0, closed = 0;
        for (int i = 0; i < 7 * (int)RK; ++i) {
            int hi = -0x7fffffff, qa = 0;
            for (int q = 0; q < 7; ++q) {
                const int e = __builtin_amdgcn_readlane(eff, q);
                if (!((closed >> q) & 1u) && e > hi) { hi = e; qa = q; }
            }
            if (hi <= 0) break;
            const u32 j = __builtin_amdgcn_readlane(kq, qa);
            const u32 has = (u32)(pk >> (5 * ((qa - lane - 1) & 7))) & 31u;
            const u32 cm = (u32)__builtin_amdgcn_ballot_w64(lane < 7 && lane != qa && !kq && eq < REMAX && j < has);
            int lw = 0x7fffffff, qb = 0;
            for (int q = 0; q < 7; ++q) {
                const int e = __builtin_amdgcn_readlane(eff, q);
                if (((cm >> q) & 1u) && e < lw) { lw = e; qb = q; }
            }
            if (!cm || j >= RK || hi - lw < 4 * (1 + RHYST_)) { closed |= 1u << qa; continue; }
            const u32 eb = __builtin_amdgcn_readlane(eq, qb);
            u32 dn = (u32)(hi - lw) / 16u + 1u;
            dn = min(min(dn, 4u), min(min(RK, (u32)__builtin_amdgcn_readlane(has, qb)) - j, REMAX - eb));
            if ((u32)lane < dn)
                __hip_atomic_store((GAS unsigned char*)(myws + OFF_TKT + qb * 256 + 132 + eb + lane), (unsigned char)((qa << 5) | (j + lane)),
                                   __ATOMIC_RELAXED, __HIP_MEMORY_SCOPE_AGENT);
            closed |= 1u << qb;
            if (lane == qb) { eq += dn; eff += 4 * (int)dn; }
            if (lane == qa) { kq += dn; eff -= 4 * (int)dn; }
        }
        STAMP(1);
        vm_wait0();
        if (lane < 7)
            __hip_atomic_store(tk + 32, kq | (eq << 5) | (keep << 10) | (call << 15), __ATOMIC_RELAXED, __HIP_MEMORY_SCOPE_AGENT);
        return;
    }
    if (__builtin_expect(b < NGEMM, 1)) {
        const int tj = b & 7, ti0 = b >> 4, half = (b >> 3) & 1;
        const int npass = pack_passes(b);
        constexpr int EBAR = 3;
        if (wave == 8) {
            u32 cv;
            char* wsl;
            asm volatile("global_load_dword %0, %2, off sc1\n\ts_nop 3\n\tglobal_load_dwordx2 %1, %3, %4 offset:%5"
                         : "=&v"(cv), "=&v"(wsl)
                         : "v"(ctr), "v"((lane & 7) * 8), "s"(__builtin_amdgcn_kernarg_segment_ptr()), "n"(offsetof(Params, ws)) : "memory");
            bar();
            put_enc_table<false>(L_ETIN, lane);
            const u32 call = landed_first<0>(cv) + 1u;
            asm volatile("" : "+v"(wsl));
            bar();
            int isw, grp, lp, kb, seq;
            msg_of(b, isw, grp, lp, kb, seq);
            if (seq < 2 * PER_LINK && lane < NR && lane != rank && (isw ? ((lane & 3) == grp) : ((lane >> 2) == grp)))
                __hip_atomic_store((GAS u32*)(wsl + OFF_FLAG + ((size_t)rank * NMSG + seq) * 128 + 68), call, __ATOMIC_RELAXED, __HIP_MEMORY_SCOPE_SYSTEM);
            if (lane == 0) __hip_atomic_store(ctr, call, __ATOMIC_RELAXED, __HIP_MEMORY_SCOPE_AGENT);
            for (int z = 0; z < npass; ++z) {
                if (z + 1 < npass) {
                    msg_of(pack_msg(b, z + 1), isw, grp, lp, kb, seq);
                    const char* sb = isw ? pw + ((size_t)kb * 64 * 4096 + (grp * 8 + lp) * 128) * 2 : px + ((size_t)(grp * 16 + lp) * 128 * 512 + kb * 64) * 2;
                    const u32 vo = isw ? (lane >> 4) * 8192 + (lane & 15) * 16 : (lane & 15) * 1024 + (lane >> 4) * 16;
                    const u32 sw = isw ? 65536u : 16384u, sh = isw ? 32768u : 64u;
#pragma unroll
                    for (int j = 0; j < 16; ++j) dma1s(sb + (j >> 1) * sw + (j & 1) * sh, vo, lds0 + (z + 1) * IMG + j * 1024);
                }
                bar();
                vm_wait0();
                bar();
                msg_of(pack_msg(b, z), isw, grp, lp, kb, seq);
                const u32 lo = lds_ld(L_TOT + 32);
                if (seq < (int)NEARF) {
                    if (lane == 0)
                        __hip_atomic_store((GAS u64*)(myws + OFF_PKIN + (size_t)((isw ? 256 + (grp * 8 + lp) * 8 : (grp * 16 + lp) * 8) + kb) * GSLOT + SFLAG),
                                           ((u64)call << 32) | lo, __ATOMIC_RELAXED, __HIP_MEMORY_SCOPE_SYSTEM);
                } else {
                    announce(myws, wsl, rank, seq, ((u64)call << 32) | lo, (isw ? 0x11u << grp : 0xfu << (4 * grp)) & ~(1u << rank),
                             INBATCH ? (isw ? 32 + grp * 8 + lp : grp * 16 + lp) : -1, lane);
                }
                if (lane == 0 && (isw ? (grp == (rank & 3)) : (grp == (rank >> 2))))
                    __hip_atomic_fetch_add((GAS u32*)(myws + OFF_CNT + (size_t)((isw ? 16 : 0) + lp) * 256 + rank * 4), 1u,
                                           __ATOMIC_RELAXED, __HIP_MEMORY_SCOPE_AGENT);
                STAMP(1 + z);
            }
            const u32 want = call * 8u;
            u32 next_known = 0u;
            for (int tile = 0; tile < 2; ++tile) {
                const char* cp = myws + OFF_CNT + (size_t)((lane & 8) ? 16 + tj : ti0 + 8 * tile) * 256 + (lane & 7) * 4;
                if (tile == 0) {
                    wait_counters((GAS u32*)(myws + OFF_CNT + (size_t)((lane & 1) ? 16 + tj : ti0) * 256 + rank * 4), want, tmo, call);
                    bar(); bar();
                    put_enc_table<true>(L_ETOUT, lane);
                }
                u32 dummy = 0;
                const u32 pfs = (rank + 1 + (b >> 4)) & 7;
                bool pf = tile == 0 && (b >> 3) < 14;
                auto prefetch = [&](u32 wmask) INL {
                    if (pf && ((wmask >> pfs) & 1u)) {
                        const char* p = myws + OFF_UW + ((size_t)tj * 64 + pfs * 8 + half * 4) * IMG + lane * 128;
#pragma unroll
                        for (int j = 0; j < 8; ++j)
                            asm volatile("global_load_dword %0, %1, off sc1" : "+v"(dummy) : "v"(p + j * 8192) : "memory");
                        pf = false;
                    }
                };
                u32 known = next_known, wm8 = 0u, left = 0xffu & ~(1u << rank), order = 0u, cw = 0u;
                bool asked = false;
                auto look = [&]() INL {
                    const u32 mk = (u32)__builtin_amdgcn_ballot_w64(cw >= want);
                    wm8 = (mk >> 8) & 0xffu; known = mk & wm8;
                };
                for (int g = 1; g < 8; ++g) {
                    for (int i = 0; i < (g == 1 ? 8 - NST : 7); ++i) bar();
                    if (left) {
                        if (asked) { asm volatile("s_waitcnt vmcnt(0)" : "+v"(cw) :: "memory"); look(); }
                        if (!(known & left)) {
                            const u64 t0 = __builtin_amdgcn_s_memrealtime();
                            for (u32 it = 1;; ++it) {
                                asm volatile("global_load_dword %0, %1, off sc0 sc1\n\ts_waitcnt vmcnt(0)" : "=&v"(cw) : "v"(cp) : "memory");
                                look();
                                if (known & left) break;
                                prefetch(wm8);
                                if ((it & 15u) == 0u
                                    && (__builtin_amdgcn_readfirstlane(__hip_atomic_load(tmo, __ATOMIC_RELAXED, __HIP_MEMORY_SCOPE_AGENT)) == call
                                        || __builtin_amdgcn_s_memrealtime() - t0 > DEADLINE)) {
                                    __hip_atomic_store(tmo, call, __ATOMIC_RELAXED, __HIP_MEMORY_SCOPE_AGENT);
                                    known = 0xffu;
                                    break;
                                }
                                __builtin_amdgcn_s_sleep(2);
                            }
                        }
                        int gg = g;
                        do {
                            const u32 av = known & left, r2 = ((av | (av << 8)) >> tj) & 0xffu;
                            const u32 p = (__builtin_ctz(r2) + tj) & 7u;
                            order |= p << (4 * (gg - 1)); left &= ~(1u << p); ++gg;
                        } while (left && (known & left) == left);
                        lds_st(L_SRC + tile * 4, left ? order : order | 0x80000000u);
                    }
                    if (g == 1) STAMP(6 + 6 * tile);
                    bar();
                    if (tile == 1 && !left) break;
                    prefetch(wm8);
                    asked = left != 0u;
                    if (asked) asm volatile("global_load_dword %0, %1, off sc0 sc1" : "=v"(cw) : "v"(cp) : "memory");
                }
                if (tile == 0) {
                    const u32 ord = SCHED.ord[0][b];
                    for (int i = 0; i < 62 - (56 - NST) - 3; ++i) bar();
                    u32 pk;
                    asm volatile("global_load_dword %0, %1, off sc0 sc1" : "=v"(pk)
                                 : "v"(myws + OFF_CNT + (size_t)(ti0 + 8) * 256 + (lane & 7) * 4) : "memory");
                    for (int i = 0; i < 4; ++i) bar();
                    asm volatile("s_waitcnt vmcnt(0)" : "+v"(pk), "+v"(dummy), "+v"(cw) :: "memory");
                    next_known = (u32)__builtin_amdgcn_ballot_w64(pk >= want) & 0xffu;
                    if (!((next_known >> rank) & 1u))
                        wait_counters((GAS u32*)(myws + OFF_CNT + (size_t)(ti0 + 8) * 256 + rank * 4), want, tmo, call);
                    for (int i = 0; i < EBAR; ++i) bar();
                    const u32 lo = lds_ld(L_TOT + 32);
                    announce(myws, wsl, rank, OUT0 + ord, ((u64)call << 32) | lo, 0xffu & ~(1u << rank), ord >= OBATCH ? 64 + (int)(ord >> 3) : -1, lane);
                }
            }
            return;
        }
        if (wave == 0) STAMP(0);
        u32 cv;
        char* wsl;
        asm volatile("global_load_dword %0, %2, off sc1\n\ts_nop 3\n\tglobal_load_dwordx2 %1, %3, %4 offset:%5"
                     : "=&v"(cv), "=&v"(wsl)
                     : "v"(ctr), "v"((lane & 7) * 8), "s"(__builtin_amdgcn_kernarg_segment_ptr()), "n"(offsetof(Params, ws)) : "memory");
        auto fetch = [&](int z, int rho) INL {
            int isw, grp, lp, kb, seq;
            msg_of(rho, isw, grp, lp, kb, seq);
            const u32 dst = lds0 + z * IMG + wave * 2048;
            if (!isw) {
                const char* s = px + ((size_t)((grp * 16 + lp) * 128 + wave * 16 + (lane & 15)) * 512 + kb * 64 + (lane >> 4) * 8) * 2;
                dma1(s, dst); dma1(s + 64, dst + 1024);
            } else {
                const char* s = pw + ((size_t)(kb * 64 + wave * 8 + (lane >> 4)) * 4096 + (grp * 8 + lp) * 128 + (lane & 15) * 8) * 2;
                dma1(s, dst); dma1(s + 4 * 8192, dst + 1024);
            }
        };
        [[maybe_unused]] u64 tq[3];
        fetch(0, b);
        const u32 call = landed_first<0>(cv) + 1u;
        int isw, grp, lp, kb, seq;
        char* slot;
        u32 d[8];
        u32x2 sample;
        auto take = [&](int z) INL {
            msg_of(pack_msg(b, z), isw, grp, lp, kb, seq);
            const u32 src = lds0 + z * IMG;
            if (!isw) {
                const u32x4 v0 = lds_ld16(src + wave * 2048 + lane * 16), v1 = lds_ld16(src + wave * 2048 + 1024 + lane * 16);
                d[0] = v0.x; d[1] = v0.y; d[2] = v0.z; d[3] = v0.w; d[4] = v1.x; d[5] = v1.y; d[6] = v1.z; d[7] = v1.w;
            } else {
                const u32 a = src + ((lane >> 4) * 8 + ((lane & 15) >> 2)) * 256 + (32 * (wave >> 1) + 8 * (lane & 3) + 4 * (wave & 1)) * 2;
                const u32x2 a0 = __builtin_bit_cast(u32x2, __builtin_amdgcn_ds_read_tr16_b64_v4i16((LAS s16x4*)(a)));
                const u32x2 a1 = __builtin_bit_cast(u32x2, __builtin_amdgcn_ds_read_tr16_b64_v4i16((LAS s16x4*)(a + 4 * 256)));
                const u32x2 a2 = __builtin_bit_cast(u32x2, __builtin_amdgcn_ds_read_tr16_b64_v4i16((LAS s16x4*)(a + 32 * 256)));
                const u32x2 a3 = __builtin_bit_cast(u32x2, __builtin_amdgcn_ds_read_tr16_b64_v4i16((LAS s16x4*)(a + 36 * 256)));
                d[0] = a0.x; d[1] = a0.y; d[2] = a1.x; d[3] = a1.y; d[4] = a2.x; d[5] = a2.y; d[6] = a3.x; d[7] = a3.y;
            }
            sample = *(const LAS u32x2*)(src + lane * 248);
            slot = myws + OFF_PKIN + (size_t)((isw ? 256 + (grp * 8 + lp) * 8 : (grp * 16 + lp) * 8) + kb) * GSLOT;
            store_input_bytes(slot, d, wave, lane);
        };
        [[maybe_unused]] u64 ts[4], tsave[4] = {};
        bar();
        TS(0);
        take(0);
        asm volatile("s_waitcnt vmcnt(0)" : "+v"(wsl) :: "memory");
        bar();
        for (int z = 0;;) {
            TS(1);
            const u32 e0 = input_e0(sample);
            encode_block<false>(slot, d, e0, L_ETIN, L_VAR + wave * 1024, L_TOT, wave, lane);
            bar();
            TS(2);
            const u32 lo = encode_finish(slot, d, L_VAR + wave * 1024, L_TOT, wave, lane) | (e0 << 16);
            TS(3);
            const bool own = isw ? (grp == (rank & 3)) : (grp == (rank >> 2));
            if (own) {
                char* img = myws + (isw ? OFF_UW : OFF_UX) + (size_t)(lp * 64 + rank * 8 + kb) * IMG + wave * 2048 + lane * 16;
                st16_wt(img, u32x4{d[0], d[1], d[2], d[3]}); st16_wt(img + 1024, u32x4{d[4], d[5], d[6], d[7]});
            }
            if (threadIdx.x == 0) lds_st(L_TOT + 32, lo);
            vm_wait0();
            bar();
            if (++z == npass) break;
            TS(0);
            take(z);
        }
        u32 ords = __builtin_amdgcn_readfirstlane((u32)SCHED.ord[0][b] | ((u32)SCHED.ord[1][b] << 8));
        asm volatile("s_waitcnt vmcnt(0)" : "+s"(ords) :: "memory");
        const int kh = wave >> 2, wn = wave & 1, wm = (wave >> 1) & 1;
        const char* const wb = myws + OFF_UW + (size_t)tj * 64 * IMG;
        const u32 voffx = wave * 1024 + lane * 16, voffw = wave * 2048 + lane * 16;
        const u32 xa = lds0 + (wm * 4 + kh) * 1024 + lane * 16;
        const u32 wa = lds0 + 8192 + (wn * 8 + kh) * 1024 + lane * 16;
#define SB __builtin_amdgcn_sched_barrier(0)
        u32 lo_last = 0;
        for (int tile = 0; tile < 2; ++tile) {
            const int ti = ti0 + 8 * tile;
            const char* const xb = myws + OFF_UX + (size_t)ti * 64 * IMG + half * 8192;
            f32x4 acc[4][2];
#pragma unroll
            for (int a = 0; a < 4; ++a) { acc[a][0] = f32x4{0.f, 0.f, 0.f, 0.f}; acc[a][1] = acc[a][0]; }
            bf16x8 wA[4], xA[2], wB[4], xB[2];
            auto mma = [&](int a, const bf16x8* wf, const bf16x8* xf) INL {
                acc[a][0] = __builtin_amdgcn_mfma_f32_16x16x32_bf16(wf[a], xf[0], acc[a][0], 0, 0, 0);
                acc[a][1] = __builtin_amdgcn_mfma_f32_16x16x32_bf16(wf[a], xf[1], acc[a][1], 0, 0, 0);
            };
            u32 buf = 0;
            u32 srcs = 0;
            auto step = [&](int kt, auto dma_tag, auto vm_tag, const bf16x8* wc, const bf16x8* xc, bf16x8* wx, bf16x8* xx) INL {
                constexpr bool DMA = decltype(dma_tag)::value;
                constexpr int VM = decltype(vm_tag)::value;
                const u32 d = buf;
                const u32 nb = buf + STG == NST * STG ? 0u : buf + STG;
                const u32 ra = xa + nb, rb = wa + nb;
                buf = nb;
                asm volatile("s_waitcnt lgkmcnt(0)\n\ts_waitcnt vmcnt(%0)\n\ts_barrier" :: "n"(VM) : "memory"); SB;
                const u32 st = kt + NST;
                if (DMA && (st & 7u) == 0u && (int)srcs >= 0) srcs = __builtin_amdgcn_readfirstlane(lds_ld(L_SRC + tile * 4));
                const u32 kb = (st < 8u ? (u32)rank : (srcs >> (4u * ((st >> 3) - 1u))) & 7u) * 8u + (st & 7u);
                const char* gx = xb + (size_t)kb * IMG; const char* gw = wb + (size_t)kb * IMG;
                SB;
                if (DMA) dma_a(gx, voffx, d + wave * 1024); SB;
                mma(0, wc, xc); SB;
                xx[0] = *(const LAS bf16x8*)(ra); wx[0] = *(const LAS bf16x8*)(rb); SB;
                mma(1, wc, xc); SB;
                if (DMA) dma_a(gw, voffw, d + 8192 + wave * 2048); SB;
                xx[1] = *(const LAS bf16x8*)(ra + 2048); wx[1] = *(const LAS bf16x8*)(rb + 2048); SB;
                mma(2, wc, xc); SB;
                wx[2] = *(const LAS bf16x8*)(rb + 4096); wx[3] = *(const LAS bf16x8*)(rb + 6144); SB;
                if (DMA) dma_b(gw, voffw); SB;
                mma(3, wc, xc); SB;
            };
            using T_ = std::true_type; using F_ = std::false_type;
            auto fill = [&](const char* xp, int s0, int s1) INL {
                for (int st = s0; st < s1; ++st) {
                    const u32 kb = (u32)(rank * 8 + st);
                    dma_a(xp + (size_t)kb * IMG, voffx, st * STG + wave * 1024);
                    dma_a(wb + (size_t)kb * IMG, voffw, st * STG + 8192 + wave * 2048);
                    dma_b(wb + (size_t)kb * IMG, voffw);
                }
            };
            if (tile == 0) {
                bar();
                fill(xb, 0, NST);
                asm volatile("s_waitcnt vmcnt(%0)\n\ts_barrier" :: "n"((NST - 1) * 3) : "memory");
            } else {
                fill(xb, 4, NST);
            }
#pragma unroll
            for (int a = 0; a < 4; ++a) wA[a] = *(const LAS bf16x8*)(wa + a * 2048);
            xA[0] = *(const LAS bf16x8*)(xa); xA[1] = *(const LAS bf16x8*)(xa + 2048);
            static_assert(NST == 6, "tail below");
            using VMS = std::integral_constant<int, (NST - 2) * 3>;
            for (int kt = 0; kt < 64 - NST; kt += 2) {
                step(kt, T_{}, VMS{}, wA, xA, wB, xB);
                step(kt + 1, T_{}, VMS{}, wB, xB, wA, xA);
            }
            step(58, F_{}, std::integral_constant<int, 12>{}, wA, xA, wB, xB);
            step(59, F_{}, std::integral_constant<int, 9>{}, wB, xB, wA, xA);
            step(60, F_{}, std::integral_constant<int, 6>{}, wA, xA, wB, xB);
            step(61, F_{}, std::integral_constant<int, 3>{}, wB, xB, wA, xA);
            step(62, F_{}, std::integral_constant<int, 0>{}, wA, xA, wB, xB);
            u32 tv;
            asm volatile("global_load_dword %0, %1, off sc1" : "=v"(tv) : "v"(tmo) : "memory");
            bar();
            mma(0, wB, xB); mma(1, wB, xB); mma(2, wB, xB); mma(3, wB, xB);
            if (wave == 0) STAMP(7 + 6 * tile);
            {
                const u32 wr = L_XCH + wave * 4096 + lane * 16;
#pragma unroll
                for (int a = 0; a < 4; ++a) *(LAS f32x4*)(wr + a * 1024) = kh ? acc[a][0] : acc[a][1];
            }
            bar();
            {
                const u32 rd = L_XCH + (wave ^ 4) * 4096 + lane * 16;
#pragma unroll
                for (int a = 0; a < 4; ++a) {
                    const f32x4 y = (kh ? acc[a][1] : acc[a][0]) + *(const LAS f32x4*)(rd + a * 1024);
                    d[a * 2] = pk_bf16(gelu(y.x), gelu(y.y));
                    d[a * 2 + 1] = pk_bf16(gelu(y.z), gelu(y.w));
                }
            }
            if (landed_first<0>(tv) == call) {
#pragma unroll
                for (int j = 0; j < 8; ++j) d[j] = 0x7fc07fc0u;
            }
            if (tile == 0) fill(xb + (size_t)8 * 64 * IMG, 0, 4);
            if (wave == 0) STAMP(8 + 6 * tile);
            const u32 ord = (ords >> (8 * tile)) & 0xffu;
            char* const oslot = myws + OFF_PKOUT + (size_t)ord * GSLOT;
            encode_block<true>(oslot, d, 0u, L_ETOUT, L_VAR + wave * 1024, L_TOT, wave, lane);
            char* o = P.out + ((size_t)((rank >> 2) * 2048 + ti * 128 + half * 64 + wm * 32 + kh * 16 + (lane & 15)) * 4096
                               + (rank & 3) * 1024 + tj * 128 + wn * 64 + (lane >> 4) * 8) * 2;
            st16_out(o, u32x4{d[0], d[1], d[2], d[3]});
            st16_out(o + 64, u32x4{d[4], d[5], d[6], d[7]});
            bar();
            const u32 lo = encode_finish(oslot, d, L_VAR + wave * 1024, L_TOT, wave, lane);
            if (wave == 0) STAMP(9 + 6 * tile);
            if (threadIdx.x == 0) lds_st(L_TOT + 32, lo);
            vm_wait0();
            bar();
            if (wave == 0) STAMP(10 + 6 * tile);
            lo_last = lo;
        }
        if (wave == 0) {
            const u32 ord = ords >> 8;
            announce(myws, wsl, rank, OUT0 + ord, ((u64)call << 32) | lo_last, 0xffu & ~(1u << rank), ord >= OBATCH ? 64 + (int)(ord >> 3) : -1, lane);
        }
    } else {
        const int pu = b - NGEMM;
        const int u = pu / 7;
        const int src = (rank + 1 + pu % 7) & 7;
        const u32 call = __builtin_amdgcn_readfirstlane(__hip_atomic_load(ctr, __ATOMIC_RELAXED, __HIP_MEMORY_SCOPE_AGENT)) + 1u;
        if (wave == 0) lds_st(lds0 + L_CTL + lane * 4, 0u);
        if (wave == 1) put_dec_table<false>(L_DTBL, lane);
        if (wave == 8) STAMP(0);
        vm_wait0();
        bar();
        if (threadIdx.x == 0) __hip_atomic_store(ctr, call, __ATOMIC_RELAXED, __HIP_MEMORY_SCOPE_AGENT);
        if (wave == 8) {
            __builtin_amdgcn_s_setprio(3);
            vm_wait0();
            const char* const rws = P.ws[src];
            const char* const fbase = myws + OFF_FLAG + (size_t)src * NMSG * 128;
            char* const tkt = myws + OFF_TKT + (size_t)(pu % 7) * 256;
            const u32 tbase = (call - 1u) * TICKETS - 3 * PER_LINK;
            u32 m = u, n = 0, np = 0;
            u32 org = src, cpy = 0u;
            u32 plan = 0u;
            u32 fl_lo = 0u, fl_hi = 0u;
            u32 pub = 0u;
            bool dead = false;
            [[maybe_unused]] u32 nst = 0;
            auto publish = [&](u32 k) INL {
                if (np > pub + k) { pub = np - k; lds_st(lds0 + L_FULL, pub); }
            };
            auto drain = [&](auto top) INL {
                constexpr int TOP = decltype(top)::value;
                static_for<TOP + 1>([&](auto i) INL {
                    constexpr int K = TOP - decltype(i)::value;
                    asm volatile("s_waitcnt vmcnt(%0)" :: "n"(K) : "memory");
                    publish(K);
                });
            };
            auto poll = [&](const char* fp) INL {
                const u64 t0 = __builtin_amdgcn_s_memrealtime();
                for (u32 it = 1;; ++it) {
                    u32x2 fl;
                    asm volatile("global_load_dwordx2 %0, %1, off sc0 sc1\n\ts_waitcnt vmcnt(0)" : "=&v"(fl) : "v"(fp) : "memory");
                    fl_lo = __builtin_amdgcn_readfirstlane(fl.x); fl_hi = __builtin_amdgcn_readfirstlane(fl.y);
                    if (fl_hi >= call) break;
                    if ((it & 15u) == 0u
                        && (__builtin_amdgcn_readfirstlane(__hip_atomic_load(tmo, __ATOMIC_RELAXED, __HIP_MEMORY_SCOPE_AGENT)) == call
                            || __builtin_amdgcn_s_memrealtime() - t0 > DEADLINE)) {
                        __hip_atomic_store(tmo, call, __ATOMIC_RELAXED, __HIP_MEMORY_SCOPE_AGENT);
                        dead = true;
                        break;
                    }
                    __builtin_amdgcn_s_sleep(1);
                }
            };
            auto remote_slot = [&](u32 mm) INL {
                const u32 t = mm / PER_LINK, v = mm % PER_LINK;
                size_t moff;
                if (t < IN_ROUNDS) {
                    const u32 kb = v & 7;
                    u32 mid;
                    if (t < 8) mid = v >= 8 ? ((rank >> 2) * 16 + t) * 8 + kb : 256 + ((rank & 3) * 8 + t) * 8 + kb;
                    else mid = ((rank >> 2) * 16 + 8 + 2 * (t - 8) + (v >> 3)) * 8 + kb;
                    moff = OFF_PKIN + (size_t)mid * GSLOT;
                } else if (cpy) {
                    moff = OFF_RLY + (size_t)(((org - src - 1u) & 7u) * RN + mm - RW0) * GSLOT;
                } else {
                    moff = OFF_PKOUT + (size_t)(mm - OUT0) * GSLOT;
                }
                return rws + moff;
            };
            auto flag_of = [&](u32 mm) INL { return mm < NEARF || cpy ? remote_slot(mm) + SFLAG : fbase + (mm & ~7u) * 128 + (mm & 7u) * 8; };
            auto resolve = [&](u32 pos) INL {
                m = pos;
                if (pos < RW0) return;
                if (!plan) {
                    plan = __builtin_amdgcn_readfirstlane(lds_ld(lds0 + L_PLAN + 32));
                    if (!plan) {
                        const u64 t0 = __builtin_amdgcn_s_memrealtime();
                        while (!(plan = fetch_plan(tkt, call, lane))) {
                            if (__builtin_amdgcn_s_memrealtime() - t0 > DEADLINE) {
                                __hip_atomic_store(tmo, call, __ATOMIC_RELAXED, __HIP_MEMORY_SCOPE_AGENT);
                                dead = true; plan = 0x8000u;
                                break;
                            }
                            __builtin_amdgcn_s_sleep(2);
                        }
                    }
                }
                const u32 kk = plan & 31u, ee = (plan >> 5) & 31u, x = pos - RW0, cls = 6u - pu % 7;
                m = x < 6u * kk ? RW0 + x / 6u * 7u + x % 6u + (x % 6u >= cls) : pos + kk;
                if (m >= NMSG + ee) {
                    m -= ee;
                    if (m == NMSG + PER_LINK - 1 && lane == 0)
                        __hip_atomic_fetch_add((GAS u32*)tkt, REMAX + 1 + kk - ee, __ATOMIC_RELAXED, __HIP_MEMORY_SCOPE_AGENT);
                } else if (m >= NMSG) {
                    const u32 z = m - NMSG, w = (__builtin_amdgcn_readfirstlane(lds_ld(lds0 + L_PLAN + (z & ~3u))) >> (8u * (z & 3u))) & 0xffu;
                    const u32 ql = (w >> 5) < 7u ? w >> 5 : 0u, j = (w & 31u) < RK ? w & 31u : 0u;
                    org = (rank + 1u + ql) & 7u; cpy = 1u;
                    m = RW0 + 7u * j + 6u - ql;
                }
            };
            u32 f2_lo = 0u, f2_hi = 0u, f3_lo = 0u, f3_hi = 0u;
            LSTAMP(L_STMP, nst, b, 59);
            poll(fbase + m * 128 + 64);
            LSTAMP(L_STMP, nst, b, 60);
            fl_hi = 0u;
            if (!dead) {
                u32 e2;
                asm volatile("global_load_dword %0, %1, off offset:%2 sc0 sc1" : "=v"(e2) : "v"(fbase + m * 128), "n"(PER_LINK * 128 + 68) : "memory");
                const char* sp = remote_slot(m) + lane * 16;
#pragma unroll
                for (int k = 0; k < 8; ++k) {
                    dma1_sys(sp + k * 1024, lds0 + L_RING + k * 1024);
                    asm volatile("s_waitcnt vmcnt(%0)" :: "n"(WIN - 1) : "memory");
                }
                np = 8u;
                if (landed_first<WIN - 1>(e2) < call) { poll(fbase + (m + PER_LINK) * 128 + 64); fl_hi = 0u; }
            }
            if (!dead) {
                u32x2 fa, fb;
                asm volatile("global_load_dwordx2 %0, %2, off sc0 sc1\n\tglobal_load_dwordx2 %1, %3, off sc0 sc1"
                             : "=&v"(fa), "=&v"(fb) : "v"(flag_of(m)), "v"(flag_of(m + PER_LINK)) : "memory");
                const char* sp = remote_slot(m + PER_LINK) + lane * 16;
#pragma unroll
                for (int k = 0; k < WIN - 1; ++k) {
                    dma1_sys(sp + k * 1024, lds0 + L_RING + SLOT + k * 1024);
                    if (k < WIN - 2) asm volatile("s_waitcnt vmcnt(%0)" :: "n"(WIN + 1) : "memory");
                }
                fl_lo = landed_first<WIN - 1>(fa.x); fl_hi = landed_first<WIN - 1>(fa.y);
                f2_lo = landed_first<WIN - 1>(fb.x); f2_hi = landed_first<WIN - 1>(fb.y);
                np += WIN - 1;
                LSTAMP(L_STMP, nst, b, 61);
            }
            for (;; ++n) {
                const u32 s = n % NSLOT;
                if (m < NMSG && !dead && fl_hi < call) {
                    drain(std::integral_constant<int, WIN - 1>{});
                    poll(flag_of(m));
                }
                if (n < 2) LSTAMP(L_STMP, nst, b, 62 + n);
                if (n >= NSLOT) {
                    const u32 need = 8u * (n / NSLOT);
                    if (__builtin_amdgcn_readfirstlane(lds_ld(lds0 + L_FREE + s * 4)) < need) {
                        drain(std::integral_constant<int, WIN - 1>{});
                        while (__builtin_amdgcn_readfirstlane(lds_ld(lds0 + L_FREE + s * 4)) < need) __builtin_amdgcn_s_sleep(1);
                    }
                }
                if (m >= NMSG) {
                    lds_st(lds0 + L_MSG + s * 4, NMSG);
                    lds_st(lds0 + L_DESC, n + 1);
                    break;
                }
                const u32 lo = dead ? F_BAD : fl_lo;
                lds_st(lds0 + L_LEN + s * 4, lo);
                const u32 kc = !cpy && !dead && m - RW0 < 7u * ((plan >> 10) & 31u) && (m - RW0) % 7u != 6u - pu % 7;
                lds_st(lds0 + L_MSG + s * 4, m | (org << 9) | (kc << 12) | (np << 16));
                lds_st(lds0 + L_DESC, n + 1);
                const u32 vb = var_bytes(lo), nv = (vb + 1023u) >> 10;
                [[maybe_unused]] const u32 t = m / PER_LINK, v = m % PER_LINK;
                const char* sp = remote_slot(m) + ((lo & F_RAW) ? RAW_OFF : 0u) + lane * 16;
                if (n < 2) {
                    if (!dead) {
                        if (lo & F_RAW) vm_wait0();
                        for (u32 o = (lo & F_RAW) ? 0u : 8192u; o < 8192u + vb; o += 1024u) {
                            if (o + lane * 16 < 8192u + vb) dma1_sys(sp + o, lds0 + L_RING + n * SLOT + o);
                            ++np;
                            asm volatile("s_waitcnt vmcnt(%0)" :: "n"(WIN - 1) : "memory");
                            publish(WIN - 1);
                        }
                        if (n == 0) {
                            u32x2 fc;
                            asm volatile("global_load_dwordx2 %0, %1, off sc0 sc1" : "=v"(fc) : "v"(flag_of(m + 2 * PER_LINK)) : "memory");
                            sp = remote_slot(m + PER_LINK) + lane * 16;
#pragma unroll
                            for (int k = WIN - 1; k < 8; ++k) {
                                dma1_sys(sp + k * 1024, lds0 + L_RING + SLOT + k * 1024);
                                ++np;
                                asm volatile("s_waitcnt vmcnt(%0)" :: "n"(WIN - 1) : "memory");
                                publish(WIN - 1);
                            }
                            f3_lo = landed_first<WIN - 1>(fc.x); f3_hi = landed_first<WIN - 1>(fc.y);
                        }
                    }
                    m += PER_LINK; fl_lo = f2_lo; fl_hi = f2_hi; f2_lo = f3_lo; f2_hi = f3_hi;
                    LSTAMP(L_STMP, nst, NGEMM + v * 7 + pu % 7, 8 + t);
                    continue;
                }
                u32 nt = 1u;
                u64 keep;
                asm volatile("s_mov_b64 %1, exec\n\ts_mov_b64 exec, 1\n\tglobal_atomic_add %0, %2, %0, off sc0\n\ts_mov_b64 exec, %1"
                             : "+v"(nt), "=&s"(keep) : "v"(tkt) : "memory");
                if (dead) {
                    resolve(landed_first<0>(nt) - tbase);
                    publish(0);
                    continue;
                }
                const u32 d = lds0 + L_RING + s * SLOT;
                [[maybe_unused]] const u32 was_cpy = cpy;
                asm volatile("s_waitcnt vmcnt(%0)" :: "n"(WIN) : "memory");
                publish(WIN - 1);
                {
                    const u32 o = 8192u + (nv - 1u) * 1024u;
                    if (o + lane * 16 < 8192u + vb) dma1_sys(sp + o, d + o);
                    ++np;
                }
                u32 left = nv + 7u, o = nv > 1u ? 8192u + (nv - 2u) * 1024u : 0u;
                auto piece = [&]() INL {
                    dma1_sys(sp + o, d + o);
                    o = o > 8192u ? o - 1024u : (o == 8192u ? 0u : o + 1024u);
                    ++np; --left;
                };
#pragma unroll
                for (int k = 0; k < WIN - 1; ++k) {
                    asm volatile("s_waitcnt vmcnt(%0)" :: "n"(WIN) : "memory");
                    publish(WIN - 1);
                    piece();
                }
                const u32 pos = landed_first<WIN - 1>(nt) - tbase;
                publish(WIN - 1);
                piece();
                resolve(pos);
                if (m >= NMSG + ENDK) {
                    while (left) {
                        asm volatile("s_waitcnt vmcnt(%0)" :: "n"(WIN2 - 1) : "memory");
                        publish(WIN2 - 1);
                        piece();
                    }
                    drain(std::integral_constant<int, WIN2 - 1>{});
                } else if (m >= NMSG) {
                    while (left) {
                        asm volatile("s_waitcnt vmcnt(%0)" :: "n"(WIN - 1) : "memory");
                        publish(WIN - 1);
                        piece();
                    }
                    drain(std::integral_constant<int, WIN - 1>{});
                } else {
                    u32x2 fn;
                    asm volatile("global_load_dwordx2 %0, %1, off sc0 sc1" : "=v"(fn) : "v"(flag_of(m)) : "memory");
#pragma unroll
                    for (int k = 0; k < WIN - 1; ++k) {
                        asm volatile("s_waitcnt vmcnt(%0)" :: "n"(WIN) : "memory");
                        publish(WIN - 1);
                        piece();
                    }
                    fl_lo = landed_first<WIN - 1>(fn.x); fl_hi = landed_first<WIN - 1>(fn.y);
                    publish(WIN - 1);
                    piece();
                    while (left) {
                        asm volatile("s_waitcnt vmcnt(%0)" :: "n"(WIN - 1) : "memory");
                        publish(WIN - 1);
                        piece();
                    }
                }
                if (!was_cpy) LSTAMP(L_STMP, nst, NGEMM + v * 7 + pu % 7, 8 + t);
            }
            STAMP(1);
            LFLUSH(L_STMP, nst);
        } else {
            if (wave == 2) {
                put_dec_table<true>(L_DTBL + 256, lane);
                *(LAS u32*)(L_UNORD + lane * 4) = ((const u32*)SCHED.unord)[lane];
                lgkm_wait0();
                lds_st(lds0 + L_TRDY, 1u);
            }
            [[maybe_unused]] u32 nst = 0;
            bool tables = false, plan = false;
            for (u32 n = 0;; ++n) {
                const u32 s = n % NSLOT;
                while (__builtin_amdgcn_readfirstlane(lds_ld(lds0 + L_DESC)) <= n) __builtin_amdgcn_s_sleep(1);
                const u32 md = __builtin_amdgcn_readfirstlane(lds_ld(lds0 + L_MSG + s * 4)), m = md & 0x1ffu, org = (md >> 9) & 7u;
                if (m >= NMSG) break;
                if (wave == 1 && m >= 10 * PER_LINK && !plan) plan = fetch_plan(myws + OFF_TKT + (size_t)(pu % 7) * 256, call, lane);
                const u32 t = m / PER_LINK, v = m % PER_LINK;
                const u32 lo = __builtin_amdgcn_readfirstlane(lds_ld(lds0 + L_LEN + s * 4));
                const u32 sl = lds0 + L_RING + s * SLOT;
                if (m >= OUT0 && !tables) {
                    while (!__builtin_amdgcn_readfirstlane(lds_ld(lds0 + L_TRDY))) __builtin_amdgcn_s_sleep(1);
                    tables = true;
                }
                if (!(lo & F_BAD)) {
                    const u32 nv = (var_bytes(lo) + 1023u) >> 10;
                    const u32 need = (md >> 16) + (n >= 2 ? nv + wave + 1u : ((lo & F_RAW) ? 16u : nv));
                    while (__builtin_amdgcn_readfirstlane(lds_ld(lds0 + L_FULL)) < need) __builtin_amdgcn_s_sleep(1);
                }
                if (t == 3 && wave == 7) LSTAMP(L_STMP + 1024 + wave * 512, nst, NGEMM + v * 7 + pu % 7, 56);
                const bool keep = md & 0x1000u;
                char* const cslot = myws + OFF_RLY + (size_t)((pu % 7) * RN + m - RW0) * GSLOT;
                if (keep) {
                    char* p = cslot + ((lo & F_RAW) ? RAW_OFF : 0u) + wave * 1024 + lane * 16;
                    st16_wt(p, lds_ld16(sl + wave * 1024 + lane * 16));
                    if ((u32)(wave * 1024 + lane * 16) < var_bytes(lo)) st16_wt(p + 8192, lds_ld16(sl + 8192 + wave * 1024 + lane * 16));
                }
                u32x4 v0, v1;
                if (lo & F_BAD) {
                    v0 = u32x4{0x7fc07fc0u, 0x7fc07fc0u, 0x7fc07fc0u, 0x7fc07fc0u}; v1 = v0;
                } else if (lo & F_RAW) {
                    v0 = lds_ld16(sl + 8192 + wave * 1024 + lane * 16);
                    v1 = lds_ld16(sl + wave * 1024 + lane * 16);
                } else if (t < IN_ROUNDS) {
                    decode_block<false>(sl, (lo >> 16) & 0xffu, L_DTBL, wave, lane, v0, v1);
                } else {
                    decode_block<true>(sl, 0u, L_DTBL + 256, wave, lane, v0, v1);
                }
                asm volatile("s_waitcnt lgkmcnt(0)" : "+v"(v0), "+v"(v1) :: "memory");
                if (lane == 0) asm volatile("ds_add_u32 %0, %1" :: "v"(lds0 + L_FREE + s * 4), "v"(1u) : "memory");
                if (t == 3 && wave == 7) LSTAMP(L_STMP + 1024 + wave * 512, nst, NGEMM + v * 7 + pu % 7, 57);
                if (t < IN_ROUNDS) {
                    const u32 kb = v & 7;
                    const bool isw = t < 8 && v < 8;
                    const u32 lp = t < 8 ? t : 8 + 2 * (t - 8) + (v >> 3);
                    char* img = myws + (isw ? OFF_UW : OFF_UX) + (size_t)(lp * 64 + src * 8 + kb) * IMG + wave * 2048 + lane * 16;
                    st16_wt(img, v0); st16_wt(img + 1024, v1);
                    vm_wait0();
                    u32 old = 0;
                    if (lane == 0) {
                        asm volatile("ds_add_rtn_u32 %0, %1, %2\n\ts_waitcnt lgkmcnt(0)" : "=&v"(old) : "v"(lds0 + L_DONE + s * 4), "v"(1u) : "memory");
                        if ((old & 7u) == 7u)
                            __hip_atomic_fetch_add((GAS u32*)(myws + OFF_CNT + (size_t)((isw ? 16 : 0) + lp) * 256 + src * 4), 1u,
                                                   __ATOMIC_RELAXED, __HIP_MEMORY_SCOPE_AGENT);
                    }
                } else {
                    const int g = __builtin_amdgcn_readfirstlane(*(const LAS unsigned char*)(L_UNORD + m - OUT0));
                    const int kh = wave >> 2, wn = wave & 1, wm = (wave >> 1) & 1;
                    const int ti = ((g >> 4) & 7) + 8 * (g >> 7), hf = (g >> 3) & 1, tj = g & 7;
                    if (keep) {
                        vm_wait0();
                        if (lane == 0) {
                            u32 old;
                            asm volatile("ds_add_rtn_u32 %0, %1, %2\n\ts_waitcnt lgkmcnt(0)" : "=&v"(old) : "v"(lds0 + L_DONE + s * 4), "v"(1u) : "memory");
                            if ((old & 7u) == 7u)
                                __hip_atomic_store((GAS u64*)(cslot + SFLAG), ((u64)call << 32) | lo, __ATOMIC_RELAXED, __HIP_MEMORY_SCOPE_SYSTEM);
                        }
                    }
                    char* op = P.out + ((size_t)((org >> 2) * 2048 + ti * 128 + hf * 64 + wm * 32 + kh * 16 + (lane & 15)) * 4096
                                        + (org & 3) * 1024 + tj * 128 + wn * 64 + (lane >> 4) * 8) * 2;
                    st16_out(op, v0); st16_out(op + 64, v1);
                }
            }
            STAMP(2);
            LFLUSH(L_STMP + 1024 + wave * 512, nst);
        }
    }
}

extern "C" void kernel_launch(int rank, int nranks, ncclComm_t comm,
                              hipStream_t stream,
                              void* const* d_in, const int* in_sizes, int n_in,
                              void* d_out, int out_size,
                              const int* in_dtypes, int out_dtype,
                              void* const* d_out_all,
                              void* const* d_ws_all, size_t ws_bytes) {
    (void)comm; (void)in_dtypes; (void)out_dtype; (void)d_out_all;
    if (nranks != NR || (unsigned)rank >= (unsigned)NR || n_in != 2 || in_sizes[0] != 4096 * 512 || in_sizes[1] != 512 * 4096
        || out_size != 4096 * 4096 || ws_bytes < WS_NEED) {
        fprintf(stderr, "kernel_launch: unexpected problem shape\n");
        return;
    }
    static bool attr_set[NR] = {};
    if (!attr_set[rank]) {
        HIP_CHECK(hipFuncSetAttribute((const void*)fused_gemm_allgather, hipFuncAttributeMaxDynamicSharedMemorySize, LDS_BYTES));
        attr_set[rank] = true;
    }
    Params P;
    for (int p = 0; p < NR; ++p) P.ws[p] = (char*)d_ws_all[p];
    P.x = (const char*)d_in[0];
    P.w = (const char*)d_in[1];
    P.out = (char*)d_out;
    P.myws = P.ws[rank];
    P.rank = rank;
    P.pad = 0;
    hipLaunchKernelGGL(fused_gemm_allgather, dim3(GRID), dim3(THREADS), LDS_BYTES, stream, P);
    HIP_CHECK(hipGetLastError());
}
```

```cpp
#include <hip/hip_runtime.h>
#include <rccl/rccl.h>
#include <cstdio>
#include <cstdint>
#include <cstddef>
#include <type_traits>
#pragma clang diagnostic ignored "-Wint-to-pointer-cast"

#define HIP_CHECK(expr) do { hipError_t _e = (expr); if (_e != hipSuccess) { \
    fprintf(stderr, "HIP error %s at %s:%d\n", hipGetErrorString(_e), __FILE__, __LINE__); \
    return; } } while (0)

typedef unsigned u32;
typedef unsigned long long u64;
typedef u32 u32x4 __attribute__((ext_vector_type(4)));
typedef u32 u32x2 __attribute__((ext_vector_type(2)));
typedef float f32x4 __attribute__((ext_vector_type(4)));
typedef float f32x2 __attribute__((ext_vector_type(2)));
typedef __bf16 bf16x8 __attribute__((ext_vector_type(8)));
typedef __bf16 bf16x2 __attribute__((ext_vector_type(2)));
typedef short s16x4 __attribute__((ext_vector_type(4)));
#define INL __attribute__((always_inline))
#define LAS __attribute__((address_space(3)))
#define GAS __attribute__((address_space(1)))

constexpr int NR = 8;
constexpr int NGEMM = 128;
constexpr int PER_LINK = 16;
constexpr int NPULL = 7 * PER_LINK;
constexpr int NROLE = NGEMM + NPULL;
constexpr int GRID = 256;
constexpr int THREADS = 576;
constexpr int IN_ROUNDS = 12, OUT_ROUNDS = 16, ROUNDS = IN_ROUNDS + OUT_ROUNDS;
constexpr u32 OUT0 = IN_ROUNDS * PER_LINK;
constexpr u32 NMSG = ROUNDS * PER_LINK;
constexpr u32 SLOT = 16384 + 512;
constexpr u32 GSLOT = 2 * SLOT;
constexpr u32 NEARF = 3 * PER_LINK, SFLAG = SLOT - 128;
constexpr u32 OBATCH = 96;
static_assert(OBATCH % 8 == 0 && NEARF % 8 == 0, "whole groups");
constexpr u32 RK = 20, RP = 3, RW0 = OUT0 + 112, RN = RP * RK, REMAX = 31;
constexpr u32 TICKETS = NMSG - 2 * PER_LINK + REMAX + 1;
static_assert(RK < 32 && RW0 + RN <= NMSG, "window");
constexpr u32 IMG = 16384;
constexpr int NSLOT = 8;
constexpr int WIN = 4;
static_assert(2 * WIN + 1 <= 9, "a message has at least 9 pieces");
constexpr unsigned long long DEADLINE = 100000000ull;

constexpr size_t OFF_CTR = 0;
constexpr size_t OFF_TMO = 32768;
constexpr size_t OFF_CNT = 36864;
constexpr size_t OFF_TKT = 45056;
constexpr size_t OFF_KEEP = 47104;
constexpr size_t OFF_GRP = 49152;
constexpr size_t OFF_FLAG = 65536;
constexpr size_t OFF_PKIN = 1u << 20;
constexpr size_t OFF_PKOUT = 18u << 20;
constexpr size_t OFF_UX = 28u << 20;
constexpr size_t OFF_UW = 44u << 20;
constexpr size_t OFF_RLY = 52u << 20;
constexpr size_t WS_NEED = 68u << 20;
static_assert(OFF_RLY + (size_t)7 * RN * GSLOT <= WS_NEED, "copies");
static_assert(OFF_FLAG + (size_t)NR * NMSG * 128 <= OFF_PKIN, "flags");
static_assert(OFF_PKIN + 512 * (size_t)GSLOT <= OFF_PKOUT, "pkin");
static_assert(OFF_PKOUT + 256 * (size_t)GSLOT <= OFF_UX, "pkout");

constexpr u32 L_RING = 0;
constexpr u32 L_CTL = NSLOT * SLOT;
constexpr u32 L_FULL = L_CTL;
constexpr u32 L_DESC = L_CTL + 4;
constexpr u32 L_PLAN = L_CTL + 192;
constexpr u32 L_TRDY = L_CTL + 240;
constexpr u32 L_FREE = L_CTL + 16;
constexpr u32 L_DONE = L_CTL + 144;
constexpr u32 L_LEN = L_CTL + 16 + 8 * NSLOT;
constexpr u32 L_MSG = L_CTL + 16 + 12 * NSLOT;
constexpr u32 L_DTBL = L_CTL + 256;
constexpr u32 L_UNORD = L_DTBL + 768;
constexpr int NST = 6;
constexpr u32 STG = 8192 + IMG;
constexpr u32 L_XCH = 4 * STG;
constexpr u32 L_VAR = L_XCH + 32768;
constexpr u32 L_ETIN = L_VAR + 8192;
constexpr u32 L_ETOUT = NST * STG;
constexpr u32 L_TOT = L_ETOUT + 2048;
constexpr u32 L_SRC = L_TOT + 48;
constexpr u32 L_SCR = L_TOT + 64;
constexpr u32 LDS_BYTES = L_SCR + 512;
static_assert(LDS_BYTES >= L_UNORD + 256 && LDS_BYTES <= 163840 && L_ETIN + 4096 <= L_ETOUT, "lds");

constexpr int pack_passes(int b) {
    return (b >> 4) <= 1 && (b & 7) <= 1 ? 2 : ((b & 7) == 7 ? 5 : 4);
}
constexpr int pack_msg(int b, int z) {
    const int early = b < 32 ? 2 * (b >> 3) + ((b & 7) < 2 ? (b & 7) : 2) : 8;
    return z < 2 ? z * 128 + b : (z < 4 ? 256 + (z - 2) * 120 + b - early : 496 + (b >> 3));
}
__device__ __forceinline__ void msg_of(int rho, int& isw, int& grp, int& lp, int& kb, int& seq) {
    if (rho < 384) {
        const int t = rho / 48, r = rho % 48;
        isw = r < 32; grp = isw ? r >> 3 : (r - 32) >> 3; kb = r & 7; lp = t; seq = t * 16 + (isw ? 0 : 8) + kb;
    } else {
        const int t = 8 + ((rho - 384) >> 5), r = rho & 31, pos = (r >> 3) & 1;
        isw = 0; grp = r >> 4; kb = r & 7; lp = 8 + 2 * (t - 8) + pos; seq = t * 16 + pos * 8 + kb;
    }
}
struct Sched { alignas(4) unsigned char unord[256]; unsigned char ord[2][128]; bool ok; };
constexpr Sched make_sched() {
    Sched s{};
    int r = 0;
    s.ok = true;
    for (int z = 0; z < 5; ++z)
        for (int b = 0; b < 128; ++b)
            if (pack_passes(b) > z && pack_msg(b, z) != r++) s.ok = false;
    if (r != 512) s.ok = false;
    int key[256] = {};
    for (int b = 0; b < 128; ++b) {
        const int i = b >> 4, j = b & 7, sh = i > j ? i : j;
        const int t1 = 30 * sh + 170, arr2 = 30 * (8 + (i >> 1));
        key[b] = t1 * 256 + b;
        key[128 + b] = ((t1 > arr2 ? t1 : arr2) + 170) * 256 + 128 + b;
    }
    for (int a = 0; a < 256; ++a) {
        int pos = 0;
        for (int c = 0; c < 256; ++c) pos += key[c] < key[a];
        s.ord[a >> 7][a & 127] = (unsigned char)pos;
        s.unord[pos] = (unsigned char)a;
    }
    return s;
}
__device__ constexpr Sched SCHED = make_sched();
static_assert(make_sched().ok, "every input message is packed once");

constexpr u32 F_RAW = 0x80000000u;
constexpr u32 F_BAD = 0x40000000u;

__device__ __forceinline__ u32 var_bytes(u32 lo) {
    const u32 v = lo & 0xffffu;
    return (lo & F_RAW) || v > 8192u ? 8192u : (v < 16u ? 16u : v);
}
template <int N, int I = 0, class F> __device__ __forceinline__ void static_for(F f) {
    if constexpr (I < N) { f(std::integral_constant<int, I>{}); static_for<N, I + 1>(f); }
}

struct Params {
    const char* x;
    const char* w;
    char* myws;
    char* out;
    int rank;
    int pad;
    char* ws[NR];
};


__device__ __forceinline__ void st16_wt(char* p, u32x4 v) {
    asm volatile("global_store_dwordx4 %0, %1, off sc0 sc1\n\ts_nop 1" :: "v"(p), "v"(v) : "memory");
}
__device__ __forceinline__ void st16_out(char* p, u32x4 v) {
    asm volatile("global_store_dwordx4 %0, %1, off sc0 sc1 nt\n\ts_nop 1" :: "v"(p), "v"(v) : "memory");
}
__device__ __forceinline__ void st8_wt(char* p, u32x2 v) {
    asm volatile("global_store_dwordx2 %0, %1, off sc0 sc1" :: "v"(p), "v"(v) : "memory");
}
__device__ __forceinline__ void st4_wt(char* p, u32 v) {
    asm volatile("global_store_dword %0, %1, off sc0 sc1" :: "v"(p), "v"(v) : "memory");
}
__device__ __forceinline__ void vm_wait0() { asm volatile("s_waitcnt vmcnt(0)" ::: "memory"); }
__device__ __forceinline__ void lgkm_wait0() { asm volatile("s_waitcnt lgkmcnt(0)" ::: "memory"); }
__device__ __forceinline__ void bar() { asm volatile("s_waitcnt lgkmcnt(0)\n\ts_barrier" ::: "memory"); }
__device__ __forceinline__ void lds_st(u32 addr, u32 v) { asm volatile("ds_write_b32 %0, %1" :: "v"(addr), "v"(v) : "memory"); }
__device__ __forceinline__ u32 lds_ld(u32 addr) {
    u32 v; asm volatile("ds_read_b32 %0, %1\n\ts_waitcnt lgkmcnt(0)" : "=v"(v) : "v"(addr) : "memory"); return v;
}
__device__ __forceinline__ u32x4 lds_ld16(u32 addr) { return *(const LAS u32x4*)addr; }
template <int N> __device__ __forceinline__ u32 landed_first(u32 v) {
    u32 r; asm volatile("s_waitcnt vmcnt(%2)\n\tv_readfirstlane_b32 %0, %1\n\ts_nop 3" : "=s"(r) : "v"(v), "n"(N) : "memory"); return r;
}
__device__ __forceinline__ void dma1(const char* src, u32 lds_dst) {
    u32 keep;
    asm volatile("s_mov_b32 %0, m0\n\ts_mov_b32 m0, %2\n\ts_nop 0\n\tglobal_load_lds_dwordx4 %1, off\n\ts_mov_b32 m0, %0"
                 : "=&s"(keep) : "v"(src), "s"(lds_dst) : "memory");
}
__device__ __forceinline__ void dma1_sys(const char* src, u32 lds_dst) {
    u32 keep;
    asm volatile("s_mov_b32 %0, m0\n\ts_mov_b32 m0, %2\n\ts_nop 0\n\tglobal_load_lds_dwordx4 %1, off sc0 sc1\n\ts_mov_b32 m0, %0"
                 : "=&s"(keep) : "v"(src), "s"(lds_dst) : "memory");
}
__device__ __forceinline__ void dma1_sys_part(const char* src, u32 lds_dst, u32 bytes) {
    const u64 mask = bytes > 1008u ? ~0ull : (1ull << ((bytes + 15u) >> 4)) - 1ull;
    u64 keep;
    asm volatile("s_mov_b64 %0, exec\n\ts_mov_b32 m0, %2\n\ts_mov_b64 exec, %3\n\tglobal_load_lds_dwordx4 %1, off sc0 sc1\n\ts_mov_b64 exec, %0"
                 : "=&s"(keep) : "v"(src), "s"(__builtin_amdgcn_readfirstlane(lds_dst)), "s"(mask) : "memory");
}
__device__ __forceinline__ void dma_dword(const char* src, u32 lds_dst) {
    asm volatile("s_mov_b32 m0, %1\n\ts_nop 0\n\tglobal_load_lds_dword %0, off sc0 sc1" :: "v"(src), "s"(lds_dst) : "memory");
}
__device__ __forceinline__ void dma1s(const char* sbase, u32 voff, u32 lds_dst) {
    u64 t;
    asm volatile("s_mov_b32 m0, %3\n\ts_mov_b64 %0, %2\n\tglobal_load_lds_dwordx4 %1, %0" : "=&s"(t) : "v"(voff), "s"(sbase), "s"(lds_dst) : "memory");
}
__device__ __forceinline__ void dma_a(const char* sbase, u32 voff, u32 lds_dst) {
    u64 t;
    asm volatile("s_mov_b32 m0, %3\n\ts_mov_b64 %0, %2\n\tglobal_load_lds_dwordx4 %1, %0 sc0 sc1" : "=&s"(t) : "v"(voff), "s"(sbase), "s"(lds_dst) : "memory");
}
__device__ __forceinline__ void dma_b(const char* sbase, u32 voff) {
    u64 t;
    asm volatile("s_mov_b64 %0, %2\n\tglobal_load_lds_dwordx4 %1, %0 offset:1024 sc0 sc1" : "=&s"(t) : "v"(voff), "s"(sbase) : "memory");
}
__device__ __forceinline__ void announce(char* myws, char* wsl, int rank, u32 seq, u64 val, u32 destmask, int grp, int lane) {
    const size_t line = OFF_FLAG + ((size_t)rank * NMSG + (seq & ~7u)) * 128;
    if (grp < 0) {
        if (lane == 0)
            __hip_atomic_store((GAS u64*)(myws + OFF_PKOUT + (size_t)(seq - OUT0) * GSLOT + SFLAG), val, __ATOMIC_RELAXED, __HIP_MEMORY_SCOPE_SYSTEM);
        return;
    }
    char* g = myws + OFF_GRP + (size_t)grp * 128;
    u32 old = 0;
    if (lane == 0) {
        __hip_atomic_store((GAS u64*)(g + (seq & 7u) * 8), val, __ATOMIC_RELAXED, __HIP_MEMORY_SCOPE_AGENT);
        asm volatile("s_waitcnt vmcnt(0)" ::: "memory");
        old = __hip_atomic_fetch_add((GAS u32*)(g + 64), 1u, __ATOMIC_RELAXED, __HIP_MEMORY_SCOPE_AGENT);
    }
    if ((__builtin_amdgcn_readfirstlane(old) & 7u) == 7u) {
        const u64 e = __hip_atomic_load((GAS u64*)(g + (lane & 7) * 8), __ATOMIC_RELAXED, __HIP_MEMORY_SCOPE_AGENT);
        const u64 wl = (u64)wsl;
        char* wp = (char*)((u64)(u32)__builtin_amdgcn_ds_bpermute((lane >> 3) * 4, (int)(u32)wl)
                           | (u64)(u32)__builtin_amdgcn_ds_bpermute((lane >> 3) * 4, (int)(u32)(wl >> 32)) << 32);
        if ((destmask >> (lane >> 3)) & 1u)
            __hip_atomic_store((GAS u64*)(wp + line + (lane & 7) * 8), e, __ATOMIC_RELAXED, __HIP_MEMORY_SCOPE_SYSTEM);
    }
}
__device__ __forceinline__ u32 fetch_plan(char* tkt, u32 call, int lane) {
    const u32 h = __builtin_amdgcn_readfirstlane(__hip_atomic_load((GAS u32*)(tkt + 128), __ATOMIC_RELAXED, __HIP_MEMORY_SCOPE_AGENT));
    if ((h >> 15) != (call & 0x1ffffu)) return 0u;
    const u32 w = __hip_atomic_load((GAS u32*)(tkt + 132 + (lane & 7) * 4), __ATOMIC_RELAXED, __HIP_MEMORY_SCOPE_AGENT);
    if (lane < 8) lds_st(L_PLAN + lane * 4, w);
    lds_st(L_PLAN + 32, (h & 0x7fffu) | 0x8000u);
    return (h & 0x7fffu) | 0x8000u;
}
__device__ __forceinline__ void wait_counters(GAS u32* c, u32 want, GAS u32* tmo, u32 call) {
    const u64 t0 = __builtin_amdgcn_s_memrealtime();
    for (u32 it = 1; !__all(__hip_atomic_load(c, __ATOMIC_RELAXED, __HIP_MEMORY_SCOPE_SYSTEM) >= want); ++it) {
        if ((it & 15u) == 0u
            && (__builtin_amdgcn_readfirstlane(__hip_atomic_load(tmo, __ATOMIC_RELAXED, __HIP_MEMORY_SCOPE_AGENT)) == call
                || __builtin_amdgcn_s_memrealtime() - t0 > DEADLINE)) {
            __hip_atomic_store(tmo, call, __ATOMIC_RELAXED, __HIP_MEMORY_SCOPE_AGENT);
            break;
        }
        __builtin_amdgcn_s_sleep(2);
    }
    asm volatile("s_waitcnt vmcnt(0)" ::: "memory");
}

__device__ __forceinline__ float gelu(float y) {
    const float k0 = -2.0f * 0.7978845608028654f * 1.4426950408889634f;
    const float u = y * y;
    const float z = y * __builtin_fmaf(u, k0 * 0.044715f, k0);
    return y * __builtin_amdgcn_rcpf(1.0f + __builtin_amdgcn_exp2f(z));
}
__device__ __forceinline__ u32 pk_bf16(float a, float b) {
    f32x2 f = {a, b};
    return __builtin_bit_cast(u32, __builtin_convertvector(f, bf16x2));
}

constexpr u64 IN_REL0 = 0x1234567'8d9abcull, IN_REL1 = 0x123d456'789cabull;
constexpr u32 OUT_E0 = 114, OUT_NSYM = 184;
__device__ constexpr unsigned char OUT_RANK[192] = {
    210, 209, 208, 207, 104, 206, 96, 105, 205, 97, 204, 98, 203, 106, 99, 202, 100, 201, 88, 101, 107, 200, 89,
    90, 102, 91, 103, 92, 108, 93, 94, 95, 109, 112, 80, 81, 82, 83, 84, 110, 85, 86, 87, 199, 198, 111,
    197, 196, 195, 194, 193, 192, 113, 72, 73, 74, 75, 76, 77, 78, 79, 114, 65, 191, 64, 66, 190, 67, 189,
    68, 188, 69, 187, 70, 71, 186, 185, 184, 115, 57, 58, 56, 59, 60, 61, 62, 63, 183, 182, 181, 180, 179,
    178, 177, 176, 116, 49, 50, 51, 52, 53, 48, 54, 55, 175, 174, 173, 172, 171, 170, 169, 168, 41, 42, 43,
    44, 45, 46, 47, 40, 117, 167, 166, 165, 164, 163, 162, 161, 160, 33, 34, 35, 36, 37, 38, 39, 32, 118,
    159, 158, 157, 156, 155, 154, 153, 152, 25, 26, 27, 28, 29, 30, 31, 24, 151, 150, 149, 147, 148, 146, 145,
    144, 119, 17, 18, 19, 20, 21, 23, 22, 16, 137, 138, 139, 140, 141, 142, 143, 136, 11, 9, 10, 12, 13};
__device__ __forceinline__ constexpr u32x2 enc_words(u32 c1, u32 q) {
    const u32 c2 = q < 3u ? q : 3u, c3 = q < 3u ? 0u : (q < 6u ? q - 3u : 3u), c4 = q < 6u ? 0u : q - 6u;
    return u32x2{(c1 << 28) | c2 | (c3 << 8) | (c4 << 16), 2u | (q >= 3u ? 2u << 8 : 0u) | (q >= 6u ? 3u << 16 : 0u)};
}
template <bool OUT> __device__ __forceinline__ void put_enc_table(u32 l, int lane) {
    const u32x2 esc = enc_words(OUT ? 15u : 0u, 13u);
#pragma unroll
    for (int j = 0; j < (OUT ? 2 : 4); ++j)
        *(LAS u32x4*)(l + j * 1024 + lane * 16) = u32x4{esc.x, esc.y | (1u << 24), esc.x, esc.y | (1u << 24)};
    if (OUT) {
#pragma unroll
        for (u32 r = lane; r < 192u; r += 64u)
            if (r < OUT_NSYM) *(LAS u32x2*)(l + OUT_RANK[r] * 8u) = r < 3u ? u32x2{r << 28, 0u} : enc_words(3u + (r - 3u) % 13u, (r - 3u) / 13u);
    } else {
        const u32 q = lane & 15u, m = ((u32)lane >> 4) & 1u;
        if (lane < 32 && q < 13u) *(LAS u32x2*)(l + ((((u32)((m ? IN_REL1 : IN_REL0) >> (4u * q)) & 15u) << 1) | m) * 8u) = enc_words(0u, q);
    }
}
template <bool OUT> __device__ __forceinline__ void put_dec_table(u32 l, int lane) {
    if (OUT) {
#pragma unroll
        for (u32 i = lane; i < 256u; i += 64u) {
            const u32 c1 = i >> 4, q = i & 15u, r = c1 < 3u ? c1 : 13u * q + c1, x = OUT_RANK[r < OUT_NSYM ? r : 0u];
            *(LAS unsigned short*)(l + i * 2u) =
                (unsigned short)(r < OUT_NSYM ? ((x >> 7) << 15) | ((((x >> 3) & 15u) + OUT_E0) << 7) | ((x & 7u) << 4) : 1u);
        }
    } else {
        const u32 c1 = (u32)lane >> 4, q = lane & 15u;
        *(LAS unsigned short*)(l + c1 * 64u + q * 2u) = (unsigned short)(
            q < 13u ? ((c1 >> 1) << 15) | (((u32)(((c1 & 1u) ? IN_REL1 : IN_REL0) >> (4u * q)) & 15u) << 7) | ((c1 & 1u) << 6) : 1u);
    }
}

__device__ __forceinline__ u32 wave_scan(u32 v) {
    v += (u32)__builtin_amdgcn_update_dpp(0, (int)v, 0x111, 0xf, 0xf, false);
    v += (u32)__builtin_amdgcn_update_dpp(0, (int)v, 0x112, 0xf, 0xf, false);
    v += (u32)__builtin_amdgcn_update_dpp(0, (int)v, 0x114, 0xf, 0xf, false);
    v += (u32)__builtin_amdgcn_update_dpp(0, (int)v, 0x118, 0xf, 0xf, false);
    v += (u32)__builtin_amdgcn_update_dpp(0, (int)v, 0x142, 0xa, 0xf, false);
    v += (u32)__builtin_amdgcn_update_dpp(0, (int)v, 0x143, 0xc, 0xf, false);
    return v;
}
__device__ __forceinline__ u32 ubfe(u32 v, u32 off, u32 w) { return __builtin_amdgcn_ubfe(v, off, w); }
__device__ __forceinline__ void lds_or_bits(u32 base, u32 pos, u32 bits) {
    if (bits) {
        LAS u32* p = (LAS u32*)(base + (pos >> 5) * 4u);
        __hip_atomic_fetch_or(p, bits << (pos & 31u), __ATOMIC_RELAXED, __HIP_MEMORY_SCOPE_WORKGROUP);
        __hip_atomic_fetch_or(p + 1, (bits >> 1) >> (~pos & 31u), __ATOMIC_RELAXED, __HIP_MEMORY_SCOPE_WORKGROUP);
    }
}
__device__ __forceinline__ u32 lds_get_bits(u32 base, u32 pos) {
    const LAS u32* p = (const LAS u32*)(base + (pos >> 5) * 4u);
    return __builtin_amdgcn_alignbit(p[1], p[0], pos & 31u);
}
__device__ __forceinline__ u32 low_mask_even(u32 n) { return ~((0xffffffffu << (n >> 1)) << (n >> 1)); }

#define SDWA_SHL(B) \
    __device__ __forceinline__ u32 shl_byte##B(u32 sh, u32 v) {     \
        u32 r; asm("v_lshlrev_b32_sdwa %0, %1, %2 dst_sel:DWORD dst_unused:UNUSED_PAD src0_sel:BYTE_" #B " src1_sel:BYTE_" #B \
                   : "=v"(r) : "v"(sh), "v"(v)); return r; }
SDWA_SHL(0) SDWA_SHL(1) SDWA_SHL(2)

template <int B> __device__ __forceinline__ u32 and_byte(u32 v, u32 m) {
    u32 r;
    if constexpr (B == 0) asm("v_and_b32_sdwa %0, %1, %2 dst_sel:DWORD dst_unused:UNUSED_PAD src0_sel:BYTE_0 src1_sel:DWORD" : "=v"(r) : "v"(v), "v"(m));
    if constexpr (B == 1) asm("v_and_b32_sdwa %0, %1, %2 dst_sel:DWORD dst_unused:UNUSED_PAD src0_sel:BYTE_1 src1_sel:DWORD" : "=v"(r) : "v"(v), "v"(m));
    if constexpr (B == 2) asm("v_and_b32_sdwa %0, %1, %2 dst_sel:DWORD dst_unused:UNUSED_PAD src0_sel:BYTE_2 src1_sel:DWORD" : "=v"(r) : "v"(v), "v"(m));
    if constexpr (B == 3) asm("v_and_b32_sdwa %0, %1, %2 dst_sel:DWORD dst_unused:UNUSED_PAD src0_sel:BYTE_3 src1_sel:DWORD" : "=v"(r) : "v"(v), "v"(m));
    return r;
}

__device__ __forceinline__ void store_input_bytes(char* slot, const u32 (&d)[8], int wave, int lane) {
    u32 bb[4];
#pragma unroll
    for (int j = 0; j < 4; ++j) {
        const u32 lowb = __builtin_amdgcn_perm(d[2 * j + 1], d[2 * j], 0x06040200u);
        const u32 higb = __builtin_amdgcn_perm(d[2 * j + 1], d[2 * j], 0x07050301u);
        bb[j] = (lowb & 0x7f7f7f7fu) | (higb & 0x80808080u);
    }
    st16_wt(slot + wave * 1024 + lane * 16, u32x4{bb[0], bb[1], bb[2], bb[3]});
}
constexpr u32 RAW_OFF = SLOT;
__device__ __forceinline__ u32 input_e0(u32x2 sample) {
    const u32 s2 = ((sample.x >> 7) & 0x00ff00ffu) + ((sample.y >> 7) & 0x00ff00ffu);
    const u32 tot = __builtin_amdgcn_readlane(wave_scan((s2 & 0xffffu) + (s2 >> 16)), 63);
    const u32 anchor = (tot + 504u) >> 8;
    return anchor < 12u ? 0u : (anchor > 252u ? 240u : anchor - 12u);
}
template <bool OUT>
__device__ __forceinline__ void encode_block(char* slot, const u32 (&d)[8], u32 e0, u32 ltbl, u32 lpriv, u32 ltot, int wave, int lane) {
    *(LAS u32x4*)(lpriv + lane * 16) = u32x4{0u, 0u, 0u, 0u};
    u32 en[16], in[16];
    typedef unsigned short u16x2 __attribute__((ext_vector_type(2)));
#pragma unroll
    for (int j = 0; j < 8; ++j) {
        u32 i0, i1;
        if (OUT) {
            const u16x2 sub = {OUT_E0 << 6, OUT_E0 << 6}, cap = {15 << 6, 15 << 6};
            const u16x2 t = __builtin_elementwise_min(__builtin_bit_cast(u16x2, (d[j] >> 1) & 0x3ff83ff8u) - sub, cap);
            const u32 x = __builtin_bit_cast(u32, t) | ((d[j] >> 5) & 0x04000400u);
            i0 = x & 0xffffu; i1 = x >> 16;
        } else {
            const u16x2 sub = {(unsigned short)(e0 << 7), (unsigned short)(e0 << 7)};
            const u32 x = __builtin_bit_cast(u32, __builtin_bit_cast(u16x2, d[j]) - sub);
            i0 = (x >> 3) & 0xff8u; i1 = (x >> 19) & 0xff8u;
        }
        const u32x2 a = *(const LAS u32x2*)(ltbl + i0), b = *(const LAS u32x2*)(ltbl + i1);
        en[2 * j] = a.x; in[2 * j] = a.y; en[2 * j + 1] = b.x; in[2 * j + 1] = b.y;
    }
    __builtin_amdgcn_sched_barrier(0);
    if (OUT) {
        u32 bb[4];
#pragma unroll
        for (int j = 0; j < 4; ++j) {
            const u32 lowb = __builtin_amdgcn_perm(d[2 * j + 1], d[2 * j], 0x06040200u);
            const u32 c01 = __builtin_amdgcn_perm(en[4 * j + 1], en[4 * j], 0x0c0c0703u);
            const u32 c23 = __builtin_amdgcn_perm(en[4 * j + 3], en[4 * j + 2], 0x07030c0cu);
            bb[j] = (lowb & 0x0f0f0f0fu) | c01 | c23;
        }
        constexpr u32 H = 0xf0f0f0f0u;
        char* p = slot + wave * 512 + lane * 8;
        st8_wt(p, u32x2{(bb[0] & H) | ((bb[1] >> 4) & ~H), (bb[2] & H) | ((bb[3] >> 4) & ~H)});
        st8_wt(p + 4096, u32x2{(bb[0] & ~H) | ((bb[1] << 4) & H), (bb[2] & ~H) | ((bb[3] << 4) & H)});
    }
    u32 f2 = 0, f3 = 0, f4[2] = {0, 0}, pos = 0, n4a = 0;
#pragma unroll
    for (int i = 0; i < 16; i += 2) {
        if (i == 8) { n4a = ubfe(pos, 16, 8); pos &= 0xff00ffffu; }
        const u32 pos1 = pos + in[i];
        f2 |= shl_byte0(pos, en[i]) | shl_byte0(pos1, en[i + 1]);
        f3 |= shl_byte1(pos, en[i]) | shl_byte1(pos1, en[i + 1]);
        f4[i >> 3] |= shl_byte2(pos, en[i]) | shl_byte2(pos1, en[i + 1]);
        pos = pos1 + in[i + 1];
    }
    const u32 n4 = n4a + ubfe(pos, 16, 8), ne = pos >> 24;
    const u32 ca = (pos & 0xffu) | ((pos << 8) & 0xff0000u), cb = n4 | (ne << 16);
    const u32 sa = wave_scan(ca), sb = wave_scan(cb);
    const u32 ta = __builtin_amdgcn_readlane(sa, 63), tb = __builtin_amdgcn_readlane(sb, 63);
    const u32 N2 = ta & 0xffffu, N3 = ta >> 16, NE = tb >> 16;
    const u32 escpos = (N2 + N3 + (tb & 0xffffu) + 15u) & ~15u;
    const u32 dw = (escpos + 16u * NE + 31u) >> 5;
    if (lane == 0) *(LAS u32*)(ltot + wave * 4) = dw <= 252u ? dw : 0x1000u;
    if (dw <= 252u) {
        const u32 xa = sa - ca, xb = sb - cb;
        if (OUT) lds_or_bits(lpriv, xa & 0xffffu, f2);
        else *(LAS u32*)(lpriv + lane * 4) = f2;
        lds_or_bits(lpriv, N2 + (xa >> 16), f3);
        const u32 p4 = N2 + N3 + (xb & 0xffffu);
        if (!__builtin_amdgcn_ballot_w64(n4 > 32u)) {
            lds_or_bits(lpriv, p4, f4[0] | (f4[1] << n4a));
        } else {
            lds_or_bits(lpriv, p4, f4[0]);
            lds_or_bits(lpriv, p4 + n4a, f4[1]);
        }
        if (NE) {
            u32 a = lpriv + (escpos >> 3) + (xb >> 16) * 2u;
#pragma unroll
            for (int j = 0; j < 4; ++j)
                if ((in[4 * j] | in[4 * j + 1] | in[4 * j + 2] | in[4 * j + 3]) >> 24) {
#pragma unroll
                    for (int i = 4 * j; i < 4 * j + 4; ++i)
                        if (in[i] >> 24) { *(LAS unsigned short*)a = (unsigned short)(d[i >> 1] >> (16 * (i & 1))); a += 2u; }
                }
        }
    }
}
__device__ __forceinline__ u32 encode_finish(char* slot, const u32 (&d)[8], u32 lpriv, u32 ltot, int wave, int lane) {
    const u32 mine = *(const LAS u32*)(ltot + (lane & 7) * 4);
    const u32 w0 = *(const LAS u32*)(lpriv + lane * 4), w1 = *(const LAS u32*)(lpriv + 256 + lane * 4);
    u32 inc = mine;
    inc += (u32)__builtin_amdgcn_update_dpp(0, (int)inc, 0x111, 0xf, 0xf, false);
    inc += (u32)__builtin_amdgcn_update_dpp(0, (int)inc, 0x112, 0xf, 0xf, false);
    inc += (u32)__builtin_amdgcn_update_dpp(0, (int)inc, 0x114, 0xf, 0xf, false);
    const u32 total = __builtin_amdgcn_readlane(inc, 7);
    if (total >= 0x1000u) {
        char* p = slot + RAW_OFF + wave * 1024 + lane * 16;
        st16_wt(p + 8192, u32x4{d[0], d[1], d[2], d[3]}); st16_wt(p, u32x4{d[4], d[5], d[6], d[7]});
        return F_RAW;
    }
    const u32 dw = __builtin_amdgcn_readlane(mine, wave), start = __builtin_amdgcn_readlane(inc, wave) - dw;
    char* p = slot + 8192 + 16 + (start + lane) * 4u;
    if ((u32)lane < dw) st4_wt(p, w0);
    if ((u32)lane + 64u < dw) st4_wt(p + 256, w1);
    if (dw > 128u) {
        if ((u32)lane + 128u < dw) st4_wt(p + 512, *(const LAS u32*)(lpriv + 512 + lane * 4));
        if ((u32)lane + 192u < dw) st4_wt(p + 768, *(const LAS u32*)(lpriv + 768 + lane * 4));
    }
    if (wave == 0 && lane < 8 && !(lane & 1)) st4_wt(slot + 8192 + lane * 2, (inc - mine) | (inc << 16));
    return (16u + total * 4u + 15u) & ~15u;
}

template <bool OUT, class F>
__device__ __forceinline__ void decode_block(u32 sl, u32 e0, u32 ltbl, int wave, int lane, u32x4& v0, u32x4& v1, F wait_low) {
    u32x4 B;
    if (OUT) { const u32x2 C = *(const LAS u32x2*)(sl + wave * 512 + lane * 8); B = u32x4{C.x, C.x << 4, C.y, C.y << 4}; }
    else B = lds_ld16(sl + wave * 1024 + lane * 16);
    const u32 hdr = __builtin_amdgcn_readfirstlane(*(const LAS unsigned short*)(sl + 8192 + wave * 2));
    const u32 base = sl + 8192 + 16 + (hdr > 2040u ? 2040u : hdr) * 4u;
    const u32 eadd = e0 * 0x00800080u;
    const u32 bb[4] = {B.x, B.y, B.z, B.w};
    u32 F2, N2, W2[4];
    if (OUT) {
        u32 c = 0;
#pragma unroll
        for (int j = 0; j < 4; ++j) {
            W2[j] = ((((bb[j] >> 4) & 0x0f0f0f0fu) + 0x0d0d0d0du) >> 3) & 0x02020202u;
            c += __builtin_popcount(W2[j]);
        }
        const u32 n2 = 2u * c, s2 = wave_scan(n2);
        N2 = __builtin_amdgcn_readlane(s2, 63);
        F2 = lds_get_bits(base, s2 - n2) & low_mask_even(n2);
    } else {
        F2 = *(const LAS u32*)(base + lane * 4);
        N2 = 2048u;
    }
    const u32 W3 = F2 & (F2 << 1) & 0xaaaaaaaau;
    const u32 n3 = 2u * __builtin_popcount(W3), s3 = wave_scan(n3);
    const u32 N3 = __builtin_amdgcn_readlane(s3, 63);
    const u32 F3 = lds_get_bits(base, N2 + s3 - n3) & low_mask_even(n3);
    const u32 K4 = F3 & (F3 >> 1) & 0x55555555u, W4 = K4 * 3u;
    const u32 n4 = 3u * __builtin_popcount(K4), s4 = wave_scan(n4);
    const u32 N4 = __builtin_amdgcn_readlane(s4, 63);
    const u32 P4 = N2 + N3 + s4 - n4;
    const LAS u32* q4 = (const LAS u32*)(base + (P4 >> 5) * 4u);
    const u32 q40 = q4[0], q41 = q4[1], q42 = q4[2];
    u32 F4 = __builtin_amdgcn_alignbit(q41, q40, P4 & 31u);
    const u32 F4h = __builtin_amdgcn_alignbit(q42, q41, P4 & 31u);
    u32 t[16], p2 = 0, p3 = 0, p4 = 0;
    const u32 cm = OUT ? 0xf0u : 0xc0u;
    static_for<16>([&](auto ii) INL {
        constexpr int i = decltype(ii)::value;
        if (i == 8) { F4 = __builtin_amdgcn_alignbit(F4h, F4, p4); p4 = 0; }
        const u32 c1 = and_byte<i & 3>(bb[i >> 2], cm);
        u32 c2, w3;
        if (OUT) {
            const u32 w2 = W2[i >> 2] >> (8 * (i & 3)); c2 = ubfe(F2, p2, w2); w3 = ubfe(W3, p2, w2); p2 += w2;
        }
        else { c2 = ubfe(F2, 2 * i, 2); w3 = ubfe(W3, 2 * i, 2); }
        const u32 c3 = ubfe(F3, p3, w3), w4 = ubfe(W4, p3, w3); p3 += w3;
        const u32 c4 = ubfe(F4, p4, w4); p4 += w4;
        t[i] = OUT ? (c1 + c2 + c3 + c4) << 1 : c1 + ((c2 + c3 + c4) << 1);
    });
    u32 o[8];
#pragma unroll
    for (int j = 0; j < 8; ++j) {
        typedef unsigned short u16x2 __attribute__((ext_vector_type(2)));
        const u16x2 pr = {*(const LAS unsigned short*)(ltbl + t[2 * j]), *(const LAS unsigned short*)(ltbl + t[2 * j + 1])};
        o[j] = __builtin_bit_cast(u32, pr) + (OUT ? 0u : eadd);
    }
    const u32 any = (o[0] | o[1] | o[2]) | (o[3] | o[4] | o[5]) | (o[6] | o[7]);
    if (__builtin_amdgcn_ballot_w64((any & 0x00010001u) != 0u)) {
        u32 c = 0;
#pragma unroll
        for (int j = 0; j < 8; ++j) c += o[j] & 0x00010001u;
        const u32 ne = (c & 0xffffu) + (c >> 16);
        u32 a = base + (((N2 + N3 + N4 + 15u) & ~15u) >> 3) + (wave_scan(ne) - ne) * 2u;
        const u32 amax = sl + SLOT - 4u;
#pragma unroll
        for (int j = 0; j < 8; ++j) {
            if (o[j] & 0x00010001u) {
                a = a < amax ? a : amax;
                if (o[j] & 1u) { o[j] = (o[j] & 0xffff0000u) | *(const LAS unsigned short*)a; a += 2u; }
                if (o[j] & 0x10000u) { o[j] = (o[j] & 0xffffu) | ((u32)*(const LAS unsigned short*)a << 16); a += 2u; }
            }
        }
    }
    u32 lb[4];
    if (OUT) {
        asm volatile("" : "+v"(o[0]), "+v"(o[1]), "+v"(o[2]), "+v"(o[3]), "+v"(o[4]), "+v"(o[5]), "+v"(o[6]), "+v"(o[7]));
        wait_low();
        const u32x2 R = *(const LAS u32x2*)(sl + 4096 + wave * 512 + lane * 8);
        lb[0] = R.x & 0x0f0f0f0fu; lb[1] = (R.x >> 4) & 0x0f0f0f0fu; lb[2] = R.y & 0x0f0f0f0fu; lb[3] = (R.y >> 4) & 0x0f0f0f0fu;
    } else {
#pragma unroll
        for (int j = 0; j < 4; ++j) lb[j] = bb[j] & 0x3f3f3f3fu;
    }
#pragma unroll
    for (int j = 0; j < 8; ++j) o[j] |= __builtin_amdgcn_perm(0u, lb[j >> 1], (j & 1) ? 0x0c030c02u : 0x0c010c00u);
    v0 = u32x4{o[0], o[1], o[2], o[3]};
    v1 = u32x4{o[4], o[5], o[6], o[7]};
}

__global__ void __launch_bounds__(THREADS, 1) fused_gemm_allgather(Params P) {
    const int lane = threadIdx.x & 63;
    const int wave = __builtin_amdgcn_readfirstlane(threadIdx.x >> 6);
    int b = blockIdx.x;
    {
        const int x = b & 7, y = b >> 3;
        if (x >= 4 && y >= 28 && b != 4 + 28 * 8) return;
        b = y >= 28 && x >= 4 ? NROLE : x < 4 ? (y >> 1) * 8 + 2 * x + (y & 1) : NGEMM + (x - 4) * 28 + y;
    }
    const int rank = P.rank;
    char* myws = P.myws;
    const char* px = P.x;
    const char* pw = P.w;
    asm volatile("" : "+s"(myws), "+s"(px), "+s"(pw));
    constexpr u32 lds0 = 0;
    GAS u32* const ctr = (GAS u32*)(myws + OFF_CTR + (size_t)b * 128);
    GAS u32* const tmo = (GAS u32*)(myws + OFF_TMO);

    if (b == NROLE) {
        if (wave) return;
        const u32 call = __builtin_amdgcn_readfirstlane(__hip_atomic_load(ctr, __ATOMIC_RELAXED, __HIP_MEMORY_SCOPE_AGENT)) + 1u;
        if (lane == 0) __hip_atomic_store(ctr, call, __ATOMIC_RELAXED, __HIP_MEMORY_SCOPE_AGENT);
        GAS u32* tk = (GAS u32*)(myws + OFF_TKT + (size_t)(lane < 7 ? lane : 0) * 256);
        u32 sum = 0, cnt = 0;
        const u64 t0 = __builtin_amdgcn_s_memrealtime();
        for (;;) {
            const u32 pos = __hip_atomic_load(tk, __ATOMIC_RELAXED, __HIP_MEMORY_SCOPE_AGENT) - (call - 1u) * TICKETS + 3 * PER_LINK;
            if (__any(pos >= OUT0)) break;
            if (__any(pos >= 7 * PER_LINK)) { sum += pos; ++cnt; }
            if (__builtin_amdgcn_s_memrealtime() - t0 > DEADLINE) { cnt = 0; break; }
            __builtin_amdgcn_s_sleep(8);
        }
        int eff = cnt ? -(int)((4u * sum) / cnt) : 0, mean = 0;
        for (int q = 0; q < 7; ++q) mean += __builtin_amdgcn_readlane(eff, q);
        eff -= mean / 7;
        const int kw = (eff + 3) / 4 + 2;
        const u32 keep = eff < 2 ? 0u : (kw > (int)RK ? RK : (u32)kw);
        u64 kv = (u64)(call & 0xfffffu) << 40;
        for (int q = 0; q < 7; ++q) kv |= (u64)(u32)__builtin_amdgcn_readlane(keep, q) << (5 * q);
        if (lane == 0) __hip_atomic_store((GAS u64*)(myws + OFF_KEEP), kv, __ATOMIC_RELAXED, __HIP_MEMORY_SCOPE_SYSTEM);
        u64 pk = 0;
        for (;;) {
            if (lane < 7 && eff < 0) pk = __hip_atomic_load((GAS u64*)(P.ws[(rank + 1 + lane) & 7] + OFF_KEEP), __ATOMIC_RELAXED, __HIP_MEMORY_SCOPE_SYSTEM);
            if (!__any(lane < 7 && eff < 0 && (u32)(pk >> 40) != (call & 0xfffffu))) break;
            if (__builtin_amdgcn_s_memrealtime() - t0 > DEADLINE) { pk = 0; break; }
            __builtin_amdgcn_s_sleep(8);
        }
        u32 kq = 0, eq = 0, closed = 0;
        for (int i = 0; i < 7 * (int)RK; ++i) {
            int hi = -0x7fffffff, qa = 0;
            for (int q = 0; q < 7; ++q) {
                const int e = __builtin_amdgcn_readlane(eff, q);
                if (!((closed >> q) & 1u) && e > hi) { hi = e; qa = q; }
            }
            if (hi <= 0) break;
            const u32 j = __builtin_amdgcn_readlane(kq, qa);
            const u32 has = (u32)(pk >> (5 * ((qa - lane - 1) & 7))) & 31u;
            const u32 cm = (u32)__builtin_amdgcn_ballot_w64(lane < 7 && lane != qa && !kq && eq < REMAX && j < has
                                                            && (u32)((qa - lane - 1) & 7) % RP != (u32)qa % RP);
            int lw = 0x7fffffff, qb = 0;
            for (int q = 0; q < 7; ++q) {
                const int e = __builtin_amdgcn_readlane(eff, q);
                if (((cm >> q) & 1u) && e < lw) { lw = e; qb = q; }
            }
            if (!cm || j >= RK || hi - lw < 8) { closed |= 1u << qa; continue; }
            const u32 eb = __builtin_amdgcn_readlane(eq, qb);
            u32 dn = (u32)(hi - lw) / 16u + 1u;
            dn = min(min(dn, 4u), min(min(RK, (u32)__builtin_amdgcn_readlane(has, qb)) - j, REMAX - eb));
            if ((u32)lane < dn)
                __hip_atomic_store((GAS unsigned char*)(myws + OFF_TKT + qb * 256 + 132 + eb + lane), (unsigned char)((qa << 5) | (j + lane)),
                                   __ATOMIC_RELAXED, __HIP_MEMORY_SCOPE_AGENT);
            closed |= 1u << qb;
            if (lane == qb) { eq += dn; eff += 4 * (int)dn; }
            if (lane == qa) { kq += dn; eff -= 4 * (int)dn; }
        }
        vm_wait0();
        if (lane < 7)
            __hip_atomic_store(tk + 32, kq | (eq << 5) | (keep << 10) | (call << 15), __ATOMIC_RELAXED, __HIP_MEMORY_SCOPE_AGENT);
        return;
    }
    if (__builtin_expect(b < NGEMM, 1)) {
        const int tj = b & 7, ti0 = b >> 4, half = (b >> 3) & 1;
        const int npass = pack_passes(b);
        constexpr int EBAR = 3;
        if (wave == 8) {
            u32 cv;
            char* wsl;
            asm volatile("global_load_dword %0, %2, off sc1\n\ts_nop 3\n\tglobal_load_dwordx2 %1, %3, %4 offset:%5"
                         : "=&v"(cv), "=&v"(wsl)
                         : "v"(ctr), "v"((lane & 7) * 8), "s"(__builtin_amdgcn_kernarg_segment_ptr()), "n"(offsetof(Params, ws)) : "memory");
            bar();
            put_enc_table<false>(L_ETIN, lane);
            const u32 call = landed_first<0>(cv) + 1u;
            asm volatile("" : "+v"(wsl));
            bar();
            int isw, grp, lp, kb, seq;
            msg_of(b, isw, grp, lp, kb, seq);
            if (seq < 2 * PER_LINK && lane < NR && lane != rank && (isw ? ((lane & 3) == grp) : ((lane >> 2) == grp)))
                __hip_atomic_store((GAS u32*)(wsl + OFF_FLAG + ((size_t)rank * NMSG + seq) * 128 + 68), call, __ATOMIC_RELAXED, __HIP_MEMORY_SCOPE_SYSTEM);
            if (lane == 0) __hip_atomic_store(ctr, call, __ATOMIC_RELAXED, __HIP_MEMORY_SCOPE_AGENT);
            for (int z = 0; z < npass; ++z) {
                if (z + 1 < npass) {
                    msg_of(pack_msg(b, z + 1), isw, grp, lp, kb, seq);
                    const char* sb = isw ? pw + ((size_t)kb * 64 * 4096 + (grp * 8 + lp) * 128) * 2 : px + ((size_t)(grp * 16 + lp) * 128 * 512 + kb * 64) * 2;
                    const u32 vo = isw ? (lane >> 4) * 8192 + (lane & 15) * 16 : (lane & 15) * 1024 + (lane >> 4) * 16;
                    const u32 sw = isw ? 65536u : 16384u, sh = isw ? 32768u : 64u;
#pragma unroll
                    for (int j = 0; j < 16; ++j) dma1s(sb + (j >> 1) * sw + (j & 1) * sh, vo, lds0 + (z + 1) * IMG + j * 1024);
                }
                bar();
                vm_wait0();
                bar();
                msg_of(pack_msg(b, z), isw, grp, lp, kb, seq);
                const u32 lo = lds_ld(L_TOT + 32);
                if (seq < (int)NEARF) {
                    if (lane == 0)
                        __hip_atomic_store((GAS u64*)(myws + OFF_PKIN + (size_t)((isw ? 256 + (grp * 8 + lp) * 8 : (grp * 16 + lp) * 8) + kb) * GSLOT + SFLAG),
                                           ((u64)call << 32) | lo, __ATOMIC_RELAXED, __HIP_MEMORY_SCOPE_SYSTEM);
                } else {
                    announce(myws, wsl, rank, seq, ((u64)call << 32) | lo, (isw ? 0x11u << grp : 0xfu << (4 * grp)) & ~(1u << rank),
                             isw ? 32 + grp * 8 + lp : grp * 16 + lp, lane);
                }
                if (lane == 0 && (isw ? (grp == (rank & 3)) : (grp == (rank >> 2))))
                    __hip_atomic_fetch_add((GAS u32*)(myws + OFF_CNT + (size_t)((isw ? 16 : 0) + lp) * 256 + rank * 4), 1u,
                                           __ATOMIC_RELAXED, __HIP_MEMORY_SCOPE_AGENT);
            }
            const u32 want = call * 8u;
            u32 next_known = 0u;
            for (int tile = 0; tile < 2; ++tile) {
                const char* cp = myws + OFF_CNT + (size_t)((lane & 8) ? 16 + tj : ti0 + 8 * tile) * 256 + (lane & 7) * 4;
                if (tile == 0) {
                    wait_counters((GAS u32*)(myws + OFF_CNT + (size_t)((lane & 1) ? 16 + tj : ti0) * 256 + rank * 4), want, tmo, call);
                    bar(); bar();
                    put_enc_table<true>(L_ETOUT, lane);
                }
                const u32 pfs = (rank + 1 + (b >> 4)) & 7;
                bool pf = tile == 0 && (b >> 3) < 14;
                auto prefetch = [&](u32 wmask) INL {
                    if (pf && ((wmask >> pfs) & 1u)) {
                        const char* p = myws + OFF_UW + ((size_t)tj * 64 + pfs * 8 + half * 4) * IMG + lane * 128;
#pragma unroll
                        for (int j = 0; j < 8; ++j) dma_dword(p + j * 8192, L_SCR + 256);
                        pf = false;
                    }
                };
                u32 known = next_known, wm8 = 0u, left = 0xffu & ~(1u << rank), order = 0u, cw = 0u;
                bool asked = false;
                auto look = [&]() INL {
                    const u32 mk = (u32)__builtin_amdgcn_ballot_w64(cw >= want);
                    wm8 = (mk >> 8) & 0xffu; known = mk & wm8;
                };
                for (int g = 1; g < 8; ++g) {
                    for (int i = 0; i < (g == 1 ? 8 - NST : 7); ++i) bar();
                    if (left) {
                        if (asked) { vm_wait0(); cw = lds_ld(L_SCR + lane * 4); look(); }
                        if (!(known & left)) {
                            const u64 t0 = __builtin_amdgcn_s_memrealtime();
                            for (u32 it = 1;; ++it) {
                                asm volatile("global_load_dword %0, %1, off sc0 sc1\n\ts_waitcnt vmcnt(0)" : "=&v"(cw) : "v"(cp) : "memory");
                                look();
                                if (known & left) break;
                                prefetch(wm8);
                                if ((it & 15u) == 0u
                                    && (__builtin_amdgcn_readfirstlane(__hip_atomic_load(tmo, __ATOMIC_RELAXED, __HIP_MEMORY_SCOPE_AGENT)) == call
                                        || __builtin_amdgcn_s_memrealtime() - t0 > DEADLINE)) {
                                    __hip_atomic_store(tmo, call, __ATOMIC_RELAXED, __HIP_MEMORY_SCOPE_AGENT);
                                    known = 0xffu;
                                    break;
                                }
                                __builtin_amdgcn_s_sleep(2);
                            }
                        }
                        int gg = g;
                        do {
                            const u32 av = known & left, r2 = ((av | (av << 8)) >> tj) & 0xffu;
                            const u32 p = (__builtin_ctz(r2) + tj) & 7u;
                            order |= p << (4 * (gg - 1)); left &= ~(1u << p); ++gg;
                        } while (left && (known & left) == left);
                        lds_st(L_SRC + tile * 4, left ? order : order | 0x80000000u);
                    }
                    bar();
                    if (tile == 1 && !left) break;
                    prefetch(wm8);
                    asked = left != 0u;
                    if (asked) dma_dword(cp, L_SCR);
                }
                if (tile == 0) {
                    const u32 ord = SCHED.ord[0][b];
                    for (int i = 0; i < 62 - (56 - NST) - 3; ++i) bar();
                    dma_dword(myws + OFF_CNT + (size_t)(ti0 + 8) * 256 + (lane & 7) * 4, L_SCR);
                    for (int i = 0; i < 4; ++i) bar();
                    vm_wait0();
                    next_known = (u32)__builtin_amdgcn_ballot_w64(lds_ld(L_SCR + lane * 4) >= want) & 0xffu;
                    if (!((next_known >> rank) & 1u))
                        wait_counters((GAS u32*)(myws + OFF_CNT + (size_t)(ti0 + 8) * 256 + rank * 4), want, tmo, call);
                    for (int i = 0; i < EBAR; ++i) bar();
                    const u32 lo = lds_ld(L_TOT + 32);
                    announce(myws, wsl, rank, OUT0 + ord, ((u64)call << 32) | lo, 0xffu & ~(1u << rank), ord >= OBATCH ? 64 + (int)(ord >> 3) : -1, lane);
                }
            }
            return;
        }
        u32 cv;
        char* wsl;
        asm volatile("global_load_dword %0, %2, off sc1\n\ts_nop 3\n\tglobal_load_dwordx2 %1, %3, %4 offset:%5"
                     : "=&v"(cv), "=&v"(wsl)
                     : "v"(ctr), "v"((lane & 7) * 8), "s"(__builtin_amdgcn_kernarg_segment_ptr()), "n"(offsetof(Params, ws)) : "memory");
        auto fetch = [&](int z, int rho) INL {
            int isw, grp, lp, kb, seq;
            msg_of(rho, isw, grp, lp, kb, seq);
            const u32 dst = lds0 + z * IMG + wave * 2048;
            if (!isw) {
                const char* s = px + ((size_t)((grp * 16 + lp) * 128 + wave * 16 + (lane & 15)) * 512 + kb * 64 + (lane >> 4) * 8) * 2;
                dma1(s, dst); dma1(s + 64, dst + 1024);
            } else {
                const char* s = pw + ((size_t)(kb * 64 + wave * 8 + (lane >> 4)) * 4096 + (grp * 8 + lp) * 128 + (lane & 15) * 8) * 2;
                dma1(s, dst); dma1(s + 4 * 8192, dst + 1024);
            }
        };
        fetch(0, b);
        const u32 call = landed_first<0>(cv) + 1u;
        int isw, grp, lp, kb, seq;
        char* slot;
        u32 d[8];
        u32x2 sample;
        auto take = [&](int z) INL {
            msg_of(pack_msg(b, z), isw, grp, lp, kb, seq);
            const u32 src = lds0 + z * IMG;
            if (!isw) {
                const u32x4 v0 = lds_ld16(src + wave * 2048 + lane * 16), v1 = lds_ld16(src + wave * 2048 + 1024 + lane * 16);
                d[0] = v0.x; d[1] = v0.y; d[2] = v0.z; d[3] = v0.w; d[4] = v1.x; d[5] = v1.y; d[6] = v1.z; d[7] = v1.w;
            } else {
                const u32 a = src + ((lane >> 4) * 8 + ((lane & 15) >> 2)) * 256 + (32 * (wave >> 1) + 8 * (lane & 3) + 4 * (wave & 1)) * 2;
                const u32x2 a0 = __builtin_bit_cast(u32x2, __builtin_amdgcn_ds_read_tr16_b64_v4i16((LAS s16x4*)(a)));
                const u32x2 a1 = __builtin_bit_cast(u32x2, __builtin_amdgcn_ds_read_tr16_b64_v4i16((LAS s16x4*)(a + 4 * 256)));
                const u32x2 a2 = __builtin_bit_cast(u32x2, __builtin_amdgcn_ds_read_tr16_b64_v4i16((LAS s16x4*)(a + 32 * 256)));
                const u32x2 a3 = __builtin_bit_cast(u32x2, __builtin_amdgcn_ds_read_tr16_b64_v4i16((LAS s16x4*)(a + 36 * 256)));
                d[0] = a0.x; d[1] = a0.y; d[2] = a1.x; d[3] = a1.y; d[4] = a2.x; d[5] = a2.y; d[6] = a3.x; d[7] = a3.y;
            }
            sample = *(const LAS u32x2*)(src + lane * 248);
            slot = myws + OFF_PKIN + (size_t)((isw ? 256 + (grp * 8 + lp) * 8 : (grp * 16 + lp) * 8) + kb) * GSLOT;
            store_input_bytes(slot, d, wave, lane);
        };
        bar();
        take(0);
        asm volatile("s_waitcnt vmcnt(0)" : "+v"(wsl) :: "memory");
        bar();
        for (int z = 0;;) {
            const u32 e0 = input_e0(sample);
            encode_block<false>(slot, d, e0, L_ETIN, L_VAR + wave * 1024, L_TOT, wave, lane);
            bar();
            const u32 lo = encode_finish(slot, d, L_VAR + wave * 1024, L_TOT, wave, lane) | (e0 << 16);
            const bool own = isw ? (grp == (rank & 3)) : (grp == (rank >> 2));
            if (own) {
                char* img = myws + (isw ? OFF_UW : OFF_UX) + (size_t)(lp * 64 + rank * 8 + kb) * IMG + wave * 2048 + lane * 16;
                st16_wt(img, u32x4{d[0], d[1], d[2], d[3]}); st16_wt(img + 1024, u32x4{d[4], d[5], d[6], d[7]});
            }
            if (threadIdx.x == 0) lds_st(L_TOT + 32, lo);
            vm_wait0();
            bar();
            if (++z == npass) break;
            take(z);
        }
        u32 ords = __builtin_amdgcn_readfirstlane((u32)SCHED.ord[0][b] | ((u32)SCHED.ord[1][b] << 8));
        asm volatile("s_waitcnt vmcnt(0)" : "+s"(ords) :: "memory");
        const int kh = wave >> 2, wn = wave & 1, wm = (wave >> 1) & 1;
        const char* const wb = myws + OFF_UW + (size_t)tj * 64 * IMG;
        const u32 voffx = wave * 1024 + lane * 16, voffw = wave * 2048 + lane * 16;
        const u32 xa = lds0 + (wm * 4 + kh) * 1024 + lane * 16;
        const u32 wa = lds0 + 8192 + (wn * 8 + kh) * 1024 + lane * 16;
#define SB __builtin_amdgcn_sched_barrier(0)
        u32 lo_last = 0;
        for (int tile = 0; tile < 2; ++tile) {
            const int ti = ti0 + 8 * tile;
            const char* const xb = myws + OFF_UX + (size_t)ti * 64 * IMG + half * 8192;
            f32x4 acc[4][2];
#pragma unroll
            for (int a = 0; a < 4; ++a) { acc[a][0] = f32x4{0.f, 0.f, 0.f, 0.f}; acc[a][1] = acc[a][0]; }
            bf16x8 wA[4], xA[2], wB[4], xB[2];
            auto mma = [&](int a, const bf16x8* wf, const bf16x8* xf) INL {
                acc[a][0] = __builtin_amdgcn_mfma_f32_16x16x32_bf16(wf[a], xf[0], acc[a][0], 0, 0, 0);
                acc[a][1] = __builtin_amdgcn_mfma_f32_16x16x32_bf16(wf[a], xf[1], acc[a][1], 0, 0, 0);
            };
            u32 buf = 0;
            u32 srcs = 0;
            const char* sx = xb + (size_t)rank * 8 * IMG;
            const char* sw = wb + (size_t)rank * 8 * IMG;
            auto step = [&](int kt, auto dma_tag, auto vm_tag, const bf16x8* wc, const bf16x8* xc, bf16x8* wx, bf16x8* xx) INL {
                constexpr bool DMA = decltype(dma_tag)::value != 0, EVEN = decltype(dma_tag)::value == 2;
                constexpr int VM = decltype(vm_tag)::value;
                const u32 d = buf;
                const u32 nb = buf + STG == NST * STG ? 0u : buf + STG;
                const u32 ra = xa + nb, rb = wa + nb;
                buf = nb;
                const u32 st = kt + NST;
                const char* gx = sx + (st & 7u) * IMG; const char* gw = sw + (st & 7u) * IMG;
                asm volatile("" : "+s"(gx), "+s"(gw));
                asm volatile("s_waitcnt lgkmcnt(0)\n\ts_waitcnt vmcnt(%0)\n\ts_barrier" :: "n"(VM) : "memory"); SB;
                if (EVEN && DMA && (st & 7u) == 0u) {
                    if ((int)srcs >= 0) srcs = __builtin_amdgcn_readfirstlane(lds_ld(L_SRC + tile * 4));
                    const u32 so = ((srcs >> (4u * ((st >> 3) - 1u))) & 7u) * (8u * IMG);
                    gx = sx = xb + so; gw = sw = wb + so;
                }
                SB;
                if (DMA) dma_a(gx, voffx, d + wave * 1024); SB;
                mma(0, wc, xc); SB;
                xx[0] = *(const LAS bf16x8*)(ra); wx[0] = *(const LAS bf16x8*)(rb); SB;
                mma(1, wc, xc); SB;
                if (DMA) dma_a(gw, voffw, d + 8192 + wave * 2048); SB;
                xx[1] = *(const LAS bf16x8*)(ra + 2048); wx[1] = *(const LAS bf16x8*)(rb + 2048); SB;
                mma(2, wc, xc); SB;
                wx[2] = *(const LAS bf16x8*)(rb + 4096); wx[3] = *(const LAS bf16x8*)(rb + 6144); SB;
                if (DMA) dma_b(gw, voffw); SB;
                mma(3, wc, xc); SB;
            };
            using F_ = std::integral_constant<int, 0>;
            auto fill = [&](const char* xp, int s0, int s1) INL {
                for (int st = s0; st < s1; ++st) {
                    const u32 kb = (u32)(rank * 8 + st);
                    dma_a(xp + (size_t)kb * IMG, voffx, st * STG + wave * 1024);
                    dma_a(wb + (size_t)kb * IMG, voffw, st * STG + 8192 + wave * 2048);
                    dma_b(wb + (size_t)kb * IMG, voffw);
                }
            };
            if (tile == 0) {
                bar();
                fill(xb, 0, NST);
                asm volatile("s_waitcnt vmcnt(%0)\n\ts_barrier" :: "n"((NST - 1) * 3) : "memory");
            } else {
                fill(xb, 4, NST);
            }
#pragma unroll
            for (int a = 0; a < 4; ++a) wA[a] = *(const LAS bf16x8*)(wa + a * 2048);
            xA[0] = *(const LAS bf16x8*)(xa); xA[1] = *(const LAS bf16x8*)(xa + 2048);
            static_assert(NST == 6, "tail below");
            using VMS = std::integral_constant<int, (NST - 2) * 3>;
            for (int kt = 0; kt < 64 - NST; kt += 2) {
                step(kt, std::integral_constant<int, 2>{}, VMS{}, wA, xA, wB, xB);
                step(kt + 1, std::integral_constant<int, 1>{}, VMS{}, wB, xB, wA, xA);
            }
            step(58, F_{}, std::integral_constant<int, 12>{}, wA, xA, wB, xB);
            step(59, F_{}, std::integral_constant<int, 9>{}, wB, xB, wA, xA);
            step(60, F_{}, std::integral_constant<int, 6>{}, wA, xA, wB, xB);
            step(61, F_{}, std::integral_constant<int, 3>{}, wB, xB, wA, xA);
            step(62, F_{}, std::integral_constant<int, 0>{}, wA, xA, wB, xB);
            u32 tv;
            asm volatile("global_load_dword %0, %1, off sc1" : "=v"(tv) : "v"(tmo) : "memory");
            bar();
            mma(0, wB, xB); mma(1, wB, xB); mma(2, wB, xB); mma(3, wB, xB);
            {
                const u32 wr = L_XCH + wave * 4096 + lane * 16;
#pragma unroll
                for (int a = 0; a < 4; ++a) *(LAS f32x4*)(wr + a * 1024) = kh ? acc[a][0] : acc[a][1];
            }
            bar();
            {
                const u32 rd = L_XCH + (wave ^ 4) * 4096 + lane * 16;
#pragma unroll
                for (int a = 0; a < 4; ++a) {
                    const f32x4 y = (kh ? acc[a][1] : acc[a][0]) + *(const LAS f32x4*)(rd + a * 1024);
                    d[a * 2] = pk_bf16(gelu(y.x), gelu(y.y));
                    d[a * 2 + 1] = pk_bf16(gelu(y.z), gelu(y.w));
                }
            }
            if (landed_first<0>(tv) == call) {
#pragma unroll
                for (int j = 0; j < 8; ++j) d[j] = 0x7fc07fc0u;
            }
            if (tile == 0) fill(xb + (size_t)8 * 64 * IMG, 0, 4);
            const u32 ord = (ords >> (8 * tile)) & 0xffu;
            char* const oslot = myws + OFF_PKOUT + (size_t)ord * GSLOT;
            encode_block<true>(oslot, d, 0u, L_ETOUT, L_VAR + wave * 1024, L_TOT, wave, lane);
            char* o = P.out + ((size_t)((rank >> 2) * 2048 + ti * 128 + half * 64 + wm * 32 + kh * 16 + (lane & 15)) * 4096
                               + (rank & 3) * 1024 + tj * 128 + wn * 64 + (lane >> 4) * 8) * 2;
            st16_out(o, u32x4{d[0], d[1], d[2], d[3]});
            st16_out(o + 64, u32x4{d[4], d[5], d[6], d[7]});
            bar();
            const u32 lo = encode_finish(oslot, d, L_VAR + wave * 1024, L_TOT, wave, lane);
            if (threadIdx.x == 0) lds_st(L_TOT + 32, lo);
            vm_wait0();
            bar();
            lo_last = lo;
        }
        if (wave == 0) {
            const u32 ord = ords >> 8;
            announce(myws, wsl, rank, OUT0 + ord, ((u64)call << 32) | lo_last, 0xffu & ~(1u << rank), ord >= OBATCH ? 64 + (int)(ord >> 3) : -1, lane);
        }
    } else {
        const int pu = b - NGEMM;
        const int u = pu / 7;
        const int src = (rank + 1 + pu % 7) & 7;
        const u32 call = __builtin_amdgcn_readfirstlane(__hip_atomic_load(ctr, __ATOMIC_RELAXED, __HIP_MEMORY_SCOPE_AGENT)) + 1u;
        if (wave == 0) lds_st(lds0 + L_CTL + lane * 4, 0u);
        if (wave == 1) put_dec_table<false>(L_DTBL, lane);
        vm_wait0();
        bar();
        if (threadIdx.x == 0) __hip_atomic_store(ctr, call, __ATOMIC_RELAXED, __HIP_MEMORY_SCOPE_AGENT);
        if (wave == 8) {
            __builtin_amdgcn_s_setprio(3);
            vm_wait0();
            const char* const rws = P.ws[src];
            const char* const fbase = myws + OFF_FLAG + (size_t)src * NMSG * 128;
            char* const tkt = myws + OFF_TKT + (size_t)(pu % 7) * 256;
            const u32 tbase = (call - 1u) * TICKETS - 3 * PER_LINK;
            u32 m = u, n = 0, np = 0;
            u32 org = src, cpy = 0u;
            u32 plan = 0u;
            u32 fl_lo = 0u, fl_hi = 0u;
            u32 pub = 0u;
            bool dead = false;
            auto publish = [&](u32 k) INL {
                if (np > pub + k) { pub = np - k; lds_st(lds0 + L_FULL, pub); }
            };
            auto drain = [&](auto top) INL {
                constexpr int TOP = decltype(top)::value;
                static_for<TOP + 1>([&](auto i) INL {
                    constexpr int K = TOP - decltype(i)::value;
                    asm volatile("s_waitcnt vmcnt(%0)" :: "n"(K) : "memory");
                    publish(K);
                });
            };
            auto poll = [&](const char* fp) INL {
                const u64 t0 = __builtin_amdgcn_s_memrealtime();
                for (u32 it = 1;; ++it) {
                    u32x2 fl;
                    asm volatile("global_load_dwordx2 %0, %1, off sc0 sc1\n\ts_waitcnt vmcnt(0)" : "=&v"(fl) : "v"(fp) : "memory");
                    fl_lo = __builtin_amdgcn_readfirstlane(fl.x); fl_hi = __builtin_amdgcn_readfirstlane(fl.y);
                    if (fl_hi >= call) break;
                    if ((it & 15u) == 0u
                        && (__builtin_amdgcn_readfirstlane(__hip_atomic_load(tmo, __ATOMIC_RELAXED, __HIP_MEMORY_SCOPE_AGENT)) == call
                            || __builtin_amdgcn_s_memrealtime() - t0 > DEADLINE)) {
                        __hip_atomic_store(tmo, call, __ATOMIC_RELAXED, __HIP_MEMORY_SCOPE_AGENT);
                        dead = true;
                        break;
                    }
                    __builtin_amdgcn_s_sleep(1);
                }
            };
            auto remote_slot = [&](u32 mm) INL {
                const u32 t = mm / PER_LINK, v = mm % PER_LINK;
                size_t moff;
                if (t < IN_ROUNDS) {
                    const u32 kb = v & 7;
                    u32 mid;
                    if (t < 8) mid = v >= 8 ? ((rank >> 2) * 16 + t) * 8 + kb : 256 + ((rank & 3) * 8 + t) * 8 + kb;
                    else mid = ((rank >> 2) * 16 + 8 + 2 * (t - 8) + (v >> 3)) * 8 + kb;
                    moff = OFF_PKIN + (size_t)mid * GSLOT;
                } else if (cpy) {
                    moff = OFF_RLY + (size_t)(((org - src - 1u) & 7u) * RN + mm - RW0) * GSLOT;
                } else {
                    moff = OFF_PKOUT + (size_t)(mm - OUT0) * GSLOT;
                }
                return rws + moff;
            };
            auto flag_of = [&](u32 mm) INL { return mm < NEARF || cpy || mm - OUT0 < OBATCH ? remote_slot(mm) + SFLAG : fbase + (mm & ~7u) * 128 + (mm & 7u) * 8; };
            auto resolve = [&](u32 pos) INL {
                m = pos;
                if (pos < RW0) return;
                if (!plan) {
                    plan = __builtin_amdgcn_readfirstlane(lds_ld(lds0 + L_PLAN + 32));
                    if (!plan) {
                        const u64 t0 = __builtin_amdgcn_s_memrealtime();
                        while (!(plan = fetch_plan(tkt, call, lane))) {
                            if (__builtin_amdgcn_s_memrealtime() - t0 > DEADLINE) {
                                __hip_atomic_store(tmo, call, __ATOMIC_RELAXED, __HIP_MEMORY_SCOPE_AGENT);
                                dead = true; plan = 0x8000u;
                                break;
                            }
                            __builtin_amdgcn_s_sleep(2);
                        }
                    }
                }
                const u32 kk = plan & 31u, ee = (plan >> 5) & 31u, x = pos - RW0, cls = pu % 7 % RP;
                m = x < (RP - 1u) * kk ? RW0 + x / (RP - 1u) * RP + x % (RP - 1u) + (x % (RP - 1u) >= cls) : pos + kk;
                if (m >= NMSG + ee) {
                    m -= ee;
                    if (m == NMSG + PER_LINK - 1 && lane == 0)
                        __hip_atomic_fetch_add((GAS u32*)tkt, REMAX + 1 + kk - ee, __ATOMIC_RELAXED, __HIP_MEMORY_SCOPE_AGENT);
                } else if (m >= NMSG) {
                    const u32 z = m - NMSG, w = (__builtin_amdgcn_readfirstlane(lds_ld(lds0 + L_PLAN + (z & ~3u))) >> (8u * (z & 3u))) & 0xffu;
                    const u32 ql = (w >> 5) < 7u ? w >> 5 : 0u, j = (w & 31u) < RK ? w & 31u : 0u;
                    org = (rank + 1u + ql) & 7u; cpy = 1u;
                    m = RW0 + RP * j + ql % RP;
                }
            };
            u32 f2_lo = 0u, f2_hi = 0u, f3_lo = 0u, f3_hi = 0u;
            poll(fbase + m * 128 + 64);
            fl_hi = 0u;
            if (!dead) {
                u32 e2;
                asm volatile("global_load_dword %0, %1, off offset:%2 sc0 sc1" : "=v"(e2) : "v"(fbase + m * 128), "n"(PER_LINK * 128 + 68) : "memory");
                const char* sp = remote_slot(m) + lane * 16;
#pragma unroll
                for (int k = 0; k < 8; ++k) {
                    dma1_sys(sp + k * 1024, lds0 + L_RING + k * 1024);
                    asm volatile("s_waitcnt vmcnt(%0)" :: "n"(WIN - 1) : "memory");
                }
                np = 8u;
                if (landed_first<WIN - 1>(e2) < call) { poll(fbase + (m + PER_LINK) * 128 + 64); fl_hi = 0u; }
            }
            if (!dead) {
                u32x2 fa, fb;
                asm volatile("global_load_dwordx2 %0, %2, off sc0 sc1\n\tglobal_load_dwordx2 %1, %3, off sc0 sc1"
                             : "=&v"(fa), "=&v"(fb) : "v"(flag_of(m)), "v"(flag_of(m + PER_LINK)) : "memory");
                const char* sp = remote_slot(m + PER_LINK) + lane * 16;
#pragma unroll
                for (int k = 0; k < WIN - 1; ++k) {
                    dma1_sys(sp + k * 1024, lds0 + L_RING + SLOT + k * 1024);
                    if (k < WIN - 2) asm volatile("s_waitcnt vmcnt(%0)" :: "n"(WIN + 1) : "memory");
                }
                fl_lo = landed_first<WIN - 1>(fa.x); fl_hi = landed_first<WIN - 1>(fa.y);
                f2_lo = landed_first<WIN - 1>(fb.x); f2_hi = landed_first<WIN - 1>(fb.y);
                np += WIN - 1;
            }
            for (;; ++n) {
                const u32 s = n % NSLOT;
                if (m < NMSG && !dead && fl_hi < call) {
                    drain(std::integral_constant<int, WIN - 1>{});
                    poll(flag_of(m));
                }
                if (n >= NSLOT) {
                    const u32 need = 8u * (n / NSLOT);
                    if (__builtin_amdgcn_readfirstlane(lds_ld(lds0 + L_FREE + s * 4)) < need) {
                        drain(std::integral_constant<int, WIN - 1>{});
                        while (__builtin_amdgcn_readfirstlane(lds_ld(lds0 + L_FREE + s * 4)) < need) __builtin_amdgcn_s_sleep(1);
                    }
                }
                if (m >= NMSG) {
                    lds_st(lds0 + L_MSG + s * 4, NMSG);
                    lds_st(lds0 + L_DESC, n + 1);
                    break;
                }
                const u32 lo = dead ? F_BAD : fl_lo;
                lds_st(lds0 + L_LEN + s * 4, lo);
                const u32 kc = !cpy && !dead && m - RW0 < RP * ((plan >> 10) & 31u) && (m - RW0) % RP != pu % 7 % RP;
                lds_st(lds0 + L_MSG + s * 4, m | (org << 9) | (kc << 12) | (np << 16));
                lds_st(lds0 + L_DESC, n + 1);
                const u32 vb = var_bytes(lo), nv = (vb + 1023u) >> 10;
                const char* sp = remote_slot(m) + ((lo & F_RAW) ? RAW_OFF : 0u) + lane * 16;
                if (n < 2) {
                    if (!dead) {
                        if (lo & F_RAW) vm_wait0();
                        for (u32 o = (lo & F_RAW) ? 0u : 8192u; o < 8192u + vb; o += 1024u) {
                            dma1_sys_part(sp + o, lds0 + L_RING + (n ? SLOT : 0u) + o, 8192u + vb - o);
                            ++np;
                            asm volatile("s_waitcnt vmcnt(%0)" :: "n"(WIN - 1) : "memory");
                            publish(WIN - 1);
                        }
                        if (n == 0) {
                            u32x2 fc;
                            asm volatile("global_load_dwordx2 %0, %1, off sc0 sc1" : "=v"(fc) : "v"(flag_of(m + 2 * PER_LINK)) : "memory");
                            sp = remote_slot(m + PER_LINK) + lane * 16;
#pragma unroll
                            for (int k = WIN - 1; k < 8; ++k) {
                                dma1_sys(sp + k * 1024, lds0 + L_RING + SLOT + k * 1024);
                                ++np;
                                asm volatile("s_waitcnt vmcnt(%0)" :: "n"(WIN - 1) : "memory");
                                publish(WIN - 1);
                            }
                            f3_lo = landed_first<WIN - 1>(fc.x); f3_hi = landed_first<WIN - 1>(fc.y);
                        }
                    }
                    m += PER_LINK; fl_lo = f2_lo; fl_hi = f2_hi; f2_lo = f3_lo; f2_hi = f3_hi;
                    continue;
                }
                u32 nt = 1u;
                u64 keep;
                asm volatile("s_mov_b64 %1, exec\n\ts_mov_b64 exec, 1\n\tglobal_atomic_add %0, %2, %0, off sc0\n\ts_mov_b64 exec, %1"
                             : "+v"(nt), "=&s"(keep) : "v"(tkt) : "memory");
                if (dead) {
                    resolve(landed_first<0>(nt) - tbase);
                    publish(0);
                    continue;
                }
                const u32 d = __builtin_amdgcn_readfirstlane(lds0 + L_RING + s * SLOT);
                asm volatile("s_waitcnt vmcnt(%0)" :: "n"(WIN) : "memory");
                publish(WIN - 1);
                {
                    const u32 o = 8192u + (nv - 1u) * 1024u;
                    dma1_sys_part(sp + o, d + o, 8192u + vb - o);
                    ++np;
                }
                u32 left = nv + 7u, o = nv > 1u ? 8192u + (nv - 2u) * 1024u : 0u;
                auto piece = [&]() INL {
                    dma1_sys(sp + o, d + o);
                    o = o > 8192u ? o - 1024u : (o == 8192u ? 0u : o + 1024u);
                    ++np; --left;
                };
#pragma unroll
                for (int k = 0; k < WIN - 1; ++k) {
                    asm volatile("s_waitcnt vmcnt(%0)" :: "n"(WIN) : "memory");
                    publish(WIN - 1);
                    piece();
                }
                const u32 pos = landed_first<WIN - 1>(nt) - tbase;
                publish(WIN - 1);
                piece();
                resolve(pos);
                if (m >= NMSG) {
                    while (left) {
                        asm volatile("s_waitcnt vmcnt(%0)" :: "n"(WIN - 1) : "memory");
                        publish(WIN - 1);
                        piece();
                    }
                    drain(std::integral_constant<int, WIN - 1>{});
                } else {
                    u32x2 fn;
                    asm volatile("global_load_dwordx2 %0, %1, off sc0 sc1" : "=v"(fn) : "v"(flag_of(m)) : "memory");
#pragma unroll
                    for (int k = 0; k < WIN - 1; ++k) {
                        asm volatile("s_waitcnt vmcnt(%0)" :: "n"(WIN) : "memory");
                        publish(WIN - 1);
                        piece();
                    }
                    fl_lo = landed_first<WIN - 1>(fn.x); fl_hi = landed_first<WIN - 1>(fn.y);
                    publish(WIN - 1);
                    piece();
                    if (fl_hi < call && left >= 1u) {
                        while (left > 1u) {
                            asm volatile("s_waitcnt vmcnt(%0)" :: "n"(WIN - 1) : "memory");
                            publish(WIN - 1);
                            piece();
                        }
                        u32x2 fr;
                        asm volatile("global_load_dwordx2 %0, %1, off sc0 sc1" : "=v"(fr) : "v"(flag_of(m)) : "memory");
                        asm volatile("s_waitcnt vmcnt(%0)" :: "n"(WIN) : "memory");
                        publish(WIN - 1);
                        piece();
                        asm volatile("s_waitcnt vmcnt(3)" ::: "memory"); publish(2);
                        asm volatile("s_waitcnt vmcnt(2)" ::: "memory"); publish(1);
                        fl_lo = landed_first<1>(fr.x); fl_hi = landed_first<1>(fr.y);
                    }
                    while (left) {
                        asm volatile("s_waitcnt vmcnt(%0)" :: "n"(WIN - 1) : "memory");
                        publish(WIN - 1);
                        piece();
                    }
                }
            }
        } else {
            if (wave == 2) {
                put_dec_table<true>(L_DTBL + 256, lane);
                *(LAS u32*)(L_UNORD + lane * 4) = ((const u32*)SCHED.unord)[lane];
                lgkm_wait0();
                lds_st(lds0 + L_TRDY, 1u);
            }
            bool tables = false, plan = false;
            char* pend = nullptr;
            u32 pend_n = 0, pend_lo = 0;
            auto report = [&]() INL {
                if (!pend) return;
                vm_wait0();
                u32 old = 0;
                if (lane == 0) {
                    asm volatile("ds_add_rtn_u32 %0, %1, %2\n\ts_waitcnt lgkmcnt(0)" : "=&v"(old) : "v"(lds0 + L_DONE + pend_n % 12u * 4), "v"(1u) : "memory");
                    if ((old & 7u) == 7u) {
                        if (pend_lo) __hip_atomic_store((GAS u64*)pend, ((u64)call << 32) | pend_lo, __ATOMIC_RELAXED, __HIP_MEMORY_SCOPE_SYSTEM);
                        else __hip_atomic_fetch_add((GAS u32*)pend, 1u, __ATOMIC_RELAXED, __HIP_MEMORY_SCOPE_AGENT);
                    }
                }
                pend = nullptr;
            };
            for (u32 n = 0;; ++n) {
                const u32 s = n % NSLOT;
                if (__builtin_amdgcn_readfirstlane(lds_ld(lds0 + L_DESC)) <= n) {
                    report();
                    while (__builtin_amdgcn_readfirstlane(lds_ld(lds0 + L_DESC)) <= n) __builtin_amdgcn_s_sleep(1);
                }
                const u32 md = __builtin_amdgcn_readfirstlane(lds_ld(lds0 + L_MSG + s * 4)), m = md & 0x1ffu, org = (md >> 9) & 7u;
                if (m >= NMSG) break;
                if (wave == 1 && m >= OUT0 && !plan) plan = fetch_plan(myws + OFF_TKT + (size_t)(pu % 7) * 256, call, lane);
                const u32 t = m / PER_LINK, v = m % PER_LINK;
                const u32 lo = __builtin_amdgcn_readfirstlane(lds_ld(lds0 + L_LEN + s * 4));
                const u32 sl = lds0 + L_RING + s * SLOT;
                if (m >= OUT0 && !tables) {
                    while (!__builtin_amdgcn_readfirstlane(lds_ld(lds0 + L_TRDY))) __builtin_amdgcn_s_sleep(1);
                    tables = true;
                }
                const u32 nv = (var_bytes(lo) + 1023u) >> 10;
                const bool nib = m >= OUT0 && !(lo & F_RAW) && !(md & 0x1000u);
                const u32 need = (md >> 16) + (n >= 2 ? nv + 1u + (nib ? wave >> 1 : (m >= OUT0 && !(lo & F_RAW) ? 4 + (wave >> 1) > wave ? 4 + (wave >> 1) : wave : wave))
                                                       : ((lo & F_RAW) ? 16u : nv));
                const u32 need2 = (md >> 16) + nv + 5u + (wave >> 1);
                if (!(lo & F_BAD) && __builtin_amdgcn_readfirstlane(lds_ld(lds0 + L_FULL)) < need) {
                    report();
                    while (__builtin_amdgcn_readfirstlane(lds_ld(lds0 + L_FULL)) < need) __builtin_amdgcn_s_sleep(1);
                }
                auto wait_low = [&]() INL {
                    while (__builtin_amdgcn_readfirstlane(lds_ld(lds0 + L_FULL)) < need2) __builtin_amdgcn_s_sleep(1);
                };
                const bool keep = md & 0x1000u;
                char* const cslot = myws + OFF_RLY + (size_t)((pu % 7) * RN + m - RW0) * GSLOT;
                if (keep) {
                    report();
                    char* p = cslot + ((lo & F_RAW) ? RAW_OFF : 0u) + wave * 1024 + lane * 16;
                    st16_wt(p, lds_ld16(sl + wave * 1024 + lane * 16));
                    if ((u32)(wave * 1024 + lane * 16) < var_bytes(lo)) st16_wt(p + 8192, lds_ld16(sl + 8192 + wave * 1024 + lane * 16));
                }
                u32x4 v0, v1;
                if (lo & F_BAD) {
                    v0 = u32x4{0x7fc07fc0u, 0x7fc07fc0u, 0x7fc07fc0u, 0x7fc07fc0u}; v1 = v0;
                } else if (lo & F_RAW) {
                    v0 = lds_ld16(sl + 8192 + wave * 1024 + lane * 16);
                    v1 = lds_ld16(sl + wave * 1024 + lane * 16);
                } else if (t < IN_ROUNDS) {
                    decode_block<false>(sl, (lo >> 16) & 0xffu, L_DTBL, wave, lane, v0, v1, wait_low);
                } else {
                    decode_block<true>(sl, 0u, L_DTBL + 256, wave, lane, v0, v1, wait_low);
                }
                asm volatile("s_waitcnt lgkmcnt(0)" : "+v"(v0), "+v"(v1) :: "memory");
                if (lane == 0) asm volatile("ds_add_u32 %0, %1" :: "v"(lds0 + L_FREE + s * 4), "v"(1u) : "memory");
                if (t < IN_ROUNDS) {
                    const u32 kb = v & 7;
                    const bool isw = t < 8 && v < 8;
                    const u32 lp = t < 8 ? t : 8 + 2 * (t - 8) + (v >> 3);
                    char* img = myws + (isw ? OFF_UW : OFF_UX) + (size_t)(lp * 64 + src * 8 + kb) * IMG + wave * 2048 + lane * 16;
                    report();
                    st16_wt(img, v0); st16_wt(img + 1024, v1);
                    pend = myws + OFF_CNT + (size_t)((isw ? 16 : 0) + lp) * 256 + src * 4; pend_n = n; pend_lo = 0u;
                } else {
                    const int g = __builtin_amdgcn_readfirstlane(*(const LAS unsigned char*)(L_UNORD + m - OUT0));
                    const int kh = wave >> 2, wn = wave & 1, wm = (wave >> 1) & 1;
                    const int ti = ((g >> 4) & 7) + 8 * (g >> 7), hf = (g >> 3) & 1, tj = g & 7;
                    report();
                    if (keep) { pend = cslot + SFLAG; pend_n = n; pend_lo = lo; }
                    char* op = P.out + ((size_t)((org >> 2) * 2048 + ti * 128 + hf * 64 + wm * 32 + kh * 16 + (lane & 15)) * 4096
                                        + (org & 3) * 1024 + tj * 128 + wn * 64 + (lane >> 4) * 8) * 2;
                    st16_out(op, v0); st16_out(op + 64, v1);
                }
            }
            report();
        }
    }
}

extern "C" void kernel_launch(int rank, int nranks, ncclComm_t comm,
                              hipStream_t stream,
                              void* const* d_in, const int* in_sizes, int n_in,
                              void* d_out, int out_size,
                              const int* in_dtypes, int out_dtype,
                              void* const* d_out_all,
                              void* const* d_ws_all, size_t ws_bytes) {
    (void)comm; (void)in_dtypes; (void)out_dtype; (void)d_out_all;
    if (nranks != NR || (unsigned)rank >= (unsigned)NR || n_in != 2 || in_sizes[0] != 4096 * 512 || in_sizes[1] != 512 * 4096
        || out_size != 4096 * 4096 || ws_bytes < WS_NEED) {
        fprintf(stderr, "kernel_launch: unexpected problem shape\n");
        return;
    }
    static bool attr_set[NR] = {};
    if (!attr_set[rank]) {
        HIP_CHECK(hipFuncSetAttribute((const void*)fused_gemm_allgather, hipFuncAttributeMaxDynamicSharedMemorySize, LDS_BYTES));
        attr_set[rank] = true;
    }
    Params P;
    for (int p = 0; p < NR; ++p) P.ws[p] = (char*)d_ws_all[p];
    P.x = (const char*)d_in[0];
    P.w = (const char*)d_in[1];
    P.out = (char*)d_out;
    P.myws = P.ws[rank];
    P.rank = rank;
    P.pad = 0;
    hipLaunchKernelGGL(fused_gemm_allgather, dim3(GRID), dim3(THREADS), LDS_BYTES, stream, P);
    HIP_CHECK(hipGetLastError());
}
```

```cpp
#include <hip/hip_runtime.h>
#include <rccl/rccl.h>
#include <cstdio>
#include <cstdint>
#include <cstddef>
#include <type_traits>
#pragma clang diagnostic ignored "-Wint-to-pointer-cast"

#define HIP_CHECK(expr) do { hipError_t _e = (expr); if (_e != hipSuccess) { \
    fprintf(stderr, "HIP error %s at %s:%d\n", hipGetErrorString(_e), __FILE__, __LINE__); \
    return; } } while (0)

typedef unsigned u32;
typedef unsigned long long u64;
typedef u32 u32x4 __attribute__((ext_vector_type(4)));
typedef u32 u32x2 __attribute__((ext_vector_type(2)));
typedef float f32x4 __attribute__((ext_vector_type(4)));
typedef float f32x2 __attribute__((ext_vector_type(2)));
typedef __bf16 bf16x8 __attribute__((ext_vector_type(8)));
typedef __bf16 bf16x2 __attribute__((ext_vector_type(2)));
typedef short s16x4 __attribute__((ext_vector_type(4)));
#define INL __attribute__((always_inline))
#define LAS __attribute__((address_space(3)))
#define GAS __attribute__((address_space(1)))

constexpr int NR = 8;
constexpr int NGEMM = 128;
constexpr int PER_LINK = 16;
constexpr int NPULL = 7 * PER_LINK;
constexpr int NROLE = NGEMM + NPULL;
constexpr int GRID = 256;
constexpr int THREADS = 576;
constexpr int IN_ROUNDS = 12, OUT_ROUNDS = 16, ROUNDS = IN_ROUNDS + OUT_ROUNDS;
constexpr u32 OUT0 = IN_ROUNDS * PER_LINK;
constexpr u32 NMSG = ROUNDS * PER_LINK;
constexpr u32 SLOT = 16384 + 512;
constexpr u32 GSLOT = 2 * SLOT;
constexpr u32 NEARF = 3 * PER_LINK, SFLAG = SLOT - 128;
constexpr u32 OBATCH = 96;
static_assert(OBATCH % 8 == 0 && NEARF % 8 == 0, "whole groups");
constexpr u32 RK = 20, RP = 3, RW0 = OUT0 + 112, RN = RP * RK, REMAX = 31;
constexpr u32 TICKETS = NMSG - 2 * PER_LINK + REMAX + 1;
static_assert(RK < 32 && RW0 + RN <= NMSG, "window");
constexpr u32 IMG = 16384;
constexpr int NSLOT = 8;
constexpr int WIN = 4;
static_assert(2 * WIN + 1 <= 9, "a message has at least 9 pieces");
constexpr unsigned long long DEADLINE = 100000000ull;

constexpr size_t OFF_CTR = 0;
constexpr size_t OFF_TMO = 32768;
constexpr size_t OFF_CNT = 36864;
constexpr size_t OFF_TKT = 45056;
constexpr size_t OFF_KEEP = 47104;
constexpr size_t OFF_GRP = 49152;
constexpr size_t OFF_FLAG = 65536;
constexpr size_t OFF_PKIN = 1u << 20;
constexpr size_t OFF_PKOUT = 18u << 20;
constexpr size_t OFF_UX = 28u << 20;
constexpr size_t OFF_UW = 44u << 20;
constexpr size_t OFF_RLY = 52u << 20;
constexpr size_t WS_NEED = 68u << 20;
static_assert(OFF_RLY + (size_t)7 * RN * GSLOT <= WS_NEED, "copies");
static_assert(OFF_FLAG + (size_t)NR * NMSG * 128 <= OFF_PKIN, "flags");
static_assert(OFF_PKIN + 512 * (size_t)GSLOT <= OFF_PKOUT, "pkin");
static_assert(OFF_PKOUT + 256 * (size_t)GSLOT <= OFF_UX, "pkout");

constexpr u32 L_RING = 0;
constexpr u32 L_CTL = NSLOT * SLOT;
constexpr u32 L_FULL = L_CTL;
constexpr u32 L_DESC = L_CTL + 4;
constexpr u32 L_PLAN = L_CTL + 192;
constexpr u32 L_TRDY = L_CTL + 240;
constexpr u32 L_FREE = L_CTL + 16;
constexpr u32 L_DONE = L_CTL + 144;
constexpr u32 L_LEN = L_CTL + 16 + 8 * NSLOT;
constexpr u32 L_MSG = L_CTL + 16 + 12 * NSLOT;
constexpr u32 L_DTBL = L_CTL + 256;
constexpr u32 L_UNORD = L_DTBL + 768;
constexpr int NST = 6;
constexpr u32 STG = 8192 + IMG;
constexpr u32 L_XCH = 4 * STG;
constexpr u32 L_VAR = L_XCH + 32768;
constexpr u32 L_ETIN = L_VAR + 8192;
constexpr u32 L_ETOUT = NST * STG;
constexpr u32 L_TOT = L_ETOUT + 2048;
constexpr u32 L_SRC = L_TOT + 48;
constexpr u32 L_SCR = L_TOT + 64;
constexpr u32 LDS_BYTES = L_SCR + 512;
static_assert(LDS_BYTES >= L_UNORD + 256 && LDS_BYTES <= 163840 && L_ETIN + 4096 <= L_ETOUT, "lds");

constexpr int pack_passes(int b) {
    return (b >> 4) <= 1 && (b & 7) <= 1 ? 2 : ((b & 7) == 7 ? 5 : 4);
}
constexpr int pack_msg(int b, int z) {
    const int early = b < 32 ? 2 * (b >> 3) + ((b & 7) < 2 ? (b & 7) : 2) : 8;
    return z < 2 ? z * 128 + b : (z < 4 ? 256 + (z - 2) * 120 + b - early : 496 + (b >> 3));
}
__device__ __forceinline__ void msg_of(int rho, int& isw, int& grp, int& lp, int& kb, int& seq) {
    if (rho < 384) {
        const int t = rho / 48, r = rho % 48;
        isw = r < 32; grp = isw ? r >> 3 : (r - 32) >> 3; kb = r & 7; lp = t; seq = t * 16 + (isw ? 0 : 8) + kb;
    } else {
        const int t = 8 + ((rho - 384) >> 5), r = rho & 31, pos = (r >> 3) & 1;
        isw = 0; grp = r >> 4; kb = r & 7; lp = 8 + 2 * (t - 8) + pos; seq = t * 16 + pos * 8 + kb;
    }
}
struct Sched { alignas(4) unsigned char unord[256]; unsigned char ord[2][128]; bool ok; };
constexpr Sched make_sched() {
    Sched s{};
    int r = 0;
    s.ok = true;
    for (int z = 0; z < 5; ++z)
        for (int b = 0; b < 128; ++b)
            if (pack_passes(b) > z && pack_msg(b, z) != r++) s.ok = false;
    if (r != 512) s.ok = false;
    int key[256] = {};
    for (int b = 0; b < 128; ++b) {
        const int i = b >> 4, j = b & 7, sh = i > j ? i : j;
        const int t1 = 30 * sh + 170, arr2 = 30 * (8 + (i >> 1));
        key[b] = t1 * 256 + b;
        key[128 + b] = ((t1 > arr2 ? t1 : arr2) + 170) * 256 + 128 + b;
    }
    for (int a = 0; a < 256; ++a) {
        int pos = 0;
        for (int c = 0; c < 256; ++c) pos += key[c] < key[a];
        s.ord[a >> 7][a & 127] = (unsigned char)pos;
        s.unord[pos] = (unsigned char)a;
    }
    return s;
}
__device__ constexpr Sched SCHED = make_sched();
static_assert(make_sched().ok, "every input message is packed once");

constexpr u32 F_RAW = 0x80000000u;
constexpr u32 F_BAD = 0x40000000u;

__device__ __forceinline__ u32 var_bytes(u32 lo) {
    const u32 v = lo & 0xffffu;
    return (lo & F_RAW) || v > 8192u ? 8192u : (v < 16u ? 16u : v);
}
template <int N, int I = 0, class F> __device__ __forceinline__ void static_for(F f) {
    if constexpr (I < N) { f(std::integral_constant<int, I>{}); static_for<N, I + 1>(f); }
}

struct Params {
    const char* x;
    const char* w;
    char* myws;
    char* out;
    int rank;
    int pad;
    char* ws[NR];
};


__device__ __forceinline__ void st16_wt(char* p, u32x4 v) {
    asm volatile("global_store_dwordx4 %0, %1, off sc0 sc1\n\ts_nop 1" :: "v"(p), "v"(v) : "memory");
}
__device__ __forceinline__ void st16_out(char* p, u32x4 v) {
    asm volatile("global_store_dwordx4 %0, %1, off sc0 sc1 nt\n\ts_nop 1" :: "v"(p), "v"(v) : "memory");
}
__device__ __forceinline__ void st8_wt(char* p, u32x2 v) {
    asm volatile("global_store_dwordx2 %0, %1, off sc0 sc1" :: "v"(p), "v"(v) : "memory");
}
__device__ __forceinline__ void st4_wt(char* p, u32 v) {
    asm volatile("global_store_dword %0, %1, off sc0 sc1" :: "v"(p), "v"(v) : "memory");
}
__device__ __forceinline__ void vm_wait0() { asm volatile("s_waitcnt vmcnt(0)" ::: "memory"); }
__device__ __forceinline__ void lgkm_wait0() { asm volatile("s_waitcnt lgkmcnt(0)" ::: "memory"); }
__device__ __forceinline__ void bar() { asm volatile("s_waitcnt lgkmcnt(0)\n\ts_barrier" ::: "memory"); }
__device__ __forceinline__ void lds_st(u32 addr, u32 v) { asm volatile("ds_write_b32 %0, %1" :: "v"(addr), "v"(v) : "memory"); }
__device__ __forceinline__ u32 lds_ld(u32 addr) {
    u32 v; asm volatile("ds_read_b32 %0, %1\n\ts_waitcnt lgkmcnt(0)" : "=v"(v) : "v"(addr) : "memory"); return v;
}
__device__ __forceinline__ u32x4 lds_ld16(u32 addr) { return *(const LAS u32x4*)addr; }
template <int N> __device__ __forceinline__ u32 landed_first(u32 v) {
    u32 r; asm volatile("s_waitcnt vmcnt(%2)\n\tv_readfirstlane_b32 %0, %1\n\ts_nop 3" : "=s"(r) : "v"(v), "n"(N) : "memory"); return r;
}
__device__ __forceinline__ void dma1(const char* src, u32 lds_dst) {
    u32 keep;
    asm volatile("s_mov_b32 %0, m0\n\ts_mov_b32 m0, %2\n\ts_nop 0\n\tglobal_load_lds_dwordx4 %1, off\n\ts_mov_b32 m0, %0"
                 : "=&s"(keep) : "v"(src), "s"(lds_dst) : "memory");
}
__device__ __forceinline__ void dma1_sys(const char* src, u32 lds_dst) {
    u32 keep;
    asm volatile("s_mov_b32 %0, m0\n\ts_mov_b32 m0, %2\n\ts_nop 0\n\tglobal_load_lds_dwordx4 %1, off sc0 sc1\n\ts_mov_b32 m0, %0"
                 : "=&s"(keep) : "v"(src), "s"(lds_dst) : "memory");
}
__device__ __forceinline__ void dma1_sys_part(const char* src, u32 lds_dst, u32 bytes) {
    const u64 mask = bytes > 1008u ? ~0ull : (1ull << ((bytes + 15u) >> 4)) - 1ull;
    u64 keep;
    asm volatile("s_mov_b64 %0, exec\n\ts_mov_b32 m0, %2\n\ts_mov_b64 exec, %3\n\tglobal_load_lds_dwordx4 %1, off sc0 sc1\n\ts_mov_b64 exec, %0"
                 : "=&s"(keep) : "v"(src), "s"(__builtin_amdgcn_readfirstlane(lds_dst)), "s"(mask) : "memory");
}
__device__ __forceinline__ void dma_dword(const char* src, u32 lds_dst) {
    asm volatile("s_mov_b32 m0, %1\n\ts_nop 0\n\tglobal_load_lds_dword %0, off sc0 sc1" :: "v"(src), "s"(lds_dst) : "memory");
}
__device__ __forceinline__ void dma1s(const char* sbase, u32 voff, u32 lds_dst) {
    u64 t;
    asm volatile("s_mov_b32 m0, %3\n\ts_mov_b64 %0, %2\n\tglobal_load_lds_dwordx4 %1, %0" : "=&s"(t) : "v"(voff), "s"(sbase), "s"(lds_dst) : "memory");
}
__device__ __forceinline__ void dma_a(const char* sbase, u32 voff, u32 lds_dst) {
    u64 t;
    asm volatile("s_mov_b32 m0, %3\n\ts_mov_b64 %0, %2\n\tglobal_load_lds_dwordx4 %1, %0 sc0 sc1" : "=&s"(t) : "v"(voff), "s"(sbase), "s"(lds_dst) : "memory");
}
__device__ __forceinline__ void dma_b(const char* sbase, u32 voff) {
    u64 t;
    asm volatile("s_mov_b64 %0, %2\n\tglobal_load_lds_dwordx4 %1, %0 offset:1024 sc0 sc1" : "=&s"(t) : "v"(voff), "s"(sbase) : "memory");
}
__device__ __forceinline__ void announce(char* myws, char* wsl, int rank, u32 seq, u64 val, u32 destmask, int grp, int lane) {
    const size_t line = OFF_FLAG + ((size_t)rank * NMSG + (seq & ~7u)) * 128;
    if (grp < 0) {
        if (lane == 0)
            __hip_atomic_store((GAS u64*)(myws + OFF_PKOUT + (size_t)(seq - OUT0) * GSLOT + SFLAG), val, __ATOMIC_RELAXED, __HIP_MEMORY_SCOPE_SYSTEM);
        return;
    }
    char* g = myws + OFF_GRP + (size_t)grp * 128;
    u32 old = 0;
    if (lane == 0) {
        __hip_atomic_store((GAS u64*)(g + (seq & 7u) * 8), val, __ATOMIC_RELAXED, __HIP_MEMORY_SCOPE_AGENT);
        asm volatile("s_waitcnt vmcnt(0)" ::: "memory");
        old = __hip_atomic_fetch_add((GAS u32*)(g + 64), 1u, __ATOMIC_RELAXED, __HIP_MEMORY_SCOPE_AGENT);
    }
    if ((__builtin_amdgcn_readfirstlane(old) & 7u) == 7u) {
        const u64 e = __hip_atomic_load((GAS u64*)(g + (lane & 7) * 8), __ATOMIC_RELAXED, __HIP_MEMORY_SCOPE_AGENT);
        const u64 wl = (u64)wsl;
        char* wp = (char*)((u64)(u32)__builtin_amdgcn_ds_bpermute((lane >> 3) * 4, (int)(u32)wl)
                           | (u64)(u32)__builtin_amdgcn_ds_bpermute((lane >> 3) * 4, (int)(u32)(wl >> 32)) << 32);
        if ((destmask >> (lane >> 3)) & 1u)
            __hip_atomic_store((GAS u64*)(wp + line + (lane & 7) * 8), e, __ATOMIC_RELAXED, __HIP_MEMORY_SCOPE_SYSTEM);
    }
}
__device__ __forceinline__ u32 fetch_plan(char* tkt, u32 call, int lane) {
    const u32 h = __builtin_amdgcn_readfirstlane(__hip_atomic_load((GAS u32*)(tkt + 128), __ATOMIC_RELAXED, __HIP_MEMORY_SCOPE_AGENT));
    if ((h >> 15) != (call & 0x1ffffu)) return 0u;
    const u32 w = __hip_atomic_load((GAS u32*)(tkt + 132 + (lane & 7) * 4), __ATOMIC_RELAXED, __HIP_MEMORY_SCOPE_AGENT);
    if (lane < 8) lds_st(L_PLAN + lane * 4, w);
    lds_st(L_PLAN + 32, (h & 0x7fffu) | 0x8000u);
    return (h & 0x7fffu) | 0x8000u;
}
__device__ __forceinline__ void wait_counters(GAS u32* c, u32 want, GAS u32* tmo, u32 call) {
    const u64 t0 = __builtin_amdgcn_s_memrealtime();
    for (u32 it = 1; !__all(__hip_atomic_load(c, __ATOMIC_RELAXED, __HIP_MEMORY_SCOPE_SYSTEM) >= want); ++it) {
        if ((it & 15u) == 0u
            && (__builtin_amdgcn_readfirstlane(__hip_atomic_load(tmo, __ATOMIC_RELAXED, __HIP_MEMORY_SCOPE_AGENT)) == call
                || __builtin_amdgcn_s_memrealtime() - t0 > DEADLINE)) {
            __hip_atomic_store(tmo, call, __ATOMIC_RELAXED, __HIP_MEMORY_SCOPE_AGENT);
            break;
        }
        __builtin_amdgcn_s_sleep(2);
    }
    asm volatile("s_waitcnt vmcnt(0)" ::: "memory");
}

__device__ __forceinline__ float gelu(float y) {
    const float k0 = -2.0f * 0.7978845608028654f * 1.4426950408889634f;
    const float u = y * y;
    const float z = y * __builtin_fmaf(u, k0 * 0.044715f, k0);
    return y * __builtin_amdgcn_rcpf(1.0f + __builtin_amdgcn_exp2f(z));
}
__device__ __forceinline__ u32 pk_bf16(float a, float b) {
    f32x2 f = {a, b};
    return __builtin_bit_cast(u32, __builtin_convertvector(f, bf16x2));
}

constexpr u64 IN_REL0 = 0x1234567'8d9abcull, IN_REL1 = 0x123d456'789cabull;
constexpr u32 OUT_E0 = 114, OUT_NSYM = 184;
__device__ constexpr unsigned char OUT_RANK[192] = {
    210, 209, 208, 207, 104, 206, 96, 105, 205, 97, 204, 98, 203, 106, 99, 202, 100, 201, 88, 101, 107, 200, 89,
    90, 102, 91, 103, 92, 108, 93, 94, 95, 109, 112, 80, 81, 82, 83, 84, 110, 85, 86, 87, 199, 198, 111,
    197, 196, 195, 194, 193, 192, 113, 72, 73, 74, 75, 76, 77, 78, 79, 114, 65, 191, 64, 66, 190, 67, 189,
    68, 188, 69, 187, 70, 71, 186, 185, 184, 115, 57, 58, 56, 59, 60, 61, 62, 63, 183, 182, 181, 180, 179,
    178, 177, 176, 116, 49, 50, 51, 52, 53, 48, 54, 55, 175, 174, 173, 172, 171, 170, 169, 168, 41, 42, 43,
    44, 45, 46, 47, 40, 117, 167, 166, 165, 164, 163, 162, 161, 160, 33, 34, 35, 36, 37, 38, 39, 32, 118,
    159, 158, 157, 156, 155, 154, 153, 152, 25, 26, 27, 28, 29, 30, 31, 24, 151, 150, 149, 147, 148, 146, 145,
    144, 119, 17, 18, 19, 20, 21, 23, 22, 16, 137, 138, 139, 140, 141, 142, 143, 136, 11, 9, 10, 12, 13};
__device__ __forceinline__ constexpr u32x2 enc_words(u32 c1, u32 q) {
    const u32 c2 = q < 3u ? q : 3u, c3 = q < 3u ? 0u : (q < 6u ? q - 3u : 3u), c4 = q < 6u ? 0u : q - 6u;
    return u32x2{(c1 << 28) | c2 | (c3 << 8) | (c4 << 16), 2u | (q >= 3u ? 2u << 8 : 0u) | (q >= 6u ? 3u << 16 : 0u)};
}
template <bool OUT> __device__ __forceinline__ void put_enc_table(u32 l, int lane) {
    const u32x2 esc = enc_words(OUT ? 15u : 0u, 13u);
#pragma unroll
    for (int j = 0; j < (OUT ? 2 : 4); ++j)
        *(LAS u32x4*)(l + j * 1024 + lane * 16) = u32x4{esc.x, esc.y | (1u << 24), esc.x, esc.y | (1u << 24)};
    if (OUT) {
#pragma unroll
        for (u32 r = lane; r < 192u; r += 64u)
            if (r < OUT_NSYM) *(LAS u32x2*)(l + OUT_RANK[r] * 8u) = r < 3u ? u32x2{r << 28, 0u} : enc_words(3u + (r - 3u) % 13u, (r - 3u) / 13u);
    } else {
        const u32 q = lane & 15u, m = ((u32)lane >> 4) & 1u;
        if (lane < 32 && q < 13u) *(LAS u32x2*)(l + ((((u32)((m ? IN_REL1 : IN_REL0) >> (4u * q)) & 15u) << 1) | m) * 8u) = enc_words(0u, q);
    }
}
template <bool OUT> __device__ __forceinline__ void put_dec_table(u32 l, int lane) {
    if (OUT) {
#pragma unroll
        for (u32 i = lane; i < 256u; i += 64u) {
            const u32 c1 = i >> 4, q = i & 15u, r = c1 < 3u ? c1 : 13u * q + c1, x = OUT_RANK[r < OUT_NSYM ? r : 0u];
            *(LAS unsigned short*)(l + i * 2u) =
                (unsigned short)(r < OUT_NSYM ? ((x >> 7) << 15) | ((((x >> 3) & 15u) + OUT_E0) << 7) | ((x & 7u) << 4) : 1u);
        }
    } else {
        const u32 c1 = (u32)lane >> 4, q = lane & 15u;
        *(LAS unsigned short*)(l + c1 * 64u + q * 2u) = (unsigned short)(
            q < 13u ? ((c1 >> 1) << 15) | (((u32)(((c1 & 1u) ? IN_REL1 : IN_REL0) >> (4u * q)) & 15u) << 7) | ((c1 & 1u) << 6) : 1u);
    }
}

__device__ __forceinline__ u32 wave_scan(u32 v) {
    v += (u32)__builtin_amdgcn_update_dpp(0, (int)v, 0x111, 0xf, 0xf, false);
    v += (u32)__builtin_amdgcn_update_dpp(0, (int)v, 0x112, 0xf, 0xf, false);
    v += (u32)__builtin_amdgcn_update_dpp(0, (int)v, 0x114, 0xf, 0xf, false);
    v += (u32)__builtin_amdgcn_update_dpp(0, (int)v, 0x118, 0xf, 0xf, false);
    v += (u32)__builtin_amdgcn_update_dpp(0, (int)v, 0x142, 0xa, 0xf, false);
    v += (u32)__builtin_amdgcn_update_dpp(0, (int)v, 0x143, 0xc, 0xf, false);
    return v;
}
__device__ __forceinline__ u32 ubfe(u32 v, u32 off, u32 w) { return __builtin_amdgcn_ubfe(v, off, w); }
__device__ __forceinline__ void lds_or_bits(u32 base, u32 pos, u32 bits) {
    if (bits) {
        LAS u32* p = (LAS u32*)(base + (pos >> 5) * 4u);
        __hip_atomic_fetch_or(p, bits << (pos & 31u), __ATOMIC_RELAXED, __HIP_MEMORY_SCOPE_WORKGROUP);
        __hip_atomic_fetch_or(p + 1, (bits >> 1) >> (~pos & 31u), __ATOMIC_RELAXED, __HIP_MEMORY_SCOPE_WORKGROUP);
    }
}
__device__ __forceinline__ u32 lds_get_bits(u32 base, u32 pos) {
    const LAS u32* p = (const LAS u32*)(base + (pos >> 5) * 4u);
    return __builtin_amdgcn_alignbit(p[1], p[0], pos & 31u);
}
__device__ __forceinline__ u32 low_mask_even(u32 n) { return ~((0xffffffffu << (n >> 1)) << (n >> 1)); }

#define SDWA_SHL(B) \
    __device__ __forceinline__ u32 shl_byte##B(u32 sh, u32 v) {     \
        u32 r; asm("v_lshlrev_b32_sdwa %0, %1, %2 dst_sel:DWORD dst_unused:UNUSED_PAD src0_sel:BYTE_" #B " src1_sel:BYTE_" #B \
                   : "=v"(r) : "v"(sh), "v"(v)); return r; }
SDWA_SHL(0) SDWA_SHL(1) SDWA_SHL(2)

template <int B> __device__ __forceinline__ u32 and_byte(u32 v, u32 m) {
    u32 r;
    if constexpr (B == 0) asm("v_and_b32_sdwa %0, %1, %2 dst_sel:DWORD dst_unused:UNUSED_PAD src0_sel:BYTE_0 src1_sel:DWORD" : "=v"(r) : "v"(v), "v"(m));
    if constexpr (B == 1) asm("v_and_b32_sdwa %0, %1, %2 dst_sel:DWORD dst_unused:UNUSED_PAD src0_sel:BYTE_1 src1_sel:DWORD" : "=v"(r) : "v"(v), "v"(m));
    if constexpr (B == 2) asm("v_and_b32_sdwa %0, %1, %2 dst_sel:DWORD dst_unused:UNUSED_PAD src0_sel:BYTE_2 src1_sel:DWORD" : "=v"(r) : "v"(v), "v"(m));
    if constexpr (B == 3) asm("v_and_b32_sdwa %0, %1, %2 dst_sel:DWORD dst_unused:UNUSED_PAD src0_sel:BYTE_3 src1_sel:DWORD" : "=v"(r) : "v"(v), "v"(m));
    return r;
}

__device__ __forceinline__ void store_input_bytes(char* slot, const u32 (&d)[8], int wave, int lane) {
    u32 bb[4];
#pragma unroll
    for (int j = 0; j < 4; ++j) {
        const u32 lowb = __builtin_amdgcn_perm(d[2 * j + 1], d[2 * j], 0x06040200u);
        const u32 higb = __builtin_amdgcn_perm(d[2 * j + 1], d[2 * j], 0x07050301u);
        bb[j] = (lowb & 0x7f7f7f7fu) | (higb & 0x80808080u);
    }
    st16_wt(slot + wave * 1024 + lane * 16, u32x4{bb[0], bb[1], bb[2], bb[3]});
}
constexpr u32 RAW_OFF = SLOT;
__device__ __forceinline__ u32 input_e0(u32x2 sample) {
    const u32 s2 = ((sample.x >> 7) & 0x00ff00ffu) + ((sample.y >> 7) & 0x00ff00ffu);
    const u32 tot = __builtin_amdgcn_readlane(wave_scan((s2 & 0xffffu) + (s2 >> 16)), 63);
    const u32 anchor = (tot + 504u) >> 8;
    return anchor < 12u ? 0u : (anchor > 252u ? 240u : anchor - 12u);
}
template <bool OUT>
__device__ __forceinline__ void encode_block(char* slot, const u32 (&d)[8], u32 e0, u32 ltbl, u32 lpriv, u32 ltot, int wave, int lane) {
    *(LAS u32x4*)(lpriv + lane * 16) = u32x4{0u, 0u, 0u, 0u};
    u32 en[16], in[16];
    typedef unsigned short u16x2 __attribute__((ext_vector_type(2)));
#pragma unroll
    for (int j = 0; j < 8; ++j) {
        u32 i0, i1;
        if (OUT) {
            const u16x2 sub = {OUT_E0 << 6, OUT_E0 << 6}, cap = {15 << 6, 15 << 6};
            const u16x2 t = __builtin_elementwise_min(__builtin_bit_cast(u16x2, (d[j] >> 1) & 0x3ff83ff8u) - sub, cap);
            const u32 x = __builtin_bit_cast(u32, t) | ((d[j] >> 5) & 0x04000400u);
            i0 = x & 0xffffu; i1 = x >> 16;
        } else {
            const u16x2 sub = {(unsigned short)(e0 << 7), (unsigned short)(e0 << 7)};
            const u32 x = __builtin_bit_cast(u32, __builtin_bit_cast(u16x2, d[j]) - sub);
            i0 = (x >> 3) & 0xff8u; i1 = (x >> 19) & 0xff8u;
        }
        const u32x2 a = *(const LAS u32x2*)(ltbl + i0), b = *(const LAS u32x2*)(ltbl + i1);
        en[2 * j] = a.x; in[2 * j] = a.y; en[2 * j + 1] = b.x; in[2 * j + 1] = b.y;
    }
    __builtin_amdgcn_sched_barrier(0);
    if (OUT) {
        u32 bb[4];
#pragma unroll
        for (int j = 0; j < 4; ++j) {
            const u32 lowb = __builtin_amdgcn_perm(d[2 * j + 1], d[2 * j], 0x06040200u);
            const u32 c01 = __builtin_amdgcn_perm(en[4 * j + 1], en[4 * j], 0x0c0c0703u);
            const u32 c23 = __builtin_amdgcn_perm(en[4 * j + 3], en[4 * j + 2], 0x07030c0cu);
            bb[j] = (lowb & 0x0f0f0f0fu) | c01 | c23;
        }
        constexpr u32 H = 0xf0f0f0f0u;
        char* p = slot + wave * 512 + lane * 8;
        st8_wt(p, u32x2{(bb[0] & H) | ((bb[1] >> 4) & ~H), (bb[2] & H) | ((bb[3] >> 4) & ~H)});
        st8_wt(p + 4096, u32x2{(bb[0] & ~H) | ((bb[1] << 4) & H), (bb[2] & ~H) | ((bb[3] << 4) & H)});
    }
    u32 f2 = 0, f3 = 0, f4[2] = {0, 0}, pos = 0, n4a = 0;
#pragma unroll
    for (int i = 0; i < 16; i += 2) {
        if (i == 8) { n4a = ubfe(pos, 16, 8); pos &= 0xff00ffffu; }
        const u32 pos1 = pos + in[i];
        f2 |= shl_byte0(pos, en[i]) | shl_byte0(pos1, en[i + 1]);
        f3 |= shl_byte1(pos, en[i]) | shl_byte1(pos1, en[i + 1]);
        f4[i >> 3] |= shl_byte2(pos, en[i]) | shl_byte2(pos1, en[i + 1]);
        pos = pos1 + in[i + 1];
    }
    const u32 n4 = n4a + ubfe(pos, 16, 8), ne = pos >> 24;
    const u32 ca = (pos & 0xffu) | ((pos << 8) & 0xff0000u), cb = n4 | (ne << 16);
    const u32 sa = wave_scan(ca), sb = wave_scan(cb);
    const u32 ta = __builtin_amdgcn_readlane(sa, 63), tb = __builtin_amdgcn_readlane(sb, 63);
    const u32 N2 = ta & 0xffffu, N3 = ta >> 16, NE = tb >> 16;
    const u32 escpos = (N2 + N3 + (tb & 0xffffu) + 15u) & ~15u;
    const u32 dw = (escpos + 16u * NE + 31u) >> 5;
    if (lane == 0) *(LAS u32*)(ltot + wave * 4) = dw <= 252u ? dw : 0x1000u;
    if (dw <= 252u) {
        const u32 xa = sa - ca, xb = sb - cb;
        if (OUT) lds_or_bits(lpriv, xa & 0xffffu, f2);
        else *(LAS u32*)(lpriv + lane * 4) = f2;
        lds_or_bits(lpriv, N2 + (xa >> 16), f3);
        const u32 p4 = N2 + N3 + (xb & 0xffffu);
        if (!__builtin_amdgcn_ballot_w64(n4 > 32u)) {
            lds_or_bits(lpriv, p4, f4[0] | (f4[1] << n4a));
        } else {
            lds_or_bits(lpriv, p4, f4[0]);
            lds_or_bits(lpriv, p4 + n4a, f4[1]);
        }
        if (NE) {
            u32 a = lpriv + (escpos >> 3) + (xb >> 16) * 2u;
#pragma unroll
            for (int j = 0; j < 4; ++j)
                if ((in[4 * j] | in[4 * j + 1] | in[4 * j + 2] | in[4 * j + 3]) >> 24) {
#pragma unroll
                    for (int i = 4 * j; i < 4 * j + 4; ++i)
                        if (in[i] >> 24) { *(LAS unsigned short*)a = (unsigned short)(d[i >> 1] >> (16 * (i & 1))); a += 2u; }
                }
        }
    }
}
__device__ __forceinline__ u32 encode_finish(char* slot, const u32 (&d)[8], u32 lpriv, u32 ltot, int wave, int lane) {
    const u32 mine = *(const LAS u32*)(ltot + (lane & 7) * 4);
    const u32 w0 = *(const LAS u32*)(lpriv + lane * 4), w1 = *(const LAS u32*)(lpriv + 256 + lane * 4);
    u32 inc = mine;
    inc += (u32)__builtin_amdgcn_update_dpp(0, (int)inc, 0x111, 0xf, 0xf, false);
    inc += (u32)__builtin_amdgcn_update_dpp(0, (int)inc, 0x112, 0xf, 0xf, false);
    inc += (u32)__builtin_amdgcn_update_dpp(0, (int)inc, 0x114, 0xf, 0xf, false);
    const u32 total = __builtin_amdgcn_readlane(inc, 7);
    if (total >= 0x1000u) {
        char* p = slot + RAW_OFF + wave * 1024 + lane * 16;
        st16_wt(p + 8192, u32x4{d[0], d[1], d[2], d[3]}); st16_wt(p, u32x4{d[4], d[5], d[6], d[7]});
        return F_RAW;
    }
    const u32 dw = __builtin_amdgcn_readlane(mine, wave), start = __builtin_amdgcn_readlane(inc, wave) - dw;
    char* p = slot + 8192 + 16 + (start + lane) * 4u;
    if ((u32)lane < dw) st4_wt(p, w0);
    if ((u32)lane + 64u < dw) st4_wt(p + 256, w1);
    if (dw > 128u) {
        if ((u32)lane + 128u < dw) st4_wt(p + 512, *(const LAS u32*)(lpriv + 512 + lane * 4));
        if ((u32)lane + 192u < dw) st4_wt(p + 768, *(const LAS u32*)(lpriv + 768 + lane * 4));
    }
    if (wave == 0 && lane < 8 && !(lane & 1)) st4_wt(slot + 8192 + lane * 2, (inc - mine) | (inc << 16));
    return (16u + total * 4u + 15u) & ~15u;
}

template <bool OUT, class F>
__device__ __forceinline__ void decode_block(u32 sl, u32 e0, u32 ltbl, int wave, int lane, u32x4& v0, u32x4& v1, F wait_low) {
    u32x4 B;
    if (OUT) { const u32x2 C = *(const LAS u32x2*)(sl + wave * 512 + lane * 8); B = u32x4{C.x, C.x << 4, C.y, C.y << 4}; }
    else B = lds_ld16(sl + wave * 1024 + lane * 16);
    const u32 hdr = __builtin_amdgcn_readfirstlane(*(const LAS unsigned short*)(sl + 8192 + wave * 2));
    const u32 base = sl + 8192 + 16 + (hdr > 2040u ? 2040u : hdr) * 4u;
    const u32 eadd = e0 * 0x00800080u;
    const u32 bb[4] = {B.x, B.y, B.z, B.w};
    u32 F2, N2, W2[4];
    if (OUT) {
        u32 c = 0;
#pragma unroll
        for (int j = 0; j < 4; ++j) {
            W2[j] = ((((bb[j] >> 4) & 0x0f0f0f0fu) + 0x0d0d0d0du) >> 3) & 0x02020202u;
            c += __builtin_popcount(W2[j]);
        }
        const u32 n2 = 2u * c, s2 = wave_scan(n2);
        N2 = __builtin_amdgcn_readlane(s2, 63);
        F2 = lds_get_bits(base, s2 - n2) & low_mask_even(n2);
    } else {
        F2 = *(const LAS u32*)(base + lane * 4);
        N2 = 2048u;
    }
    const u32 W3 = F2 & (F2 << 1) & 0xaaaaaaaau;
    const u32 n3 = 2u * __builtin_popcount(W3), s3 = wave_scan(n3);
    const u32 N3 = __builtin_amdgcn_readlane(s3, 63);
    const u32 F3 = lds_get_bits(base, N2 + s3 - n3) & low_mask_even(n3);
    const u32 K4 = F3 & (F3 >> 1) & 0x55555555u, W4 = K4 * 3u;
    const u32 n4 = 3u * __builtin_popcount(K4), s4 = wave_scan(n4);
    const u32 N4 = __builtin_amdgcn_readlane(s4, 63);
    const u32 P4 = N2 + N3 + s4 - n4;
    const LAS u32* q4 = (const LAS u32*)(base + (P4 >> 5) * 4u);
    const u32 q40 = q4[0], q41 = q4[1], q42 = q4[2];
    u32 F4 = __builtin_amdgcn_alignbit(q41, q40, P4 & 31u);
    const u32 F4h = __builtin_amdgcn_alignbit(q42, q41, P4 & 31u);
    u32 t[16], p2 = 0, p3 = 0, p4 = 0;
    const u32 cm = OUT ? 0xf0u : 0xc0u;
    static_for<16>([&](auto ii) INL {
        constexpr int i = decltype(ii)::value;
        if (i == 8) { F4 = __builtin_amdgcn_alignbit(F4h, F4, p4); p4 = 0; }
        const u32 c1 = and_byte<i & 3>(bb[i >> 2], cm);
        u32 c2, w3;
        if (OUT) {
            const u32 w2 = W2[i >> 2] >> (8 * (i & 3)); c2 = ubfe(F2, p2, w2); w3 = ubfe(W3, p2, w2); p2 += w2;
        }
        else { c2 = ubfe(F2, 2 * i, 2); w3 = ubfe(W3, 2 * i, 2); }
        const u32 c3 = ubfe(F3, p3, w3), w4 = ubfe(W4, p3, w3); p3 += w3;
        const u32 c4 = ubfe(F4, p4, w4); p4 += w4;
        t[i] = OUT ? (c1 + c2 + c3 + c4) << 1 : c1 + ((c2 + c3 + c4) << 1);
    });
    u32 o[8];
#pragma unroll
    for (int j = 0; j < 8; ++j) {
        typedef unsigned short u16x2 __attribute__((ext_vector_type(2)));
        const u16x2 pr = {*(const LAS unsigned short*)(ltbl + t[2 * j]), *(const LAS unsigned short*)(ltbl + t[2 * j + 1])};
        o[j] = __builtin_bit_cast(u32, pr) + (OUT ? 0u : eadd);
    }
    const u32 any = (o[0] | o[1] | o[2]) | (o[3] | o[4] | o[5]) | (o[6] | o[7]);
    if (__builtin_amdgcn_ballot_w64((any & 0x00010001u) != 0u)) {
        u32 c = 0;
#pragma unroll
        for (int j = 0; j < 8; ++j) c += o[j] & 0x00010001u;
        const u32 ne = (c & 0xffffu) + (c >> 16);
        u32 a = base + (((N2 + N3 + N4 + 15u) & ~15u) >> 3) + (wave_scan(ne) - ne) * 2u;
        const u32 amax = sl + SLOT - 4u;
#pragma unroll
        for (int j = 0; j < 8; ++j) {
            if (o[j] & 0x00010001u) {
                a = a < amax ? a : amax;
                if (o[j] & 1u) { o[j] = (o[j] & 0xffff0000u) | *(const LAS unsigned short*)a; a += 2u; }
                if (o[j] & 0x10000u) { o[j] = (o[j] & 0xffffu) | ((u32)*(const LAS unsigned short*)a << 16); a += 2u; }
            }
        }
    }
    u32 lb[4];
    if (OUT) {
        asm volatile("" : "+v"(o[0]), "+v"(o[1]), "+v"(o[2]), "+v"(o[3]), "+v"(o[4]), "+v"(o[5]), "+v"(o[6]), "+v"(o[7]));
        wait_low();
        const u32x2 R = *(const LAS u32x2*)(sl + 4096 + wave * 512 + lane * 8);
        lb[0] = R.x & 0x0f0f0f0fu; lb[1] = (R.x >> 4) & 0x0f0f0f0fu; lb[2] = R.y & 0x0f0f0f0fu; lb[3] = (R.y >> 4) & 0x0f0f0f0fu;
    } else {
#pragma unroll
        for (int j = 0; j < 4; ++j) lb[j] = bb[j] & 0x3f3f3f3fu;
    }
#pragma unroll
    for (int j = 0; j < 8; ++j) o[j] |= __builtin_amdgcn_perm(0u, lb[j >> 1], (j & 1) ? 0x0c030c02u : 0x0c010c00u);
    v0 = u32x4{o[0], o[1], o[2], o[3]};
    v1 = u32x4{o[4], o[5], o[6], o[7]};
}

__global__ void __launch_bounds__(THREADS, 1) fused_gemm_allgather(Params P) {
    const int lane = threadIdx.x & 63;
    const int wave = __builtin_amdgcn_readfirstlane(threadIdx.x >> 6);
    int b = blockIdx.x;
    {
        const int x = b & 7, y = b >> 3;
        if (x >= 4 && y >= 28 && b != 4 + 28 * 8) return;
        b = y >= 28 && x >= 4 ? NROLE : x < 4 ? (y >> 1) * 8 + 2 * x + (y & 1) : NGEMM + (x - 4) * 28 + y;
    }
    const int rank = P.rank;
    char* myws = P.myws;
    const char* px = P.x;
    const char* pw = P.w;
    asm volatile("" : "+s"(myws), "+s"(px), "+s"(pw));
    constexpr u32 lds0 = 0;
    GAS u32* const ctr = (GAS u32*)(myws + OFF_CTR + (size_t)b * 128);
    GAS u32* const tmo = (GAS u32*)(myws + OFF_TMO);

    if (b == NROLE) {
        if (wave) return;
        const u32 call = __builtin_amdgcn_readfirstlane(__hip_atomic_load(ctr, __ATOMIC_RELAXED, __HIP_MEMORY_SCOPE_AGENT)) + 1u;
        if (lane == 0) __hip_atomic_store(ctr, call, __ATOMIC_RELAXED, __HIP_MEMORY_SCOPE_AGENT);
        GAS u32* tk = (GAS u32*)(myws + OFF_TKT + (size_t)(lane < 7 ? lane : 0) * 256);
        u32 sum = 0, cnt = 0;
        const u64 t0 = __builtin_amdgcn_s_memrealtime();
        for (;;) {
            const u32 pos = __hip_atomic_load(tk, __ATOMIC_RELAXED, __HIP_MEMORY_SCOPE_AGENT) - (call - 1u) * TICKETS + 3 * PER_LINK;
            if (__any(pos >= OUT0)) break;
            if (__any(pos >= 7 * PER_LINK)) { sum += pos; ++cnt; }
            if (__builtin_amdgcn_s_memrealtime() - t0 > DEADLINE) { cnt = 0; break; }
            __builtin_amdgcn_s_sleep(8);
        }
        int eff = cnt ? -(int)((4u * sum) / cnt) : 0, mean = 0;
        for (int q = 0; q < 7; ++q) mean += __builtin_amdgcn_readlane(eff, q);
        eff -= mean / 7;
        const int kw = (eff + 3) / 4 + 2;
        const u32 keep = eff < 2 ? 0u : (kw > (int)RK ? RK : (u32)kw);
        u64 kv = (u64)(call & 0xfffffu) << 40;
        for (int q = 0; q < 7; ++q) kv |= (u64)(u32)__builtin_amdgcn_readlane(keep, q) << (5 * q);
        if (lane == 0) __hip_atomic_store((GAS u64*)(myws + OFF_KEEP), kv, __ATOMIC_RELAXED, __HIP_MEMORY_SCOPE_SYSTEM);
        u64 pk = 0;
        for (;;) {
            if (lane < 7 && eff < 0) pk = __hip_atomic_load((GAS u64*)(P.ws[(rank + 1 + lane) & 7] + OFF_KEEP), __ATOMIC_RELAXED, __HIP_MEMORY_SCOPE_SYSTEM);
            if (!__any(lane < 7 && eff < 0 && (u32)(pk >> 40) != (call & 0xfffffu))) break;
            if (__builtin_amdgcn_s_memrealtime() - t0 > DEADLINE) { pk = 0; break; }
            __builtin_amdgcn_s_sleep(8);
        }
        u32 kq = 0, eq = 0, closed = 0;
        for (int i = 0; i < 7 * (int)RK; ++i) {
            int hi = -0x7fffffff, qa = 0;
            for (int q = 0; q < 7; ++q) {
                const int e = __builtin_amdgcn_readlane(eff, q);
                if (!((closed >> q) & 1u) && e > hi) { hi = e; qa = q; }
            }
            if (hi <= 0) break;
            const u32 j = __builtin_amdgcn_readlane(kq, qa);
            const u32 has = (u32)(pk >> (5 * ((qa - lane - 1) & 7))) & 31u;
            const u32 cm = (u32)__builtin_amdgcn_ballot_w64(lane < 7 && lane != qa && !kq && eq < REMAX && j < has
                                                            && (u32)((qa - lane - 1) & 7) % RP != (u32)qa % RP);
            int lw = 0x7fffffff, qb = 0;
            for (int q = 0; q < 7; ++q) {
                const int e = __builtin_amdgcn_readlane(eff, q);
                if (((cm >> q) & 1u) && e < lw) { lw = e; qb = q; }
            }
            if (!cm || j >= RK || hi - lw < 8) { closed |= 1u << qa; continue; }
            const u32 eb = __builtin_amdgcn_readlane(eq, qb);
            u32 dn = (u32)(hi - lw) / 16u + 1u;
            dn = min(min(dn, 4u), min(min(RK, (u32)__builtin_amdgcn_readlane(has, qb)) - j, REMAX - eb));
            if ((u32)lane < dn)
                __hip_atomic_store((GAS unsigned char*)(myws + OFF_TKT + qb * 256 + 132 + eb + lane), (unsigned char)((qa << 5) | (j + lane)),
                                   __ATOMIC_RELAXED, __HIP_MEMORY_SCOPE_AGENT);
            closed |= 1u << qb;
            if (lane == qb) { eq += dn; eff += 4 * (int)dn; }
            if (lane == qa) { kq += dn; eff -= 4 * (int)dn; }
        }
        vm_wait0();
        if (lane < 7)
            __hip_atomic_store(tk + 32, kq | (eq << 5) | (keep << 10) | (call << 15), __ATOMIC_RELAXED, __HIP_MEMORY_SCOPE_AGENT);
        return;
    }
    if (__builtin_expect(b < NGEMM, 1)) {
        const int tj = b & 7, ti0 = b >> 4, half = (b >> 3) & 1;
        const int npass = pack_passes(b);
        constexpr int EBAR = 3;
        if (wave == 8) {
            u32 cv;
            char* wsl;
            asm volatile("global_load_dword %0, %2, off sc1\n\ts_nop 3\n\tglobal_load_dwordx2 %1, %3, %4 offset:%5"
                         : "=&v"(cv), "=&v"(wsl)
                         : "v"(ctr), "v"((lane & 7) * 8), "s"(__builtin_amdgcn_kernarg_segment_ptr()), "n"(offsetof(Params, ws)) : "memory");
            bar();
            put_enc_table<false>(L_ETIN, lane);
            const u32 call = landed_first<0>(cv) + 1u;
            asm volatile("" : "+v"(wsl));
            int isw, grp, lp, kb, seq;
            msg_of(b, isw, grp, lp, kb, seq);
            char* ef = wsl + OFF_FLAG + ((size_t)rank * NMSG + seq) * 128 + 68;
            u32 ev = seq < 2 * PER_LINK && lane < NR && lane != rank && (isw ? ((lane & 3) == grp) : ((lane >> 2) == grp)) ? call : 0u;
            asm volatile("" : "+v"(ef), "+v"(ev));
            bar();
            if (ev) st4_wt(ef, ev);
            if (lane == 0) __hip_atomic_store(ctr, call, __ATOMIC_RELAXED, __HIP_MEMORY_SCOPE_AGENT);
            for (int z = 0; z < npass; ++z) {
                if (z + 1 < npass) {
                    msg_of(pack_msg(b, z + 1), isw, grp, lp, kb, seq);
                    const char* sb = isw ? pw + ((size_t)kb * 64 * 4096 + (grp * 8 + lp) * 128) * 2 : px + ((size_t)(grp * 16 + lp) * 128 * 512 + kb * 64) * 2;
                    const u32 vo = isw ? (lane >> 4) * 8192 + (lane & 15) * 16 : (lane & 15) * 1024 + (lane >> 4) * 16;
                    const u32 sw = isw ? 65536u : 16384u, sh = isw ? 32768u : 64u;
#pragma unroll
                    for (int j = 0; j < 16; ++j) dma1s(sb + (j >> 1) * sw + (j & 1) * sh, vo, lds0 + (z + 1) * IMG + j * 1024);
                }
                bar();
                vm_wait0();
                bar();
                msg_of(pack_msg(b, z), isw, grp, lp, kb, seq);
                const u32 lo = lds_ld(L_TOT + 32);
                if (seq < (int)NEARF) {
                    if (lane == 0)
                        __hip_atomic_store((GAS u64*)(myws + OFF_PKIN + (size_t)((isw ? 256 + (grp * 8 + lp) * 8 : (grp * 16 + lp) * 8) + kb) * GSLOT + SFLAG),
                                           ((u64)call << 32) | lo, __ATOMIC_RELAXED, __HIP_MEMORY_SCOPE_SYSTEM);
                } else {
                    announce(myws, wsl, rank, seq, ((u64)call << 32) | lo, (isw ? 0x11u << grp : 0xfu << (4 * grp)) & ~(1u << rank),
                             isw ? 32 + grp * 8 + lp : grp * 16 + lp, lane);
                }
                if (lane == 0 && (isw ? (grp == (rank & 3)) : (grp == (rank >> 2))))
                    __hip_atomic_fetch_add((GAS u32*)(myws + OFF_CNT + (size_t)((isw ? 16 : 0) + lp) * 256 + rank * 4), 1u,
                                           __ATOMIC_RELAXED, __HIP_MEMORY_SCOPE_AGENT);
            }
            const u32 want = call * 8u;
            u32 next_known = 0u;
            for (int tile = 0; tile < 2; ++tile) {
                const char* cp = myws + OFF_CNT + (size_t)((lane & 8) ? 16 + tj : ti0 + 8 * tile) * 256 + (lane & 7) * 4;
                if (tile == 0) {
                    wait_counters((GAS u32*)(myws + OFF_CNT + (size_t)((lane & 1) ? 16 + tj : ti0) * 256 + rank * 4), want, tmo, call);
                    bar(); bar();
                    put_enc_table<true>(L_ETOUT, lane);
                }
                const u32 pfs = (rank + 1 + (b >> 4)) & 7;
                bool pf = tile == 0 && (b >> 3) < 14;
                auto prefetch = [&](u32 wmask) INL {
                    if (pf && ((wmask >> pfs) & 1u)) {
                        const char* p = myws + OFF_UW + ((size_t)tj * 64 + pfs * 8 + half * 4) * IMG + lane * 128;
#pragma unroll
                        for (int j = 0; j < 8; ++j) dma_dword(p + j * 8192, L_SCR + 256);
                        pf = false;
                    }
                };
                u32 known = next_known, wm8 = 0u, left = 0xffu & ~(1u << rank), order = 0u, cw = 0u;
                bool asked = false;
                auto look = [&]() INL {
                    const u32 mk = (u32)__builtin_amdgcn_ballot_w64(cw >= want);
                    wm8 = (mk >> 8) & 0xffu; known = mk & wm8;
                };
                for (int g = 1; g < 8; ++g) {
                    for (int i = 0; i < (g == 1 ? 8 - NST : 7); ++i) bar();
                    if (left) {
                        if (asked) { vm_wait0(); cw = lds_ld(L_SCR + lane * 4); look(); }
                        if (!(known & left)) {
                            const u64 t0 = __builtin_amdgcn_s_memrealtime();
                            for (u32 it = 1;; ++it) {
                                asm volatile("global_load_dword %0, %1, off sc0 sc1\n\ts_waitcnt vmcnt(0)" : "=&v"(cw) : "v"(cp) : "memory");
                                look();
                                if (known & left) break;
                                prefetch(wm8);
                                if ((it & 15u) == 0u
                                    && (__builtin_amdgcn_readfirstlane(__hip_atomic_load(tmo, __ATOMIC_RELAXED, __HIP_MEMORY_SCOPE_AGENT)) == call
                                        || __builtin_amdgcn_s_memrealtime() - t0 > DEADLINE)) {
                                    __hip_atomic_store(tmo, call, __ATOMIC_RELAXED, __HIP_MEMORY_SCOPE_AGENT);
                                    known = 0xffu;
                                    break;
                                }
                                __builtin_amdgcn_s_sleep(2);
                            }
                        }
                        int gg = g;
                        do {
                            const u32 av = known & left, r2 = ((av | (av << 8)) >> tj) & 0xffu;
                            const u32 p = (__builtin_ctz(r2) + tj) & 7u;
                            order |= p << (4 * (gg - 1)); left &= ~(1u << p); ++gg;
                        } while (left && (known & left) == left);
                        lds_st(L_SRC + tile * 4, left ? order : order | 0x80000000u);
                    }
                    bar();
                    if (tile == 1 && !left) break;
                    prefetch(wm8);
                    asked = left != 0u;
                    if (asked) dma_dword(cp, L_SCR);
                }
                if (tile == 0) {
                    const u32 ord = SCHED.ord[0][b];
                    for (int i = 0; i < 62 - (56 - NST) - 3; ++i) bar();
                    dma_dword(myws + OFF_CNT + (size_t)(ti0 + 8) * 256 + (lane & 7) * 4, L_SCR);
                    for (int i = 0; i < 4; ++i) bar();
                    vm_wait0();
                    next_known = (u32)__builtin_amdgcn_ballot_w64(lds_ld(L_SCR + lane * 4) >= want) & 0xffu;
                    if (!((next_known >> rank) & 1u))
                        wait_counters((GAS u32*)(myws + OFF_CNT + (size_t)(ti0 + 8) * 256 + rank * 4), want, tmo, call);
                    for (int i = 0; i < EBAR; ++i) bar();
                    const u32 lo = lds_ld(L_TOT + 32);
                    announce(myws, wsl, rank, OUT0 + ord, ((u64)call << 32) | lo, 0xffu & ~(1u << rank), ord >= OBATCH ? 64 + (int)(ord >> 3) : -1, lane);
                }
            }
            return;
        }
        u32 cv;
        char* wsl;
        asm volatile("global_load_dword %0, %2, off sc1\n\ts_nop 3\n\tglobal_load_dwordx2 %1, %3, %4 offset:%5"
                     : "=&v"(cv), "=&v"(wsl)
                     : "v"(ctr), "v"((lane & 7) * 8), "s"(__builtin_amdgcn_kernarg_segment_ptr()), "n"(offsetof(Params, ws)) : "memory");
        auto fetch = [&](int z, int rho) INL {
            int isw, grp, lp, kb, seq;
            msg_of(rho, isw, grp, lp, kb, seq);
            const u32 dst = lds0 + z * IMG + wave * 2048;
            if (!isw) {
                const char* s = px + ((size_t)((grp * 16 + lp) * 128 + wave * 16 + (lane & 15)) * 512 + kb * 64 + (lane >> 4) * 8) * 2;
                dma1(s, dst); dma1(s + 64, dst + 1024);
            } else {
                const char* s = pw + ((size_t)(kb * 64 + wave * 8 + (lane >> 4)) * 4096 + (grp * 8 + lp) * 128 + (lane & 15) * 8) * 2;
                dma1(s, dst); dma1(s + 4 * 8192, dst + 1024);
            }
        };
        fetch(0, b);
        const u32 call = landed_first<0>(cv) + 1u;
        int isw, grp, lp, kb, seq;
        char* slot;
        u32 d[8];
        u32x2 sample;
        auto take = [&](int z) INL {
            msg_of(pack_msg(b, z), isw, grp, lp, kb, seq);
            const u32 src = lds0 + z * IMG;
            if (!isw) {
                const u32x4 v0 = lds_ld16(src + wave * 2048 + lane * 16), v1 = lds_ld16(src + wave * 2048 + 1024 + lane * 16);
                d[0] = v0.x; d[1] = v0.y; d[2] = v0.z; d[3] = v0.w; d[4] = v1.x; d[5] = v1.y; d[6] = v1.z; d[7] = v1.w;
            } else {
                const u32 a = src + ((lane >> 4) * 8 + ((lane & 15) >> 2)) * 256 + (32 * (wave >> 1) + 8 * (lane & 3) + 4 * (wave & 1)) * 2;
                const u32x2 a0 = __builtin_bit_cast(u32x2, __builtin_amdgcn_ds_read_tr16_b64_v4i16((LAS s16x4*)(a)));
                const u32x2 a1 = __builtin_bit_cast(u32x2, __builtin_amdgcn_ds_read_tr16_b64_v4i16((LAS s16x4*)(a + 4 * 256)));
                const u32x2 a2 = __builtin_bit_cast(u32x2, __builtin_amdgcn_ds_read_tr16_b64_v4i16((LAS s16x4*)(a + 32 * 256)));
                const u32x2 a3 = __builtin_bit_cast(u32x2, __builtin_amdgcn_ds_read_tr16_b64_v4i16((LAS s16x4*)(a + 36 * 256)));
                d[0] = a0.x; d[1] = a0.y; d[2] = a1.x; d[3] = a1.y; d[4] = a2.x; d[5] = a2.y; d[6] = a3.x; d[7] = a3.y;
            }
            sample = *(const LAS u32x2*)(src + lane * 248);
            slot = myws + OFF_PKIN + (size_t)((isw ? 256 + (grp * 8 + lp) * 8 : (grp * 16 + lp) * 8) + kb) * GSLOT;
            store_input_bytes(slot, d, wave, lane);
        };
        bar();
        take(0);
        asm volatile("s_waitcnt vmcnt(0)" : "+v"(wsl) :: "memory");
        bar();
        for (int z = 0;;) {
            const u32 e0 = input_e0(sample);
            encode_block<false>(slot, d, e0, L_ETIN, L_VAR + wave * 1024, L_TOT, wave, lane);
            bar();
            const u32 lo = encode_finish(slot, d, L_VAR + wave * 1024, L_TOT, wave, lane) | (e0 << 16);
            const bool own = isw ? (grp == (rank & 3)) : (grp == (rank >> 2));
            if (own) {
                char* img = myws + (isw ? OFF_UW : OFF_UX) + (size_t)(lp * 64 + rank * 8 + kb) * IMG + wave * 2048 + lane * 16;
                st16_wt(img, u32x4{d[0], d[1], d[2], d[3]}); st16_wt(img + 1024, u32x4{d[4], d[5], d[6], d[7]});
            }
            if (threadIdx.x == 0) lds_st(L_TOT + 32, lo);
            vm_wait0();
            bar();
            if (++z == npass) break;
            take(z);
        }
        u32 ords = __builtin_amdgcn_readfirstlane((u32)SCHED.ord[0][b] | ((u32)SCHED.ord[1][b] << 8));
        asm volatile("s_waitcnt vmcnt(0)" : "+s"(ords) :: "memory");
        const int kh = wave >> 2, wn = wave & 1, wm = (wave >> 1) & 1;
        const char* const wb = myws + OFF_UW + (size_t)tj * 64 * IMG;
        const u32 voffx = wave * 1024 + lane * 16, voffw = wave * 2048 + lane * 16;
        const u32 xa = lds0 + (wm * 4 + kh) * 1024 + lane * 16;
        const u32 wa = lds0 + 8192 + (wn * 8 + kh) * 1024 + lane * 16;
#define SB __builtin_amdgcn_sched_barrier(0)
        u32 lo_last = 0;
        for (int tile = 0; tile < 2; ++tile) {
            const int ti = ti0 + 8 * tile;
            const char* const xb = myws + OFF_UX + (size_t)ti * 64 * IMG + half * 8192;
            f32x4 acc[4][2];
#pragma unroll
            for (int a = 0; a < 4; ++a) { acc[a][0] = f32x4{0.f, 0.f, 0.f, 0.f}; acc[a][1] = acc[a][0]; }
            bf16x8 wA[4], xA[2], wB[4], xB[2];
            auto mma = [&](int a, const bf16x8* wf, const bf16x8* xf) INL {
                acc[a][0] = __builtin_amdgcn_mfma_f32_16x16x32_bf16(wf[a], xf[0], acc[a][0], 0, 0, 0);
                acc[a][1] = __builtin_amdgcn_mfma_f32_16x16x32_bf16(wf[a], xf[1], acc[a][1], 0, 0, 0);
            };
            u32 buf = 0;
            u32 srcs = 0;
            const char* sx = xb + (size_t)rank * 8 * IMG;
            const char* sw = wb + (size_t)rank * 8 * IMG;
            auto step = [&](int kt, auto dma_tag, auto vm_tag, const bf16x8* wc, const bf16x8* xc, bf16x8* wx, bf16x8* xx) INL {
                constexpr bool DMA = decltype(dma_tag)::value != 0, EVEN = decltype(dma_tag)::value == 2;
                constexpr int VM = decltype(vm_tag)::value;
                const u32 d = buf;
                const u32 nb = buf + STG == NST * STG ? 0u : buf + STG;
                const u32 ra = xa + nb, rb = wa + nb;
                buf = nb;
                const u32 st = kt + NST;
                const char* gx = sx + (st & 7u) * IMG; const char* gw = sw + (st & 7u) * IMG;
                asm volatile("" : "+s"(gx), "+s"(gw));
                asm volatile("s_waitcnt lgkmcnt(0)\n\ts_waitcnt vmcnt(%0)\n\ts_barrier" :: "n"(VM) : "memory"); SB;
                if (EVEN && DMA && (st & 7u) == 0u) {
                    if ((int)srcs >= 0) srcs = __builtin_amdgcn_readfirstlane(lds_ld(L_SRC + tile * 4));
                    const u32 so = ((srcs >> (4u * ((st >> 3) - 1u))) & 7u) * (8u * IMG);
                    gx = sx = xb + so; gw = sw = wb + so;
                }
                SB;
                if (DMA) dma_a(gx, voffx, d + wave * 1024); SB;
                mma(0, wc, xc); SB;
                xx[0] = *(const LAS bf16x8*)(ra); wx[0] = *(const LAS bf16x8*)(rb); SB;
                mma(1, wc, xc); SB;
                if (DMA) dma_a(gw, voffw, d + 8192 + wave * 2048); SB;
                xx[1] = *(const LAS bf16x8*)(ra + 2048); wx[1] = *(const LAS bf16x8*)(rb + 2048); SB;
                mma(2, wc, xc); SB;
                wx[2] = *(const LAS bf16x8*)(rb + 4096); wx[3] = *(const LAS bf16x8*)(rb + 6144); SB;
                if (DMA) dma_b(gw, voffw); SB;
                mma(3, wc, xc); SB;
            };
            using F_ = std::integral_constant<int, 0>;
            auto fill = [&](const char* xp, int s0, int s1) INL {
                for (int st = s0; st < s1; ++st) {
                    const u32 kb = (u32)(rank * 8 + st);
                    dma_a(xp + (size_t)kb * IMG, voffx, st * STG + wave * 1024);
                    dma_a(wb + (size_t)kb * IMG, voffw, st * STG + 8192 + wave * 2048);
                    dma_b(wb + (size_t)kb * IMG, voffw);
                }
            };
            if (tile == 0) {
                bar();
                fill(xb, 0, NST);
                asm volatile("s_waitcnt vmcnt(%0)\n\ts_barrier" :: "n"((NST - 1) * 3) : "memory");
            } else {
                fill(xb, 4, NST);
            }
#pragma unroll
            for (int a = 0; a < 4; ++a) wA[a] = *(const LAS bf16x8*)(wa + a * 2048);
            xA[0] = *(const LAS bf16x8*)(xa); xA[1] = *(const LAS bf16x8*)(xa + 2048);
            static_assert(NST == 6, "tail below");
            using VMS = std::integral_constant<int, (NST - 2) * 3>;
            for (int kt = 0; kt < 64 - NST; kt += 2) {
                step(kt, std::integral_constant<int, 2>{}, VMS{}, wA, xA, wB, xB);
                step(kt + 1, std::integral_constant<int, 1>{}, VMS{}, wB, xB, wA, xA);
            }
            step(58, F_{}, std::integral_constant<int, 12>{}, wA, xA, wB, xB);
            step(59, F_{}, std::integral_constant<int, 9>{}, wB, xB, wA, xA);
            step(60, F_{}, std::integral_constant<int, 6>{}, wA, xA, wB, xB);
            step(61, F_{}, std::integral_constant<int, 3>{}, wB, xB, wA, xA);
            step(62, F_{}, std::integral_constant<int, 0>{}, wA, xA, wB, xB);
            u32 tv;
            asm volatile("global_load_dword %0, %1, off sc1" : "=v"(tv) : "v"(tmo) : "memory");
            bar();
            mma(0, wB, xB); mma(1, wB, xB); mma(2, wB, xB); mma(3, wB, xB);
            {
                const u32 wr = L_XCH + wave * 4096 + lane * 16;
#pragma unroll
                for (int a = 0; a < 4; ++a) *(LAS f32x4*)(wr + a * 1024) = kh ? acc[a][0] : acc[a][1];
            }
            bar();
            {
                const u32 rd = L_XCH + (wave ^ 4) * 4096 + lane * 16;
#pragma unroll
                for (int a = 0; a < 4; ++a) {
                    const f32x4 y = (kh ? acc[a][1] : acc[a][0]) + *(const LAS f32x4*)(rd + a * 1024);
                    d[a * 2] = pk_bf16(gelu(y.x), gelu(y.y));
                    d[a * 2 + 1] = pk_bf16(gelu(y.z), gelu(y.w));
                }
            }
            if (landed_first<0>(tv) == call) {
#pragma unroll
                for (int j = 0; j < 8; ++j) d[j] = 0x7fc07fc0u;
            }
            if (tile == 0) fill(xb + (size_t)8 * 64 * IMG, 0, 4);
            const u32 ord = (ords >> (8 * tile)) & 0xffu;
            char* const oslot = myws + OFF_PKOUT + (size_t)ord * GSLOT;
            encode_block<true>(oslot, d, 0u, L_ETOUT, L_VAR + wave * 1024, L_TOT, wave, lane);
            char* o = P.out + ((size_t)((rank >> 2) * 2048 + ti * 128 + half * 64 + wm * 32 + kh * 16 + (lane & 15)) * 4096
                               + (rank & 3) * 1024 + tj * 128 + wn * 64 + (lane >> 4) * 8) * 2;
            st16_out(o, u32x4{d[0], d[1], d[2], d[3]});
            st16_out(o + 64, u32x4{d[4], d[5], d[6], d[7]});
            bar();
            const u32 lo = encode_finish(oslot, d, L_VAR + wave * 1024, L_TOT, wave, lane);
            if (threadIdx.x == 0) lds_st(L_TOT + 32, lo);
            vm_wait0();
            bar();
            lo_last = lo;
        }
        if (wave == 0) {
            const u32 ord = ords >> 8;
            announce(myws, wsl, rank, OUT0 + ord, ((u64)call << 32) | lo_last, 0xffu & ~(1u << rank), ord >= OBATCH ? 64 + (int)(ord >> 3) : -1, lane);
        }
    } else {
        const int pu = b - NGEMM;
        const int u = pu / 7;
        const int src = (rank + 1 + pu % 7) & 7;
        const u32 call = __builtin_amdgcn_readfirstlane(__hip_atomic_load(ctr, __ATOMIC_RELAXED, __HIP_MEMORY_SCOPE_AGENT)) + 1u;
        if (wave == 0) lds_st(lds0 + L_CTL + lane * 4, 0u);
        if (wave == 1) put_dec_table<false>(L_DTBL, lane);
        vm_wait0();
        bar();
        if (threadIdx.x == 0) __hip_atomic_store(ctr, call, __ATOMIC_RELAXED, __HIP_MEMORY_SCOPE_AGENT);
        if (wave == 8) {
            __builtin_amdgcn_s_setprio(3);
            vm_wait0();
            const char* const rws = P.ws[src];
            const char* const fbase = myws + OFF_FLAG + (size_t)src * NMSG * 128;
            char* const tkt = myws + OFF_TKT + (size_t)(pu % 7) * 256;
            const u32 tbase = (call - 1u) * TICKETS - 3 * PER_LINK;
            u32 m = u, n = 0, np = 0;
            u32 org = src, cpy = 0u;
            u32 plan = 0u;
            u32 fl_lo = 0u, fl_hi = 0u;
            u32 pub = 0u;
            bool dead = false;
            auto publish = [&](u32 k) INL {
                if (np > pub + k) { pub = np - k; lds_st(lds0 + L_FULL, pub); }
            };
            auto drain = [&](auto top) INL {
                constexpr int TOP = decltype(top)::value;
                static_for<TOP + 1>([&](auto i) INL {
                    constexpr int K = TOP - decltype(i)::value;
                    asm volatile("s_waitcnt vmcnt(%0)" :: "n"(K) : "memory");
                    publish(K);
                });
            };
            auto poll = [&](const char* fp) INL {
                const u64 t0 = __builtin_amdgcn_s_memrealtime();
                for (u32 it = 1;; ++it) {
                    u32x2 fl;
                    asm volatile("global_load_dwordx2 %0, %1, off sc0 sc1\n\ts_waitcnt vmcnt(0)" : "=&v"(fl) : "v"(fp) : "memory");
                    fl_lo = __builtin_amdgcn_readfirstlane(fl.x); fl_hi = __builtin_amdgcn_readfirstlane(fl.y);
                    if (fl_hi >= call) break;
                    if ((it & 15u) == 0u
                        && (__builtin_amdgcn_readfirstlane(__hip_atomic_load(tmo, __ATOMIC_RELAXED, __HIP_MEMORY_SCOPE_AGENT)) == call
                            || __builtin_amdgcn_s_memrealtime() - t0 > DEADLINE)) {
                        __hip_atomic_store(tmo, call, __ATOMIC_RELAXED, __HIP_MEMORY_SCOPE_AGENT);
                        dead = true;
                        break;
                    }
                    __builtin_amdgcn_s_sleep(1);
                }
            };
            auto remote_slot = [&](u32 mm) INL {
                const u32 t = mm / PER_LINK, v = mm % PER_LINK;
                size_t moff;
                if (t < IN_ROUNDS) {
                    const u32 kb = v & 7;
                    u32 mid;
                    if (t < 8) mid = v >= 8 ? ((rank >> 2) * 16 + t) * 8 + kb : 256 + ((rank & 3) * 8 + t) * 8 + kb;
                    else mid = ((rank >> 2) * 16 + 8 + 2 * (t - 8) + (v >> 3)) * 8 + kb;
                    moff = OFF_PKIN + (size_t)mid * GSLOT;
                } else if (cpy) {
                    moff = OFF_RLY + (size_t)(((org - src - 1u) & 7u) * RN + mm - RW0) * GSLOT;
                } else {
                    moff = OFF_PKOUT + (size_t)(mm - OUT0) * GSLOT;
                }
                return rws + moff;
            };
            auto flag_of = [&](u32 mm) INL { return mm < NEARF || cpy || mm - OUT0 < OBATCH ? remote_slot(mm) + SFLAG : fbase + (mm & ~7u) * 128 + (mm & 7u) * 8; };
            auto resolve = [&](u32 pos) INL {
                m = pos;
                if (pos < RW0) return;
                if (!plan) {
                    plan = __builtin_amdgcn_readfirstlane(lds_ld(lds0 + L_PLAN + 32));
                    if (!plan) {
                        const u64 t0 = __builtin_amdgcn_s_memrealtime();
                        while (!(plan = fetch_plan(tkt, call, lane))) {
                            if (__builtin_amdgcn_s_memrealtime() - t0 > DEADLINE) {
                                __hip_atomic_store(tmo, call, __ATOMIC_RELAXED, __HIP_MEMORY_SCOPE_AGENT);
                                dead = true; plan = 0x8000u;
                                break;
                            }
                            __builtin_amdgcn_s_sleep(2);
                        }
                    }
                }
                const u32 kk = plan & 31u, ee = (plan >> 5) & 31u, x = pos - RW0, cls = pu % 7 % RP;
                m = x < (RP - 1u) * kk ? RW0 + x / (RP - 1u) * RP + x % (RP - 1u) + (x % (RP - 1u) >= cls) : pos + kk;
                if (m >= NMSG + ee) {
                    m -= ee;
                    if (m == NMSG + PER_LINK - 1 && lane == 0)
                        __hip_atomic_fetch_add((GAS u32*)tkt, REMAX + 1 + kk - ee, __ATOMIC_RELAXED, __HIP_MEMORY_SCOPE_AGENT);
                } else if (m >= NMSG) {
                    const u32 z = m - NMSG, w = (__builtin_amdgcn_readfirstlane(lds_ld(lds0 + L_PLAN + (z & ~3u))) >> (8u * (z & 3u))) & 0xffu;
                    const u32 ql = (w >> 5) < 7u ? w >> 5 : 0u, j = (w & 31u) < RK ? w & 31u : 0u;
                    org = (rank + 1u + ql) & 7u; cpy = 1u;
                    m = RW0 + RP * j + ql % RP;
                }
            };
            u32 f2_lo = 0u, f2_hi = 0u, f3_lo = 0u, f3_hi = 0u;
            {
                const char* fp = fbase + m * 128 + 68;
                constexpr u32 scr = L_RING + (NSLOT - 1) * SLOT;
                u32 it = 0;
                for (; it < 3u; ++it) { dma_dword(fp, scr + it * 256); __builtin_amdgcn_s_sleep(4); }
                for (; it < 256u; ++it) {
                    dma_dword(fp, scr + (it & 3u) * 256);
                    asm volatile("s_waitcnt vmcnt(3)" ::: "memory");
                    if (__builtin_amdgcn_readfirstlane(lds_ld(scr + ((it + 1u) & 3u) * 256)) >= call) break;
                }
                if (it == 256u) poll(fbase + m * 128 + 64);
            }
            fl_hi = 0u;
            if (!dead) {
                u32 e2;
                asm volatile("global_load_dword %0, %1, off offset:%2 sc0 sc1" : "=v"(e2) : "v"(fbase + m * 128), "n"(PER_LINK * 128 + 68) : "memory");
                const char* sp = remote_slot(m) + lane * 16;
#pragma unroll
                for (int k = 0; k < 8; ++k) {
                    dma1_sys(sp + k * 1024, lds0 + L_RING + k * 1024);
                    if (k >= WIN - 1) asm volatile("s_waitcnt vmcnt(%0)" :: "n"(WIN - 1) : "memory");
                }
                np = 8u;
                if (landed_first<WIN - 1>(e2) < call) { poll(fbase + (m + PER_LINK) * 128 + 64); fl_hi = 0u; }
            }
            if (!dead) {
                u32x2 fa, fb;
                asm volatile("global_load_dwordx2 %0, %2, off sc0 sc1\n\tglobal_load_dwordx2 %1, %3, off sc0 sc1"
                             : "=&v"(fa), "=&v"(fb) : "v"(flag_of(m)), "v"(flag_of(m + PER_LINK)) : "memory");
                const char* sp = remote_slot(m + PER_LINK) + lane * 16;
#pragma unroll
                for (int k = 0; k < WIN - 1; ++k) {
                    dma1_sys(sp + k * 1024, lds0 + L_RING + SLOT + k * 1024);
                    if (k < WIN - 2) asm volatile("s_waitcnt vmcnt(%0)" :: "n"(WIN + 1) : "memory");
                }
                fl_lo = landed_first<WIN - 1>(fa.x); fl_hi = landed_first<WIN - 1>(fa.y);
                f2_lo = landed_first<WIN - 1>(fb.x); f2_hi = landed_first<WIN - 1>(fb.y);
                np += WIN - 1;
            }
            for (;; ++n) {
                const u32 s = n % NSLOT;
                if (m < NMSG && !dead && fl_hi < call) {
                    drain(std::integral_constant<int, WIN - 1>{});
                    poll(flag_of(m));
                }
                if (n >= NSLOT) {
                    const u32 need = 8u * (n / NSLOT);
                    if (__builtin_amdgcn_readfirstlane(lds_ld(lds0 + L_FREE + s * 4)) < need) {
                        drain(std::integral_constant<int, WIN - 1>{});
                        while (__builtin_amdgcn_readfirstlane(lds_ld(lds0 + L_FREE + s * 4)) < need) __builtin_amdgcn_s_sleep(1);
                    }
                }
                if (m >= NMSG) {
                    lds_st(lds0 + L_MSG + s * 4, NMSG);
                    lds_st(lds0 + L_DESC, n + 1);
                    break;
                }
                const u32 lo = dead ? F_BAD : fl_lo;
                lds_st(lds0 + L_LEN + s * 4, lo);
                const u32 kc = !cpy && !dead && m - RW0 < RP * ((plan >> 10) & 31u) && (m - RW0) % RP != pu % 7 % RP;
                lds_st(lds0 + L_MSG + s * 4, m | (org << 9) | (kc << 12) | (np << 16));
                lds_st(lds0 + L_DESC, n + 1);
                const u32 vb = var_bytes(lo), nv = (vb + 1023u) >> 10;
                const char* sp = remote_slot(m) + ((lo & F_RAW) ? RAW_OFF : 0u) + lane * 16;
                if (n < 2) {
                    if (!dead) {
                        if (lo & F_RAW) vm_wait0();
                        for (u32 o = (lo & F_RAW) ? 0u : 8192u; o < 8192u + vb; o += 1024u) {
                            dma1_sys_part(sp + o, lds0 + L_RING + (n ? SLOT : 0u) + o, 8192u + vb - o);
                            ++np;
                            asm volatile("s_waitcnt vmcnt(%0)" :: "n"(WIN - 1) : "memory");
                            publish(WIN - 1);
                        }
                        if (n == 0) {
                            u32x2 fc;
                            asm volatile("global_load_dwordx2 %0, %1, off sc0 sc1" : "=v"(fc) : "v"(flag_of(m + 2 * PER_LINK)) : "memory");
                            sp = remote_slot(m + PER_LINK) + lane * 16;
#pragma unroll
                            for (int k = WIN - 1; k < 8; ++k) {
                                dma1_sys(sp + k * 1024, lds0 + L_RING + SLOT + k * 1024);
                                ++np;
                                asm volatile("s_waitcnt vmcnt(%0)" :: "n"(WIN - 1) : "memory");
                                publish(WIN - 1);
                            }
                            f3_lo = landed_first<WIN - 1>(fc.x); f3_hi = landed_first<WIN - 1>(fc.y);
                        }
                    }
                    m += PER_LINK; fl_lo = f2_lo; fl_hi = f2_hi; f2_lo = f3_lo; f2_hi = f3_hi;
                    continue;
                }
                u32 nt = 1u;
                u64 keep;
                asm volatile("s_mov_b64 %1, exec\n\ts_mov_b64 exec, 1\n\tglobal_atomic_add %0, %2, %0, off sc0\n\ts_mov_b64 exec, %1"
                             : "+v"(nt), "=&s"(keep) : "v"(tkt) : "memory");
                if (dead) {
                    resolve(landed_first<0>(nt) - tbase);
                    publish(0);
                    continue;
                }
                const u32 d = __builtin_amdgcn_readfirstlane(lds0 + L_RING + s * SLOT);
                asm volatile("s_waitcnt vmcnt(%0)" :: "n"(WIN) : "memory");
                publish(WIN - 1);
                {
                    const u32 o = 8192u + (nv - 1u) * 1024u;
                    dma1_sys_part(sp + o, d + o, 8192u + vb - o);
                    ++np;
                }
                u32 left = nv + 7u, o = nv > 1u ? 8192u + (nv - 2u) * 1024u : 0u;
                auto piece = [&]() INL {
                    dma1_sys(sp + o, d + o);
                    o = o > 8192u ? o - 1024u : (o == 8192u ? 0u : o + 1024u);
                    ++np; --left;
                };
#pragma unroll
                for (int k = 0; k < WIN - 1; ++k) {
                    asm volatile("s_waitcnt vmcnt(%0)" :: "n"(WIN) : "memory");
                    publish(WIN - 1);
                    piece();
                }
                const u32 pos = landed_first<WIN - 1>(nt) - tbase;
                publish(WIN - 1);
                piece();
                resolve(pos);
                if (m >= NMSG) {
                    while (left) {
                        asm volatile("s_waitcnt vmcnt(%0)" :: "n"(WIN - 1) : "memory");
                        publish(WIN - 1);
                        piece();
                    }
                    drain(std::integral_constant<int, WIN - 1>{});
                } else {
                    u32x2 fn;
                    asm volatile("global_load_dwordx2 %0, %1, off sc0 sc1" : "=v"(fn) : "v"(flag_of(m)) : "memory");
#pragma unroll
                    for (int k = 0; k < WIN - 1; ++k) {
                        asm volatile("s_waitcnt vmcnt(%0)" :: "n"(WIN) : "memory");
                        publish(WIN - 1);
                        piece();
                    }
                    fl_lo = landed_first<WIN - 1>(fn.x); fl_hi = landed_first<WIN - 1>(fn.y);
                    publish(WIN - 1);
                    piece();
                    if (fl_hi < call && left >= 1u) {
                        while (left > 1u) {
                            asm volatile("s_waitcnt vmcnt(%0)" :: "n"(WIN - 1) : "memory");
                            publish(WIN - 1);
                            piece();
                        }
                        u32x2 fr;
                        asm volatile("global_load_dwordx2 %0, %1, off sc0 sc1" : "=v"(fr) : "v"(flag_of(m)) : "memory");
                        asm volatile("s_waitcnt vmcnt(%0)" :: "n"(WIN) : "memory");
                        publish(WIN - 1);
                        piece();
                        asm volatile("s_waitcnt vmcnt(3)" ::: "memory"); publish(2);
                        asm volatile("s_waitcnt vmcnt(2)" ::: "memory"); publish(1);
                        fl_lo = landed_first<1>(fr.x); fl_hi = landed_first<1>(fr.y);
                    }
                    while (left) {
                        asm volatile("s_waitcnt vmcnt(%0)" :: "n"(WIN - 1) : "memory");
                        publish(WIN - 1);
                        piece();
                    }
                }
            }
        } else {
            if (wave == 2) {
                put_dec_table<true>(L_DTBL + 256, lane);
                *(LAS u32*)(L_UNORD + lane * 4) = ((const u32*)SCHED.unord)[lane];
                lgkm_wait0();
                lds_st(lds0 + L_TRDY, 1u);
            }
            bool tables = false, plan = false;
            char* pend = nullptr;
            u32 pend_n = 0, pend_lo = 0;
            auto report = [&]() INL {
                if (!pend) return;
                vm_wait0();
                u32 old = 0;
                if (lane == 0) {
                    asm volatile("ds_add_rtn_u32 %0, %1, %2\n\ts_waitcnt lgkmcnt(0)" : "=&v"(old) : "v"(lds0 + L_DONE + pend_n % 12u * 4), "v"(1u) : "memory");
                    if ((old & 7u) == 7u) {
                        if (pend_lo) __hip_atomic_store((GAS u64*)pend, ((u64)call << 32) | pend_lo, __ATOMIC_RELAXED, __HIP_MEMORY_SCOPE_SYSTEM);
                        else __hip_atomic_fetch_add((GAS u32*)pend, 1u, __ATOMIC_RELAXED, __HIP_MEMORY_SCOPE_AGENT);
                    }
                }
                pend = nullptr;
            };
            for (u32 n = 0;; ++n) {
                const u32 s = n % NSLOT;
                if (__builtin_amdgcn_readfirstlane(lds_ld(lds0 + L_DESC)) <= n) {
                    report();
                    while (__builtin_amdgcn_readfirstlane(lds_ld(lds0 + L_DESC)) <= n) __builtin_amdgcn_s_sleep(1);
                }
                const u32 md = __builtin_amdgcn_readfirstlane(lds_ld(lds0 + L_MSG + s * 4)), m = md & 0x1ffu, org = (md >> 9) & 7u;
                if (m >= NMSG) break;
                if (wave == 1 && m >= OUT0 && !plan) plan = fetch_plan(myws + OFF_TKT + (size_t)(pu % 7) * 256, call, lane);
                const u32 t = m / PER_LINK, v = m % PER_LINK;
                const u32 lo = __builtin_amdgcn_readfirstlane(lds_ld(lds0 + L_LEN + s * 4));
                const u32 sl = lds0 + L_RING + s * SLOT;
                if (m >= OUT0 && !tables) {
                    while (!__builtin_amdgcn_readfirstlane(lds_ld(lds0 + L_TRDY))) __builtin_amdgcn_s_sleep(1);
                    tables = true;
                }
                const u32 nv = (var_bytes(lo) + 1023u) >> 10;
                const bool nib = m >= OUT0 && !(lo & F_RAW) && !(md & 0x1000u);
                const u32 need = (md >> 16) + (n >= 2 ? nv + 1u + (nib ? wave >> 1 : (m >= OUT0 && !(lo & F_RAW) ? 4 + (wave >> 1) > wave ? 4 + (wave >> 1) : wave : wave))
                                                       : ((lo & F_RAW) ? 16u : nv));
                const u32 need2 = (md >> 16) + nv + 5u + (wave >> 1);
                if (!(lo & F_BAD) && __builtin_amdgcn_readfirstlane(lds_ld(lds0 + L_FULL)) < need) {
                    report();
                    while (__builtin_amdgcn_readfirstlane(lds_ld(lds0 + L_FULL)) < need) __builtin_amdgcn_s_sleep(1);
                }
                auto wait_low = [&]() INL {
                    while (__builtin_amdgcn_readfirstlane(lds_ld(lds0 + L_FULL)) < need2) __builtin_amdgcn_s_sleep(1);
                };
                const bool keep = md & 0x1000u;
                char* const cslot = myws + OFF_RLY + (size_t)((pu % 7) * RN + m - RW0) * GSLOT;
                if (keep) {
                    report();
                    char* p = cslot + ((lo & F_RAW) ? RAW_OFF : 0u) + wave * 1024 + lane * 16;
                    st16_wt(p, lds_ld16(sl + wave * 1024 + lane * 16));
                    if ((u32)(wave * 1024 + lane * 16) < var_bytes(lo)) st16_wt(p + 8192, lds_ld16(sl + 8192 + wave * 1024 + lane * 16));
                }
                u32x4 v0, v1;
                if (lo & F_BAD) {
                    v0 = u32x4{0x7fc07fc0u, 0x7fc07fc0u, 0x7fc07fc0u, 0x7fc07fc0u}; v1 = v0;
                } else if (lo & F_RAW) {
                    v0 = lds_ld16(sl + 8192 + wave * 1024 + lane * 16);
                    v1 = lds_ld16(sl + wave * 1024 + lane * 16);
                } else if (t < IN_ROUNDS) {
                    decode_block<false>(sl, (lo >> 16) & 0xffu, L_DTBL, wave, lane, v0, v1, wait_low);
                } else {
                    decode_block<true>(sl, 0u, L_DTBL + 256, wave, lane, v0, v1, wait_low);
                }
                asm volatile("s_waitcnt lgkmcnt(0)" : "+v"(v0), "+v"(v1) :: "memory");
                if (lane == 0) asm volatile("ds_add_u32 %0, %1" :: "v"(lds0 + L_FREE + s * 4), "v"(1u) : "memory");
                if (t < IN_ROUNDS) {
                    const u32 kb = v & 7;
                    const bool isw = t < 8 && v < 8;
                    const u32 lp = t < 8 ? t : 8 + 2 * (t - 8) + (v >> 3);
                    char* img = myws + (isw ? OFF_UW : OFF_UX) + (size_t)(lp * 64 + src * 8 + kb) * IMG + wave * 2048 + lane * 16;
                    report();
                    st16_wt(img, v0); st16_wt(img + 1024, v1);
                    pend = myws + OFF_CNT + (size_t)((isw ? 16 : 0) + lp) * 256 + src * 4; pend_n = n; pend_lo = 0u;
                } else {
                    const int g = __builtin_amdgcn_readfirstlane(*(const LAS unsigned char*)(L_UNORD + m - OUT0));
                    const int kh = wave >> 2, wn = wave & 1, wm = (wave >> 1) & 1;
                    const int ti = ((g >> 4) & 7) + 8 * (g >> 7), hf = (g >> 3) & 1, tj = g & 7;
                    report();
                    if (keep) { pend = cslot + SFLAG; pend_n = n; pend_lo = lo; }
                    char* op = P.out + ((size_t)((org >> 2) * 2048 + ti * 128 + hf * 64 + wm * 32 + kh * 16 + (lane & 15)) * 4096
                                        + (org & 3) * 1024 + tj * 128 + wn * 64 + (lane >> 4) * 8) * 2;
                    st16_out(op, v0); st16_out(op + 64, v1);
                }
            }
            report();
        }
    }
}

extern "C" void kernel_launch(int rank, int nranks, ncclComm_t comm,
                              hipStream_t stream,
                              void* const* d_in, const int* in_sizes, int n_in,
                              void* d_out, int out_size,
                              const int* in_dtypes, int out_dtype,
                              void* const* d_out_all,
                              void* const* d_ws_all, size_t ws_bytes) {
    (void)comm; (void)in_dtypes; (void)out_dtype; (void)d_out_all;
    if (nranks != NR || (unsigned)rank >= (unsigned)NR || n_in != 2 || in_sizes[0] != 4096 * 512 || in_sizes[1] != 512 * 4096
        || out_size != 4096 * 4096 || ws_bytes < WS_NEED) {
        fprintf(stderr, "kernel_launch: unexpected problem shape\n");
        return;
    }
    static bool attr_set[NR] = {};
    if (!attr_set[rank]) {
        HIP_CHECK(hipFuncSetAttribute((const void*)fused_gemm_allgather, hipFuncAttributeMaxDynamicSharedMemorySize, LDS_BYTES));
        attr_set[rank] = true;
    }
    Params P;
    for (int p = 0; p < NR; ++p) P.ws[p] = (char*)d_ws_all[p];
    P.x = (const char*)d_in[0];
    P.w = (const char*)d_in[1];
    P.out = (char*)d_out;
    P.myws = P.ws[rank];
    P.rank = rank;
    P.pad = 0;
    hipLaunchKernelGGL(fused_gemm_allgather, dim3(GRID), dim3(THREADS), LDS_BYTES, stream, P);
    HIP_CHECK(hipGetLastError());
}
```
